```python
import numpy as np

kernel_name = "gpu_gemm_ar_m4096_k4096_n16384_f32_q8"


ABSMAX_RTOL = 0.02

COMPUTE_DTYPE = "float32"
EPILOGUE = "quant_int8"
IN_QUANT = None
OUT_QUANT = "int8"
IN_DTYPES = {"x": "float32", "w_mat": "float32"}
WORLD_SIZE = 8

def setup_inputs():

    x = np.random.randn(4096, 4096).astype(np.float32)
    w_mat = (np.random.randn(4096, 16384) / np.sqrt(4096)).astype(np.float32)
    return {"x": x, "w_mat": w_mat}

def _quant_dequant(y, amax):
    scale = amax / 127.0
    q = np.clip(np.round(y / scale), -127.0, 127.0)
    return (q * scale).astype(np.float32)
def _epilogue(y):
    return _quant_dequant(y, np.abs(y).max())

def reference(x, w_mat):
    y = (x @ w_mat).astype(np.float32)
    return _epilogue(y)
```

```cpp
#include <hip/hip_runtime.h>
#include <rccl/rccl.h>
#include <cstdio>
#include <cstdint>

#define STAMPS 0

#define LAS __attribute__((address_space(3)))
typedef unsigned u32;
typedef unsigned long long u64;
typedef u32 u32x4 __attribute__((ext_vector_type(4)));
typedef u32 u32x2 __attribute__((ext_vector_type(2)));
typedef float f32x4 __attribute__((ext_vector_type(4)));
typedef short bf16x8 __attribute__((ext_vector_type(8)));

constexpr int NR = 8, GM = 4096, GN = 16384, KS = 512, NS = 2048;
constexpr int GRID = 256, BLOCK = 512;
constexpr int PANEL = 32768;
constexpr int PSLOT = 40960;
constexpr int OSLOT = 8320;
constexpr int MAXEXC = 13;
constexpr size_t W_TICKET = 0, W_TMO = 128;
constexpr size_t W_ACNT = 4096;
constexpr size_t W_BCNT = 8192;
constexpr size_t W_AMAX = 16384;
constexpr size_t W_OFLAG = 32768;
constexpr size_t W_HDR = 65536;
constexpr size_t W_DBG = 512 << 10;
constexpr size_t W_A = 1ull << 20;
constexpr size_t W_B = 33ull << 20;
constexpr size_t W_PX = 49ull << 20;
constexpr size_t W_PW = 54ull << 20;
constexpr size_t W_PO = 74ull << 20;
constexpr size_t W_END = 84ull << 20;
constexpr u64 DEADLINE = 40000000ull;

#define NPW 8
#define PW1 4
#define PW2 2

constexpr int LDS_STAGE = 131072;
constexpr int LDS_MISC = LDS_STAGE;
constexpr int LDS_BYTES = LDS_STAGE + 4096;

struct Params {
    const float* x;
    const float* w;
    signed char* out;
    char* ws;
    char* const* ws_all;
    int rank;
    int pad;
};

#define WAIT_V(n) asm volatile("s_waitcnt vmcnt(" #n ")" ::: "memory")
#define WAIT_L(n) asm volatile("s_waitcnt lgkmcnt(" #n ")" ::: "memory")
#define BAR __builtin_amdgcn_s_barrier()
#define SCHED __builtin_amdgcn_sched_barrier(0)
#define CFENCE asm volatile("" ::: "memory")

template <int OFF> __device__ __forceinline__ void st16_sys(void* p, u32x4 x) {
    asm volatile("global_store_dwordx4 %0, %1, off offset:%2 sc0 sc1\n\ts_nop 1" :: "v"(p), "v"(x), "i"(OFF) : "memory");
}
template <int OFF> __device__ __forceinline__ void st16_plain(void* p, u32x4 x) {
    asm volatile("global_store_dwordx4 %0, %1, off offset:%2\n\ts_nop 1" :: "v"(p), "v"(x), "i"(OFF) : "memory");
}
template <int OFF> __device__ __forceinline__ void ld16_sys(u32x4& x, const void* p) {
    asm volatile("global_load_dwordx4 %0, %1, off offset:%2 sc0 sc1" : "=v"(x) : "v"(p), "i"(OFF) : "memory");
}
__device__ __forceinline__ void ld8_sys(u32x2& x, const void* p) {
    asm volatile("global_load_dwordx2 %0, %1, off sc0 sc1" : "=v"(x) : "v"(p) : "memory");
}
__device__ __forceinline__ void ld2_sys(u32& x, const void* p) {
    asm volatile("global_load_ushort %0, %1, off sc0 sc1" : "=v"(x) : "v"(p) : "memory");
}
__device__ __forceinline__ void st8_sys(void* p, u32x2 x) {
    asm volatile("global_store_dwordx2 %0, %1, off sc0 sc1\n\ts_nop 1" :: "v"(p), "v"(x) : "memory");
}
__device__ __forceinline__ void st4_sys(void* p, u32 x) {
    asm volatile("global_store_dword %0, %1, off sc0 sc1\n\ts_nop 1" :: "v"(p), "v"(x) : "memory");
}
__device__ __forceinline__ void add4_agent(void* p, u32 x) {
    asm volatile("global_atomic_add %0, %1, off sc1\n\ts_nop 1" :: "v"(p), "v"(x) : "memory");
}
__device__ __forceinline__ u32 ld4_wait(const void* p) {
    u32 x; asm volatile("global_load_dword %0, %1, off sc0 sc1\n\ts_waitcnt vmcnt(0)" : "=&v"(x) : "v"(p) : "memory"); return x;
}
__device__ __forceinline__ u32x2 ld8_wait(const void* p) {
    u32x2 x; asm volatile("global_load_dwordx2 %0, %1, off sc0 sc1\n\ts_waitcnt vmcnt(0)" : "=&v"(x) : "v"(p) : "memory"); return x;
}
__device__ __forceinline__ u32x4 ld16_wait(const void* p) {
    u32x4 x; asm volatile("global_load_dwordx4 %0, %1, off sc0 sc1\n\ts_waitcnt vmcnt(0)" : "=&v"(x) : "v"(p) : "memory"); return x;
}
__device__ __forceinline__ u32 cvt_pk_bf16(float lo, float hi) {
    u32 r; asm("v_cvt_pk_bf16_f32 %0, %1, %2" : "=v"(r) : "v"(lo), "v"(hi)); return r;
}
__device__ __forceinline__ u32 bfi(u32 m, u32 a, u32 b) { return (a & m) | (b & ~m); }
__device__ __forceinline__ u32 rfl(u32 x) { return (u32)__builtin_amdgcn_readfirstlane((int)x); }
__device__ __forceinline__ u64 now() { return __builtin_amdgcn_s_memrealtime(); }

__device__ __forceinline__ bool spin_fail(char* ws, u32 call, u64 t0) {
    const u32 t = rfl(ld4_wait(ws + W_TMO));
    if (t == call) return true;
    if (now() - t0 > DEADLINE) { st4_sys(ws + W_TMO, call); WAIT_V(0); return true; }
    __builtin_amdgcn_s_sleep(2);
    return false;
}
__device__ __forceinline__ bool timed_out(char* ws, u32 call) { return rfl(ld4_wait(ws + W_TMO)) == call; }

__device__ __forceinline__ float wave_max(float v) {
#pragma unroll
    for (int o = 32; o >= 1; o >>= 1) v = fmaxf(v, __shfl_xor(v, o));
    return v;
}
__device__ __forceinline__ int rot_of(int j) { return (j * 16) / 7; }
__device__ __forceinline__ int kt_of(int o, int rank, int j0) {
    if (o < 8) return rank * 8 + o;
    if (o > 63) o = 63;
    const int oo = o - 8, r = oo / 7, s = oo - r * 7;
    int j = j0 - s; if (j < 0) j += 7;
    return ((rank + 1 + j) & 7) * 8 + r;
}

#define STAMP(i) do {} while (0)

__device__ __forceinline__ void pack_panel(const Params& p, LAS unsigned char* lds, int q, u32 call, int wid, int lane) {
    const int fr = lane & 15, fq = lane >> 4;
    const int r = q / 80, i = q - r * 80;
    const bool isB = i < 64;
    const int dest = isB ? ((p.rank + 1 + (i >> 3)) & 7) : p.rank;
    const int pn = i & 7, pm = i - 64;
    f32x4 f[8];
    u32 u[16];
    if (isB) {
        const int wc = wid >> 1, ks = wid & 1;
        const float* base = p.w + (size_t)(r * 64 + 32 * ks + 4 * fq) * GN + dest * NS + pn * 256 + wc * 64 + 4 * fr;
#pragma unroll
        for (int j = 0; j < 8; ++j) f[j] = __builtin_nontemporal_load((const f32x4*)(base + (size_t)(16 * (j >> 2) + (j & 3)) * GN));
#pragma unroll
        for (int c = 0; c < 4; ++c)
#pragma unroll
            for (int t = 0; t < 4; ++t) u[4 * c + t] = cvt_pk_bf16(f[2 * t][c], f[2 * t + 1][c]);
    } else {
        const float* base = p.x + (size_t)(pm * 256 + wid * 32 + fr) * KS + r * 64 + 4 * fq;
#pragma unroll
        for (int j = 0; j < 8; ++j) f[j] = __builtin_nontemporal_load((const f32x4*)(base + (j >> 2) * 16 * KS + 16 * (j & 3)));
#pragma unroll
        for (int c = 0; c < 4; ++c) {
            u[4 * c + 0] = cvt_pk_bf16(f[2 * c][0], f[2 * c][1]); u[4 * c + 1] = cvt_pk_bf16(f[2 * c][2], f[2 * c][3]);
            u[4 * c + 2] = cvt_pk_bf16(f[2 * c + 1][0], f[2 * c + 1][1]); u[4 * c + 3] = cvt_pk_bf16(f[2 * c + 1][2], f[2 * c + 1][3]);
        }
    }
    const bool own = dest == p.rank;
    const bool exported = !(isB && own);
    const int kt = p.rank * 8 + r;
    if (own) {
        char* d = p.ws + (isB ? W_B + (size_t)(pn * 64 + kt) * PANEL : W_A + (size_t)(pm * 64 + kt) * PANEL) + wid * 4096 + lane * 16;
        st16_sys<0>(d, (u32x4){u[0], u[1], u[2], u[3]}); st16_sys<1024>(d, (u32x4){u[4], u[5], u[6], u[7]});
        st16_sys<2048>(d, (u32x4){u[8], u[9], u[10], u[11]}); st16_sys<3072>(d, (u32x4){u[12], u[13], u[14], u[15]});
    }
    volatile LAS u32* hd = (volatile LAS u32*)(lds + LDS_MISC + 256);
    if (exported) {
        char* slot = p.ws + (isB ? W_PW + (size_t)((dest * 8 + r) * 8 + pn) * PSLOT : W_PX + (size_t)(r * 16 + pm) * PSLOT);
        u32 lo[8], hi[8];
        u32 mx = 0;
#pragma unroll
        for (int k = 0; k < 8; ++k) {
            lo[k] = __builtin_amdgcn_perm(u[2 * k + 1], u[2 * k], 0x06040200u);
            hi[k] = __builtin_amdgcn_perm(u[2 * k + 1], u[2 * k], 0x07050301u);
        }
#pragma unroll
        for (int k = 0; k < 16; ++k) { mx = max(mx, u[k] & 0x7f000000u); mx = max(mx, (u[k] << 16) & 0x7f000000u); }
#pragma unroll
        for (int o = 32; o >= 1; o >>= 1) mx = max(mx, (u32)__shfl_xor((int)mx, o));
        const u32 top = rfl(mx) >> 24;
        const u32 base = top >= 7 ? top - 7 : 0;
        const u32 b4 = base * 0x01010101u;
        u32 t[8], bad = 0;
#pragma unroll
        for (int k = 0; k < 8; ++k) { t[k] = (hi[k] | 0x80808080u) - b4; bad |= (t[k] & 0xf8f8f8f8u) ^ 0x80808080u; }
        u32 nb[4];
#pragma unroll
        for (int k = 0; k < 4; ++k)
            nb[k] = bfi(0x0f0f0f0fu, bfi(0x07070707u, t[2 * k], hi[2 * k] >> 4), bfi(0x70707070u, t[2 * k + 1] << 4, hi[2 * k + 1]));
        char* d = slot + wid * 3072 + lane * 16;
        st16_sys<0>(d, (u32x4){lo[0], lo[1], lo[2], lo[3]}); st16_sys<1024>(d, (u32x4){lo[4], lo[5], lo[6], lo[7]});
        st16_sys<2048>(d, (u32x4){nb[0], nb[1], nb[2], nb[3]});
        if (lane == 0) ((volatile LAS unsigned char*)hd)[wid] = (unsigned char)base;
        if (bad) {
#pragma unroll
            for (int k = 0; k < 8; ++k) {
                const u32 bk = (t[k] & 0xf8f8f8f8u) ^ 0x80808080u;
#pragma unroll
                for (int b = 0; b < 4; ++b)
                    if ((bk >> (8 * b)) & 0xffu) {
                        const u32 s = __hip_atomic_fetch_add((LAS u32*)(hd + 2), 1u, __ATOMIC_RELAXED, __HIP_MEMORY_SCOPE_WORKGROUP);
                        if (s < (u32)MAXEXC) hd[3 + s] = (u32)((((wid * 64 + lane) * 8 + k) * 4 + b) << 8) | ((hi[k] >> (8 * b)) & 0x7fu);
                    }
            }
        }
        WAIT_L(0); BAR; CFENCE;
        const u32 cnt = rfl(hd[2]);
        if (cnt > (u32)MAXEXC) {
            char* dr = slot + 24576 + wid * 2048 + lane * 16;
            st16_sys<0>(dr, (u32x4){hi[0], hi[1], hi[2], hi[3]}); st16_sys<1024>(dr, (u32x4){hi[4], hi[5], hi[6], hi[7]});
        }
    }
    WAIT_V(0); BAR; CFENCE;
    const int tid = wid * 64 + lane;
    if (exported && tid < 128) {
        const u32 pr = (u32)(tid >> 4), g = (u32)(tid & 15);
        const bool send = isB ? pr == (u32)dest : pr != (u32)p.rank;
        const u32 val = hd[g];
        if (send && pr < (u32)NR) {
            char* dws = p.ws_all[pr];
            st8_sys(dws + W_HDR + (size_t)((p.rank * 8 + r) * 24 + (isB ? pn : 8 + pm)) * 128 + g * 8, (u32x2){val, call});
        }
    }
    if (own && tid == 128) add4_agent(p.ws + (isB ? W_BCNT + (size_t)(pn * 64 + kt) * 4 : W_ACNT + (size_t)(pm * 64 + kt) * 4), 8u);
    WAIT_L(0); BAR; CFENCE;
    if (tid == 0) hd[2] = 0;
    WAIT_L(0); BAR; CFENCE;
}

__device__ __forceinline__ void pull_inputs(const Params& p, u32 call, int j, int g, int nw, int lane) {
    const int peer = (p.rank + 1 + j) & 7;
    const char* pws = p.ws_all[peer];
    const int rot = rot_of(j);
    const int NU = 192 * 8;
    auto hdr_ptr = [&](int v) {
        const int P = v >> 3, r = P / 24, i = P - r * 24;
        const int idx = i < 8 ? ((i + j) & 7) : 8 + ((i - 8 + rot) & 15);
        return p.ws + W_HDR + (size_t)((peer * 8 + r) * 24 + idx) * 128 + (lane & 15) * 8;
    };
    int v = g;
    u32x2 h = ld8_wait(hdr_ptr(v));
    char* prev_cnt = nullptr;
    for (; v < NU; v += nw) {
        if (!__all(h.y == call)) {
            const u64 t0 = now();
            for (;;) {
                h = ld8_wait(hdr_ptr(v));
                if (__all(h.y == call)) break;
                if (spin_fail(p.ws, call, t0)) return;
            }
        }
        const int P = v >> 3, unit = v & 7, r = P / 24, i = P - r * 24;
        const bool isB = i < 8;
        const int pn = (i + j) & 7, pm = (i - 8 + rot) & 15, kt = peer * 8 + r;
        const char* slot = pws + (isB ? W_PW + (size_t)((p.rank * 8 + r) * 8 + pn) * PSLOT : W_PX + (size_t)(r * 16 + pm) * PSLOT);
        const char* s = slot + unit * 3072 + lane * 16;
        u32x4 l0, l1, nb; u32x2 hn;
        ld16_sys<0>(l0, s); ld16_sys<1024>(l1, s); ld16_sys<2048>(nb, s);
        ld8_sys(hn, hdr_ptr(v + nw < NU ? v + nw : v));
        WAIT_V(4);
        if (prev_cnt != nullptr && lane == 0) add4_agent(prev_cnt, 1u);
        asm volatile("s_waitcnt vmcnt(0)" : "+v"(l0), "+v"(l1), "+v"(nb), "+v"(hn) :: "memory");
        const u32 bw = (u32)__builtin_amdgcn_readlane((int)h.x, unit >> 2);
        const u32 base = (bw >> (8 * (unit & 3))) & 0x7fu;
        const u32 cnt = (u32)__builtin_amdgcn_readlane((int)h.x, 2);
        const u32 b4 = base * 0x01010101u;
        u32 hi[8];
#pragma unroll
        for (int k = 0; k < 4; ++k) {
            const u32 n = nb[k];
            hi[2 * k] = bfi(0x80808080u, n << 4, (n & 0x07070707u) + b4);
            hi[2 * k + 1] = bfi(0x80808080u, n, ((n >> 4) & 0x07070707u) + b4);
        }
        if (cnt > (u32)MAXEXC) {
            const char* sr = slot + 24576 + unit * 2048 + lane * 16;
            const u32x4 a = ld16_wait(sr), b = ld16_wait(sr + 1024);
#pragma unroll
            for (int k = 0; k < 4; ++k) { hi[k] = a[k]; hi[4 + k] = b[k]; }
        } else {
            for (u32 e = 0; e < cnt; ++e) {
                const u32 en = (u32)__builtin_amdgcn_readlane((int)h.x, (int)(3 + e));
                if ((en >> 19) == (u32)unit) {
                    const u32 sh = ((en >> 8) & 3u) * 8u, kk = (en >> 10) & 7u;
                    const bool me = ((en >> 13) & 63u) == (u32)lane;
#pragma unroll
                    for (int k = 0; k < 8; ++k)
                        if (me && kk == (u32)k) hi[k] = bfi(0x7fu << sh, (en & 0x7fu) << sh, hi[k]);
                }
            }
        }
        u32 u[16];
#pragma unroll
        for (int k = 0; k < 8; ++k) {
            const u32 lo = k < 4 ? l0[k & 3] : l1[k & 3];
            u[2 * k] = __builtin_amdgcn_perm(hi[k], lo, 0x05010400u);
            u[2 * k + 1] = __builtin_amdgcn_perm(hi[k], lo, 0x07030602u);
        }
        char* d = p.ws + (isB ? W_B + (size_t)(pn * 64 + kt) * PANEL : W_A + (size_t)(pm * 64 + kt) * PANEL) + unit * 4096 + lane * 16;
        st16_sys<0>(d, (u32x4){u[0], u[1], u[2], u[3]}); st16_sys<1024>(d, (u32x4){u[4], u[5], u[6], u[7]});
        st16_sys<2048>(d, (u32x4){u[8], u[9], u[10], u[11]}); st16_sys<3072>(d, (u32x4){u[12], u[13], u[14], u[15]});
        prev_cnt = p.ws + (isB ? W_BCNT + (size_t)(pn * 64 + kt) * 4 : W_ACNT + (size_t)(pm * 64 + kt) * 4);
        h = hn;
    }
    WAIT_V(0);
    if (prev_cnt != nullptr && lane == 0) add4_agent(prev_cnt, 1u);
}

__device__ __forceinline__ void pull_outputs(const Params& p, LAS unsigned char* lds, u32 call, int j, int g, int nw, int wid, int lane) {
    const int peer = (p.rank + 1 + j) & 7;
    const char* pws = p.ws_all[peer];
    const int fr = lane & 15, fq = lane >> 4;
    LAS u32* bm = (LAS u32*)(lds + wid * 1024);
    const char* fl = p.ws + W_OFLAG + (size_t)peer * 512 + lane * 8;
    u64 m0 = 0, m1 = 0;
    bool dead = false;
    for (int v = g; v < 1024; v += nw) {
        const int tile = v >> 3, unit = v & 7;
        if (!dead && !((((tile & 1) ? m1 : m0) >> (tile >> 1)) & 1ull)) {
            const u64 t0 = now();
            for (;;) {
                const u32x2 f = ld8_wait(fl);
                m0 = __ballot(f.x >= call); m1 = __ballot(f.y >= call);
                if ((((tile & 1) ? m1 : m0) >> (tile >> 1)) & 1ull) break;
                if (spin_fail(p.ws, call, t0)) break;
            }
            dead = timed_out(p.ws, call);
        }
        const int pm = tile >> 3, pn = tile & 7, wr = unit >> 2, wc = unit & 3;
        signed char* o = p.out + (size_t)(pm * 256 + wr * 128 + fr) * GN + peer * NS + pn * 256 + wc * 64 + fq * 16;
        u32 q[32];
        if (!dead) {
            const char* slot = pws + W_PO + (size_t)v * OSLOT;
            const char* s = slot + lane * 16;
            u32x4 w4[7]; u32 en;
            ld2_sys(en, slot + 7168 + lane * 2);
            ld16_sys<0>(w4[0], s); ld16_sys<1024>(w4[1], s); ld16_sys<2048>(w4[2], s); ld16_sys<3072>(w4[3], s);
            ld16_sys<0>(w4[4], s + 4096); ld16_sys<1024>(w4[5], s + 4096); ld16_sys<2048>(w4[6], s + 4096);
            *(LAS u32x4*)(bm + lane * 4) = (u32x4){0u, 0u, 0u, 0u};
            asm volatile("s_waitcnt vmcnt(7)" : "+v"(en) :: "memory");
            const u32 cnt = rfl(en);
            u32x4 mk;
            if (cnt > 63u) {
                mk = ld16_wait(slot + 7296 + lane * 16);
            } else {
                if (lane >= 1 && (u32)lane <= cnt)
                    __hip_atomic_fetch_or(bm + ((en >> 5) & 255u), 1u << (en & 31u), __ATOMIC_RELAXED, __HIP_MEMORY_SCOPE_WORKGROUP);
                CFENCE;
                mk = *(volatile LAS u32x4*)(bm + lane * 4);
            }
            asm volatile("s_waitcnt vmcnt(0)" : "+v"(w4[0]), "+v"(w4[1]), "+v"(w4[2]), "+v"(w4[3]), "+v"(w4[4]), "+v"(w4[5]), "+v"(w4[6]) :: "memory");
#pragma unroll
            for (int gq = 0; gq < 4; ++gq) {
                u32 last = 0;
#pragma unroll
                for (int i = 0; i < 7; ++i) {
                    const int k = gq * 7 + i;
                    const u32 w = w4[k >> 2][k & 3];
                    q[gq * 8 + i] = bfi(0x7f7f7f7fu, w, (w << 1) ^ (mk[gq] << (7 - i)));
                    last |= (w >> (7 - i)) & (0x01010101u << i);
                }
                q[gq * 8 + 7] = bfi(0x7f7f7f7fu, last, (last << 1) ^ mk[gq]);
            }
        } else {
#pragma unroll
            for (int k = 0; k < 32; ++k) q[k] = 0x80808080u;
        }
#pragma unroll
        for (int k = 0; k < 8; ++k)
            st16_plain<0>(o + (size_t)k * 16 * GN, (u32x4){q[4 * k], q[4 * k + 1], q[4 * k + 2], q[4 * k + 3]});
    }
}

__device__ __forceinline__ void gemm_tile(const Params& p, LAS unsigned char* lds, u32 call, int pm, int pn, int wid, int lane) {
    const int wr = wid >> 2, wc = wid & 3;
    int j0 = 0;
#pragma unroll
    for (int j = 1; j < 7; ++j) if (rot_of(j) <= pm) j0 = j;
    const char* Ab = p.ws + W_A + (size_t)pm * 64 * PANEL;
    const char* Bb = p.ws + W_B + (size_t)pn * 64 * PANEL;
    const u32 voff = (u32)(wid * 4096 + lane * 16);
    const u32 ldsw = (u32)wid * 2048u;
    const int aoff = wr * 8192 + lane * 16, boff = wc * 4096 + lane * 16;
    u64 ready = 0;
    const u32 need = call * 8u;
#define SA(b, h) (((b) * 2 + (h)) * 16384)
#define SB(b, h) ((4 + (b) * 2 + (h)) * 16384)
#define STAGE(buf, gbase, h) do { \
        __builtin_amdgcn_global_load_lds((const u32*)((gbase) + voff + (h) * 2048), (LAS u32*)(lds + (buf) + ldsw), 16, 0, 17); \
        __builtin_amdgcn_global_load_lds((const u32*)((gbase) + voff + (h) * 2048), (LAS u32*)(lds + (buf) + ldsw), 16, 1024, 17); } while (0)
#define LDA(dst, b, h) do { _Pragma("unroll") for (int m = 0; m < 4; ++m) _Pragma("unroll") for (int k = 0; k < 2; ++k) \
        dst[m][k] = *(const LAS bf16x8*)(lds + SA(b, h) + aoff + m * 2048 + k * 1024); } while (0)
#define LDB(dst, b, h) do { _Pragma("unroll") for (int n = 0; n < 2; ++n) _Pragma("unroll") for (int k = 0; k < 2; ++k) \
        dst[n][k] = *(const LAS bf16x8*)(lds + SB(b, h) + boff + k * 2048 + n * 1024); } while (0)
#define MMA(ai, bj, At, Bt) do { __builtin_amdgcn_s_setprio(1); \
        _Pragma("unroll") for (int m = 0; m < 4; ++m) _Pragma("unroll") for (int n = 0; n < 2; ++n) _Pragma("unroll") for (int k = 0; k < 2; ++k) \
            acc[ai][bj][m][n] = __builtin_amdgcn_mfma_f32_16x16x32_bf16(Bt[n][k], At[m][k], acc[ai][bj][m][n], 0, 0, 0); \
        __builtin_amdgcn_s_setprio(0); } while (0)
#define CHECK(o) do { if (wid == 4 && (o) < 64 && !((ready >> (o)) & 1ull)) { \
        const u64 t0_ = now(); int ln_ = (int)__builtin_amdgcn_mbcnt_hi(~0u, __builtin_amdgcn_mbcnt_lo(~0u, 0u)); \
        const int kt_ = kt_of(ln_, p.rank, j0); \
        for (;;) { \
            const u32 a_ = ld4_wait(p.ws + W_ACNT + (size_t)(pm * 64 + kt_) * 4), b_ = ld4_wait(p.ws + W_BCNT + (size_t)(pn * 64 + kt_) * 4); \
            ready = __ballot(a_ >= need && b_ >= need); \
            if ((ready >> (o)) & 1ull) break; \
            if (spin_fail(p.ws, call, t0_)) { ready = ~0ull; break; } \
        } } } while (0)

    f32x4 acc[2][2][4][2];
#pragma unroll
    for (int a = 0; a < 2; ++a)
#pragma unroll
        for (int b = 0; b < 2; ++b)
#pragma unroll
            for (int m = 0; m < 4; ++m)
#pragma unroll
                for (int n = 0; n < 2; ++n) acc[a][b][m][n] = (f32x4){0.f, 0.f, 0.f, 0.f};
    bf16x8 At[4][2], B0[2][2], B1[2][2];

    CHECK(0); CHECK(1);
    BAR; CFENCE;
    const char* a1; const char* a2; const char* b2; const char* a3; const char* b3;
    {
        const int k0 = kt_of(0, p.rank, j0), k1 = kt_of(1, p.rank, j0);
        const char* a0 = Ab + (size_t)k0 * PANEL; const char* b0 = Bb + (size_t)k0 * PANEL;
        a1 = Ab + (size_t)k1 * PANEL; const char* b1 = Bb + (size_t)k1 * PANEL;
        STAGE(SB(0, 0), b0, 0); STAGE(SA(0, 0), a0, 0); STAGE(SB(0, 1), b0, 1); STAGE(SA(0, 1), a0, 1);
        if (wr == 1) BAR;
        WAIT_V(4); BAR;
        STAGE(SB(1, 0), b1, 0); STAGE(SA(1, 0), a1, 0); STAGE(SB(1, 1), b1, 1);
        WAIT_V(6); BAR;
    }
    for (int t = 0; t < 64; t += 2) {
        const int k2 = kt_of(t + 2, p.rank, j0), k3 = kt_of(t + 3, p.rank, j0);
        a2 = Ab + (size_t)k2 * PANEL; b2 = Bb + (size_t)k2 * PANEL; a3 = Ab + (size_t)k3 * PANEL; b3 = Bb + (size_t)k3 * PANEL;
        LDB(B0, 0, 0); SCHED; LDA(At, 0, 0); STAGE(SA(1, 1), a1, 1);
        CHECK(t + 2);
        WAIT_L(8); BAR; WAIT_L(0); MMA(0, 0, At, B0); BAR; SCHED;
        LDB(B1, 0, 1); STAGE(SB(0, 0), b2, 0);
        BAR; WAIT_L(0); MMA(0, 1, At, B1); BAR;
        LDA(At, 0, 1); STAGE(SA(0, 0), a2, 0);
        BAR; WAIT_L(0); MMA(1, 0, At, B0); BAR; SCHED;
        STAGE(SB(0, 1), b2, 1);
        WAIT_V(6); BAR; MMA(1, 1, At, B1); BAR;
        LDB(B0, 1, 0); SCHED; LDA(At, 1, 0); STAGE(SA(0, 1), a2, 1);
        CHECK(t + 3);
        WAIT_L(8); BAR; WAIT_L(0); MMA(0, 0, At, B0); BAR; SCHED;
        LDB(B1, 1, 1); STAGE(SB(1, 0), b3, 0);
        BAR; WAIT_L(0); MMA(0, 1, At, B1); BAR;
        LDA(At, 1, 1); STAGE(SA(1, 0), a3, 0);
        BAR; WAIT_L(0); MMA(1, 0, At, B0); BAR; SCHED;
        STAGE(SB(1, 1), b3, 1);
        WAIT_V(6); BAR; MMA(1, 1, At, B1); BAR;
        a1 = a3;
    }
    if (wr == 0) BAR;
    STAMP(2);
    int tz = threadIdx.x; asm volatile("" : "+v"(tz));
    lane = tz & 63;
    const int fr = lane & 15, fq = lane >> 4;
    const int tile = pm * 8 + pn;
    float mx = 0.f;
#pragma unroll
    for (int a = 0; a < 2; ++a)
#pragma unroll
        for (int b = 0; b < 2; ++b)
#pragma unroll
            for (int m = 0; m < 4; ++m)
#pragma unroll
                for (int n = 0; n < 2; ++n)
#pragma unroll
                    for (int r = 0; r < 4; ++r) mx = fmaxf(mx, fabsf(acc[a][b][m][n][r]));
    mx = wave_max(mx);
    volatile LAS float* red = (volatile LAS float*)(lds + LDS_MISC + 512);
    if (lane == 0) red[wid] = mx;
    WAIT_L(0); BAR; CFENCE;
    if (tz < NR) {
        float t = red[0];
#pragma unroll
        for (int k = 1; k < 8; ++k) t = fmaxf(t, red[k]);
        char* dws = p.ws_all[tz];
        st8_sys(dws + W_AMAX + (size_t)(p.rank * 128 + tile) * 8, (u32x2){__float_as_uint(t), call});
    }
    {
        const char* src = p.ws + W_AMAX + tz * 16;
        const u64 t0 = now();
        u32x4 gq;
        for (;;) {
            gq = ld16_wait(src);
            if (__all(gq.y == call && gq.w == call)) break;
            if (spin_fail(p.ws, call, t0)) break;
        }
        mx = wave_max(fmaxf(__uint_as_float(gq.x), __uint_as_float(gq.z)));
        if (lane == 0) red[8 + wid] = mx;
        WAIT_L(0); BAR; CFENCE;
        mx = red[8];
#pragma unroll
        for (int k = 1; k < 8; ++k) mx = fmaxf(mx, red[8 + k]);
    }
    STAMP(3);
    const bool dead = timed_out(p.ws, call);
    const float inv = mx > 0.f ? 127.0f / mx : 0.f;
    u32 q[32];
#pragma unroll
    for (int m = 0; m < 4; ++m)
#pragma unroll
        for (int a = 0; a < 2; ++a)
#pragma unroll
            for (int r = 0; r < 4; ++r) {
                const u32 t0 = __float_as_uint(fmaf(acc[a][0][m][0][r], inv, 12582912.f)), t1 = __float_as_uint(fmaf(acc[a][0][m][1][r], inv, 12582912.f));
                const u32 t2 = __float_as_uint(fmaf(acc[a][1][m][0][r], inv, 12582912.f)), t3 = __float_as_uint(fmaf(acc[a][1][m][1][r], inv, 12582912.f));
                const u32 v = __builtin_amdgcn_perm(t1, t0, 0x0c0c0400u) | __builtin_amdgcn_perm(t3, t2, 0x04000c0cu);
                q[(m * 2 + a) * 4 + r] = dead ? 0x80808080u : v;
            }
    char* slot = p.ws + W_PO + (size_t)(tile * 8 + wid) * OSLOT;
    {
        u32 w[28], mk[4];
#pragma unroll
        for (int g = 0; g < 4; ++g) {
            mk[g] = 0;
#pragma unroll
            for (int i = 0; i < 8; ++i) {
                const u32 v = q[g * 8 + i];
                mk[g] |= ((v ^ (v << 1)) & 0x80808080u) >> (7 - i);
                if (i < 7) w[g * 7 + i] = bfi(0x7f7f7f7fu, v, q[g * 8 + 7] << (7 - i));
            }
        }
        char* d = slot + lane * 16;
        st16_sys<0>(d, (u32x4){w[0], w[1], w[2], w[3]}); st16_sys<1024>(d, (u32x4){w[4], w[5], w[6], w[7]});
        st16_sys<2048>(d, (u32x4){w[8], w[9], w[10], w[11]}); st16_sys<3072>(d, (u32x4){w[12], w[13], w[14], w[15]});
        st16_sys<0>(d + 4096, (u32x4){w[16], w[17], w[18], w[19]}); st16_sys<1024>(d + 4096, (u32x4){w[20], w[21], w[22], w[23]});
        st16_sys<2048>(d + 4096, (u32x4){w[24], w[25], w[26], w[27]});
        volatile LAS u32* ctr = (volatile LAS u32*)(lds + LDS_MISC + 1024 + wid * 4);
        volatile LAS unsigned short* list = (volatile LAS unsigned short*)(lds + LDS_MISC + 2048 + wid * 128);
        if (lane == 0) *ctr = 1u;
        CFENCE;
        const u32 c = __popc(mk[0]) + __popc(mk[1]) + __popc(mk[2]) + __popc(mk[3]);
        if (c) {
            u32 s = __hip_atomic_fetch_add((LAS u32*)ctr, c, __ATOMIC_RELAXED, __HIP_MEMORY_SCOPE_WORKGROUP);
#pragma unroll
            for (int g = 0; g < 4; ++g) {
                u32 mm = mk[g];
                while (mm && s < 64u) {
                    const u32 b = (u32)__builtin_ctz(mm);
                    list[s] = (unsigned short)((lane * 4 + g) * 32 + b);
                    mm &= mm - 1; ++s;
                }
            }
        }
        CFENCE;
        const u32 total = rfl(*ctr) - 1u;
        if (lane == 0) list[0] = (unsigned short)(total > 63u ? 0xffffu : total);
        CFENCE;
        if (lane < 8) st16_sys<0>(slot + 7168 + lane * 16, *(volatile LAS u32x4*)((volatile LAS unsigned char*)list + lane * 16));
        if (total > 63u) st16_sys<0>(slot + 7296 + lane * 16, (u32x4){mk[0], mk[1], mk[2], mk[3]});
    }
    WAIT_V(0); BAR; CFENCE;
    if (tz < NR && tz != p.rank) {
        char* dws = p.ws_all[tz];
        st4_sys(dws + W_OFLAG + (size_t)(p.rank * 128 + tile) * 4, call);
    }
    STAMP(4);
    signed char* o = p.out + (size_t)(pm * 256 + wr * 128 + fr) * GN + p.rank * NS + pn * 256 + wc * 64 + fq * 16;
#pragma unroll
    for (int k = 0; k < 8; ++k)
        st16_plain<0>(o + (size_t)k * 16 * GN, (u32x4){q[4 * k], q[4 * k + 1], q[4 * k + 2], q[4 * k + 3]});
#undef SA
#undef SB
#undef STAGE
#undef LDA
#undef LDB
#undef MMA
#undef CHECK
}

__global__ void __launch_bounds__(BLOCK, 2) fused_gemm_q8(Params p) {
    __shared__ __attribute__((aligned(16))) unsigned char smem[LDS_BYTES];
    LAS unsigned char* lds = (LAS unsigned char*)smem;
    const int tid = threadIdx.x, wid = (int)rfl((u32)tid >> 6), lane = tid & 63;
    const int xcd = blockIdx.x & 7, idx = blockIdx.x >> 3;
    STAMP(0);
    volatile LAS u32* misc = (volatile LAS u32*)(lds + LDS_MISC);
    if (tid == 0) {
        const u32 t = __hip_atomic_fetch_add((u32*)(p.ws + W_TICKET), 1u, __ATOMIC_RELAXED, __HIP_MEMORY_SCOPE_AGENT);
        misc[0] = t / (u32)GRID + 1u;
        misc[64 + 2] = 0u;
    }
    WAIT_V(0); WAIT_L(0); BAR; CFENCE;
    const u32 call = rfl(misc[0]);

    const bool is_gemm = idx < 16;
    const int c = idx - 16;
    const bool is_pull = !is_gemm && xcd < 7 && c < NPW;
    constexpr int NSPARE = 7 * (16 - NPW) + 16;
    int q = blockIdx.x, step, first2;
    if (is_gemm) { first2 = 256 + (int)blockIdx.x; step = 1024; }
    else if (is_pull) { first2 = 1024; step = 1024; }
    else { first2 = 384 + (xcd < 7 ? xcd * (16 - NPW) + (c - NPW) : 7 * (16 - NPW) + c); step = NSPARE; }
    for (bool firstq = true; q < 640; firstq = false) {
        pack_panel(p, lds, q, call, wid, lane);
        q = firstq ? first2 : q + step;
    }
    STAMP(1);
    if (is_gemm) {
        gemm_tile(p, lds, call, (xcd >> 1) * 4 + (idx >> 2), (xcd & 1) * 4 + (idx & 3), wid, lane);
        STAMP(5);
    } else if (is_pull) {
        if (wid < PW1) pull_inputs(p, call, xcd, wid * NPW + c, NPW * PW1, lane);
        STAMP(2);
        if (wid < PW2) pull_outputs(p, lds, call, xcd, c * PW2 + wid, NPW * PW2, wid, lane);
        STAMP(5);
    }
}

extern "C" void kernel_launch(int rank, int nranks, ncclComm_t comm, hipStream_t stream,
                              void* const* d_in, const int* in_sizes, int n_in,
                              void* d_out, int out_size,
                              const int* in_dtypes, int out_dtype,
                              void* const* d_out_all,
                              void* const* d_ws_all, size_t ws_bytes) {
    (void)comm; (void)in_dtypes; (void)out_dtype; (void)d_out_all;
    if (nranks != NR || n_in < 2 || ws_bytes < W_END || in_sizes[0] != GM * KS || in_sizes[1] != KS * GN || out_size != GM * GN ||
        (unsigned)rank >= (unsigned)NR) {
        fprintf(stderr, "unexpected problem geometry\n");
        return;
    }
    Params p;
    p.x = (const float*)d_in[0];
    p.w = (const float*)d_in[1];
    p.out = (signed char*)d_out;
    p.ws = (char*)d_ws_all[rank];
    p.ws_all = (char* const*)d_ws_all;
    p.rank = rank;
    p.pad = 0;
    hipLaunchKernelGGL(fused_gemm_q8, dim3(GRID), dim3(BLOCK), 0, stream, p);
}
```

```cpp
#include <hip/hip_runtime.h>
#include <rccl/rccl.h>
#include <cstdio>
#include <cstdint>

#define STAMPS 0

#define LAS __attribute__((address_space(3)))
typedef unsigned u32;
typedef unsigned long long u64;
typedef u32 u32x4 __attribute__((ext_vector_type(4)));
typedef u32 u32x2 __attribute__((ext_vector_type(2)));
typedef float f32x4 __attribute__((ext_vector_type(4)));
typedef short bf16x8 __attribute__((ext_vector_type(8)));

constexpr int NR = 8, GM = 4096, GN = 16384, KS = 512, NS = 2048;
constexpr int GRID = 256, BLOCK = 512;
constexpr int PANEL = 32768;
constexpr int USLOT = 8192;
constexpr size_t W_TMO = 128;
constexpr size_t W_Q = 1024;
constexpr size_t W_ACNT = 4096;
constexpr size_t W_BCNT = 8192;
constexpr size_t W_XCNT = 10240;
constexpr size_t W_AMAX = 16384;
constexpr size_t W_HI = 32768;
constexpr size_t W_HX = 81920;
constexpr size_t W_HO = 98304;
constexpr size_t W_SEQ = 256 << 10;
constexpr size_t W_DBG = 512 << 10;
constexpr size_t W_A = 1ull << 20;
constexpr size_t W_B = 33ull << 20;
constexpr size_t W_X = 49ull << 20;
constexpr size_t W_PX = 65ull << 20;
constexpr size_t W_PW = 69ull << 20;
constexpr size_t W_PO = 85ull << 20;
constexpr size_t W_END = 93ull << 20;
constexpr u64 DEADLINE = 40000000ull;

#define NPW 8
#define DEPTH 6
#define RUNAHEAD 2
#define NEAR 1
#define NTLD 0
#define SPDLY 0
constexpr int ND = 6, NSLOT = 12;
constexpr int NSPARE = 7 * (16 - NPW) + 16;
#define FC 3
constexpr int NVP = (7 * FC + 3) / 4;
static_assert(DEPTH <= 11 && FC >= 2 && FC <= 4, "roles");

constexpr int LDS_BM = 98304;
constexpr int LDS_ENC = 114688;
constexpr int LDS_MISC = 131072;
constexpr int LDS_BYTES = LDS_MISC + 2048;

struct Params {
    const float* x;
    const float* w;
    signed char* out;
    char* ws;
    char* const* ws_all;
    int rank;
    int pad;
};

#define WAIT_V(n) asm volatile("s_waitcnt vmcnt(" #n ")" ::: "memory")
#define WAIT_L(n) asm volatile("s_waitcnt lgkmcnt(" #n ")" ::: "memory")
#define BAR __builtin_amdgcn_s_barrier()
#define SCHED __builtin_amdgcn_sched_barrier(0)
#define CFENCE asm volatile("" ::: "memory")

template <int OFF> __device__ __forceinline__ void st16_sys(void* p, u32x4 x) {
    asm volatile("global_store_dwordx4 %0, %1, off offset:%2 sc0 sc1\n\ts_nop 1" :: "v"(p), "v"(x), "i"(OFF) : "memory");
}
__device__ __forceinline__ void st16_plain(void* p, u32x4 x) {
    asm volatile("global_store_dwordx4 %0, %1, off\n\ts_nop 1" :: "v"(p), "v"(x) : "memory");
}
__device__ __forceinline__ void ld8_sys(u32x2& x, const void* p) {
    asm volatile("global_load_dwordx2 %0, %1, off sc0 sc1" : "=v"(x) : "v"(p) : "memory");
}
__device__ __forceinline__ void st8_sys(void* p, u32x2 x) {
    asm volatile("global_store_dwordx2 %0, %1, off sc0 sc1\n\ts_nop 1" :: "v"(p), "v"(x) : "memory");
}
__device__ __forceinline__ void st4_sys(void* p, u32 x) {
    asm volatile("global_store_dword %0, %1, off sc0 sc1\n\ts_nop 1" :: "v"(p), "v"(x) : "memory");
}
__device__ __forceinline__ void add4_sys(void* p, u32 x) {
    asm volatile("global_atomic_add %0, %1, off sc1\n\ts_nop 1" :: "v"(p), "v"(x) : "memory");
}
__device__ __forceinline__ void ld16_nt(f32x4& x, const void* p) {
    asm volatile("global_load_dwordx4 %0, %1, off" : "=v"(x) : "v"(p) : "memory");
}
__device__ __forceinline__ void ticket_issue(u32& x, void* p) {
    u64 sv;
    asm volatile("s_mov_b64 %1, exec\n\ts_mov_b64 exec, 1\n\tglobal_atomic_add %0, %2, %3, off sc0 sc1\n\ts_mov_b64 exec, %1"
                 : "=&v"(x), "=&s"(sv) : "v"(p), "v"(1u) : "memory");
}
__device__ __forceinline__ void poll16(u32x4& x, u32& t, const void* p, const void* tp) {
    asm volatile("global_load_dwordx4 %0, %2, off sc0 sc1\n\tglobal_load_dword %1, %3, off sc0 sc1\n\ts_waitcnt vmcnt(0)"
                 : "=&v"(x), "=&v"(t) : "v"(p), "v"(tp) : "memory");
}
__device__ __forceinline__ void poll8(u32x2& x, u32& t, const void* p, const void* tp) {
    asm volatile("global_load_dwordx2 %0, %2, off sc0 sc1\n\tglobal_load_dword %1, %3, off sc0 sc1\n\ts_waitcnt vmcnt(0)"
                 : "=&v"(x), "=&v"(t) : "v"(p), "v"(tp) : "memory");
}
__device__ __forceinline__ void poll4x2(u32& x, u32& y, u32& t, const void* p, const void* q, const void* tp) {
    asm volatile("global_load_dword %0, %3, off sc0 sc1\n\tglobal_load_dword %1, %4, off sc0 sc1\n\tglobal_load_dword %2, %5, off sc0 sc1\n\ts_waitcnt vmcnt(0)"
                 : "=&v"(x), "=&v"(y), "=&v"(t) : "v"(p), "v"(q), "v"(tp) : "memory");
}
__device__ __forceinline__ u32 cvt_pk_bf16(float lo, float hi) {
    u32 r; asm("v_cvt_pk_bf16_f32 %0, %1, %2" : "=v"(r) : "v"(lo), "v"(hi)); return r;
}
__device__ __forceinline__ u32 bfi(u32 m, u32 a, u32 b) { return (a & m) | (b & ~m); }
__device__ __forceinline__ u32 rfl(u32 x) { return (u32)__builtin_amdgcn_readfirstlane((int)x); }
__device__ __forceinline__ u32 rdl(u32 x, int l) { return (u32)__builtin_amdgcn_readlane((int)x, l); }
__device__ __forceinline__ u64 now() { return __builtin_amdgcn_s_memrealtime(); }

__device__ __forceinline__ bool spin_fail(char* ws, u32 call, u64 t0, u32 t) {
    if (rfl(t) == call) return true;
    if (now() - t0 > DEADLINE) { st4_sys(ws + W_TMO, call); WAIT_V(0); return true; }
    __builtin_amdgcn_s_sleep(1);
    return false;
}
#define DPP0(v, ctl, rm, bc) (u32)__builtin_amdgcn_update_dpp(0, (int)(v), ctl, rm, 0xf, bc)
__device__ __forceinline__ u32 scan_add(u32 v) {
    v += DPP0(v, 0x111, 0xf, true); v += DPP0(v, 0x112, 0xf, true); v += DPP0(v, 0x114, 0xf, true); v += DPP0(v, 0x118, 0xf, true);
    v += DPP0(v, 0x142, 0xa, false); v += DPP0(v, 0x143, 0xc, false);
    return v;
}
__device__ __forceinline__ u32 wave_umax(u32 v) {
    v = max(v, DPP0(v, 0x111, 0xf, true)); v = max(v, DPP0(v, 0x112, 0xf, true)); v = max(v, DPP0(v, 0x114, 0xf, true)); v = max(v, DPP0(v, 0x118, 0xf, true));
    v = max(v, DPP0(v, 0x142, 0xa, false)); v = max(v, DPP0(v, 0x143, 0xc, false));
    return rdl(v, 63);
}
__device__ __forceinline__ int rot_of(int j) { return (j * 16) / 7; }
__device__ __forceinline__ int kt_of(int o, int rank, int j0) {
    if (o < 8) return rank * 8 + o;
    const int r = (o - 8) / 7, s = o - 8 - r * 7;
    int j = j0 - s; if (j < 0) j += 7;
    return ((rank + 1 + j) & 7) * 8 + r;
}
__device__ __forceinline__ int peer_of(int x, int rank) {
    const u32 ga = 0x6320u, gb = 0x7541u;
    const bool ina = (0x4du >> rank) & 1u;
    const u32 own = ina ? ga : gb, oth = ina ? gb : ga;
    if (x >= 4) return (int)((oth >> (4 * (x - 4))) & 7u);
    if (x == 1) return rank;
    const int want = x == 0 ? 0 : x - 1;
    int n = 0, res = rank;
#pragma unroll
    for (int i = 0; i < 4; ++i) { const int q = (int)((own >> (4 * i)) & 7u); if (q != rank) { if (n == want) res = q; ++n; } }
    return res;
}
__device__ __forceinline__ char* peer_ws(LAS unsigned char* lds, u32 pr) {
    return (char*)((volatile LAS u64*)(lds + LDS_MISC + 64))[pr & 7u];
}

#define STAMP(i) do {} while (0)

__device__ __forceinline__ u32 nib_flags(u32 m) { u32 t = m | (m >> 1); t |= t >> 2; return t & 0x11111111u; }
__device__ __forceinline__ void sparse_put(const u32 (&m)[4], u32 fl, u32 off, LAS u32* st) {
#pragma unroll
    for (int w = 0; w < 4; ++w) {
        const u32 t = (fl >> w) & 0x11111111u, pin = t * 0x11111111u, pex = pin << 4;
        const u32 pe = (pex & 0x0f0f0f0fu) << 2, po = (pex >> 2) & 0x3c3c3c3cu;
        u32 c = m[w] & 15u;
#pragma unroll
        for (int i = 1; i < 8; ++i) c |= ((m[w] >> (4 * i)) & 15u) << ((((i & 1) ? po : pe) >> (8 * (i >> 1))) & 31u);
        const u64 x = (u64)c << ((off & 7u) * 4u);
        __hip_atomic_fetch_or(st + (off >> 3), (u32)x, __ATOMIC_RELAXED, __HIP_MEMORY_SCOPE_WORKGROUP);
        __hip_atomic_fetch_or(st + (off >> 3) + 1, (u32)(x >> 32), __ATOMIC_RELAXED, __HIP_MEMORY_SCOPE_WORKGROUP);
        off += pin >> 28;
    }
}
__device__ __forceinline__ void sparse_get(u32 (&m)[4], u32 fl, u32 off, const LAS u32* st) {
    u32 lo[4], hi[4], sh[4], pin[4];
#pragma unroll
    for (int w = 0; w < 4; ++w) {
        pin[w] = ((fl >> w) & 0x11111111u) * 0x11111111u;
        lo[w] = st[off >> 3]; hi[w] = st[(off >> 3) + 1]; sh[w] = (off & 7u) * 4u;
        off += pin[w] >> 28;
    }
#pragma unroll
    for (int w = 0; w < 4; ++w) {
        const u32 t = (fl >> w) & 0x11111111u, pex = pin[w] << 4;
        const u32 pe = (pex & 0x0f0f0f0fu) << 2, po = (pex >> 2) & 0x3c3c3c3cu;
        const u32 c = __builtin_amdgcn_alignbit(hi[w], lo[w], sh[w]);
        u32 r = c & 15u;
#pragma unroll
        for (int i = 1; i < 8; ++i) r |= ((c >> ((((i & 1) ? po : pe) >> (8 * (i >> 1))) & 31u)) & 15u) << (4 * i);
        m[w] = r & ((t << 4) - t);
    }
}
__device__ __forceinline__ void tail_out(char* dst, LAS unsigned char* enc, u32 n16, int lane) {
    CFENCE;
    const u32x4 a = *(volatile LAS u32x4*)(enc + lane * 16), b = *(volatile LAS u32x4*)(enc + 1024 + lane * 16);
    if ((u32)lane < n16) st16_sys<0>(dst + lane * 16, a);
    if ((u32)lane + 64u < n16) st16_sys<1024>(dst + lane * 16, b);
}
constexpr u32 H_RAW = 0x80000000u;

struct PanelId { int r, dest, pi; bool isB; };
__device__ __forceinline__ PanelId panel_of(int r, int i, int rank) {
    PanelId d; d.r = r; d.isB = i < 56 || i >= 72;
    d.dest = i < 56 ? ((rank + 1 + (i >> 3)) & 7) : rank;
    d.pi = d.isB ? (i & 7) : i - 56;
    return d;
}
__device__ __forceinline__ bool is_fill(const PanelId& d, int unit) { return d.isB && d.pi == 7 && unit >= 4 - FC; }
__device__ __forceinline__ void unit_load(const Params& p, const PanelId& d, int unit, int lane, f32x4 (&f)[16]) {
    const int fr = lane & 15, fq = lane >> 4;
    const float* b = d.isB ? p.w + (size_t)(d.r * 64 + 4 * fq) * GN + d.dest * NS + d.pi * 256 + unit * 64 + 4 * fr
                           : p.x + (size_t)(d.pi * 256 + unit * 64 + fr) * KS + d.r * 64 + 4 * fq;
    const int s1 = d.isB ? 16 * GN : 16 * KS, s2 = d.isB ? GN : 16;
#pragma unroll
    for (int j = 0; j < 16; ++j) ld16_nt(f[j], b + (j >> 2) * s1 + (j & 3) * s2);
}
#define PIN_F(f) do { asm volatile("" : "+v"(f[0]), "+v"(f[1]), "+v"(f[2]), "+v"(f[3]), "+v"(f[4]), "+v"(f[5]), "+v"(f[6]), "+v"(f[7]) :: "memory"); \
    asm volatile("" : "+v"(f[8]), "+v"(f[9]), "+v"(f[10]), "+v"(f[11]), "+v"(f[12]), "+v"(f[13]), "+v"(f[14]), "+v"(f[15]) :: "memory"); } while (0)
__device__ __forceinline__ void st_unit(char* dp, const u32 (&u)[32]) {
    st16_sys<0>(dp, (u32x4){u[0], u[1], u[2], u[3]}); st16_sys<1024>(dp, (u32x4){u[4], u[5], u[6], u[7]});
    st16_sys<2048>(dp, (u32x4){u[8], u[9], u[10], u[11]}); st16_sys<3072>(dp, (u32x4){u[12], u[13], u[14], u[15]});
    dp += 4096;
    st16_sys<0>(dp, (u32x4){u[16], u[17], u[18], u[19]}); st16_sys<1024>(dp, (u32x4){u[20], u[21], u[22], u[23]});
    st16_sys<2048>(dp, (u32x4){u[24], u[25], u[26], u[27]}); st16_sys<3072>(dp, (u32x4){u[28], u[29], u[30], u[31]});
}
__device__ __forceinline__ size_t local_off(const PanelId& d, int unit, int rank, int kt, size_t& cnt) {
    if (d.dest == rank) { cnt = (d.isB ? W_BCNT : W_ACNT) + (size_t)(d.pi * 64 + kt) * 4; return (d.isB ? W_B : W_A) + (size_t)(d.pi * 64 + kt) * PANEL + unit * USLOT; }
    const int g = ((d.dest - rank - 1) & 7) * FC + unit - (4 - FC);
    cnt = W_XCNT + (size_t)((g >> 2) * 64 + kt) * 4;
    return W_X + (size_t)((g >> 2) * 64 + kt) * PANEL + (g & 3) * USLOT;
}
__device__ __forceinline__ size_t export_off(const PanelId& d, int unit) {
    return (d.isB ? W_PW + (size_t)((d.dest * 8 + d.r) * 8 + d.pi) * PANEL : W_PX + (size_t)(d.r * 16 + d.pi) * PANEL) + unit * USLOT;
}
__device__ __forceinline__ u32 unit_pack(const Params& p, LAS unsigned char* enc, const PanelId& d, int unit, int lane, const f32x4 (&f)[16]) {
    u32 u[32];
    if (d.isB) {
#pragma unroll
        for (int h = 0; h < 2; ++h)
#pragma unroll
            for (int c = 0; c < 4; ++c)
#pragma unroll
                for (int t = 0; t < 4; ++t) u[16 * h + 4 * c + t] = cvt_pk_bf16(f[8 * h + 2 * t][c], f[8 * h + 2 * t + 1][c]);
    } else {
#pragma unroll
        for (int c = 0; c < 16; ++c) { u[2 * c] = cvt_pk_bf16(f[c][0], f[c][1]); u[2 * c + 1] = cvt_pk_bf16(f[c][2], f[c][3]); }
    }
    const bool own = d.dest == p.rank, fill = is_fill(d, unit);
    u32 hdr = 0;
    if (!(d.isB && own) || fill) {
        char* slot = p.ws + export_off(d, unit);
        *(volatile LAS u32x4*)(enc + lane * 16) = (u32x4){0u, 0u, 0u, 0u};
        *(volatile LAS u32x4*)(enc + 1024 + lane * 16) = (u32x4){0u, 0u, 0u, 0u};
        u32 hi[16], t[16], sum = 0;
#pragma unroll
        for (int k = 0; k < 16; ++k) {
            hi[k] = __builtin_amdgcn_perm(u[2 * k + 1], u[2 * k], 0x07050301u);
            t[k] = hi[k] | 0x80808080u;
            sum = __builtin_amdgcn_sad_u8(t[k], 0u, sum);
        }
        u32 a = ((rdl(scan_add(sum), 63) + 4095u) >> 12) - 128u;
        a = min(max(a, 6u), 126u);
        const u32 b4 = (a - 6u) * 0x01010101u;
        u32 bad = 0, n[8];
#pragma unroll
        for (int k = 0; k < 16; ++k) { t[k] -= b4; bad |= (t[k] & 0xf8f8f8f8u) ^ 0x80808080u; }
#pragma unroll
        for (int k = 0; k < 8; ++k)
            n[k] = __builtin_amdgcn_perm(0x06000204u, 0x080a0c0eu, t[2 * k] & 0x07070707u) | __builtin_amdgcn_perm(0x60002040u, 0x80a0c0e0u, t[2 * k + 1] & 0x07070707u)
                 | ((hi[2 * k] >> 7) & 0x01010101u) | ((hi[2 * k + 1] >> 3) & 0x10101010u);
        u32 dn[4], m[4];
#pragma unroll
        for (int h = 0; h < 2; ++h) {
            dn[2 * h] = bfi(0x33333333u, n[4 * h], n[4 * h + 2] << 2); dn[2 * h + 1] = bfi(0x33333333u, n[4 * h + 1], n[4 * h + 3] << 2);
            const u32 B = bfi(0x33333333u, n[4 * h] >> 2, n[4 * h + 2]), D = bfi(0x33333333u, n[4 * h + 1] >> 2, n[4 * h + 3]);
            m[h] = bfi(0x55555555u, B, D << 1); m[2 + h] = bfi(0x55555555u, B >> 1, D);
        }
        const u32 fl = nib_flags(m[0]) | (nib_flags(m[1]) << 1) | (nib_flags(m[2]) << 2) | (nib_flags(m[3]) << 3);
        const u32 cn = (u32)__popc(fl), inc = scan_add(cn);
        const u32 sdw = (rdl(inc, 63) + 7u) >> 3;
        u32 nexc = 0;
        u64 anyb = __ballot(bad != 0);
        if (anyb) {
            u32 bm[2] = {0u, 0u};
#pragma unroll
            for (int k = 0; k < 16; ++k) {
                const u32 y = (t[k] & 0xf8f8f8f8u) ^ 0x80808080u;
                bm[k >> 3] |= ((((y & 0x78787878u) + 0x78787878u) | y) & 0x80808080u) >> (7 - (k & 7));
            }
            if (rdl(scan_add((u32)(__popc(bm[0]) + __popc(bm[1]))), 63) > 64u) {
                hdr = H_RAW | 512u;
            } else {
                while (anyb) {
                    const int l = __builtin_ctzll(anyb);
                    const u32 w0 = rdl(bm[0], l), w1 = rdl(bm[1], l);
                    const u32 pos = (u32)__builtin_ctz(w0 ? w0 : w1), k = (w0 ? 0u : 8u) + (pos & 7u);
                    u32 hv = 0;
#pragma unroll
                    for (int kk = 0; kk < 16; ++kk) hv = k == (u32)kk ? hi[kk] : hv;
                    const u32 e = (rdl(hv, l) >> (pos & 24u)) & 0x7fu;
                    if (lane == 0) ((volatile LAS u32*)enc)[64u + sdw + nexc] = ((u32)l << 13) | (k << 9) | ((pos >> 3) << 7) | e;
                    ++nexc;
                    if (lane == l) { if (w0) bm[0] &= bm[0] - 1u; else bm[1] &= bm[1] - 1u; }
                    anyb = __ballot((bm[0] | bm[1]) != 0);
                }
            }
        }
        char* dp = slot + lane * 16;
        if (hdr) {
            st_unit(dp, u);
        } else {
            u32 lo[16];
#pragma unroll
            for (int k = 0; k < 16; ++k) lo[k] = __builtin_amdgcn_perm(u[2 * k + 1], u[2 * k], 0x06040200u);
            st16_sys<0>(dp, (u32x4){lo[0], lo[1], lo[2], lo[3]}); st16_sys<1024>(dp, (u32x4){lo[4], lo[5], lo[6], lo[7]});
            st16_sys<2048>(dp, (u32x4){lo[8], lo[9], lo[10], lo[11]}); st16_sys<3072>(dp, (u32x4){lo[12], lo[13], lo[14], lo[15]});
            st16_sys<0>(dp + 4096, (u32x4){dn[0], dn[1], dn[2], dn[3]});
            ((volatile LAS u32*)enc)[lane] = fl;
            sparse_put(m, fl, inc - cn, (LAS u32*)(enc + 256));
            const u32 n16 = (64u + sdw + nexc + 3u) >> 2;
            tail_out(slot + 5120, enc, n16, lane);
            hdr = (320u + n16) | (a << 10) | (nexc << 17);
        }
    }
    if (own || fill) {
        size_t cnt;
        st_unit(p.ws + local_off(d, unit, p.rank, p.rank * 8 + d.r, cnt) + lane * 16, u);
    }
    return hdr;
}
__device__ __forceinline__ void unit_announce(const Params& p, LAS unsigned char* lds, const PanelId& d, int unit, int pr, u32 hdr, u32 call) {
    const bool fill = is_fill(d, unit);
    if (pr == p.rank) {
        if (d.dest == p.rank || fill) { size_t cnt; local_off(d, unit, p.rank, p.rank * 8 + d.r, cnt); add4_sys(p.ws + cnt, 1u); }
    } else if (!d.isB || pr == d.dest) {
        st8_sys(peer_ws(lds, (u32)pr) + W_HI + (size_t)(((p.rank * 8 + d.r) * 24 + (d.isB ? d.pi : 8 + d.pi)) * 4 + unit) * 8, (u32x2){hdr, call});
    } else if (fill) {
        st8_sys(peer_ws(lds, (u32)pr) + W_HX + (size_t)(((p.rank * 8 + d.r) * 8 + d.dest) * 4 + unit) * 8, (u32x2){hdr, call});
    }
}

constexpr u32 T_IN = 0, T_OUT = 1, T_END = 2, T_DEAD = 3, T_SKIP = 4;
constexpr u32 NP_IN = 384, NP_X = 28 * FC, NP_ROW = 32 - FC, NP_OUT = 16 * NP_ROW, NP_ALL = NP_IN + NP_X + NP_OUT;
#define R_POSTED 0
#define R_TAKEN 1
#define R_FULL 2
#define R_DONE 4
__device__ __forceinline__ volatile LAS u32* ring_words(LAS unsigned char* lds) { return (volatile LAS u32*)(lds + LDS_MISC + 256); }
__device__ __forceinline__ volatile LAS u32x4* ring_desc(LAS unsigned char* lds, u32 n) { return (volatile LAS u32x4*)(lds + LDS_MISC + 512 + (n % (u32)NSLOT) * 32u); }

struct Pair { u32 type, so, ho, cnt, sstep, hstep, cstep, dstep; char* dst; };
__device__ __forceinline__ Pair pair_of(const Params& p, u32 z, int j, int peer, int rot) {
    Pair q; q.type = T_END; q.so = 0; q.ho = (u32)W_TMO; q.cnt = 0; q.sstep = USLOT; q.hstep = 8; q.cstep = 0; q.dstep = USLOT; q.dst = nullptr;
    size_t cnt;
    if (z < NP_IN) {
        PanelId d;
        const int P = (int)(z >> 1), r = P / 24, i = P - r * 24, unit = (int)(z & 1u) * 2;
        d.r = r; d.isB = i < 8; d.dest = p.rank; d.pi = d.isB ? ((i + j) & 7) : ((i - 8 + rot) & 15);
        q.type = T_IN; q.so = (u32)export_off(d, unit);
        q.ho = (u32)W_HI + (u32)(((peer * 8 + r) * 24 + (d.isB ? d.pi : 8 + d.pi)) * 4 + unit) * 8u;
        q.dst = p.ws + local_off(d, unit, p.rank, peer * 8 + r, cnt);
        q.cnt = (u32)cnt;
    } else if (z < NP_IN + NP_X) {
        const int y = (int)(z - NP_IN), rp = y / (7 * FC), yy = y - rp * (7 * FC), jj = yy / FC, k = yy - jj * FC;
        const bool across = (FC & 1) && k == FC - 1;
        PanelId d; d.r = 2 * rp + (across ? 0 : (k & 1)); d.isB = true; d.dest = (p.rank + 1 + jj) & 7; d.pi = 7;
        const int unit = across ? 4 - FC : 2 - (k & 2);
        PanelId e = d; e.r = d.r + (across ? 1 : 0);
        const int unit1 = across ? unit : unit + 1;
        size_t cnt1;
        const u32 l0 = (u32)local_off(d, unit, p.rank, peer * 8 + d.r, cnt), l1 = (u32)local_off(e, unit1, p.rank, peer * 8 + e.r, cnt1);
        q.type = T_IN; q.so = (u32)export_off(d, unit); q.sstep = (u32)export_off(e, unit1) - q.so;
        q.ho = (u32)W_HX + (u32)(((peer * 8 + d.r) * 8 + d.dest) * 4 + unit) * 8u; q.hstep = across ? 256u : 8u;
        q.dst = p.ws + l0; q.dstep = l1 - l0;
        q.cnt = (u32)cnt; q.cstep = (u32)cnt1 - (u32)cnt;
    } else if (z < NP_ALL) {
        const int y = (int)(z - NP_IN - NP_X), pm = y / (int)NP_ROW, i = y - pm * (int)NP_ROW;
        const int pn = i < 28 ? i >> 2 : 7, unit = (i < 28 ? (i & 3) : (i - 28) * 2) * 2;
        const bool across = FC == 3 && i >= 28;
        q.type = T_OUT; q.so = (u32)W_PO + (u32)((pm * 8 + pn) * 8 + unit) * (u32)USLOT;
        q.sstep = across ? 4u * USLOT : (u32)USLOT; q.hstep = across ? 32u : 8u; q.dstep = across ? 128u * GN : 64u;
        q.ho = (u32)W_HO + (u32)((peer * 128 + pm * 8 + pn) * 8 + unit) * 8u;
        q.dst = (char*)p.out + (size_t)(pm * 256 + (unit >> 2) * 128) * GN + peer * NS + pn * 256 + (unit & 3) * 64;
    }
    return q;
}
__device__ __forceinline__ void pull_schedule(const Params& p, LAS unsigned char* lds, u32 call, int j, int wid, int lane) {
    __builtin_amdgcn_s_setprio(2);
    const int peer = (p.rank + 1 + j) & 7, rot = rot_of(j);
    const u32 sub = (u32)lane & 1u;
    const u32 subm = 0u - sub;
    const char* pws = peer_ws(lds, (u32)peer);
    volatile LAS u32* rw = ring_words(lds);
    const u32 qbase = (call - 1u) * (NP_ALL + 2u * (u32)NPW);
    char* qp = p.ws + W_Q + j * 128;
    u32 tk, tk1;
    ticket_issue(tk, qp); ticket_issue(tk1, qp);
    asm volatile("s_waitcnt vmcnt(0)" : "+v"(tk), "+v"(tk1) :: "memory");
    u32 z = rfl(tk) - qbase, zn = rfl(tk1) - qbase, n = 0, slot = 0, dec = 0, lap = 0;
    Pair cur = pair_of(p, z, j, peer, rot);
    u32x2 h = (u32x2){0u, 0u};
    bool dead = false;
    u32 ends = 0;
    for (;;) {
        const bool more = z < NP_ALL;
        if (more && !dead && !__all(h.y == call)) {
            const u64 t0 = now();
            for (;;) {
                u32 t; poll8(h, t, p.ws + (cur.ho + (cur.hstep & subm)), p.ws + W_TMO);
                if (__all(h.y == call)) { dead = rfl(t) == call; break; }
                if (spin_fail(p.ws, call, t0, t)) { dead = true; break; }
            }
        }
        const Pair nxt = pair_of(p, zn, j, peer, rot);
        u32x2 hn;
        ld8_sys(hn, p.ws + (nxt.ho + (nxt.hstep & subm)));
        tk = 0xffffffffu;
        if (more) ticket_issue(tk, qp);
        for (;;) {
            const u32 tkn = rw[R_TAKEN], dn = rw[R_DONE + dec + sub];
            if (__all(n < tkn + (u32)RUNAHEAD && dn + (u32)(NSLOT / ND) > lap)) break;
            __builtin_amdgcn_s_sleep(1);
        }
        if (lane < 2) {
            const u32 type = !dead || !more ? cur.type : (cur.type == T_OUT ? T_DEAD : T_SKIP);
            const u64 src = (u64)(pws + (cur.so + (cur.sstep & subm))), d64 = (u64)(cur.dst + (cur.dstep & subm));
            volatile LAS u32x4* ds = (volatile LAS u32x4*)(lds + LDS_MISC + 512 + (slot + sub) * 32u);
            ds[0] = (u32x4){(u32)src, (u32)(src >> 32), h.x, type};
            ds[1] = (u32x4){(u32)d64, (u32)(d64 >> 32), cur.cnt + (cur.cstep & subm), 0u};
        }
        n += 2;
        rw[R_POSTED] = n;
        slot = slot == (u32)NSLOT - 2u ? 0u : slot + 2u;
        if (dec == (u32)ND - 2u) { dec = 0; ++lap; } else dec += 2u;
        asm volatile("s_waitcnt vmcnt(0)" : "+v"(tk), "+v"(hn) :: "memory");
        if (more) { z = zn; zn = rfl(tk) - qbase; cur = nxt; h = hn; }
        else if (++ends == (u32)ND / 2u) break;
    }
}

#define GLDS(src, off) "global_load_lds_dwordx4 " src ", off offset:" #off " sc0 sc1\n\ts_waitcnt vmcnt(%[d])\n\t"
__device__ __forceinline__ void glds_part(const void* src, u32 dst, u32 rem) {
    const u64 mask = rem >= 64u ? ~0ull : (1ull << rem) - 1ull;
    u32 keep;
    asm volatile("s_mov_b32 %0, m0\n\ts_mov_b32 m0, %2\n\ts_mov_b64 exec, %3\n\t" GLDS("%1", 0) "s_mov_b64 exec, -1\n\ts_mov_b32 m0, %0"
                 : "=&s"(keep) : "v"(src), "s"(dst), "s"(mask), [d] "i"(DEPTH - 1) : "memory");
}
__device__ __forceinline__ void pull_load(const Params& p, LAS unsigned char* lds, int wid, int lane) {
    __builtin_amdgcn_s_setprio(3);
    volatile LAS u32* rw = ring_words(lds);
    const u32 lbase = (u32)(u64)lds;
    u32 ends = 0, slot = 0, full = 0;
    u32 posted = 0;
    u32x4 ds = (u32x4){0u, 0u, 0u, 0u};
    for (u32 n = 0;; ++n) {
        volatile LAS u32x4* dsp = (volatile LAS u32x4*)(lds + LDS_MISC + 512 + slot * 32u);
        if (rfl(posted) <= n) {
            posted = rw[R_POSTED];
            if (rfl(posted) <= n) {
                WAIT_V(0);
                full = n; rw[R_FULL] = n;
                while (rfl(rw[R_POSTED]) <= n) __builtin_amdgcn_s_sleep(1);
            }
            ds = dsp[0];
        }
        const u32 type = rfl(ds.w);
        const u32 dst = lbase + slot * (u32)USLOT;
        slot = slot == (u32)NSLOT - 1u ? 0u : slot + 1u;
        if (type <= T_OUT) {
            const u32 n16 = rfl(min(max(ds.z & 1023u, 336u), 512u));
            const char* src = (const char*)(((u64)ds.y << 32) | ds.x) + lane * 16;
            u32 keep;
            asm volatile("s_mov_b32 %0, m0\n\ts_mov_b32 m0, %2\n\ts_nop 0\n\t" GLDS("%1", 0) GLDS("%1", 1024) "s_mov_b32 m0, %0"
                         : "=&s"(keep) : "v"(src), "s"(dst), [d] "i"(DEPTH - 1) : "memory");
            if (DEPTH == 8 || DEPTH == 9) { if (full + 1u < n) { full = n - 1u; rw[R_FULL] = full; } }
            asm volatile("s_mov_b32 %0, m0\n\ts_mov_b32 m0, %2\n\ts_nop 0\n\t" GLDS("%1", 2048) GLDS("%1", 3072) "s_mov_b32 m0, %0"
                         : "=&s"(keep) : "v"(src), "s"(dst), [d] "i"(DEPTH - 1) : "memory");
            if (DEPTH == 10 || DEPTH == 11) { if (full + 1u < n) { full = n - 1u; rw[R_FULL] = full; } }
            if (DEPTH <= 5) rw[R_FULL] = n;
            posted = rw[R_POSTED];
            ds = *(volatile LAS u32x4*)(lds + LDS_MISC + 512 + slot * 32u);
            src += 4096;
            glds_part(src, dst + 4096u, 64u);
            if (DEPTH == 6) rw[R_FULL] = n;
            glds_part(src + 1024, dst + 5120u, n16 - 320u);
            if (DEPTH == 7) rw[R_FULL] = n;
            if (n16 > 384u) {
                glds_part(src + 2048, dst + 6144u, n16 - 384u);
                if (DEPTH == 8) { full = n; rw[R_FULL] = n; }
                if (n16 > 448u) glds_part(src + 3072, dst + 7168u, n16 - 448u);
            }
            rw[R_TAKEN] = n + 1u;
        } else {
            WAIT_V(0);
            full = n + 1u; rw[R_FULL] = full; rw[R_TAKEN] = full;
            posted = 0;
            if (type == T_END && ++ends == (u32)ND) break;
        }
    }
}

__device__ __forceinline__ void pull_decode(const Params& p, LAS unsigned char* lds, int wid, int lane) {
    volatile LAS u32* rw = ring_words(lds);
    LAS u32* bm = (LAS u32*)(lds + LDS_BM + wid * 1024);
    u32 done = 0;
    for (u32 n = (u32)wid;; n += (u32)ND) {
        while (rfl(rw[R_FULL]) <= n) __builtin_amdgcn_s_sleep(1);
        CFENCE;
        const u32x4 d0 = ring_desc(lds, n)[0], d1 = ring_desc(lds, n)[1];
        const u32 type = rfl(d0.w), hdr = rfl(d0.z);
        if (type == T_END) break;
        char* dst = (char*)(((u64)d1.y << 32) | d1.x);
        LAS unsigned char* sl = lds + (n % (u32)NSLOT) * (u32)USLOT;
        u32 u[32];
        if (type <= T_OUT && (hdr & H_RAW)) {
#pragma unroll
            for (int k = 0; k < 8; ++k) { const u32x4 v = *(const LAS u32x4*)(sl + k * 1024 + lane * 16); u[4 * k] = v.x; u[4 * k + 1] = v.y; u[4 * k + 2] = v.z; u[4 * k + 3] = v.w; }
        } else if (type == T_IN) {
            const u32 fl = *(const LAS u32*)(sl + 5120 + lane * 4);
            const u32x4 dn = *(const LAS u32x4*)(sl + 4096 + lane * 16);
            u32x4 lo[4];
#pragma unroll
            for (int k = 0; k < 4; ++k) lo[k] = *(const LAS u32x4*)(sl + k * 1024 + lane * 16);
            const u32 cn = (u32)__popc(fl), inc = scan_add(cn);
            u32 m[4];
            sparse_get(m, fl, inc - cn, (const LAS u32*)(sl + 5376));
            const u32 b4 = (((hdr >> 10) & 0x7fu) - 6u) * 0x01010101u;
            const u32 tl = b4 + 0x07040506u, th = b4 + 0x00010203u;
            u32 hi[16];
#pragma unroll
            for (int h = 0; h < 2; ++h) {
                const u32 B = bfi(0x55555555u, m[h], m[2 + h] << 1), D = bfi(0x55555555u, m[h] >> 1, m[2 + h]);
                u32 nn[4];
                nn[0] = bfi(0x33333333u, dn[2 * h], B << 2); nn[2] = bfi(0x33333333u, dn[2 * h] >> 2, B);
                nn[1] = bfi(0x33333333u, dn[2 * h + 1], D << 2); nn[3] = bfi(0x33333333u, dn[2 * h + 1] >> 2, D);
#pragma unroll
                for (int k = 0; k < 4; ++k) {
                    hi[8 * h + 2 * k] = bfi(0x80808080u, nn[k] << 7, __builtin_amdgcn_perm(th, tl, (nn[k] >> 1) & 0x07070707u));
                    hi[8 * h + 2 * k + 1] = bfi(0x80808080u, nn[k] << 3, __builtin_amdgcn_perm(th, tl, (nn[k] >> 5) & 0x07070707u));
                }
            }
            const u32 nexc = (hdr >> 17) & 0xffu;
            if (nexc) {
                const u32 sdw = (rdl(inc, 63) + 7u) >> 3;
                const u32 ev = *(const LAS u32*)(sl + 5376 + (sdw + (u32)lane) * 4u);
                for (u32 e = 0; e < nexc; ++e) {
                    const u32 en = rdl(ev, (int)e);
                    const u32 sh = ((en >> 7) & 3u) * 8u, kk = (en >> 9) & 15u;
                    const bool me = (en >> 13) == (u32)lane;
#pragma unroll
                    for (int k = 0; k < 16; ++k)
                        if (me && kk == (u32)k) hi[k] = bfi(0x7fu << sh, (en & 0x7fu) << sh, hi[k]);
                }
            }
#pragma unroll
            for (int k = 0; k < 16; ++k) {
                u[2 * k] = __builtin_amdgcn_perm(hi[k], lo[k >> 2][k & 3], 0x05010400u);
                u[2 * k + 1] = __builtin_amdgcn_perm(hi[k], lo[k >> 2][k & 3], 0x07030602u);
            }
        } else if (type == T_OUT) {
            *(volatile LAS u32x4*)(bm + lane * 4) = (u32x4){0u, 0u, 0u, 0u};
            const u32 fl = *(const LAS u32*)(sl + 6144 + lane * 4);
            u32x4 w4[6];
#pragma unroll
            for (int k = 0; k < 6; ++k) w4[k] = *(const LAS u32x4*)(sl + k * 1024 + lane * 16);
            const u32 cn = (u32)__popc(fl), inc = scan_add(cn);
            const u32 sdw = (rdl(inc, 63) + 7u) >> 3, nexc = (hdr >> 17) & 0xffu;
            const u32 ev = *(const LAS u32*)(sl + 6400 + (sdw + (u32)lane) * 4u);
            u32 m[4];
            sparse_get(m, fl, inc - cn, (const LAS u32*)(sl + 6400));
            if (2u * (u32)lane < nexc) __hip_atomic_fetch_or(bm + ((ev >> 5) & 255u), 1u << (ev & 31u), __ATOMIC_RELAXED, __HIP_MEMORY_SCOPE_WORKGROUP);
            if (2u * (u32)lane + 1u < nexc) __hip_atomic_fetch_or(bm + ((ev >> 21) & 255u), 1u << ((ev >> 16) & 31u), __ATOMIC_RELAXED, __HIP_MEMORY_SCOPE_WORKGROUP);
            CFENCE;
            const u32x4 m7 = *(volatile LAS u32x4*)(bm + lane * 4);
#pragma unroll
            for (int g = 0; g < 8; ++g) {
                const u32 ac = (g & 1) ? bfi(0x55555555u, m[g >> 1] >> 1, m7[g >> 1] << 1) : bfi(0x55555555u, m[g >> 1], m7[g >> 1]);
                u32 r[4]; r[3] = 0;
#pragma unroll
                for (int i = 0; i < 3; ++i) { const int k = 3 * g + i; r[i] = w4[k >> 2][k & 3]; r[3] |= (r[i] >> (6 - 2 * i)) & (0x03030303u << (2 * i)); }
#pragma unroll
                for (int i = 0; i < 4; ++i) {
                    const u32 e = ac << (2 * i), y6 = (r[i] << 1) ^ e, y7 = (y6 << 1) ^ e;
                    u[4 * g + i] = bfi(0x3f3f3f3fu, r[i], bfi(0x40404040u, y6, y7));
                }
            }
        } else {
#pragma unroll
            for (int k = 0; k < 32; ++k) u[k] = 0x80808080u;
        }
        WAIT_L(0);
        ++done;
        if (lane == 0) rw[R_DONE + wid] = done;
        if (type == T_IN) {
            st_unit(dst + lane * 16, u);
            WAIT_V(0);
            if (lane == 0) add4_sys(p.ws + d1.z, 1u);
        } else if (type != T_SKIP) {
            char* o = dst + (size_t)(lane & 15) * GN + (lane >> 4) * 16;
#pragma unroll
            for (int k = 0; k < 8; ++k)
                st16_plain(o + (size_t)k * 16 * GN, (u32x4){u[4 * k], u[4 * k + 1], u[4 * k + 2], u[4 * k + 3]});
        }
    }
}

__device__ __forceinline__ void gemm_tile(const Params& p, LAS unsigned char* lds, u32 call, bool fillm, int pm, int pn, int wid, int lane) {
    const int wr = wid >> 2, wc = wid & 3;
    int j0 = 0;
#pragma unroll
    for (int j = 1; j < 7; ++j) if (rot_of(j) <= pm) j0 = j;
    const char* Ab = p.ws + W_A + (size_t)pm * 64 * PANEL;
    const char* Bb = p.ws + (fillm ? W_X : W_B) + (size_t)pn * 64 * PANEL;
    const char* cA = p.ws + W_ACNT + (size_t)pm * 256;
    const char* cB = p.ws + (fillm ? W_XCNT : W_BCNT) + (size_t)pn * 256;
    const u32 voff = (u32)(wid * 4096 + lane * 16);
    const u32 ldsw = (u32)wid * 2048u;
    const int aoff = wr * 8192 + lane * 16;
    int boff = 65536 + wc * 4096 + lane * 16;
    asm volatile("" : "+v"(boff));
    int ktv = kt_of(lane, p.rank, j0);
    asm volatile("" : "+v"(ktv));
    u64 rdyA = wid == 4 ? 0ull : ~0ull, rdyB = rdyA;
    const u32 needA = call * 4u, needB = (fillm && pn == NVP - 1) ? call * (u32)(7 * FC - 4 * (NVP - 1)) : needA;
#define SA(b, h) (((b) * 2 + (h)) * 16384)
#define SB(b, h) ((4 + (b) * 2 + (h)) * 16384)
#define STAGE(buf, gbase, h) do { \
        __builtin_amdgcn_global_load_lds((const u32*)((gbase) + voff + (h) * 2048), (LAS u32*)(lds + (buf) + ldsw), 16, 0, 17); \
        __builtin_amdgcn_global_load_lds((const u32*)((gbase) + voff + (h) * 2048), (LAS u32*)(lds + (buf) + ldsw), 16, 1024, 17); } while (0)
#define LDA(dst, b, h) do { _Pragma("unroll") for (int m = 0; m < 4; ++m) _Pragma("unroll") for (int k = 0; k < 2; ++k) \
        dst[m][k] = *(const LAS bf16x8*)(lds + SA(b, h) + aoff + m * 2048 + k * 1024); } while (0)
#define LDB(dst, b, h) do { _Pragma("unroll") for (int n = 0; n < 2; ++n) _Pragma("unroll") for (int k = 0; k < 2; ++k) \
        dst[n][k] = *(const LAS bf16x8*)(lds + (SB(b, h) - 65536) + boff + k * 2048 + n * 1024); } while (0)
#define MMA(ai, bj, At, Bt) do { __builtin_amdgcn_s_setprio(1); \
        _Pragma("unroll") for (int m = 0; m < 4; ++m) _Pragma("unroll") for (int n = 0; n < 2; ++n) _Pragma("unroll") for (int k = 0; k < 2; ++k) \
            acc[ai][bj][m][n] = __builtin_amdgcn_mfma_f32_16x16x32_bf16(Bt[n][k], At[m][k], acc[ai][bj][m][n], 0, 0, 0); \
        __builtin_amdgcn_s_setprio(0); } while (0)
#define CHECK(RDY, o) do { if (!((RDY >> (o)) & 1ull)) { \
        const u64 t0_ = now(); __builtin_amdgcn_s_waitcnt(0xC07F); \
        for (;;) { \
            u32 a_, b_, t_; poll4x2(a_, b_, t_, cA + ktv * 4, cB + ktv * 4, p.ws + W_TMO); \
            rdyA = __ballot(a_ >= needA); rdyB = __ballot(b_ >= needB); \
            if ((RDY >> (o)) & 1ull) break; \
            if (spin_fail(p.ws, call, t0_, t_)) { rdyA = rdyB = ~0ull; break; } \
        } } } while (0)
#define KOFF(o) ((size_t)__builtin_amdgcn_readlane(ktv, (o)) * PANEL)

    f32x4 acc[2][2][4][2];
#pragma unroll
    for (int a = 0; a < 2; ++a)
#pragma unroll
        for (int b = 0; b < 2; ++b)
#pragma unroll
            for (int m = 0; m < 4; ++m)
#pragma unroll
                for (int n = 0; n < 2; ++n) acc[a][b][m][n] = (f32x4){0.f, 0.f, 0.f, 0.f};
    bf16x8 At[4][2], B0[2][2], B1[2][2];

    CHECK(rdyA, 0); CHECK(rdyB, 0); CHECK(rdyB, 1); CHECK(rdyA, 1);
    BAR; CFENCE;
    size_t k1 = KOFF(1);
    {
        const size_t k0 = KOFF(0);
        STAGE(SB(0, 0), Bb + k0, 0); STAGE(SB(0, 1), Bb + k0, 1); STAGE(SA(0, 0), Ab + k0, 0);
        STAGE(SB(1, 0), Bb + k1, 0); STAGE(SA(0, 1), Ab + k0, 1); STAGE(SB(1, 1), Bb + k1, 1);
        WAIT_V(6); BAR;
        if (wr == 1) BAR;
    }
    for (int t = 0; t < 64; t += 2) {
        const int o2 = (t + 2) & 63, o3 = o2 + 1;
        const size_t k2 = KOFF(o2), k3 = KOFF(o3);
        LDB(B0, 0, 0); SCHED; LDA(At, 0, 0); STAGE(SA(1, 0), Ab + k1, 0);
        CHECK(rdyB, o2);
        WAIT_L(8); BAR; WAIT_L(0); MMA(0, 0, At, B0); BAR; SCHED;
        LDB(B1, 0, 1); STAGE(SB(0, 0), Bb + k2, 0);
        WAIT_V(6); BAR; WAIT_L(0); MMA(0, 1, At, B1); BAR;
        LDA(At, 0, 1); STAGE(SA(1, 1), Ab + k1, 1);
        BAR; WAIT_L(0); MMA(1, 0, At, B0); BAR; SCHED;
        STAGE(SB(0, 1), Bb + k2, 1);
        CHECK(rdyA, o2);
        WAIT_V(6); BAR; MMA(1, 1, At, B1); BAR;
        LDB(B0, 1, 0); SCHED; LDA(At, 1, 0); STAGE(SA(0, 0), Ab + k2, 0);
        CHECK(rdyB, o3);
        WAIT_L(8); BAR; WAIT_L(0); MMA(0, 0, At, B0); BAR; SCHED;
        LDB(B1, 1, 1); STAGE(SB(1, 0), Bb + k3, 0);
        WAIT_V(6); BAR; WAIT_L(0); MMA(0, 1, At, B1); BAR;
        LDA(At, 1, 1); STAGE(SA(0, 1), Ab + k2, 1);
        BAR; WAIT_L(0); MMA(1, 0, At, B0); BAR; SCHED;
        STAGE(SB(1, 1), Bb + k3, 1);
        CHECK(rdyA, o3);
        WAIT_V(6); BAR; MMA(1, 1, At, B1); BAR;
        k1 = k3;
    }
    if (wr == 0) BAR;
    if (!fillm) STAMP(2);
    int tz = threadIdx.x; asm volatile("" : "+v"(tz));
    lane = tz & 63;
    const int tile = pm * 8 + pn;
    volatile LAS u32* red = (volatile LAS u32*)(lds + LDS_MISC + 1024);
    if (!fillm) {
        float mx = 0.f;
#pragma unroll
        for (int a = 0; a < 2; ++a)
#pragma unroll
            for (int b = 0; b < 2; ++b)
#pragma unroll
                for (int m = 0; m < 4; ++m)
#pragma unroll
                    for (int n = 0; n < 2; ++n)
#pragma unroll
                        for (int r = 0; r < 4; ++r) mx = fmaxf(mx, fabsf(acc[a][b][m][n][r]));
        const u32 wm = wave_umax(__float_as_uint(mx));
        if (lane == 0) red[wid] = wm;
        WAIT_L(0); BAR; CFENCE;
        if (tz < NR) {
            const u32x4 r0 = *(volatile LAS u32x4*)red, r1 = *(volatile LAS u32x4*)(red + 4);
            const u32 t = max(max(max(r0.x, r0.y), max(r0.z, r0.w)), max(max(r1.x, r1.y), max(r1.z, r1.w)));
            st8_sys(peer_ws(lds, (u32)tz) + W_AMAX + (size_t)(p.rank * 128 + tile) * 8, (u32x2){t, call});
        }
    }
    bool dead = false;
    float amax;
    {
        const char* src = p.ws + W_AMAX + tz * 16;
        const u64 t0 = now();
        u32x4 gq;
        for (;;) {
            u32 t; poll16(gq, t, src, p.ws + W_TMO);
            dead = rfl(t) == call;
            if (__all(gq.y == call && gq.w == call)) break;
            if (spin_fail(p.ws, call, t0, t)) { dead = true; break; }
        }
        const u32 wm = wave_umax(max(gq.x, gq.z));
        if (lane == 0) red[8 + wid] = dead ? 0xffffffffu : wm;
        WAIT_L(0); BAR; CFENCE;
        const u32x4 r0 = *(volatile LAS u32x4*)(red + 8), r1 = *(volatile LAS u32x4*)(red + 12);
        const u32 t = rfl(max(max(max(r0.x, r0.y), max(r0.z, r0.w)), max(max(r1.x, r1.y), max(r1.z, r1.w))));
        dead = t == 0xffffffffu;
        amax = __uint_as_float(t);
    }
    if (!fillm) STAMP(3);
    const float inv = amax > 0.f ? 127.0f / amax : 0.f;
    u32 q[32];
#pragma unroll
    for (int m = 0; m < 4; ++m)
#pragma unroll
        for (int a = 0; a < 2; ++a)
#pragma unroll
            for (int r = 0; r < 4; ++r) {
                const u32 t0 = __float_as_uint(fmaf(acc[a][0][m][0][r], inv, 12582912.f)), t1 = __float_as_uint(fmaf(acc[a][0][m][1][r], inv, 12582912.f));
                const u32 t2 = __float_as_uint(fmaf(acc[a][1][m][0][r], inv, 12582912.f)), t3 = __float_as_uint(fmaf(acc[a][1][m][1][r], inv, 12582912.f));
                q[(m * 2 + a) * 4 + r] = __builtin_amdgcn_perm(t1, t0, 0x0c0c0400u) | __builtin_amdgcn_perm(t3, t2, 0x04000c0cu);
            }
    if (dead) {
#pragma unroll
        for (int k = 0; k < 32; ++k) q[k] = 0x80808080u;
    }
    int col = p.rank * NS + pn * 256 + wc * 64;
    if (!fillm) {
        char* slot = p.ws + W_PO + (size_t)(tile * 8 + wid) * USLOT;
        LAS unsigned char* enc = lds + LDS_ENC + wid * 2048;
        *(volatile LAS u32x4*)(enc + lane * 16) = (u32x4){0u, 0u, 0u, 0u};
        *(volatile LAS u32x4*)(enc + 1024 + lane * 16) = (u32x4){0u, 0u, 0u, 0u};
        u32 ac[8], m6[4], m7[4];
#pragma unroll
        for (int g = 0; g < 8; ++g) {
            ac[g] = 0;
#pragma unroll
            for (int i = 0; i < 4; ++i) { const u32 v = q[4 * g + i]; ac[g] |= ((v ^ (v << 1)) & 0xc0c0c0c0u) >> (2 * i); }
        }
#pragma unroll
        for (int h = 0; h < 4; ++h) { m6[h] = bfi(0x55555555u, ac[2 * h], ac[2 * h + 1] << 1); m7[h] = bfi(0xaaaaaaaau, ac[2 * h], ac[2 * h + 1] >> 1); }
        const u32 fl = nib_flags(m6[0]) | (nib_flags(m6[1]) << 1) | (nib_flags(m6[2]) << 2) | (nib_flags(m6[3]) << 3);
        const u32 cn = (u32)__popc(fl) | ((u32)(__popc(m7[0]) + __popc(m7[1]) + __popc(m7[2]) + __popc(m7[3])) << 16);
        const u32 inc = scan_add(cn), tot = rdl(inc, 63), exc = inc - cn;
        const u32 sdw = ((tot & 0xffffu) + 7u) >> 3, nexc = tot >> 16;
        char* dp = slot + lane * 16;
        u32 hdr;
        if (nexc > 128u) {
            st_unit(dp, q);
            hdr = H_RAW | 512u;
        } else {
            u32 w[24];
#pragma unroll
            for (int g = 0; g < 8; ++g)
#pragma unroll
                for (int i = 0; i < 3; ++i) w[3 * g + i] = bfi(0x3f3f3f3fu, q[4 * g + i], q[4 * g + 3] << (6 - 2 * i));
            st16_sys<0>(dp, (u32x4){w[0], w[1], w[2], w[3]}); st16_sys<1024>(dp, (u32x4){w[4], w[5], w[6], w[7]});
            st16_sys<2048>(dp, (u32x4){w[8], w[9], w[10], w[11]}); st16_sys<3072>(dp, (u32x4){w[12], w[13], w[14], w[15]});
            st16_sys<0>(dp + 4096, (u32x4){w[16], w[17], w[18], w[19]}); st16_sys<1024>(dp + 4096, (u32x4){w[20], w[21], w[22], w[23]});
            ((volatile LAS u32*)enc)[lane] = fl;
            sparse_put(m6, fl, exc & 0xffffu, (LAS u32*)(enc + 256));
            volatile LAS unsigned short* list = (volatile LAS unsigned short*)(enc + 256 + sdw * 4u);
            u32 s = exc >> 16;
#pragma unroll
            for (int h = 0; h < 4; ++h) {
                u32 mm = m7[h];
                while (mm) { list[s] = (unsigned short)((lane * 4 + h) * 32 + __builtin_ctz(mm)); mm &= mm - 1u; ++s; }
            }
            const u32 n16 = (64u + sdw + ((nexc + 1u) >> 1) + 3u) >> 2;
            tail_out(slot + 6144, enc, n16, lane);
            hdr = (384u + n16) | (nexc << 17);
        }
        if (lane == 0) red[16 + wid] = hdr;
        WAIT_V(0); WAIT_L(0); BAR; CFENCE;
        if (tz < 64 && (tz >> 3) != p.rank)
            st8_sys(peer_ws(lds, (u32)(tz >> 3)) + W_HO + (size_t)((p.rank * 128 + tile) * 8 + (tz & 7)) * 8, (u32x2){red[16 + (tz & 7)], call});
        STAMP(4);
    } else {
        const int g = pn * 4 + wc;
        col = g < 7 * FC ? ((p.rank + 1 + g / FC) & 7) * NS + 7 * 256 + (4 - FC + g % FC) * 64 : -1;
    }
    if (col >= 0) {
        signed char* o = p.out + (size_t)(pm * 256 + wr * 128 + (lane & 15)) * GN + col + (lane >> 4) * 16;
#pragma unroll
        for (int k = 0; k < 8; ++k)
            st16_plain(o + (size_t)k * 16 * GN, (u32x4){q[4 * k], q[4 * k + 1], q[4 * k + 2], q[4 * k + 3]});
    }
#undef SA
#undef SB
#undef STAGE
#undef LDA
#undef LDB
#undef MMA
#undef CHECK
#undef KOFF
}

__global__ void __launch_bounds__(BLOCK, 2) fused_gemm_q8(Params p) {
    __shared__ __attribute__((aligned(16))) unsigned char smem[LDS_BYTES];
    LAS unsigned char* lds = (LAS unsigned char*)smem;
    const int tid = threadIdx.x, wid = (int)rfl((u32)tid >> 6), lane = tid & 63;
    const int xcd = blockIdx.x & 7, idx = blockIdx.x >> 3;
    STAMP(0);
    volatile LAS u32* misc = (volatile LAS u32*)(lds + LDS_MISC);
    LAS unsigned char* enc = lds + LDS_ENC + wid * 2048;
    const int n0 = wid * 256 + (int)blockIdx.x;
    if (n0 < 320) {
        const PanelId d0 = panel_of(0, n0 >> 2, p.rank);
        f32x4 f[16];
        unit_load(p, d0, n0 & 3, lane, f);
        WAIT_V(0); PIN_F(f);
        const u32 hdr = unit_pack(p, enc, d0, n0 & 3, lane, f);
        BAR; CFENCE;
        const u32 call0 = misc[0];
        WAIT_V(0);
        if (lane < NR) unit_announce(p, lds, d0, n0 & 3, lane, hdr, call0);
    } else {
        if (wid == 7) {
            u32 tk; u32x2 pw;
            ticket_issue(tk, p.ws + W_SEQ + (size_t)blockIdx.x * 128);
            ld8_sys(pw, (const char*)p.ws_all + (lane & 7) * 8);
            asm volatile("s_waitcnt vmcnt(0)" : "+v"(tk), "+v"(pw) :: "memory");
            if (lane == 0) misc[0] = tk + 1u;
            if (lane < NR) *(volatile LAS u32x2*)(lds + LDS_MISC + 64 + lane * 8) = pw;
            if (lane < 16) ring_words(lds)[lane] = 0u;
            WAIT_L(0);
        }
        BAR; CFENCE;
    }
    const u32 call = rfl(misc[0]);
    STAMP(6);

    const bool is_gemm = idx < 16;
    const int c = idx - 16;
    const int jl = (peer_of(xcd, p.rank) - p.rank - 1) & 7;
    if (!is_gemm && jl < 7 && c < NPW) {
        if (wid == 7) pull_schedule(p, lds, call, jl, wid, lane);
        else if (wid == 6) pull_load(p, lds, wid, lane);
        else pull_decode(p, lds, wid, lane);
        STAMP(5);
        return;
    }
    const int pm = (xcd >> 1) * 4 + (idx >> 2), pn = (xcd & 1) * 4 + (idx & 3);
    if (is_gemm) {
        const int b = (int)blockIdx.x;
        const int e = wid >= 2 && wid < 7 ? (wid - 2) * 128 + b : (wid == 1 && b >= 64 && b < 96 ? 576 + b : -1);
        if (e >= 0) {
            const int r = e / 96, i = e - r * 96;
            const PanelId d = panel_of(1 + r, 56 + (i >> 2), p.rank);
            f32x4 f[16];
            unit_load(p, d, i & 3, lane, f);
            WAIT_V(0); PIN_F(f);
            const u32 hdr = unit_pack(p, enc, d, i & 3, lane, f);
            WAIT_V(0);
            if (lane < NR) unit_announce(p, lds, d, i & 3, lane, hdr, call);
        }
        STAMP(1);
    } else {
        const int me = jl < 7 ? jl * (16 - NPW) + (c - NPW) : 7 * (16 - NPW) + c;
        int q = me;
        PanelId d = panel_of(1 + q / 28, (q % 28) * 2 + (wid >> 2), p.rank);
        f32x4 f[16];
        if (SPDLY && me >= 28) { const u64 t0 = now(); while (now() - t0 < SPDLY) __builtin_amdgcn_s_sleep(8); }
        unit_load(p, d, wid & 3, lane, f);
        volatile LAS u32* hb = (volatile LAS u32*)(lds + LDS_MISC + 1024);
        for (int step = 0; q < 196; ++step) {
            WAIT_V(0); PIN_F(f);
            const u32 hdr = unit_pack(p, enc, d, wid & 3, lane, f);
            if (lane == 0) hb[(step & 1) * 8 + wid] = hdr;
            const int qn = q + NSPARE;
            d = panel_of(1 + qn / 28, (qn % 28) * 2 + (wid >> 2), p.rank);
            if (qn < 196) { unit_load(p, d, wid & 3, lane, f); WAIT_V(16); } else { WAIT_V(0); }
            WAIT_L(0); BAR; CFENCE;
            if (wid == 0)
                unit_announce(p, lds, panel_of(1 + q / 28, (q % 28) * 2 + ((lane >> 2) & 1), p.rank), lane & 3, lane >> 3, hb[(step & 1) * 8 + (lane & 7)], call);
            q = qn;
        }
        STAMP(1);
        return;
    }
#pragma clang loop unroll(disable)
    for (int pass = 0; pass < (pn < NVP ? 2 : 1); ++pass) {
        int tz = threadIdx.x; asm volatile("" : "+v"(tz));
        gemm_tile(p, lds, call, pass == 1, pm, pn, wid, tz & 63);
        if (pass == 0) STAMP(5); else STAMP(13);
    }
}

extern "C" void kernel_launch(int rank, int nranks, ncclComm_t comm, hipStream_t stream,
                              void* const* d_in, const int* in_sizes, int n_in,
                              void* d_out, int out_size,
                              const int* in_dtypes, int out_dtype,
                              void* const* d_out_all,
                              void* const* d_ws_all, size_t ws_bytes) {
    (void)comm; (void)in_dtypes; (void)out_dtype; (void)d_out_all;
    if (nranks != NR || n_in < 2 || ws_bytes < W_END || in_sizes[0] != GM * KS || in_sizes[1] != KS * GN || out_size != GM * GN ||
        (unsigned)rank >= (unsigned)NR) {
        fprintf(stderr, "unexpected problem geometry\n");
        return;
    }
    Params p;
    p.x = (const float*)d_in[0];
    p.w = (const float*)d_in[1];
    p.out = (signed char*)d_out;
    p.ws = (char*)d_ws_all[rank];
    p.ws_all = (char* const*)d_ws_all;
    p.rank = rank;
    p.pad = 0;
    hipLaunchKernelGGL(fused_gemm_q8, dim3(GRID), dim3(BLOCK), 0, stream, p);
}
```

```cpp
#include <hip/hip_runtime.h>
#include <rccl/rccl.h>
#include <cstdio>
#include <cstdint>

#define STAMPS 0

#define LAS __attribute__((address_space(3)))
typedef unsigned u32;
typedef unsigned long long u64;
typedef u32 u32x4 __attribute__((ext_vector_type(4)));
typedef u32 u32x2 __attribute__((ext_vector_type(2)));
typedef float f32x4 __attribute__((ext_vector_type(4)));
typedef short bf16x8 __attribute__((ext_vector_type(8)));

constexpr int NR = 8, GM = 4096, GN = 16384, KS = 512, NS = 2048;
constexpr int GRID = 256, BLOCK = 512;
constexpr int PANEL = 32768;
constexpr int USLOT = 8192;
constexpr size_t W_TMO = 128;
constexpr size_t W_ROT = 256;
constexpr size_t W_Q = 1024;
constexpr size_t W_ACNT = 4096;
constexpr size_t W_BCNT = 8192;
constexpr size_t W_XCNT = 10240;
constexpr size_t W_AMAX = 16384;
constexpr size_t W_HI = 32768;
constexpr size_t W_HX = 81920;
constexpr size_t W_HO = 98304;
constexpr size_t W_SEQ = 256 << 10;
constexpr size_t W_DBG = 512 << 10;
constexpr size_t W_A = 1ull << 20;
constexpr size_t W_B = 33ull << 20;
constexpr size_t W_X = 49ull << 20;
constexpr size_t W_PX = 65ull << 20;
constexpr size_t W_PW = 69ull << 20;
constexpr size_t W_PO = 85ull << 20;
constexpr size_t W_END = 93ull << 20;
constexpr u64 DEADLINE = 40000000ull;

#define NPW 8
#define DEPTH 6
#define RUNAHEAD 2
#define NEAR 1
#define NTLD 0
#define SPDLY 0
constexpr int ND = 6, NSLOT = 12;
constexpr int NSPARE = 7 * (16 - NPW) + 16;
#define FC 3
constexpr int NVP = (7 * FC + 3) / 4;
static_assert(DEPTH <= 11 && FC >= 2 && FC <= 4, "roles");

constexpr int LDS_BM = 98304;
constexpr int LDS_ENC = 114688;
constexpr int LDS_MISC = 131072;
constexpr int LDS_BYTES = LDS_MISC + 2048;

struct Params {
    const float* x;
    const float* w;
    signed char* out;
    char* ws;
    char* const* ws_all;
    int rank;
    int pad;
};

#define WAIT_V(n) asm volatile("s_waitcnt vmcnt(" #n ")" ::: "memory")
#define WAIT_L(n) asm volatile("s_waitcnt lgkmcnt(" #n ")" ::: "memory")
#define BAR __builtin_amdgcn_s_barrier()
#define SCHED __builtin_amdgcn_sched_barrier(0)
#define CFENCE asm volatile("" ::: "memory")

template <int OFF> __device__ __forceinline__ void st16_sys(void* p, u32x4 x) {
    asm volatile("global_store_dwordx4 %0, %1, off offset:%2 sc0 sc1\n\ts_nop 1" :: "v"(p), "v"(x), "i"(OFF) : "memory");
}
__device__ __forceinline__ void st16_plain(void* p, u32x4 x) {
    asm volatile("global_store_dwordx4 %0, %1, off\n\ts_nop 1" :: "v"(p), "v"(x) : "memory");
}
__device__ __forceinline__ void ld8_sys(u32x2& x, const void* p) {
    asm volatile("global_load_dwordx2 %0, %1, off sc0 sc1" : "=v"(x) : "v"(p) : "memory");
}
__device__ __forceinline__ void st8_sys(void* p, u32x2 x) {
    asm volatile("global_store_dwordx2 %0, %1, off sc0 sc1\n\ts_nop 1" :: "v"(p), "v"(x) : "memory");
}
__device__ __forceinline__ void st4_sys(void* p, u32 x) {
    asm volatile("global_store_dword %0, %1, off sc0 sc1\n\ts_nop 1" :: "v"(p), "v"(x) : "memory");
}
__device__ __forceinline__ void add4_sys(void* p, u32 x) {
    asm volatile("global_atomic_add %0, %1, off sc1\n\ts_nop 1" :: "v"(p), "v"(x) : "memory");
}
__device__ __forceinline__ void ld16_nt(f32x4& x, const void* p) {
    asm volatile("global_load_dwordx4 %0, %1, off" : "=v"(x) : "v"(p) : "memory");
}
__device__ __forceinline__ void ticket_issue(u32& x, void* p) {
    u64 sv;
    asm volatile("s_mov_b64 %1, exec\n\ts_mov_b64 exec, 1\n\tglobal_atomic_add %0, %2, %3, off sc0 sc1\n\ts_mov_b64 exec, %1"
                 : "=&v"(x), "=&s"(sv) : "v"(p), "v"(1u) : "memory");
}
__device__ __forceinline__ void poll16(u32x4& x, u32& t, const void* p, const void* tp) {
    asm volatile("global_load_dwordx4 %0, %2, off sc0 sc1\n\tglobal_load_dword %1, %3, off sc0 sc1\n\ts_waitcnt vmcnt(0)"
                 : "=&v"(x), "=&v"(t) : "v"(p), "v"(tp) : "memory");
}
__device__ __forceinline__ void poll8(u32x2& x, u32& t, const void* p, const void* tp) {
    asm volatile("global_load_dwordx2 %0, %2, off sc0 sc1\n\tglobal_load_dword %1, %3, off sc0 sc1\n\ts_waitcnt vmcnt(0)"
                 : "=&v"(x), "=&v"(t) : "v"(p), "v"(tp) : "memory");
}
__device__ __forceinline__ void poll4x2(u32& x, u32& y, u32& t, const void* p, const void* q, const void* tp) {
    asm volatile("global_load_dword %0, %3, off sc0 sc1\n\tglobal_load_dword %1, %4, off sc0 sc1\n\tglobal_load_dword %2, %5, off sc0 sc1\n\ts_waitcnt vmcnt(0)"
                 : "=&v"(x), "=&v"(y), "=&v"(t) : "v"(p), "v"(q), "v"(tp) : "memory");
}
__device__ __forceinline__ u32 cvt_pk_bf16(float lo, float hi) {
    u32 r; asm("v_cvt_pk_bf16_f32 %0, %1, %2" : "=v"(r) : "v"(lo), "v"(hi)); return r;
}
__device__ __forceinline__ u32 bfi(u32 m, u32 a, u32 b) { return (a & m) | (b & ~m); }
__device__ __forceinline__ u32 rfl(u32 x) { return (u32)__builtin_amdgcn_readfirstlane((int)x); }
__device__ __forceinline__ u32 rdl(u32 x, int l) { return (u32)__builtin_amdgcn_readlane((int)x, l); }
__device__ __forceinline__ u64 now() { return __builtin_amdgcn_s_memrealtime(); }

__device__ __forceinline__ bool spin_fail(char* ws, u32 call, u64 t0, u32 t) {
    if (rfl(t) == call) return true;
    if (now() - t0 > DEADLINE) { st4_sys(ws + W_TMO, call); WAIT_V(0); return true; }
    __builtin_amdgcn_s_sleep(1);
    return false;
}
#define DPP0(v, ctl, rm, bc) (u32)__builtin_amdgcn_update_dpp(0, (int)(v), ctl, rm, 0xf, bc)
__device__ __forceinline__ u32 scan_add(u32 v) {
    v += DPP0(v, 0x111, 0xf, true); v += DPP0(v, 0x112, 0xf, true); v += DPP0(v, 0x114, 0xf, true); v += DPP0(v, 0x118, 0xf, true);
    v += DPP0(v, 0x142, 0xa, false); v += DPP0(v, 0x143, 0xc, false);
    return v;
}
__device__ __forceinline__ u32 wave_umax(u32 v) {
    v = max(v, DPP0(v, 0x111, 0xf, true)); v = max(v, DPP0(v, 0x112, 0xf, true)); v = max(v, DPP0(v, 0x114, 0xf, true)); v = max(v, DPP0(v, 0x118, 0xf, true));
    v = max(v, DPP0(v, 0x142, 0xa, false)); v = max(v, DPP0(v, 0x143, 0xc, false));
    return rdl(v, 63);
}
__device__ __forceinline__ int rot_of(int j) { return (j * 16) / 7; }
__device__ __forceinline__ int kt_of(int o, int rank, int j0) {
    if (o < 8) return rank * 8 + o;
    const int r = (o - 8) / 7, s = o - 8 - r * 7;
    int j = j0 - s; if (j < 0) j += 7;
    return ((rank + 1 + j) & 7) * 8 + r;
}
__device__ __forceinline__ int peer_of(int x, int rank) {
    const u32 ga = 0x6320u, gb = 0x7541u;
    const bool ina = (0x4du >> rank) & 1u;
    const u32 own = ina ? ga : gb, oth = ina ? gb : ga;
    if (x >= 4) return (int)((oth >> (4 * (x - 4))) & 7u);
    if (x == 1) return rank;
    const int want = x == 0 ? 0 : x - 1;
    int n = 0, res = rank;
#pragma unroll
    for (int i = 0; i < 4; ++i) { const int q = (int)((own >> (4 * i)) & 7u); if (q != rank) { if (n == want) res = q; ++n; } }
    return res;
}
__device__ __forceinline__ char* peer_ws(LAS unsigned char* lds, u32 pr) {
    return (char*)((volatile LAS u64*)(lds + LDS_MISC + 64))[pr & 7u];
}

#define STAMP(i) do {} while (0)

__device__ __forceinline__ u32 nib_flags(u32 m) { u32 t = m | (m >> 1); t |= t >> 2; return t & 0x11111111u; }
__device__ __forceinline__ void sparse_put(const u32 (&m)[4], u32 fl, u32 off, LAS u32* st) {
#pragma unroll
    for (int w = 0; w < 4; ++w) {
        const u32 t = (fl >> w) & 0x11111111u, pin = t * 0x11111111u, pex = pin << 4;
        const u32 pe = (pex & 0x0f0f0f0fu) << 2, po = (pex >> 2) & 0x3c3c3c3cu;
        u32 c = m[w] & 15u;
#pragma unroll
        for (int i = 1; i < 8; ++i) c |= ((m[w] >> (4 * i)) & 15u) << ((((i & 1) ? po : pe) >> (8 * (i >> 1))) & 31u);
        const u64 x = (u64)c << ((off & 7u) * 4u);
        __hip_atomic_fetch_or(st + (off >> 3), (u32)x, __ATOMIC_RELAXED, __HIP_MEMORY_SCOPE_WORKGROUP);
        __hip_atomic_fetch_or(st + (off >> 3) + 1, (u32)(x >> 32), __ATOMIC_RELAXED, __HIP_MEMORY_SCOPE_WORKGROUP);
        off += pin >> 28;
    }
}
__device__ __forceinline__ void sparse_get(u32 (&m)[4], u32 fl, u32 off, const LAS u32* st) {
    u32 lo[4], hi[4], sh[4], pin[4];
#pragma unroll
    for (int w = 0; w < 4; ++w) {
        pin[w] = ((fl >> w) & 0x11111111u) * 0x11111111u;
        lo[w] = st[off >> 3]; hi[w] = st[(off >> 3) + 1]; sh[w] = (off & 7u) * 4u;
        off += pin[w] >> 28;
    }
#pragma unroll
    for (int w = 0; w < 4; ++w) {
        const u32 t = (fl >> w) & 0x11111111u, pex = pin[w] << 4;
        const u32 pe = (pex & 0x0f0f0f0fu) << 2, po = (pex >> 2) & 0x3c3c3c3cu;
        const u32 c = __builtin_amdgcn_alignbit(hi[w], lo[w], sh[w]);
        u32 r = c & 15u;
#pragma unroll
        for (int i = 1; i < 8; ++i) r |= ((c >> ((((i & 1) ? po : pe) >> (8 * (i >> 1))) & 31u)) & 15u) << (4 * i);
        m[w] = r & ((t << 4) - t);
    }
}
__device__ __forceinline__ void tail_out(char* dst, LAS unsigned char* enc, u32 n16, int lane) {
    CFENCE;
    const u32x4 a = *(volatile LAS u32x4*)(enc + lane * 16), b = *(volatile LAS u32x4*)(enc + 1024 + lane * 16);
    if ((u32)lane < n16) st16_sys<0>(dst + lane * 16, a);
    if ((u32)lane + 64u < n16) st16_sys<1024>(dst + lane * 16, b);
}
constexpr u32 H_RAW = 0x80000000u;

struct PanelId { int r, dest, pi; bool isB; };
__device__ __forceinline__ PanelId panel_of(int r, int i, int rank) {
    PanelId d; d.r = r; d.isB = i < 56 || i >= 72;
    d.dest = i < 56 ? ((rank + 1 + (i >> 3)) & 7) : rank;
    d.pi = d.isB ? (i & 7) : i - 56;
    return d;
}
__device__ __forceinline__ bool is_fill(const PanelId& d, int unit) { return d.isB && d.pi == 7 && unit >= 4 - FC; }
__device__ __forceinline__ void unit_load(const Params& p, const PanelId& d, int unit, int lane, f32x4 (&f)[16]) {
    const int fr = lane & 15, fq = lane >> 4;
    const float* b = d.isB ? p.w + (size_t)(d.r * 64 + 4 * fq) * GN + d.dest * NS + d.pi * 256 + unit * 64 + 4 * fr
                           : p.x + (size_t)(d.pi * 256 + unit * 64 + fr) * KS + d.r * 64 + 4 * fq;
    const int s1 = d.isB ? 16 * GN : 16 * KS, s2 = d.isB ? GN : 16;
#pragma unroll
    for (int j = 0; j < 16; ++j) ld16_nt(f[j], b + (j >> 2) * s1 + (j & 3) * s2);
}
#define PIN_F(f) do { asm volatile("" : "+v"(f[0]), "+v"(f[1]), "+v"(f[2]), "+v"(f[3]), "+v"(f[4]), "+v"(f[5]), "+v"(f[6]), "+v"(f[7]) :: "memory"); \
    asm volatile("" : "+v"(f[8]), "+v"(f[9]), "+v"(f[10]), "+v"(f[11]), "+v"(f[12]), "+v"(f[13]), "+v"(f[14]), "+v"(f[15]) :: "memory"); } while (0)
__device__ __forceinline__ void st_unit(char* dp, const u32 (&u)[32]) {
    st16_sys<0>(dp, (u32x4){u[0], u[1], u[2], u[3]}); st16_sys<1024>(dp, (u32x4){u[4], u[5], u[6], u[7]});
    st16_sys<2048>(dp, (u32x4){u[8], u[9], u[10], u[11]}); st16_sys<3072>(dp, (u32x4){u[12], u[13], u[14], u[15]});
    dp += 4096;
    st16_sys<0>(dp, (u32x4){u[16], u[17], u[18], u[19]}); st16_sys<1024>(dp, (u32x4){u[20], u[21], u[22], u[23]});
    st16_sys<2048>(dp, (u32x4){u[24], u[25], u[26], u[27]}); st16_sys<3072>(dp, (u32x4){u[28], u[29], u[30], u[31]});
}
__device__ __forceinline__ size_t local_off(const PanelId& d, int unit, int rank, int kt, size_t& cnt) {
    if (d.dest == rank) { cnt = (d.isB ? W_BCNT : W_ACNT) + (size_t)(d.pi * 64 + kt) * 4; return (d.isB ? W_B : W_A) + (size_t)(d.pi * 64 + kt) * PANEL + unit * USLOT; }
    const int g = ((d.dest - rank - 1) & 7) * FC + unit - (4 - FC);
    cnt = W_XCNT + (size_t)((g >> 2) * 64 + kt) * 4;
    return W_X + (size_t)((g >> 2) * 64 + kt) * PANEL + (g & 3) * USLOT;
}
__device__ __forceinline__ size_t export_off(const PanelId& d, int unit) {
    return (d.isB ? W_PW + (size_t)((d.dest * 8 + d.r) * 8 + d.pi) * PANEL : W_PX + (size_t)(d.r * 16 + d.pi) * PANEL) + unit * USLOT;
}
__device__ __forceinline__ u32 unit_pack(const Params& p, LAS unsigned char* enc, const PanelId& d, int unit, int lane, const f32x4 (&f)[16]) {
    u32 u[32];
    if (d.isB) {
#pragma unroll
        for (int h = 0; h < 2; ++h)
#pragma unroll
            for (int c = 0; c < 4; ++c)
#pragma unroll
                for (int t = 0; t < 4; ++t) u[16 * h + 4 * c + t] = cvt_pk_bf16(f[8 * h + 2 * t][c], f[8 * h + 2 * t + 1][c]);
    } else {
#pragma unroll
        for (int c = 0; c < 16; ++c) { u[2 * c] = cvt_pk_bf16(f[c][0], f[c][1]); u[2 * c + 1] = cvt_pk_bf16(f[c][2], f[c][3]); }
    }
    const bool own = d.dest == p.rank, fill = is_fill(d, unit);
    u32 hdr = 0;
    if (!(d.isB && own) || fill) {
        char* slot = p.ws + export_off(d, unit);
        *(volatile LAS u32x4*)(enc + lane * 16) = (u32x4){0u, 0u, 0u, 0u};
        *(volatile LAS u32x4*)(enc + 1024 + lane * 16) = (u32x4){0u, 0u, 0u, 0u};
        u32 hi[16], t[16], sum = 0;
#pragma unroll
        for (int k = 0; k < 16; ++k) {
            hi[k] = __builtin_amdgcn_perm(u[2 * k + 1], u[2 * k], 0x07050301u);
            t[k] = hi[k] | 0x80808080u;
            sum = __builtin_amdgcn_sad_u8(t[k], 0u, sum);
        }
        u32 a = ((rdl(scan_add(sum), 63) + 4095u) >> 12) - 128u;
        a = min(max(a, 6u), 126u);
        const u32 b4 = (a - 6u) * 0x01010101u;
        u32 bad = 0, n[8];
#pragma unroll
        for (int k = 0; k < 16; ++k) { t[k] -= b4; bad |= (t[k] & 0xf8f8f8f8u) ^ 0x80808080u; }
#pragma unroll
        for (int k = 0; k < 8; ++k)
            n[k] = __builtin_amdgcn_perm(0x06000204u, 0x080a0c0eu, t[2 * k] & 0x07070707u) | __builtin_amdgcn_perm(0x60002040u, 0x80a0c0e0u, t[2 * k + 1] & 0x07070707u)
                 | ((hi[2 * k] >> 7) & 0x01010101u) | ((hi[2 * k + 1] >> 3) & 0x10101010u);
        u32 dn[4], m[4];
#pragma unroll
        for (int h = 0; h < 2; ++h) {
            dn[2 * h] = bfi(0x33333333u, n[4 * h], n[4 * h + 2] << 2); dn[2 * h + 1] = bfi(0x33333333u, n[4 * h + 1], n[4 * h + 3] << 2);
            const u32 B = bfi(0x33333333u, n[4 * h] >> 2, n[4 * h + 2]), D = bfi(0x33333333u, n[4 * h + 1] >> 2, n[4 * h + 3]);
            m[h] = bfi(0x55555555u, B, D << 1); m[2 + h] = bfi(0x55555555u, B >> 1, D);
        }
        const u32 fl = nib_flags(m[0]) | (nib_flags(m[1]) << 1) | (nib_flags(m[2]) << 2) | (nib_flags(m[3]) << 3);
        const u32 cn = (u32)__popc(fl), inc = scan_add(cn);
        const u32 sdw = (rdl(inc, 63) + 7u) >> 3;
        u32 nexc = 0;
        u64 anyb = __ballot(bad != 0);
        if (anyb) {
            u32 bm[2] = {0u, 0u};
#pragma unroll
            for (int k = 0; k < 16; ++k) {
                const u32 y = (t[k] & 0xf8f8f8f8u) ^ 0x80808080u;
                bm[k >> 3] |= ((((y & 0x78787878u) + 0x78787878u) | y) & 0x80808080u) >> (7 - (k & 7));
            }
            if (rdl(scan_add((u32)(__popc(bm[0]) + __popc(bm[1]))), 63) > 64u) {
                hdr = H_RAW | 512u;
            } else {
                while (anyb) {
                    const int l = __builtin_ctzll(anyb);
                    const u32 w0 = rdl(bm[0], l), w1 = rdl(bm[1], l);
                    const u32 pos = (u32)__builtin_ctz(w0 ? w0 : w1), k = (w0 ? 0u : 8u) + (pos & 7u);
                    u32 hv = 0;
#pragma unroll
                    for (int kk = 0; kk < 16; ++kk) hv = k == (u32)kk ? hi[kk] : hv;
                    const u32 e = (rdl(hv, l) >> (pos & 24u)) & 0x7fu;
                    if (lane == 0) ((volatile LAS u32*)enc)[64u + sdw + nexc] = ((u32)l << 13) | (k << 9) | ((pos >> 3) << 7) | e;
                    ++nexc;
                    if (lane == l) { if (w0) bm[0] &= bm[0] - 1u; else bm[1] &= bm[1] - 1u; }
                    anyb = __ballot((bm[0] | bm[1]) != 0);
                }
            }
        }
        char* dp = slot + lane * 16;
        if (hdr) {
            st_unit(dp, u);
        } else {
            u32 lo[16];
#pragma unroll
            for (int k = 0; k < 16; ++k) lo[k] = __builtin_amdgcn_perm(u[2 * k + 1], u[2 * k], 0x06040200u);
            st16_sys<0>(dp, (u32x4){lo[0], lo[1], lo[2], lo[3]}); st16_sys<1024>(dp, (u32x4){lo[4], lo[5], lo[6], lo[7]});
            st16_sys<2048>(dp, (u32x4){lo[8], lo[9], lo[10], lo[11]}); st16_sys<3072>(dp, (u32x4){lo[12], lo[13], lo[14], lo[15]});
            st16_sys<0>(dp + 4096, (u32x4){dn[0], dn[1], dn[2], dn[3]});
            ((volatile LAS u32*)enc)[lane] = fl;
            sparse_put(m, fl, inc - cn, (LAS u32*)(enc + 256));
            const u32 n16 = (64u + sdw + nexc + 3u) >> 2;
            tail_out(slot + 5120, enc, n16, lane);
            hdr = (320u + n16) | (a << 10) | (nexc << 17);
        }
    }
    if (own || fill) {
        size_t cnt;
        st_unit(p.ws + local_off(d, unit, p.rank, p.rank * 8 + d.r, cnt) + lane * 16, u);
    }
    return hdr;
}
__device__ __forceinline__ void unit_announce(const Params& p, LAS unsigned char* lds, const PanelId& d, int unit, int pr, u32 hdr, u32 call) {
    const bool fill = is_fill(d, unit);
    if (pr == p.rank) {
        if (d.dest == p.rank || fill) { size_t cnt; local_off(d, unit, p.rank, p.rank * 8 + d.r, cnt); add4_sys(p.ws + cnt, 1u); }
    } else if (!d.isB || pr == d.dest) {
        st8_sys(peer_ws(lds, (u32)pr) + W_HI + (size_t)(((p.rank * 8 + d.r) * 24 + (d.isB ? d.pi : 8 + d.pi)) * 4 + unit) * 8, (u32x2){hdr, call});
    } else if (fill) {
        st8_sys(peer_ws(lds, (u32)pr) + W_HX + (size_t)(((p.rank * 8 + d.r) * 8 + d.dest) * 4 + unit) * 8, (u32x2){hdr, call});
    }
}

constexpr u32 T_IN = 0, T_OUT = 1, T_END = 2, T_DEAD = 3, T_SKIP = 4;
constexpr u32 NP_IN = 384, NP_X = 28 * FC, NP_ROW = 32 - FC, NP_OUT = 16 * NP_ROW, NP_ALL = NP_IN + NP_X + NP_OUT;
#define R_POSTED 0
#define R_TAKEN 1
#define R_FULL 2
#define R_DONE 4
__device__ __forceinline__ volatile LAS u32* ring_words(LAS unsigned char* lds) { return (volatile LAS u32*)(lds + LDS_MISC + 256); }
__device__ __forceinline__ volatile LAS u32x4* ring_desc(LAS unsigned char* lds, u32 n) { return (volatile LAS u32x4*)(lds + LDS_MISC + 512 + (n % (u32)NSLOT) * 32u); }

struct Pair { u32 type, so, ho, cnt, sstep, hstep, cstep, dstep; char* dst; };
__device__ __forceinline__ Pair pair_of(const Params& p, u32 z, int j, int peer, int rot) {
    Pair q; q.type = T_END; q.so = 0; q.ho = (u32)W_TMO; q.cnt = 0; q.sstep = USLOT; q.hstep = 8; q.cstep = 0; q.dstep = USLOT; q.dst = nullptr;
    size_t cnt;
    if (z < NP_IN) {
        PanelId d;
        const int P = (int)(z >> 1), r = P / 24, i = P - r * 24, unit = (int)(z & 1u) * 2;
        d.r = r; d.isB = i < 8; d.dest = p.rank; d.pi = d.isB ? ((i + j) & 7) : ((i - 8 + rot) & 15);
        q.type = T_IN; q.so = (u32)export_off(d, unit);
        q.ho = (u32)W_HI + (u32)(((peer * 8 + r) * 24 + (d.isB ? d.pi : 8 + d.pi)) * 4 + unit) * 8u;
        q.dst = p.ws + local_off(d, unit, p.rank, peer * 8 + r, cnt);
        q.cnt = (u32)cnt;
    } else if (z < NP_IN + NP_X) {
        const int y = (int)(z - NP_IN), rp = y / (7 * FC), yy = y - rp * (7 * FC), jj = yy / FC, k = yy - jj * FC;
        const bool across = (FC & 1) && k == FC - 1;
        PanelId d; d.r = 2 * rp + (across ? 0 : (k & 1)); d.isB = true; d.dest = (p.rank + 1 + jj) & 7; d.pi = 7;
        const int unit = across ? 4 - FC : 2 - (k & 2);
        PanelId e = d; e.r = d.r + (across ? 1 : 0);
        const int unit1 = across ? unit : unit + 1;
        size_t cnt1;
        const u32 l0 = (u32)local_off(d, unit, p.rank, peer * 8 + d.r, cnt), l1 = (u32)local_off(e, unit1, p.rank, peer * 8 + e.r, cnt1);
        q.type = T_IN; q.so = (u32)export_off(d, unit); q.sstep = (u32)export_off(e, unit1) - q.so;
        q.ho = (u32)W_HX + (u32)(((peer * 8 + d.r) * 8 + d.dest) * 4 + unit) * 8u; q.hstep = across ? 256u : 8u;
        q.dst = p.ws + l0; q.dstep = l1 - l0;
        q.cnt = (u32)cnt; q.cstep = (u32)cnt1 - (u32)cnt;
    } else if (z < NP_ALL) {
        const int y = (int)(z - NP_IN - NP_X), pm = y / (int)NP_ROW, i = y - pm * (int)NP_ROW;
        const int pn = i < 28 ? i >> 2 : 7, unit = (i < 28 ? (i & 3) : (i - 28) * 2) * 2;
        const bool across = FC == 3 && i >= 28;
        q.type = T_OUT; q.so = (u32)W_PO + (u32)((pm * 8 + pn) * 8 + unit) * (u32)USLOT;
        q.sstep = across ? 4u * USLOT : (u32)USLOT; q.hstep = across ? 32u : 8u; q.dstep = across ? 128u * GN : 64u;
        q.ho = (u32)W_HO + (u32)((peer * 128 + pm * 8 + pn) * 8 + unit) * 8u;
        q.dst = (char*)p.out + (size_t)(pm * 256 + (unit >> 2) * 128) * GN + peer * NS + pn * 256 + (unit & 3) * 64;
    }
    return q;
}
__device__ __forceinline__ void pull_schedule(const Params& p, LAS unsigned char* lds, u32 call, int j, int wid, int lane) {
    __builtin_amdgcn_s_setprio(2);
    const int peer = (p.rank + 1 + j) & 7, rot = rot_of(j);
    const u32 sub = (u32)lane & 1u;
    const u32 subm = 0u - sub;
    const char* pws = peer_ws(lds, (u32)peer);
    volatile LAS u32* rw = ring_words(lds);
    const u32 qbase = (call - 1u) * (NP_ALL + 2u * (u32)NPW);
    char* qp = p.ws + W_Q + j * 128;
    u32 tk, tk1;
    ticket_issue(tk, qp); ticket_issue(tk1, qp);
    asm volatile("s_waitcnt vmcnt(0)" : "+v"(tk), "+v"(tk1) :: "memory");
    u32 z = rfl(tk) - qbase, zn = rfl(tk1) - qbase, n = 0, slot = 0, dec = 0, lap = 0;
    Pair cur = pair_of(p, z, j, peer, rot);
    u32x2 h = (u32x2){0u, 0u};
    bool dead = false;
    u32 ends = 0;
    for (;;) {
        const bool more = z < NP_ALL;
        if (more && !dead && !__all(h.y == call)) {
            const u64 t0 = now();
            for (;;) {
                u32 t; poll8(h, t, p.ws + (cur.ho + (cur.hstep & subm)), p.ws + W_TMO);
                if (__all(h.y == call)) { dead = rfl(t) == call; break; }
                if (spin_fail(p.ws, call, t0, t)) { dead = true; break; }
            }
        }
        const Pair nxt = pair_of(p, zn, j, peer, rot);
        u32x2 hn;
        ld8_sys(hn, p.ws + (nxt.ho + (nxt.hstep & subm)));
        tk = 0xffffffffu;
        if (more) ticket_issue(tk, qp);
        for (;;) {
            const u32 tkn = rw[R_TAKEN], dn = rw[R_DONE + dec + sub];
            if (__all(n < tkn + (u32)RUNAHEAD && dn + (u32)(NSLOT / ND) > lap)) break;
            __builtin_amdgcn_s_sleep(1);
        }
        if (lane < 2) {
            const u32 type = !dead || !more ? cur.type : (cur.type == T_OUT ? T_DEAD : T_SKIP);
            const u64 src = (u64)(pws + (cur.so + (cur.sstep & subm))), d64 = (u64)(cur.dst + (cur.dstep & subm));
            volatile LAS u32x4* ds = (volatile LAS u32x4*)(lds + LDS_MISC + 512 + (slot + sub) * 32u);
            ds[0] = (u32x4){(u32)src, (u32)(src >> 32), h.x, type};
            ds[1] = (u32x4){(u32)d64, (u32)(d64 >> 32), cur.cnt + (cur.cstep & subm), 0u};
        }
        n += 2;
        rw[R_POSTED] = n;
        slot = slot == (u32)NSLOT - 2u ? 0u : slot + 2u;
        if (dec == (u32)ND - 2u) { dec = 0; ++lap; } else dec += 2u;
        asm volatile("s_waitcnt vmcnt(0)" : "+v"(tk), "+v"(hn) :: "memory");
        if (more) { z = zn; zn = rfl(tk) - qbase; cur = nxt; h = hn; }
        else if (++ends == (u32)ND / 2u) break;
    }
}

#define GLDS(src, off) "global_load_lds_dwordx4 " src ", off offset:" #off " sc0 sc1\n\ts_waitcnt vmcnt(%[d])\n\t"
__device__ __forceinline__ void glds_part(const void* src, u32 dst, u32 rem) {
    const u64 mask = rem >= 64u ? ~0ull : (1ull << rem) - 1ull;
    u32 keep;
    asm volatile("s_mov_b32 %0, m0\n\ts_mov_b32 m0, %2\n\ts_mov_b64 exec, %3\n\t" GLDS("%1", 0) "s_mov_b64 exec, -1\n\ts_mov_b32 m0, %0"
                 : "=&s"(keep) : "v"(src), "s"(dst), "s"(mask), [d] "i"(DEPTH - 1) : "memory");
}
__device__ __forceinline__ void pull_load(const Params& p, LAS unsigned char* lds, int wid, int lane) {
    __builtin_amdgcn_s_setprio(3);
    volatile LAS u32* rw = ring_words(lds);
    const u32 lbase = (u32)(u64)lds;
    u32 ends = 0, slot = 0, full = 0;
    u32 posted = 0;
    u32x4 ds = (u32x4){0u, 0u, 0u, 0u};
    for (u32 n = 0;; ++n) {
        volatile LAS u32x4* dsp = (volatile LAS u32x4*)(lds + LDS_MISC + 512 + slot * 32u);
        if (rfl(posted) <= n) {
            posted = rw[R_POSTED];
            if (rfl(posted) <= n) {
                WAIT_V(0);
                full = n; rw[R_FULL] = n;
                while (rfl(rw[R_POSTED]) <= n) __builtin_amdgcn_s_sleep(1);
            }
            ds = dsp[0];
        }
        const u32 type = rfl(ds.w);
        const u32 dst = lbase + slot * (u32)USLOT;
        slot = slot == (u32)NSLOT - 1u ? 0u : slot + 1u;
        if (type <= T_OUT) {
            const u32 n16 = rfl(min(max(ds.z & 1023u, 336u), 512u));
            const char* src = (const char*)(((u64)ds.y << 32) | ds.x) + lane * 16;
            u32 keep;
            asm volatile("s_mov_b32 %0, m0\n\ts_mov_b32 m0, %2\n\ts_nop 0\n\t" GLDS("%1", 0) GLDS("%1", 1024) "s_mov_b32 m0, %0"
                         : "=&s"(keep) : "v"(src), "s"(dst), [d] "i"(DEPTH - 1) : "memory");
            if (DEPTH == 8 || DEPTH == 9) { if (full + 1u < n) { full = n - 1u; rw[R_FULL] = full; } }
            asm volatile("s_mov_b32 %0, m0\n\ts_mov_b32 m0, %2\n\ts_nop 0\n\t" GLDS("%1", 2048) GLDS("%1", 3072) "s_mov_b32 m0, %0"
                         : "=&s"(keep) : "v"(src), "s"(dst), [d] "i"(DEPTH - 1) : "memory");
            if (DEPTH == 10 || DEPTH == 11) { if (full + 1u < n) { full = n - 1u; rw[R_FULL] = full; } }
            if (DEPTH <= 5) rw[R_FULL] = n;
            posted = rw[R_POSTED];
            ds = *(volatile LAS u32x4*)(lds + LDS_MISC + 512 + slot * 32u);
            src += 4096;
            glds_part(src, dst + 4096u, 64u);
            if (DEPTH == 6) rw[R_FULL] = n;
            glds_part(src + 1024, dst + 5120u, n16 - 320u);
            if (DEPTH == 7) rw[R_FULL] = n;
            if (n16 > 384u) {
                glds_part(src + 2048, dst + 6144u, n16 - 384u);
                if (DEPTH == 8) { full = n; rw[R_FULL] = n; }
                if (n16 > 448u) glds_part(src + 3072, dst + 7168u, n16 - 448u);
            }
            rw[R_TAKEN] = n + 1u;
        } else {
            WAIT_V(0);
            full = n + 1u; rw[R_FULL] = full; rw[R_TAKEN] = full;
            posted = 0;
            if (type == T_END && ++ends == (u32)ND) break;
        }
    }
}

__device__ __forceinline__ void pull_decode(const Params& p, LAS unsigned char* lds, int wid, int lane) {
    volatile LAS u32* rw = ring_words(lds);
    LAS u32* bm = (LAS u32*)(lds + LDS_BM + wid * 1024);
    u32 done = 0;
    for (u32 n = (u32)wid;; n += (u32)ND) {
        while (rfl(rw[R_FULL]) <= n) __builtin_amdgcn_s_sleep(1);
        CFENCE;
        const u32x4 d0 = ring_desc(lds, n)[0], d1 = ring_desc(lds, n)[1];
        const u32 type = rfl(d0.w), hdr = rfl(d0.z);
        if (type == T_END) break;
        char* dst = (char*)(((u64)d1.y << 32) | d1.x);
        LAS unsigned char* sl = lds + (n % (u32)NSLOT) * (u32)USLOT;
        u32 u[32];
        if (type <= T_OUT && (hdr & H_RAW)) {
#pragma unroll
            for (int k = 0; k < 8; ++k) { const u32x4 v = *(const LAS u32x4*)(sl + k * 1024 + lane * 16); u[4 * k] = v.x; u[4 * k + 1] = v.y; u[4 * k + 2] = v.z; u[4 * k + 3] = v.w; }
        } else if (type == T_IN) {
            const u32 fl = *(const LAS u32*)(sl + 5120 + lane * 4);
            const u32x4 dn = *(const LAS u32x4*)(sl + 4096 + lane * 16);
            u32x4 lo[4];
#pragma unroll
            for (int k = 0; k < 4; ++k) lo[k] = *(const LAS u32x4*)(sl + k * 1024 + lane * 16);
            const u32 cn = (u32)__popc(fl), inc = scan_add(cn);
            u32 m[4];
            sparse_get(m, fl, inc - cn, (const LAS u32*)(sl + 5376));
            const u32 b4 = (((hdr >> 10) & 0x7fu) - 6u) * 0x01010101u;
            const u32 tl = b4 + 0x07040506u, th = b4 + 0x00010203u;
            u32 hi[16];
#pragma unroll
            for (int h = 0; h < 2; ++h) {
                const u32 B = bfi(0x55555555u, m[h], m[2 + h] << 1), D = bfi(0x55555555u, m[h] >> 1, m[2 + h]);
                u32 nn[4];
                nn[0] = bfi(0x33333333u, dn[2 * h], B << 2); nn[2] = bfi(0x33333333u, dn[2 * h] >> 2, B);
                nn[1] = bfi(0x33333333u, dn[2 * h + 1], D << 2); nn[3] = bfi(0x33333333u, dn[2 * h + 1] >> 2, D);
#pragma unroll
                for (int k = 0; k < 4; ++k) {
                    hi[8 * h + 2 * k] = bfi(0x80808080u, nn[k] << 7, __builtin_amdgcn_perm(th, tl, (nn[k] >> 1) & 0x07070707u));
                    hi[8 * h + 2 * k + 1] = bfi(0x80808080u, nn[k] << 3, __builtin_amdgcn_perm(th, tl, (nn[k] >> 5) & 0x07070707u));
                }
            }
            const u32 nexc = (hdr >> 17) & 0xffu;
            if (nexc) {
                const u32 sdw = (rdl(inc, 63) + 7u) >> 3;
                const u32 ev = *(const LAS u32*)(sl + 5376 + (sdw + (u32)lane) * 4u);
                for (u32 e = 0; e < nexc; ++e) {
                    const u32 en = rdl(ev, (int)e);
                    const u32 sh = ((en >> 7) & 3u) * 8u, kk = (en >> 9) & 15u;
                    const bool me = (en >> 13) == (u32)lane;
#pragma unroll
                    for (int k = 0; k < 16; ++k)
                        if (me && kk == (u32)k) hi[k] = bfi(0x7fu << sh, (en & 0x7fu) << sh, hi[k]);
                }
            }
#pragma unroll
            for (int k = 0; k < 16; ++k) {
                u[2 * k] = __builtin_amdgcn_perm(hi[k], lo[k >> 2][k & 3], 0x05010400u);
                u[2 * k + 1] = __builtin_amdgcn_perm(hi[k], lo[k >> 2][k & 3], 0x07030602u);
            }
        } else if (type == T_OUT) {
            *(volatile LAS u32x4*)(bm + lane * 4) = (u32x4){0u, 0u, 0u, 0u};
            const u32 fl = *(const LAS u32*)(sl + 6144 + lane * 4);
            u32x4 w4[6];
#pragma unroll
            for (int k = 0; k < 6; ++k) w4[k] = *(const LAS u32x4*)(sl + k * 1024 + lane * 16);
            const u32 cn = (u32)__popc(fl), inc = scan_add(cn);
            const u32 sdw = (rdl(inc, 63) + 7u) >> 3, nexc = (hdr >> 17) & 0xffu;
            const u32 ev = *(const LAS u32*)(sl + 6400 + (sdw + (u32)lane) * 4u);
            u32 m[4];
            sparse_get(m, fl, inc - cn, (const LAS u32*)(sl + 6400));
            if (2u * (u32)lane < nexc) __hip_atomic_fetch_or(bm + ((ev >> 5) & 255u), 1u << (ev & 31u), __ATOMIC_RELAXED, __HIP_MEMORY_SCOPE_WORKGROUP);
            if (2u * (u32)lane + 1u < nexc) __hip_atomic_fetch_or(bm + ((ev >> 21) & 255u), 1u << ((ev >> 16) & 31u), __ATOMIC_RELAXED, __HIP_MEMORY_SCOPE_WORKGROUP);
            CFENCE;
            const u32x4 m7 = *(volatile LAS u32x4*)(bm + lane * 4);
#pragma unroll
            for (int g = 0; g < 8; ++g) {
                const u32 ac = (g & 1) ? bfi(0x55555555u, m[g >> 1] >> 1, m7[g >> 1] << 1) : bfi(0x55555555u, m[g >> 1], m7[g >> 1]);
                u32 r[4]; r[3] = 0;
#pragma unroll
                for (int i = 0; i < 3; ++i) { const int k = 3 * g + i; r[i] = w4[k >> 2][k & 3]; r[3] |= (r[i] >> (6 - 2 * i)) & (0x03030303u << (2 * i)); }
#pragma unroll
                for (int i = 0; i < 4; ++i) {
                    const u32 e = ac << (2 * i), y6 = (r[i] << 1) ^ e, y7 = (y6 << 1) ^ e;
                    u[4 * g + i] = bfi(0x3f3f3f3fu, r[i], bfi(0x40404040u, y6, y7));
                }
            }
        } else {
#pragma unroll
            for (int k = 0; k < 32; ++k) u[k] = 0x80808080u;
        }
        WAIT_L(0);
        ++done;
        if (lane == 0) rw[R_DONE + wid] = done;
        if (type == T_IN) {
            st_unit(dst + lane * 16, u);
            WAIT_V(0);
            if (lane == 0) add4_sys(p.ws + d1.z, 1u);
        } else if (type != T_SKIP) {
            char* o = dst + (size_t)(lane & 15) * GN + (lane >> 4) * 16;
#pragma unroll
            for (int k = 0; k < 8; ++k)
                st16_plain(o + (size_t)k * 16 * GN, (u32x4){u[4 * k], u[4 * k + 1], u[4 * k + 2], u[4 * k + 3]});
        }
    }
}

__device__ __forceinline__ void gemm_tile(const Params& p, LAS unsigned char* lds, u32 call, bool fillm, int pm, int pn, int wid, int lane) {
    const int wr = wid >> 2, wc = wid & 3;
    int j0 = 0;
#pragma unroll
    for (int j = 1; j < 7; ++j) if (rot_of(j) <= pm) j0 = j;
    const char* Ab = p.ws + W_A + (size_t)pm * 64 * PANEL;
    const char* Bb = p.ws + (fillm ? W_X : W_B) + (size_t)pn * 64 * PANEL;
    const char* cA = p.ws + W_ACNT + (size_t)pm * 256;
    const char* cB = p.ws + (fillm ? W_XCNT : W_BCNT) + (size_t)pn * 256;
    const u32 voff = (u32)(wid * 4096 + lane * 16);
    const u32 ldsw = (u32)wid * 2048u;
    const int aoff = wr * 8192 + lane * 16;
    int boff = 65536 + wc * 4096 + lane * 16;
    asm volatile("" : "+v"(boff));
    int ktv = kt_of(lane, p.rank, j0);
    asm volatile("" : "+v"(ktv));
    u64 rdyA = wid == 4 ? 0ull : ~0ull, rdyB = rdyA;
    const u32 needA = call * 4u, needB = (fillm && pn == NVP - 1) ? call * (u32)(7 * FC - 4 * (NVP - 1)) : needA;
#define SA(b, h) (((b) * 2 + (h)) * 16384)
#define SB(b, h) ((4 + (b) * 2 + (h)) * 16384)
#define STAGE(buf, gbase, h) do { \
        __builtin_amdgcn_global_load_lds((const u32*)((gbase) + voff + (h) * 2048), (LAS u32*)(lds + (buf) + ldsw), 16, 0, 17); \
        __builtin_amdgcn_global_load_lds((const u32*)((gbase) + voff + (h) * 2048), (LAS u32*)(lds + (buf) + ldsw), 16, 1024, 17); } while (0)
#define LDA(dst, b, h) do { _Pragma("unroll") for (int m = 0; m < 4; ++m) _Pragma("unroll") for (int k = 0; k < 2; ++k) \
        dst[m][k] = *(const LAS bf16x8*)(lds + SA(b, h) + aoff + m * 2048 + k * 1024); } while (0)
#define LDB(dst, b, h) do { _Pragma("unroll") for (int n = 0; n < 2; ++n) _Pragma("unroll") for (int k = 0; k < 2; ++k) \
        dst[n][k] = *(const LAS bf16x8*)(lds + (SB(b, h) - 65536) + boff + k * 2048 + n * 1024); } while (0)
#define MMA(ai, bj, At, Bt) do { __builtin_amdgcn_s_setprio(1); \
        _Pragma("unroll") for (int m = 0; m < 4; ++m) _Pragma("unroll") for (int n = 0; n < 2; ++n) _Pragma("unroll") for (int k = 0; k < 2; ++k) \
            acc[ai][bj][m][n] = __builtin_amdgcn_mfma_f32_16x16x32_bf16(Bt[n][k], At[m][k], acc[ai][bj][m][n], 0, 0, 0); \
        __builtin_amdgcn_s_setprio(0); } while (0)
#define CHECK(RDY, o) do { if (!((RDY >> (o)) & 1ull)) { \
        const u64 t0_ = now(); __builtin_amdgcn_s_waitcnt(0xC07F); \
        for (;;) { \
            u32 a_, b_, t_; poll4x2(a_, b_, t_, cA + ktv * 4, cB + ktv * 4, p.ws + W_TMO); \
            rdyA = __ballot(a_ >= needA); rdyB = __ballot(b_ >= needB); \
            if ((RDY >> (o)) & 1ull) break; \
            if (spin_fail(p.ws, call, t0_, t_)) { rdyA = rdyB = ~0ull; break; } \
        } } } while (0)
#define KOFF(o) ((size_t)__builtin_amdgcn_readlane(ktv, (o)) * PANEL)

    f32x4 acc[2][2][4][2];
#pragma unroll
    for (int a = 0; a < 2; ++a)
#pragma unroll
        for (int b = 0; b < 2; ++b)
#pragma unroll
            for (int m = 0; m < 4; ++m)
#pragma unroll
                for (int n = 0; n < 2; ++n) acc[a][b][m][n] = (f32x4){0.f, 0.f, 0.f, 0.f};
    bf16x8 At[4][2], B0[2][2], B1[2][2];

    CHECK(rdyA, 0); CHECK(rdyB, 0); CHECK(rdyB, 1); CHECK(rdyA, 1);
    BAR; CFENCE;
    size_t k1 = KOFF(1);
    {
        const size_t k0 = KOFF(0);
        STAGE(SB(0, 0), Bb + k0, 0); STAGE(SB(0, 1), Bb + k0, 1); STAGE(SA(0, 0), Ab + k0, 0);
        STAGE(SB(1, 0), Bb + k1, 0); STAGE(SA(0, 1), Ab + k0, 1); STAGE(SB(1, 1), Bb + k1, 1);
        WAIT_V(6); BAR;
        if (wr == 1) BAR;
    }
    for (int t = 0; t < 64; t += 2) {
        const int o2 = (t + 2) & 63, o3 = o2 + 1;
        const size_t k2 = KOFF(o2), k3 = KOFF(o3);
        LDB(B0, 0, 0); SCHED; LDA(At, 0, 0); STAGE(SA(1, 0), Ab + k1, 0);
        CHECK(rdyB, o2);
        WAIT_L(8); BAR; WAIT_L(0); MMA(0, 0, At, B0); BAR; SCHED;
        LDB(B1, 0, 1); STAGE(SB(0, 0), Bb + k2, 0);
        WAIT_V(6); BAR; WAIT_L(0); MMA(0, 1, At, B1); BAR;
        LDA(At, 0, 1); STAGE(SA(1, 1), Ab + k1, 1);
        BAR; WAIT_L(0); MMA(1, 0, At, B0); BAR; SCHED;
        STAGE(SB(0, 1), Bb + k2, 1);
        CHECK(rdyA, o2);
        WAIT_V(6); BAR; MMA(1, 1, At, B1); BAR;
        LDB(B0, 1, 0); SCHED; LDA(At, 1, 0); STAGE(SA(0, 0), Ab + k2, 0);
        CHECK(rdyB, o3);
        WAIT_L(8); BAR; WAIT_L(0); MMA(0, 0, At, B0); BAR; SCHED;
        LDB(B1, 1, 1); STAGE(SB(1, 0), Bb + k3, 0);
        WAIT_V(6); BAR; WAIT_L(0); MMA(0, 1, At, B1); BAR;
        LDA(At, 1, 1); STAGE(SA(0, 1), Ab + k2, 1);
        BAR; WAIT_L(0); MMA(1, 0, At, B0); BAR; SCHED;
        STAGE(SB(1, 1), Bb + k3, 1);
        CHECK(rdyA, o3);
        WAIT_V(6); BAR; MMA(1, 1, At, B1); BAR;
        k1 = k3;
    }
    if (wr == 0) BAR;
    if (!fillm) STAMP(2);
    int tz = threadIdx.x; asm volatile("" : "+v"(tz));
    lane = tz & 63;
    const int tile = pm * 8 + pn;
    volatile LAS u32* red = (volatile LAS u32*)(lds + LDS_MISC + 1024);
    if (!fillm) {
        float mx = 0.f;
#pragma unroll
        for (int a = 0; a < 2; ++a)
#pragma unroll
            for (int b = 0; b < 2; ++b)
#pragma unroll
                for (int m = 0; m < 4; ++m)
#pragma unroll
                    for (int n = 0; n < 2; ++n)
#pragma unroll
                        for (int r = 0; r < 4; ++r) mx = fmaxf(mx, fabsf(acc[a][b][m][n][r]));
        const u32 wm = wave_umax(__float_as_uint(mx));
        if (lane == 0) red[wid] = wm;
        WAIT_L(0); BAR; CFENCE;
        if (tz < NR) {
            const u32x4 r0 = *(volatile LAS u32x4*)red, r1 = *(volatile LAS u32x4*)(red + 4);
            const u32 t = max(max(max(r0.x, r0.y), max(r0.z, r0.w)), max(max(r1.x, r1.y), max(r1.z, r1.w)));
            st8_sys(peer_ws(lds, (u32)tz) + W_AMAX + (size_t)(p.rank * 128 + tile) * 8, (u32x2){t, call});
        }
    }
    bool dead = false;
    float amax;
    {
        const char* src = p.ws + W_AMAX + tz * 16;
        const u64 t0 = now();
        u32x4 gq;
        for (;;) {
            u32 t; poll16(gq, t, src, p.ws + W_TMO);
            dead = rfl(t) == call;
            if (__all(gq.y == call && gq.w == call)) break;
            if (spin_fail(p.ws, call, t0, t)) { dead = true; break; }
        }
        const u32 wm = wave_umax(max(gq.x, gq.z));
        if (lane == 0) red[8 + wid] = dead ? 0xffffffffu : wm;
        WAIT_L(0); BAR; CFENCE;
        const u32x4 r0 = *(volatile LAS u32x4*)(red + 8), r1 = *(volatile LAS u32x4*)(red + 12);
        const u32 t = rfl(max(max(max(r0.x, r0.y), max(r0.z, r0.w)), max(max(r1.x, r1.y), max(r1.z, r1.w))));
        dead = t == 0xffffffffu;
        amax = __uint_as_float(t);
    }
    if (!fillm) STAMP(3);
    const float inv = amax > 0.f ? 127.0f / amax : 0.f;
    u32 q[32];
#pragma unroll
    for (int m = 0; m < 4; ++m)
#pragma unroll
        for (int a = 0; a < 2; ++a)
#pragma unroll
            for (int r = 0; r < 4; ++r) {
                const u32 t0 = __float_as_uint(fmaf(acc[a][0][m][0][r], inv, 12582912.f)), t1 = __float_as_uint(fmaf(acc[a][0][m][1][r], inv, 12582912.f));
                const u32 t2 = __float_as_uint(fmaf(acc[a][1][m][0][r], inv, 12582912.f)), t3 = __float_as_uint(fmaf(acc[a][1][m][1][r], inv, 12582912.f));
                q[(m * 2 + a) * 4 + r] = __builtin_amdgcn_perm(t1, t0, 0x0c0c0400u) | __builtin_amdgcn_perm(t3, t2, 0x04000c0cu);
            }
    if (dead) {
#pragma unroll
        for (int k = 0; k < 32; ++k) q[k] = 0x80808080u;
    }
    int col = p.rank * NS + pn * 256 + wc * 64;
    if (!fillm) {
        char* slot = p.ws + W_PO + (size_t)(tile * 8 + wid) * USLOT;
        LAS unsigned char* enc = lds + LDS_ENC + wid * 2048;
        *(volatile LAS u32x4*)(enc + lane * 16) = (u32x4){0u, 0u, 0u, 0u};
        *(volatile LAS u32x4*)(enc + 1024 + lane * 16) = (u32x4){0u, 0u, 0u, 0u};
        u32 ac[8], m6[4], m7[4];
#pragma unroll
        for (int g = 0; g < 8; ++g) {
            ac[g] = 0;
#pragma unroll
            for (int i = 0; i < 4; ++i) { const u32 v = q[4 * g + i]; ac[g] |= ((v ^ (v << 1)) & 0xc0c0c0c0u) >> (2 * i); }
        }
#pragma unroll
        for (int h = 0; h < 4; ++h) { m6[h] = bfi(0x55555555u, ac[2 * h], ac[2 * h + 1] << 1); m7[h] = bfi(0xaaaaaaaau, ac[2 * h], ac[2 * h + 1] >> 1); }
        const u32 fl = nib_flags(m6[0]) | (nib_flags(m6[1]) << 1) | (nib_flags(m6[2]) << 2) | (nib_flags(m6[3]) << 3);
        const u32 cn = (u32)__popc(fl) | ((u32)(__popc(m7[0]) + __popc(m7[1]) + __popc(m7[2]) + __popc(m7[3])) << 16);
        const u32 inc = scan_add(cn), tot = rdl(inc, 63), exc = inc - cn;
        const u32 sdw = ((tot & 0xffffu) + 7u) >> 3, nexc = tot >> 16;
        char* dp = slot + lane * 16;
        u32 hdr;
        if (nexc > 128u) {
            st_unit(dp, q);
            hdr = H_RAW | 512u;
        } else {
            u32 w[24];
#pragma unroll
            for (int g = 0; g < 8; ++g)
#pragma unroll
                for (int i = 0; i < 3; ++i) w[3 * g + i] = bfi(0x3f3f3f3fu, q[4 * g + i], q[4 * g + 3] << (6 - 2 * i));
            st16_sys<0>(dp, (u32x4){w[0], w[1], w[2], w[3]}); st16_sys<1024>(dp, (u32x4){w[4], w[5], w[6], w[7]});
            st16_sys<2048>(dp, (u32x4){w[8], w[9], w[10], w[11]}); st16_sys<3072>(dp, (u32x4){w[12], w[13], w[14], w[15]});
            st16_sys<0>(dp + 4096, (u32x4){w[16], w[17], w[18], w[19]}); st16_sys<1024>(dp + 4096, (u32x4){w[20], w[21], w[22], w[23]});
            ((volatile LAS u32*)enc)[lane] = fl;
            sparse_put(m6, fl, exc & 0xffffu, (LAS u32*)(enc + 256));
            volatile LAS unsigned short* list = (volatile LAS unsigned short*)(enc + 256 + sdw * 4u);
            u32 s = exc >> 16;
#pragma unroll
            for (int h = 0; h < 4; ++h) {
                u32 mm = m7[h];
                while (mm) { list[s] = (unsigned short)((lane * 4 + h) * 32 + __builtin_ctz(mm)); mm &= mm - 1u; ++s; }
            }
            const u32 n16 = (64u + sdw + ((nexc + 1u) >> 1) + 3u) >> 2;
            tail_out(slot + 6144, enc, n16, lane);
            hdr = (384u + n16) | (nexc << 17);
        }
        if (lane == 0) red[16 + wid] = hdr;
        WAIT_V(0); WAIT_L(0); BAR; CFENCE;
        if (tz < 64 && (tz >> 3) != p.rank)
            st8_sys(peer_ws(lds, (u32)(tz >> 3)) + W_HO + (size_t)((p.rank * 128 + tile) * 8 + (tz & 7)) * 8, (u32x2){red[16 + (tz & 7)], call});
        STAMP(4);
    } else {
        const int g = pn * 4 + wc;
        col = g < 7 * FC ? ((p.rank + 1 + g / FC) & 7) * NS + 7 * 256 + (4 - FC + g % FC) * 64 : -1;
    }
    if (col >= 0) {
        signed char* o = p.out + (size_t)(pm * 256 + wr * 128 + (lane & 15)) * GN + col + (lane >> 4) * 16;
#pragma unroll
        for (int k = 0; k < 8; ++k)
            st16_plain(o + (size_t)k * 16 * GN, (u32x4){q[4 * k], q[4 * k + 1], q[4 * k + 2], q[4 * k + 3]});
    }
#undef SA
#undef SB
#undef STAGE
#undef LDA
#undef LDB
#undef MMA
#undef CHECK
#undef KOFF
}

__global__ void __launch_bounds__(BLOCK, 2) fused_gemm_q8(Params p) {
    __shared__ __attribute__((aligned(16))) unsigned char smem[LDS_BYTES];
    LAS unsigned char* lds = (LAS unsigned char*)smem;
    const int tid = threadIdx.x, wid = (int)rfl((u32)tid >> 6), lane = tid & 63;
    const int xcd = blockIdx.x & 7, idx = blockIdx.x >> 3;
    STAMP(0);
    volatile LAS u32* misc = (volatile LAS u32*)(lds + LDS_MISC);
    LAS unsigned char* enc = lds + LDS_ENC + wid * 2048;
    const int n0 = wid * 256 + (int)blockIdx.x;
    if (n0 < 320) {
        const PanelId d0 = panel_of(0, n0 >> 2, p.rank);
        f32x4 f[16];
        unit_load(p, d0, n0 & 3, lane, f);
        WAIT_V(0); PIN_F(f);
        const u32 hdr = unit_pack(p, enc, d0, n0 & 3, lane, f);
        BAR; CFENCE;
        const u32 call0 = misc[0];
        WAIT_V(0);
        if (lane < NR) unit_announce(p, lds, d0, n0 & 3, lane, hdr, call0);
    } else {
        if (wid == 7) {
            u32 tk; u32x2 pw;
            ticket_issue(tk, p.ws + W_SEQ + (size_t)blockIdx.x * 128);
            ld8_sys(pw, (const char*)p.ws_all + (lane & 7) * 8);
            asm volatile("s_waitcnt vmcnt(0)" : "+v"(tk), "+v"(pw) :: "memory");
            if (lane == 0) misc[0] = tk + 1u;
            if (lane < NR) *(volatile LAS u32x2*)(lds + LDS_MISC + 64 + lane * 8) = pw;
            if (lane < 16) ring_words(lds)[lane] = 0u;
            const u32 call7 = rfl(tk) + 1u;
            if (blockIdx.x == 0) {
                if (lane == 0) st8_sys(p.ws + W_ROT, (u32x2){(u32)__builtin_amdgcn_s_getreg(20 | (3 << 11)) & 7u, call7});
            } else if (idx >= 16) {
                const u64 t0 = now();
                u32x2 ro;
                for (;;) {
                    u32 t; poll8(ro, t, p.ws + W_ROT, p.ws + W_TMO);
                    if (rfl(ro.y) == call7) break;
                    if (spin_fail(p.ws, call7, t0, t)) { ro.x = 8u; break; }
                }
                if (lane == 0) misc[1] = ro.x;
            }
            WAIT_L(0);
        }
        BAR; CFENCE;
    }
    const u32 call = rfl(misc[0]);
    STAMP(6);

    const bool is_gemm = idx < 16;
    const int c = idx - 16;
    const u32 rot0 = is_gemm ? 0u : rfl(misc[1]);
    if (rot0 > 7u) return;
    const int jl = (peer_of((int)((xcd + rot0) & 7u), p.rank) - p.rank - 1) & 7;
    if (!is_gemm && jl < 7 && c < NPW) {
        if (wid == 7) pull_schedule(p, lds, call, jl, wid, lane);
        else if (wid == 6) pull_load(p, lds, wid, lane);
        else pull_decode(p, lds, wid, lane);
        STAMP(5);
        return;
    }
    const int pm = (xcd >> 1) * 4 + (idx >> 2), pn = (xcd & 1) * 4 + (idx & 3);
    if (is_gemm) {
        const int b = (int)blockIdx.x;
        const int e = wid >= 2 && wid < 7 ? (wid - 2) * 128 + b : (wid == 1 && b >= 64 && b < 96 ? 576 + b : -1);
        if (e >= 0) {
            const int r = e / 96, i = e - r * 96;
            const PanelId d = panel_of(1 + r, 56 + (i >> 2), p.rank);
            f32x4 f[16];
            unit_load(p, d, i & 3, lane, f);
            WAIT_V(0); PIN_F(f);
            const u32 hdr = unit_pack(p, enc, d, i & 3, lane, f);
            WAIT_V(0);
            if (lane < NR) unit_announce(p, lds, d, i & 3, lane, hdr, call);
        }
        STAMP(1);
    } else {
        const int me = jl < 7 ? jl * (16 - NPW) + (c - NPW) : 7 * (16 - NPW) + c;
        int q = me;
        PanelId d = panel_of(1 + q / 28, (q % 28) * 2 + (wid >> 2), p.rank);
        f32x4 f[16];
        if (SPDLY && me >= 28) { const u64 t0 = now(); while (now() - t0 < SPDLY) __builtin_amdgcn_s_sleep(8); }
        unit_load(p, d, wid & 3, lane, f);
        volatile LAS u32* hb = (volatile LAS u32*)(lds + LDS_MISC + 1024);
        for (int step = 0; q < 196; ++step) {
            WAIT_V(0); PIN_F(f);
            const u32 hdr = unit_pack(p, enc, d, wid & 3, lane, f);
            if (lane == 0) hb[(step & 1) * 8 + wid] = hdr;
            const int qn = q + NSPARE;
            d = panel_of(1 + qn / 28, (qn % 28) * 2 + (wid >> 2), p.rank);
            if (qn < 196) { unit_load(p, d, wid & 3, lane, f); WAIT_V(16); } else { WAIT_V(0); }
            WAIT_L(0); BAR; CFENCE;
            if (wid == 0)
                unit_announce(p, lds, panel_of(1 + q / 28, (q % 28) * 2 + ((lane >> 2) & 1), p.rank), lane & 3, lane >> 3, hb[(step & 1) * 8 + (lane & 7)], call);
            q = qn;
        }
        STAMP(1);
        return;
    }
#pragma clang loop unroll(disable)
    for (int pass = 0; pass < (pn < NVP ? 2 : 1); ++pass) {
        int tz = threadIdx.x; asm volatile("" : "+v"(tz));
        gemm_tile(p, lds, call, pass == 1, pm, pn, wid, tz & 63);
        if (pass == 0) STAMP(5); else STAMP(13);
    }
}

extern "C" void kernel_launch(int rank, int nranks, ncclComm_t comm, hipStream_t stream,
                              void* const* d_in, const int* in_sizes, int n_in,
                              void* d_out, int out_size,
                              const int* in_dtypes, int out_dtype,
                              void* const* d_out_all,
                              void* const* d_ws_all, size_t ws_bytes) {
    (void)comm; (void)in_dtypes; (void)out_dtype; (void)d_out_all;
    if (nranks != NR || n_in < 2 || ws_bytes < W_END || in_sizes[0] != GM * KS || in_sizes[1] != KS * GN || out_size != GM * GN ||
        (unsigned)rank >= (unsigned)NR) {
        fprintf(stderr, "unexpected problem geometry\n");
        return;
    }
    Params p;
    p.x = (const float*)d_in[0];
    p.w = (const float*)d_in[1];
    p.out = (signed char*)d_out;
    p.ws = (char*)d_ws_all[rank];
    p.ws_all = (char* const*)d_ws_all;
    p.rank = rank;
    p.pad = 0;
    hipLaunchKernelGGL(fused_gemm_q8, dim3(GRID), dim3(BLOCK), 0, stream, p);
}
```

```cpp
#include <hip/hip_runtime.h>
#include <rccl/rccl.h>
#include <cstdio>
#include <cstdint>

#define STAMPS 0

#define LAS __attribute__((address_space(3)))
typedef unsigned u32;
typedef unsigned long long u64;
typedef u32 u32x4 __attribute__((ext_vector_type(4)));
typedef u32 u32x2 __attribute__((ext_vector_type(2)));
typedef float f32x4 __attribute__((ext_vector_type(4)));
typedef short bf16x8 __attribute__((ext_vector_type(8)));

constexpr int NR = 8, GM = 4096, GN = 16384, KS = 512, NS = 2048;
constexpr int GRID = 256, BLOCK = 512;
constexpr int PANEL = 32768;
constexpr int USLOT = 8192;
constexpr size_t W_TMO = 128;
constexpr size_t W_ROT = 256;
constexpr size_t W_Q = 1024;
constexpr size_t W_ACNT = 4096;
constexpr size_t W_BCNT = 8192;
constexpr size_t W_XCNT = 10240;
constexpr size_t W_AMAX = 16384;
constexpr size_t W_HI = 32768;
constexpr size_t W_HX = 81920;
constexpr size_t W_HO = 98304;
constexpr size_t W_SEQ = 256 << 10;
constexpr size_t W_DBG = 512 << 10;
constexpr size_t W_A = 1ull << 20;
constexpr size_t W_B = 33ull << 20;
constexpr size_t W_X = 49ull << 20;
constexpr size_t W_PX = 65ull << 20;
constexpr size_t W_PW = 69ull << 20;
constexpr size_t W_PO = 85ull << 20;
constexpr size_t W_END = 93ull << 20;
constexpr u64 DEADLINE = 40000000ull;

#define NPW 8
#define DEPTH 8
#define DEPTH2 6
#define RUNAHEAD 2
#define NEAR 1
#define NTLD 1
#define SPD0 450
#define SPD1 700
#define CSC "sc0 sc1"
#define PSLEEP 0
#define PKT 64
#define NRAW 8
#define XNOMMA 0
#define XNOST 0
#define RATES 0
#define NEARLY 1
#define OUTST 1
constexpr int ND = 6, NSLOT = 12;
constexpr int NSPARE = 7 * (16 - NPW) + 16;
#define FC 3
constexpr int NVP = (7 * FC + 3) / 4;
static_assert(7 * 292 <= 4 * 8 * NSPARE && NPW == 8 && DEPTH <= 11 && DEPTH2 <= 11 && FC >= 2 && FC <= 4, "roles");

constexpr int LDS_BM = 98304;
constexpr int LDS_ENC = 114688;
constexpr int LDS_MISC = 131072;
constexpr int LDS_BYTES = LDS_MISC + 2048;

struct Params {
    const float* x;
    const float* w;
    signed char* out;
    char* ws;
    char* const* ws_all;
    int rank;
    int pad;
};

#define WAIT_V(n) asm volatile("s_waitcnt vmcnt(" #n ")" ::: "memory")
#define WAIT_L(n) asm volatile("s_waitcnt lgkmcnt(" #n ")" ::: "memory")
#define BAR __builtin_amdgcn_s_barrier()
#define SCHED __builtin_amdgcn_sched_barrier(0)
#define CFENCE asm volatile("" ::: "memory")

template <int OFF> __device__ __forceinline__ void st16_sys(void* p, u32x4 x) {
    asm volatile("global_store_dwordx4 %0, %1, off offset:%2 sc0 sc1\n\ts_nop 1" :: "v"(p), "v"(x), "i"(OFF) : "memory");
}
__device__ __forceinline__ void st16_plain(void* p, u32x4 x) {
    asm volatile("global_store_dwordx4 %0, %1, off sc0 sc1\n\ts_nop 1" :: "v"(p), "v"(x) : "memory");
}
__device__ __forceinline__ void ld8_sys(u32x2& x, const void* p) {
    asm volatile("global_load_dwordx2 %0, %1, off sc0 sc1" : "=v"(x) : "v"(p) : "memory");
}
__device__ __forceinline__ void st8_sys(void* p, u32x2 x) {
    asm volatile("global_store_dwordx2 %0, %1, off sc0 sc1\n\ts_nop 1" :: "v"(p), "v"(x) : "memory");
}
__device__ __forceinline__ void st4_sys(void* p, u32 x) {
    asm volatile("global_store_dword %0, %1, off sc0 sc1\n\ts_nop 1" :: "v"(p), "v"(x) : "memory");
}
__device__ __forceinline__ void add4_sys(void* p, u32 x) {
    asm volatile("global_atomic_add %0, %1, off sc1\n\ts_nop 1" :: "v"(p), "v"(x) : "memory");
}
__device__ __forceinline__ void ld16x4(f32x4& a, f32x4& b, f32x4& c, f32x4& d, const void* p0, const void* p1, const void* p2, const void* p3, int nt) {
    asm volatile("s_cmp_lg_u32 %8, 0\n\ts_cbranch_scc1 .Lnt%=\n\t"
                 "global_load_dwordx4 %0, %4, off\n\tglobal_load_dwordx4 %1, %5, off\n\tglobal_load_dwordx4 %2, %6, off\n\tglobal_load_dwordx4 %3, %7, off\n\t"
                 "s_branch .Lnd%=\n.Lnt%=:\n\t"
                 "global_load_dwordx4 %0, %4, off nt\n\tglobal_load_dwordx4 %1, %5, off nt\n\tglobal_load_dwordx4 %2, %6, off nt\n\tglobal_load_dwordx4 %3, %7, off nt\n.Lnd%=:"
                 : "=&v"(a), "=&v"(b), "=&v"(c), "=&v"(d) : "v"(p0), "v"(p1), "v"(p2), "v"(p3), "s"(nt) : "memory", "scc");
}
__device__ __forceinline__ void ticket_issue(u32& x, void* p) {
    u64 sv;
    asm volatile("s_mov_b64 %1, exec\n\ts_mov_b64 exec, 1\n\tglobal_atomic_add %0, %2, %3, off sc0 sc1\n\ts_mov_b64 exec, %1"
                 : "=&v"(x), "=&s"(sv) : "v"(p), "v"(1u) : "memory");
}
__device__ __forceinline__ void poll16(u32x4& x, u32& t, const void* p, const void* tp) {
    asm volatile("global_load_dwordx4 %0, %2, off sc0 sc1\n\tglobal_load_dword %1, %3, off sc0 sc1\n\ts_waitcnt vmcnt(0)"
                 : "=&v"(x), "=&v"(t) : "v"(p), "v"(tp) : "memory");
}
__device__ __forceinline__ void poll8(u32x2& x, u32& t, const void* p, const void* tp) {
    asm volatile("global_load_dwordx2 %0, %2, off sc0 sc1\n\tglobal_load_dword %1, %3, off sc0 sc1\n\ts_waitcnt vmcnt(0)"
                 : "=&v"(x), "=&v"(t) : "v"(p), "v"(tp) : "memory");
}
__device__ __forceinline__ void poll4x2(u32& x, u32& y, const void* p, const void* q) {
    asm volatile("global_load_dword %0, %2, off " CSC "\n\tglobal_load_dword %1, %3, off " CSC "\n\ts_waitcnt vmcnt(0)"
                 : "=&v"(x), "=&v"(y) : "v"(p), "v"(q) : "memory");
}
__device__ __forceinline__ u32 ld4_wait(const void* p) {
    u32 x; asm volatile("global_load_dword %0, %1, off sc0 sc1\n\ts_waitcnt vmcnt(0)" : "=&v"(x) : "v"(p) : "memory"); return x;
}
__device__ __forceinline__ u32 cvt_pk_bf16(float lo, float hi) {
    u32 r; asm("v_cvt_pk_bf16_f32 %0, %1, %2" : "=v"(r) : "v"(lo), "v"(hi)); return r;
}
__device__ __forceinline__ u32 bfi(u32 m, u32 a, u32 b) { return (a & m) | (b & ~m); }
__device__ __forceinline__ u32 rfl(u32 x) { return (u32)__builtin_amdgcn_readfirstlane((int)x); }
__device__ __forceinline__ u32 rdl(u32 x, int l) { return (u32)__builtin_amdgcn_readlane((int)x, l); }
__device__ __forceinline__ u64 now() { return __builtin_amdgcn_s_memrealtime(); }

__device__ __forceinline__ bool spin_fail(char* ws, u32 call, u64 t0, u32 t) {
    if (rfl(t) == call) return true;
    if (now() - t0 > DEADLINE) { st4_sys(ws + W_TMO, call); WAIT_V(0); return true; }
    __builtin_amdgcn_s_sleep(1);
    return false;
}
#define DPP0(v, ctl, rm, bc) (u32)__builtin_amdgcn_update_dpp(0, (int)(v), ctl, rm, 0xf, bc)
__device__ __forceinline__ u32 scan_add(u32 v) {
    v += DPP0(v, 0x111, 0xf, true); v += DPP0(v, 0x112, 0xf, true); v += DPP0(v, 0x114, 0xf, true); v += DPP0(v, 0x118, 0xf, true);
    v += DPP0(v, 0x142, 0xa, false); v += DPP0(v, 0x143, 0xc, false);
    return v;
}
__device__ __forceinline__ u32 wave_umax(u32 v) {
    v = max(v, DPP0(v, 0x111, 0xf, true)); v = max(v, DPP0(v, 0x112, 0xf, true)); v = max(v, DPP0(v, 0x114, 0xf, true)); v = max(v, DPP0(v, 0x118, 0xf, true));
    v = max(v, DPP0(v, 0x142, 0xa, false)); v = max(v, DPP0(v, 0x143, 0xc, false));
    return rdl(v, 63);
}
__device__ __forceinline__ int rot_of(int j) { return (j * 16) / 7; }
__device__ __forceinline__ int kt_of(int o, int rank, int j0) {
    if (o < 8) return rank * 8 + o;
    const int r = (o - 8) / 7, s = o - 8 - r * 7;
    int j = j0 - s; if (j < 0) j += 7;
    return ((rank + 1 + j) & 7) * 8 + r;
}
__device__ __forceinline__ int peer_of(int x, int rank) {
    const u32 ga = 0x6320u, gb = 0x7541u;
    const bool ina = (0x4du >> rank) & 1u;
    const u32 own = ina ? ga : gb, oth = ina ? gb : ga;
    if (x >= 4) return (int)((oth >> (4 * (x - 4))) & 7u);
    if (x == 1) return rank;
    const int want = x == 0 ? 0 : x - 1;
    int n = 0, res = rank;
#pragma unroll
    for (int i = 0; i < 4; ++i) { const int q = (int)((own >> (4 * i)) & 7u); if (q != rank) { if (n == want) res = q; ++n; } }
    return res;
}
__device__ __forceinline__ char* peer_ws(LAS unsigned char* lds, u32 pr) {
    return (char*)((volatile LAS u64*)(lds + LDS_MISC + 64))[pr & 7u];
}

#define STAMP(i) do {} while (0)

__device__ __forceinline__ u32 nib_flags(u32 m) { u32 t = m | (m >> 1); t |= t >> 2; return t & 0x11111111u; }
__device__ __forceinline__ void sparse_put(const u32 (&m)[4], u32 fl, u32 off, LAS u32* st) {
#pragma unroll
    for (int w = 0; w < 4; ++w) {
        const u32 t = (fl >> w) & 0x11111111u, pin = t * 0x11111111u, pex = pin << 4;
        const u32 pe = (pex & 0x0f0f0f0fu) << 2, po = (pex >> 2) & 0x3c3c3c3cu;
        u32 c = m[w] & 15u;
#pragma unroll
        for (int i = 1; i < 8; ++i) c |= ((m[w] >> (4 * i)) & 15u) << ((((i & 1) ? po : pe) >> (8 * (i >> 1))) & 31u);
        const u64 x = (u64)c << ((off & 7u) * 4u);
        __hip_atomic_fetch_or(st + (off >> 3), (u32)x, __ATOMIC_RELAXED, __HIP_MEMORY_SCOPE_WORKGROUP);
        __hip_atomic_fetch_or(st + (off >> 3) + 1, (u32)(x >> 32), __ATOMIC_RELAXED, __HIP_MEMORY_SCOPE_WORKGROUP);
        off += pin >> 28;
    }
}
__device__ __forceinline__ void sparse_get(u32 (&m)[4], u32 fl, u32 off, const LAS u32* st) {
    u32 lo[4], hi[4], sh[4], pin[4];
#pragma unroll
    for (int w = 0; w < 4; ++w) {
        pin[w] = ((fl >> w) & 0x11111111u) * 0x11111111u;
        lo[w] = st[off >> 3]; hi[w] = st[(off >> 3) + 1]; sh[w] = (off & 7u) * 4u;
        off += pin[w] >> 28;
    }
#pragma unroll
    for (int w = 0; w < 4; ++w) {
        const u32 t = (fl >> w) & 0x11111111u, pex = pin[w] << 4;
        const u32 pe = (pex & 0x0f0f0f0fu) << 2, po = (pex >> 2) & 0x3c3c3c3cu;
        const u32 c = __builtin_amdgcn_alignbit(hi[w], lo[w], sh[w]);
        u32 r = c & 15u;
#pragma unroll
        for (int i = 1; i < 8; ++i) r |= ((c >> ((((i & 1) ? po : pe) >> (8 * (i >> 1))) & 31u)) & 15u) << (4 * i);
        m[w] = r & ((t << 4) - t);
    }
}
__device__ __forceinline__ void tail_out(char* dst, LAS unsigned char* enc, u32 n16, int lane) {
    CFENCE;
    const u32x4 a = *(volatile LAS u32x4*)(enc + lane * 16), b = *(volatile LAS u32x4*)(enc + 1024 + lane * 16);
    if ((u32)lane < n16) st16_sys<0>(dst + lane * 16, a);
    if ((u32)lane + 64u < n16) st16_sys<1024>(dst + lane * 16, b);
}
constexpr u32 H_RAW = 0x80000000u;

struct PanelId { int r, dest, pi; bool isB; };
__device__ __forceinline__ PanelId panel_of(int r, int i, int rank) {
    PanelId d; d.r = r; d.isB = i < 56 || i >= 72;
    d.dest = i < 56 ? ((rank + 1 + (i >> 3)) & 7) : rank;
    d.pi = d.isB ? (i & 7) : i - 56;
    return d;
}
__device__ __forceinline__ bool is_fill(const PanelId& d, int unit) { return d.isB && d.pi == 7 && unit >= 4 - FC; }
__device__ __forceinline__ void unit_load(const Params& p, const PanelId& d, int unit, int lane, f32x4 (&f)[16]) {
    const int fr = lane & 15, fq = lane >> 4;
    const float* b = d.isB ? p.w + (size_t)(d.r * 64 + 4 * fq) * GN + d.dest * NS + d.pi * 256 + unit * 64 + 4 * fr
                           : p.x + (size_t)(d.pi * 256 + unit * 64 + (lane >> 2)) * KS + d.r * 64 + 4 * ((lane & 3) ^ (fq & 2));
    const int s1 = d.isB ? 16 * GN : 16 * KS, s2 = d.isB ? GN : 16;
    const int nt = (int)rfl(NTLD && d.isB ? 1u : 0u);
#pragma unroll
    for (int j = 0; j < 4; ++j) ld16x4(f[4 * j], f[4 * j + 1], f[4 * j + 2], f[4 * j + 3], b + j * s1, b + j * s1 + s2, b + j * s1 + 2 * s2, b + j * s1 + 3 * s2, nt);
}
#define PIN_F(f) do { asm volatile("" : "+v"(f[0]), "+v"(f[1]), "+v"(f[2]), "+v"(f[3]), "+v"(f[4]), "+v"(f[5]), "+v"(f[6]), "+v"(f[7]) :: "memory"); \
    asm volatile("" : "+v"(f[8]), "+v"(f[9]), "+v"(f[10]), "+v"(f[11]), "+v"(f[12]), "+v"(f[13]), "+v"(f[14]), "+v"(f[15]) :: "memory"); } while (0)
__device__ __forceinline__ void st_unit(char* dp, const u32 (&u)[32]) {
    st16_sys<0>(dp, (u32x4){u[0], u[1], u[2], u[3]}); st16_sys<1024>(dp, (u32x4){u[4], u[5], u[6], u[7]});
    st16_sys<2048>(dp, (u32x4){u[8], u[9], u[10], u[11]}); st16_sys<3072>(dp, (u32x4){u[12], u[13], u[14], u[15]});
    dp += 4096;
    st16_sys<0>(dp, (u32x4){u[16], u[17], u[18], u[19]}); st16_sys<1024>(dp, (u32x4){u[20], u[21], u[22], u[23]});
    st16_sys<2048>(dp, (u32x4){u[24], u[25], u[26], u[27]}); st16_sys<3072>(dp, (u32x4){u[28], u[29], u[30], u[31]});
}
__device__ __forceinline__ size_t local_off(const PanelId& d, int unit, int rank, int kt, size_t& cnt) {
    if (d.dest == rank) { cnt = (d.isB ? W_BCNT : W_ACNT) + (size_t)(d.pi * 64 + kt) * 4; return (d.isB ? W_B : W_A) + (size_t)(d.pi * 64 + kt) * PANEL + unit * USLOT; }
    const int g = ((d.dest - rank - 1) & 7) * FC + unit - (4 - FC);
    cnt = W_XCNT + (size_t)((g >> 2) * 64 + kt) * 4;
    return W_X + (size_t)((g >> 2) * 64 + kt) * PANEL + (g & 3) * USLOT;
}
__device__ __forceinline__ size_t export_off(const PanelId& d, int unit) {
    return (d.isB ? W_PW + (size_t)((d.dest * 8 + d.r) * 8 + d.pi) * PANEL : W_PX + (size_t)(d.r * 16 + d.pi) * PANEL) + unit * USLOT;
}
__device__ __forceinline__ u32 unit_pack(const Params& p, LAS unsigned char* enc, const PanelId& d, int unit, int lane, const f32x4 (&f)[16], int mode) {
    u32 u[32];
    if (d.isB) {
#pragma unroll
        for (int h = 0; h < 2; ++h)
#pragma unroll
            for (int c = 0; c < 4; ++c)
#pragma unroll
                for (int t = 0; t < 4; ++t) u[16 * h + 4 * c + t] = cvt_pk_bf16(f[8 * h + 2 * t][c], f[8 * h + 2 * t + 1][c]);
    } else {
#pragma unroll
        for (int c = 0; c < 16; ++c) { u[2 * c] = cvt_pk_bf16(f[c][0], f[c][1]); u[2 * c + 1] = cvt_pk_bf16(f[c][2], f[c][3]); }
    }
    const bool own = d.dest == p.rank, fill = is_fill(d, unit);
    u32 hdr = 0;
    if (mode & 4) {
        st_unit(p.ws + export_off(d, unit) + lane * 16, u);
        hdr = H_RAW | 512u;
    } else if ((mode & 1) && (!(d.isB && own) || fill)) {
        char* slot = p.ws + export_off(d, unit);
        *(volatile LAS u32x4*)(enc + lane * 16) = (u32x4){0u, 0u, 0u, 0u};
        *(volatile LAS u32x4*)(enc + 1024 + lane * 16) = (u32x4){0u, 0u, 0u, 0u};
        u32 hi[16], t[16], sum = 0;
#pragma unroll
        for (int k = 0; k < 16; ++k) {
            hi[k] = __builtin_amdgcn_perm(u[2 * k + 1], u[2 * k], 0x07050301u);
            t[k] = hi[k] | 0x80808080u;
            sum = __builtin_amdgcn_sad_u8(t[k], 0u, sum);
        }
        u32 a = ((rdl(scan_add(sum), 63) + 4095u) >> 12) - 128u;
        a = min(max(a, 6u), 126u);
        const u32 b4 = (a - 6u) * 0x01010101u;
        u32 bad = 0, n[8];
#pragma unroll
        for (int k = 0; k < 16; ++k) { t[k] -= b4; bad |= (t[k] & 0xf8f8f8f8u) ^ 0x80808080u; }
#pragma unroll
        for (int k = 0; k < 8; ++k)
            n[k] = __builtin_amdgcn_perm(0x06000204u, 0x080a0c0eu, t[2 * k] & 0x07070707u) | __builtin_amdgcn_perm(0x60002040u, 0x80a0c0e0u, t[2 * k + 1] & 0x07070707u)
                 | ((hi[2 * k] >> 7) & 0x01010101u) | ((hi[2 * k + 1] >> 3) & 0x10101010u);
        u32 dn[4], m[4];
#pragma unroll
        for (int h = 0; h < 2; ++h) {
            dn[2 * h] = bfi(0x33333333u, n[4 * h], n[4 * h + 2] << 2); dn[2 * h + 1] = bfi(0x33333333u, n[4 * h + 1], n[4 * h + 3] << 2);
            const u32 B = bfi(0x33333333u, n[4 * h] >> 2, n[4 * h + 2]), D = bfi(0x33333333u, n[4 * h + 1] >> 2, n[4 * h + 3]);
            m[h] = bfi(0x55555555u, B, D << 1); m[2 + h] = bfi(0x55555555u, B >> 1, D);
        }
        const u32 fl = nib_flags(m[0]) | (nib_flags(m[1]) << 1) | (nib_flags(m[2]) << 2) | (nib_flags(m[3]) << 3);
        const u32 cn = (u32)__popc(fl), inc = scan_add(cn);
        const u32 sdw = (rdl(inc, 63) + 7u) >> 3;
        u32 nexc = 0;
        u64 anyb = __ballot(bad != 0);
        if (anyb) {
            u32 bm[2] = {0u, 0u};
#pragma unroll
            for (int k = 0; k < 16; ++k) {
                const u32 y = (t[k] & 0xf8f8f8f8u) ^ 0x80808080u;
                bm[k >> 3] |= ((((y & 0x78787878u) + 0x78787878u) | y) & 0x80808080u) >> (7 - (k & 7));
            }
            if (rdl(scan_add((u32)(__popc(bm[0]) + __popc(bm[1]))), 63) > 64u) {
                hdr = H_RAW | 512u;
            } else {
                while (anyb) {
                    const int l = __builtin_ctzll(anyb);
                    const u32 w0 = rdl(bm[0], l), w1 = rdl(bm[1], l);
                    const u32 pos = (u32)__builtin_ctz(w0 ? w0 : w1), k = (w0 ? 0u : 8u) + (pos & 7u);
                    u32 hv = 0;
#pragma unroll
                    for (int kk = 0; kk < 16; ++kk) hv = k == (u32)kk ? hi[kk] : hv;
                    const u32 e = (rdl(hv, l) >> (pos & 24u)) & 0x7fu;
                    if (lane == 0) ((volatile LAS u32*)enc)[64u + sdw + nexc] = ((u32)l << 13) | (k << 9) | ((pos >> 3) << 7) | e;
                    ++nexc;
                    if (lane == l) { if (w0) bm[0] &= bm[0] - 1u; else bm[1] &= bm[1] - 1u; }
                    anyb = __ballot((bm[0] | bm[1]) != 0);
                }
            }
        }
        char* dp = slot + lane * 16;
        if (hdr) {
            st_unit(dp, u);
        } else {
            u32 lo[16];
#pragma unroll
            for (int k = 0; k < 16; ++k) lo[k] = __builtin_amdgcn_perm(u[2 * k + 1], u[2 * k], 0x06040200u);
            st16_sys<0>(dp, (u32x4){lo[0], lo[1], lo[2], lo[3]}); st16_sys<1024>(dp, (u32x4){lo[4], lo[5], lo[6], lo[7]});
            st16_sys<2048>(dp, (u32x4){lo[8], lo[9], lo[10], lo[11]}); st16_sys<3072>(dp, (u32x4){lo[12], lo[13], lo[14], lo[15]});
            st16_sys<0>(dp + 4096, (u32x4){dn[0], dn[1], dn[2], dn[3]});
            ((volatile LAS u32*)enc)[lane] = fl;
            sparse_put(m, fl, inc - cn, (LAS u32*)(enc + 256));
            const u32 n16 = (64u + sdw + nexc + 3u) >> 2;
            tail_out(slot + 5120, enc, n16, lane);
            hdr = (320u + n16) | (a << 10) | (nexc << 17);
        }
    }
    if (own ? (mode & 2) : fill) {
        size_t cnt;
        st_unit(p.ws + local_off(d, unit, p.rank, p.rank * 8 + d.r, cnt) + lane * 16, u);
    }
    return hdr;
}
__device__ __forceinline__ void unit_announce(const Params& p, LAS unsigned char* lds, const PanelId& d, int unit, int pr, u32 hdr, u32 call, int mode) {
    const bool fill = is_fill(d, unit);
    if (pr == p.rank) {
        if (d.dest == p.rank ? (mode & 2) : fill) { size_t cnt; local_off(d, unit, p.rank, p.rank * 8 + d.r, cnt); add4_sys(p.ws + cnt, 1u); }
    } else if (!(mode & 1)) {
    } else if (!d.isB || pr == d.dest) {
        st8_sys(peer_ws(lds, (u32)pr) + W_HI + (size_t)(((p.rank * 8 + d.r) * 24 + (d.isB ? d.pi : 8 + d.pi)) * 4 + unit) * 8, (u32x2){hdr, call});
    } else if (fill) {
        st8_sys(peer_ws(lds, (u32)pr) + W_HX + (size_t)(((p.rank * 8 + d.r) * 8 + d.dest) * 4 + unit) * 8, (u32x2){hdr, call});
    }
}

constexpr u32 T_IN = 0, T_OUT = 1, T_END = 2, T_DEAD = 3, T_SKIP = 4;
constexpr u32 NP_IN = 384, NP_X = 28 * FC, NP_ROW = 32 - FC, NP_OUT = 16 * NP_ROW, NP_ALL = NP_IN + NP_X + NP_OUT;
#define R_POSTED 0
#define R_TAKEN 1
#define R_FULL 2
#define R_DONE 4
__device__ __forceinline__ volatile LAS u32* ring_words(LAS unsigned char* lds) { return (volatile LAS u32*)(lds + LDS_MISC + 256); }
__device__ __forceinline__ volatile LAS u32x4* ring_desc(LAS unsigned char* lds, u32 n) { return (volatile LAS u32x4*)(lds + LDS_MISC + 512 + (n % (u32)NSLOT) * 32u); }

struct Pair { u32 type, so, ho, cnt, sstep, hstep, cstep, dstep; char* dst; };
__device__ __forceinline__ Pair pair_of(const Params& p, u32 z, int j, int peer, int rot) {
    Pair q; q.type = T_END; q.so = 0; q.ho = (u32)W_TMO; q.cnt = 0; q.sstep = USLOT; q.hstep = 8; q.cstep = 0; q.dstep = USLOT; q.dst = nullptr;
    size_t cnt;
    if (z < NP_IN) {
        PanelId d;
        const int P = (int)(z >> 1), r = P / 24, i = P - r * 24, unit = (int)(z & 1u) * 2;
        d.r = r; d.isB = i < 8; d.dest = p.rank; d.pi = d.isB ? i : ((i - 8 + rot) & 15);
        q.type = T_IN; q.so = (u32)export_off(d, unit);
        q.ho = (u32)W_HI + (u32)(((peer * 8 + r) * 24 + (d.isB ? d.pi : 8 + d.pi)) * 4 + unit) * 8u;
        q.dst = p.ws + local_off(d, unit, p.rank, peer * 8 + r, cnt);
        q.cnt = (u32)cnt;
    } else if (z < NP_IN + NP_X) {
        const int y = (int)(z - NP_IN), rp = y / (7 * FC), yy = y - rp * (7 * FC), jj = yy / FC, k = yy - jj * FC;
        const bool across = (FC & 1) && k == FC - 1;
        PanelId d; d.r = 2 * rp + (across ? 0 : (k & 1)); d.isB = true; d.dest = (p.rank + 1 + jj) & 7; d.pi = 7;
        const int unit = across ? 4 - FC : 2 - (k & 2);
        PanelId e = d; e.r = d.r + (across ? 1 : 0);
        const int unit1 = across ? unit : unit + 1;
        size_t cnt1;
        const u32 l0 = (u32)local_off(d, unit, p.rank, peer * 8 + d.r, cnt), l1 = (u32)local_off(e, unit1, p.rank, peer * 8 + e.r, cnt1);
        q.type = T_IN; q.so = (u32)export_off(d, unit); q.sstep = (u32)export_off(e, unit1) - q.so;
        q.ho = (u32)W_HX + (u32)(((peer * 8 + d.r) * 8 + d.dest) * 4 + unit) * 8u; q.hstep = across ? 256u : 8u;
        q.dst = p.ws + l0; q.dstep = l1 - l0;
        q.cnt = (u32)cnt; q.cstep = (u32)cnt1 - (u32)cnt;
    } else if (z < NP_ALL) {
        int y = (int)(z - NP_IN - NP_X), pm, pn, unit;
        bool across = false;
        if (y < 112) { pn = y >> 4; pm = y & 15; unit = 0; }
        else {
            y -= 112; pm = y / (int)(NP_ROW - 7); const int i = y - pm * (int)(NP_ROW - 7);
            pn = i < 21 ? i / 3 : 7; unit = i < 21 ? 2 + (i - pn * 3) * 2 : (i - 21) * 4;
            across = FC == 3 && i >= 21;
        }
        q.type = T_OUT; q.so = (u32)W_PO + (u32)((pm * 8 + pn) * 8 + unit) * (u32)USLOT;
        q.sstep = across ? 4u * USLOT : (u32)USLOT; q.hstep = across ? 32u : 8u; q.dstep = across ? 128u * GN : 64u;
        q.ho = (u32)W_HO + (u32)((peer * 128 + pm * 8 + pn) * 8 + unit) * 8u;
        q.dst = (char*)p.out + (size_t)(pm * 256 + (unit >> 2) * 128) * GN + peer * NS + pn * 256 + (unit & 3) * 64;
    }
    return q;
}
__device__ __forceinline__ void pull_schedule(const Params& p, LAS unsigned char* lds, u32 call, int j, int c, int wid, int lane) {
    __builtin_amdgcn_s_setprio(2);
    const int peer = (p.rank + 1 + j) & 7, rot = rot_of(j);
    const u32 sub = (u32)lane & 1u;
    const u32 subm = 0u - sub;
    const char* pws = peer_ws(lds, (u32)peer);
    volatile LAS u32* rw = ring_words(lds);
    const u32 qbase = (call - 1u) * NP_ALL - 2u * (u32)NPW;
    char* qp = p.ws + W_Q + j * 128;
    u32 tk;
    u32 z = (u32)c, zn = (u32)(NPW + c), n = 0, slot = 0, dec = 0, lap = 0;
    Pair cur = pair_of(p, z, j, peer, rot);
    u32x2 h = (u32x2){0u, 0u};
    bool dead = false;
    u32 ends = 0;
    for (;;) {
        const bool more = z < NP_ALL;
        if (more && !dead && !__all(h.y == call)) {
            const u64 t0 = now();
            for (;;) {
                u32 t; poll8(h, t, p.ws + (cur.ho + (cur.hstep & subm)), p.ws + W_TMO);
                if (__all(h.y == call)) { dead = rfl(t) == call; break; }
                if (spin_fail(p.ws, call, t0, t)) { dead = true; break; }
            }
        }
        const Pair nxt = pair_of(p, zn, j, peer, rot);
        u32x2 hn;
        ld8_sys(hn, p.ws + (nxt.ho + (nxt.hstep & subm)));
        tk = 0xffffffffu;
        if (more) ticket_issue(tk, qp);
        for (;;) {
            const u32 tkn = rw[R_TAKEN], dn = rw[R_DONE + dec + sub];
            if (__all(n < tkn + (u32)RUNAHEAD && dn + (u32)(NSLOT / ND) > lap)) break;
            __builtin_amdgcn_s_sleep(1);
        }
        if (lane < 2) {
            const u32 type = !dead || !more ? cur.type : (cur.type == T_OUT ? T_DEAD : T_SKIP);
            const u64 src = (u64)(pws + (cur.so + (cur.sstep & subm))), d64 = (u64)(cur.dst + (cur.dstep & subm));
            volatile LAS u32x4* ds = (volatile LAS u32x4*)(lds + LDS_MISC + 512 + (slot + sub) * 32u);
            ds[0] = (u32x4){(u32)src, (u32)(src >> 32), h.x | (STAMPS ? (z < NP_IN / 2 ? 0u : z < NP_IN ? 1u : z < NP_IN + NP_X ? 2u : z < NP_IN + NP_X + NP_OUT / 2 ? 3u : 4u) << 26 : 0u), type};
            ds[1] = (u32x4){(u32)d64, (u32)(d64 >> 32), cur.cnt + (cur.cstep & subm), 0u};
        }
        n += 2;
        rw[R_POSTED] = n;
        slot = slot == (u32)NSLOT - 2u ? 0u : slot + 2u;
        if (dec == (u32)ND - 2u) { dec = 0; ++lap; } else dec += 2u;
        asm volatile("s_waitcnt vmcnt(0)" : "+v"(tk), "+v"(hn) :: "memory");
        if (more) { z = zn; zn = rfl(tk) - qbase; cur = nxt; h = hn; }
        else if (++ends == (u32)ND / 2u) break;
    }
}

#define GLDS(src, off) "global_load_lds_dwordx4 " src ", off offset:" #off " sc0 sc1\n\ts_waitcnt vmcnt(%[d])\n\t"
template <int D> __device__ __forceinline__ void glds_part(const void* src, u32 dst, u32 rem) {
    const u64 mask = rem >= 64u ? ~0ull : (1ull << rem) - 1ull;
    u32 keep;
    asm volatile("s_mov_b32 %0, m0\n\ts_mov_b32 m0, %2\n\ts_mov_b64 exec, %3\n\t" GLDS("%1", 0) "s_mov_b64 exec, -1\n\ts_mov_b32 m0, %0"
                 : "=&s"(keep) : "v"(src), "s"(dst), "s"(mask), [d] "i"(D - 1) : "memory");
}
template <int D> __device__ __forceinline__ void load_unit(LAS unsigned char* lds, const char* src, u32 dst, u32 n16, u32 n, u32 nslot, u32& full, u32& posted, u32x4& ds) {
    volatile LAS u32* rw = ring_words(lds);
    u32 keep;
    asm volatile("s_mov_b32 %0, m0\n\ts_mov_b32 m0, %2\n\ts_nop 0\n\t" GLDS("%1", 0) GLDS("%1", 1024) "s_mov_b32 m0, %0"
                 : "=&s"(keep) : "v"(src), "s"(dst), [d] "i"(D - 1) : "memory");
    if (D <= 9 && full + 1u < n) { full = n - 1u; rw[R_FULL] = full; }
    asm volatile("s_mov_b32 %0, m0\n\ts_mov_b32 m0, %2\n\ts_nop 0\n\t" GLDS("%1", 2048) GLDS("%1", 3072) "s_mov_b32 m0, %0"
                 : "=&s"(keep) : "v"(src), "s"(dst), [d] "i"(D - 1) : "memory");
    if (D > 9 && full + 1u < n) { full = n - 1u; rw[R_FULL] = full; }
    if (D <= 5) { full = n; rw[R_FULL] = n; }
    posted = rw[R_POSTED];
    ds = *(volatile LAS u32x4*)(lds + LDS_MISC + 512 + nslot * 32u);
    src += 4096;
    glds_part<D>(src, dst + 4096u, 64u);
    if (D == 6) { full = n; rw[R_FULL] = n; }
    glds_part<D>(src + 1024, dst + 5120u, n16 - 320u);
    if (D == 7) { full = n; rw[R_FULL] = n; }
    if (n16 > 384u) {
        glds_part<D>(src + 2048, dst + 6144u, n16 - 384u);
        if (D == 8) { full = n; rw[R_FULL] = n; }
        if (n16 > 448u) glds_part<D>(src + 3072, dst + 7168u, n16 - 448u);
    }
    rw[R_TAKEN] = n + 1u;
}
__device__ __forceinline__ void pull_load(const Params& p, LAS unsigned char* lds, int wid, int lane) {
    __builtin_amdgcn_s_setprio(3);
    volatile LAS u32* rw = ring_words(lds);
    const u32 lbase = (u32)(u64)lds;
    u32 ends = 0, slot = 0, full = 0;
    u32 posted = 0;
    u32x4 ds = (u32x4){0u, 0u, 0u, 0u};
    for (u32 n = 0;; ++n) {
        volatile LAS u32x4* dsp = (volatile LAS u32x4*)(lds + LDS_MISC + 512 + slot * 32u);
        if (rfl(posted) <= n) {
            posted = rw[R_POSTED];
            if (rfl(posted) <= n) {
                WAIT_V(0);
                full = n; rw[R_FULL] = n;
                while (rfl(rw[R_POSTED]) <= n) __builtin_amdgcn_s_sleep(1);
            }
            ds = dsp[0];
        }
        const u32 type = rfl(ds.w);
        const u32 dst = lbase + slot * (u32)USLOT;
        slot = slot == (u32)NSLOT - 1u ? 0u : slot + 1u;
        if (type <= T_OUT) {
            const u32 n16 = rfl(min(max(ds.z & 1023u, 336u), 512u));
            const char* src = (const char*)(((u64)ds.y << 32) | ds.x) + lane * 16;
            if (type == T_IN) load_unit<DEPTH>(lds, src, dst, n16, n, slot, full, posted, ds);
            else load_unit<DEPTH2>(lds, src, dst, n16, n, slot, full, posted, ds);
        } else {
            WAIT_V(0);
            full = n + 1u; rw[R_FULL] = full; rw[R_TAKEN] = full;
            posted = 0;
            if (type == T_END && ++ends == (u32)ND) break;
        }
    }
}

__device__ __forceinline__ void pull_decode(const Params& p, LAS unsigned char* lds, int wid, int lane) {
    volatile LAS u32* rw = ring_words(lds);
    LAS u32* bm = (LAS u32*)(lds + LDS_BM + wid * 1024);
    u32 done = 0;
    for (u32 n = (u32)wid;; n += (u32)ND) {
        while (rfl(rw[R_FULL]) <= n) __builtin_amdgcn_s_sleep(1);
        CFENCE;
        const u32x4 d0 = ring_desc(lds, n)[0], d1 = ring_desc(lds, n)[1];
        const u32 type = rfl(d0.w), hdr = rfl(d0.z);
        if (type == T_END) break;
        char* dst = (char*)(((u64)d1.y << 32) | d1.x);
        LAS unsigned char* sl = lds + (n % (u32)NSLOT) * (u32)USLOT;
        u32 u[32];
        if (type <= T_OUT && (hdr & H_RAW)) {
#pragma unroll
            for (int k = 0; k < 8; ++k) { const u32x4 v = *(const LAS u32x4*)(sl + k * 1024 + lane * 16); u[4 * k] = v.x; u[4 * k + 1] = v.y; u[4 * k + 2] = v.z; u[4 * k + 3] = v.w; }
        } else if (type == T_IN) {
            const u32 fl = *(const LAS u32*)(sl + 5120 + lane * 4);
            const u32x4 dn = *(const LAS u32x4*)(sl + 4096 + lane * 16);
            u32x4 lo[4];
#pragma unroll
            for (int k = 0; k < 4; ++k) lo[k] = *(const LAS u32x4*)(sl + k * 1024 + lane * 16);
            const u32 cn = (u32)__popc(fl), inc = scan_add(cn);
            u32 m[4];
            sparse_get(m, fl, inc - cn, (const LAS u32*)(sl + 5376));
            const u32 b4 = (((hdr >> 10) & 0x7fu) - 6u) * 0x01010101u;
            const u32 tl = b4 + 0x07040506u, th = b4 + 0x00010203u;
            u32 hi[16];
#pragma unroll
            for (int h = 0; h < 2; ++h) {
                const u32 B = bfi(0x55555555u, m[h], m[2 + h] << 1), D = bfi(0x55555555u, m[h] >> 1, m[2 + h]);
                u32 nn[4];
                nn[0] = bfi(0x33333333u, dn[2 * h], B << 2); nn[2] = bfi(0x33333333u, dn[2 * h] >> 2, B);
                nn[1] = bfi(0x33333333u, dn[2 * h + 1], D << 2); nn[3] = bfi(0x33333333u, dn[2 * h + 1] >> 2, D);
#pragma unroll
                for (int k = 0; k < 4; ++k) {
                    hi[8 * h + 2 * k] = bfi(0x80808080u, nn[k] << 7, __builtin_amdgcn_perm(th, tl, (nn[k] >> 1) & 0x07070707u));
                    hi[8 * h + 2 * k + 1] = bfi(0x80808080u, nn[k] << 3, __builtin_amdgcn_perm(th, tl, (nn[k] >> 5) & 0x07070707u));
                }
            }
            const u32 nexc = (hdr >> 17) & 0xffu;
            if (nexc) {
                const u32 sdw = (rdl(inc, 63) + 7u) >> 3;
                const u32 ev = *(const LAS u32*)(sl + 5376 + (sdw + (u32)lane) * 4u);
                for (u32 e = 0; e < nexc; ++e) {
                    const u32 en = rdl(ev, (int)e);
                    const u32 sh = ((en >> 7) & 3u) * 8u, kk = (en >> 9) & 15u;
                    const bool me = (en >> 13) == (u32)lane;
#pragma unroll
                    for (int k = 0; k < 16; ++k)
                        if (me && kk == (u32)k) hi[k] = bfi(0x7fu << sh, (en & 0x7fu) << sh, hi[k]);
                }
            }
#pragma unroll
            for (int k = 0; k < 16; ++k) {
                u[2 * k] = __builtin_amdgcn_perm(hi[k], lo[k >> 2][k & 3], 0x05010400u);
                u[2 * k + 1] = __builtin_amdgcn_perm(hi[k], lo[k >> 2][k & 3], 0x07030602u);
            }
        } else if (type == T_OUT) {
            *(volatile LAS u32x4*)(bm + lane * 4) = (u32x4){0u, 0u, 0u, 0u};
            const u32 fl = *(const LAS u32*)(sl + 6144 + lane * 4);
            u32x4 w4[6];
#pragma unroll
            for (int k = 0; k < 6; ++k) w4[k] = *(const LAS u32x4*)(sl + k * 1024 + lane * 16);
            const u32 cn = (u32)__popc(fl), inc = scan_add(cn);
            const u32 sdw = (rdl(inc, 63) + 7u) >> 3, nexc = (hdr >> 17) & 0xffu;
            const u32 ev = *(const LAS u32*)(sl + 6400 + (sdw + (u32)lane) * 4u);
            u32 m[4];
            sparse_get(m, fl, inc - cn, (const LAS u32*)(sl + 6400));
            if (2u * (u32)lane < nexc) __hip_atomic_fetch_or(bm + ((ev >> 5) & 255u), 1u << (ev & 31u), __ATOMIC_RELAXED, __HIP_MEMORY_SCOPE_WORKGROUP);
            if (2u * (u32)lane + 1u < nexc) __hip_atomic_fetch_or(bm + ((ev >> 21) & 255u), 1u << ((ev >> 16) & 31u), __ATOMIC_RELAXED, __HIP_MEMORY_SCOPE_WORKGROUP);
            CFENCE;
            const u32x4 m7 = *(volatile LAS u32x4*)(bm + lane * 4);
#pragma unroll
            for (int g = 0; g < 8; ++g) {
                const u32 ac = (g & 1) ? bfi(0x55555555u, m[g >> 1] >> 1, m7[g >> 1] << 1) : bfi(0x55555555u, m[g >> 1], m7[g >> 1]);
                u32 r[4]; r[3] = 0;
#pragma unroll
                for (int i = 0; i < 3; ++i) { const int k = 3 * g + i; r[i] = w4[k >> 2][k & 3]; r[3] |= (r[i] >> (6 - 2 * i)) & (0x03030303u << (2 * i)); }
#pragma unroll
                for (int i = 0; i < 4; ++i) {
                    const u32 e = ac << (2 * i), y6 = (r[i] << 1) ^ e, y7 = (y6 << 1) ^ e;
                    u[4 * g + i] = bfi(0x3f3f3f3fu, r[i], bfi(0x40404040u, y6, y7));
                }
            }
        } else {
#pragma unroll
            for (int k = 0; k < 32; ++k) u[k] = 0x80808080u;
        }
        WAIT_L(0);
        ++done;
        if (lane == 0) rw[R_DONE + wid] = done;
        if (type == T_IN) {
            if (!(XNOST & 1)) st_unit(dst + lane * 16, u); else asm volatile("" :: "v"(u[0] ^ u[5] ^ u[9] ^ u[13] ^ u[17] ^ u[22] ^ u[27] ^ u[31]));
            WAIT_V(0);
            if (lane == 0) add4_sys(p.ws + d1.z, 1u);
        } else if (XNOST & 2) { asm volatile("" :: "v"(u[0] ^ u[5] ^ u[9] ^ u[13] ^ u[17] ^ u[22] ^ u[27] ^ u[31]));
        } else if (type != T_SKIP) {
            char* o = dst + (size_t)(lane & 15) * GN + (lane >> 4) * 16;
#pragma unroll
            for (int k = 0; k < 8; ++k)
                st16_plain(o + (size_t)k * 16 * GN, (u32x4){u[4 * k], u[4 * k + 1], u[4 * k + 2], u[4 * k + 3]});
        }
    }
}

__device__ __forceinline__ void gemm_tile(const Params& p, LAS unsigned char* lds, u32 call, bool fillm, int pm, int pn, int wid, int lane) {
    const int wr = wid >> 2, wc = wid & 3;
    int j0 = 0;
#pragma unroll
    for (int j = 1; j < 7; ++j) if (rot_of(j) <= pm) j0 = j;
    const char* Ab = p.ws + W_A + (size_t)pm * 64 * PANEL;
    const char* Bb = p.ws + (fillm ? W_X : W_B) + (size_t)pn * 64 * PANEL;
    const char* cA = p.ws + W_ACNT + (size_t)pm * 256;
    const char* cB = p.ws + (fillm ? W_XCNT : W_BCNT) + (size_t)pn * 256;
    const u32 voff = (u32)(wid * 4096 + lane * 16);
    const u32 ldsw = (u32)wid * 2048u;
    const int aoff = wr * 8192 + ((lane & 15) * 4 + ((lane >> 4) ^ ((lane >> 2) & 2))) * 16;
    int boff = 65536 + wc * 4096 + lane * 16;
    asm volatile("" : "+v"(boff));
    int ktv = kt_of(lane, p.rank, j0);
    asm volatile("" : "+v"(ktv));
    u64 rdyA = wid == 4 ? 0ull : ~0ull, rdyB = rdyA;
    const u32 needA = call * 4u, needB = (fillm && pn == NVP - 1) ? call * (u32)(7 * FC - 4 * (NVP - 1)) : needA;
#define SA(b, h) (((b) * 2 + (h)) * 16384)
#define SB(b, h) ((4 + (b) * 2 + (h)) * 16384)
#define STAGE(buf, gbase, h) do { if (XNOMMA & 2) break; \
        __builtin_amdgcn_global_load_lds((const u32*)((gbase) + voff + (h) * 2048), (LAS u32*)(lds + (buf) + ldsw), 16, 0, 17); \
        __builtin_amdgcn_global_load_lds((const u32*)((gbase) + voff + (h) * 2048), (LAS u32*)(lds + (buf) + ldsw), 16, 1024, 17); } while (0)
#define LDA(dst, b, h) do { _Pragma("unroll") for (int m = 0; m < 4; ++m) _Pragma("unroll") for (int k = 0; k < 2; ++k) \
        dst[m][k] = *(const LAS bf16x8*)(lds + SA(b, h) + aoff + m * 2048 + k * 1024); } while (0)
#define LDB(dst, b, h) do { _Pragma("unroll") for (int n = 0; n < 2; ++n) _Pragma("unroll") for (int k = 0; k < 2; ++k) \
        dst[n][k] = *(const LAS bf16x8*)(lds + (SB(b, h) - 65536) + boff + k * 2048 + n * 1024); } while (0)
#define MMA(ai, bj, At, Bt) do { if (XNOMMA & 1) break; __builtin_amdgcn_s_setprio(1); \
        _Pragma("unroll") for (int m = 0; m < 4; ++m) _Pragma("unroll") for (int n = 0; n < 2; ++n) _Pragma("unroll") for (int k = 0; k < 2; ++k) \
            acc[ai][bj][m][n] = __builtin_amdgcn_mfma_f32_16x16x32_bf16(Bt[n][k], At[m][k], acc[ai][bj][m][n], 0, 0, 0); \
        __builtin_amdgcn_s_setprio(0); } while (0)
#define CHECK(RDY, o) do { if (!((RDY >> (o)) & 1ull)) { \
        const u64 t0_ = now(); __builtin_amdgcn_s_waitcnt(0xC07F); \
        for (u32 it_ = 1;; ++it_) { \
            u32 a_, b_; poll4x2(a_, b_, cA + ktv * 4, cB + ktv * 4); \
            rdyA = __ballot(a_ >= needA); rdyB = __ballot(b_ >= needB); \
            if ((RDY >> (o)) & 1ull) break; \
            if (PSLEEP) __builtin_amdgcn_s_sleep(PSLEEP); \
            if (spin_fail(p.ws, call, t0_, (it_ & 63u) ? 0u : ld4_wait(p.ws + W_TMO))) { rdyA = rdyB = ~0ull; break; } \
        } if (STAMPS > 1) { wacc += now() - t0_; __builtin_amdgcn_s_waitcnt(0xC07F); } } } while (0)
#define PEEK asm volatile("s_cmp_lg_u32 %4, 4\n\ts_cbranch_scc1 .Lpk%=\n\tglobal_load_dword %0, %2, off " CSC "\n\tglobal_load_dword %1, %3, off " CSC "\n.Lpk%=:" \
        : "+v"(pa), "+v"(pb) : "v"(cA + ktv * 4), "v"(cB + ktv * 4), "s"(pkw) : "memory", "scc")
#define WAIT_PK asm volatile("s_cmp_eq_u32 %0, 4\n\ts_cbranch_scc1 .Lpw%=\n\ts_waitcnt vmcnt(6)\n.Lpw%=:\n\ts_waitcnt vmcnt(8)" :: "s"(pkw) : "memory", "scc")
#define PEEKED do { asm volatile("s_waitcnt vmcnt(6)" : "+v"(pa), "+v"(pb) :: "memory"); rdyA |= __ballot(pa >= needA); rdyB |= __ballot(pb >= needB); } while (0)
#define KOFF(o) ((size_t)__builtin_amdgcn_readlane(ktv, (o)) * PANEL)

    u64 wacc = 0;
    f32x4 acc[2][2][4][2];
#pragma unroll
    for (int a = 0; a < 2; ++a)
#pragma unroll
        for (int b = 0; b < 2; ++b)
#pragma unroll
            for (int m = 0; m < 4; ++m)
#pragma unroll
                for (int n = 0; n < 2; ++n) acc[a][b][m][n] = (f32x4){0.f, 0.f, 0.f, 0.f};
    bf16x8 At[4][2], B0[2][2], B1[2][2];
    u32 pa = 0, pb = 0;
    int pkw = 0;

    CHECK(rdyA, 0); CHECK(rdyB, 0); CHECK(rdyB, 1); CHECK(rdyA, 1);
    BAR; CFENCE;
    size_t k1 = KOFF(1);
    {
        const size_t k0 = KOFF(0);
        STAGE(SB(0, 0), Bb + k0, 0); STAGE(SB(0, 1), Bb + k0, 1); STAGE(SA(0, 0), Ab + k0, 0);
        STAGE(SB(1, 0), Bb + k1, 0); STAGE(SA(0, 1), Ab + k0, 1); PEEK; STAGE(SB(1, 1), Bb + k1, 1);
        WAIT_PK; BAR;
        if (wr == 1) BAR;
    }
    for (int t = 0; t < 64; t += 2) {
        const int o2 = (t + 2) & 63, o3 = o2 + 1;
        const size_t k2 = KOFF(o2), k3 = KOFF(o3);
        pkw = t >= PKT ? wid : 0;
        LDB(B0, 0, 0); SCHED; LDA(At, 0, 0); STAGE(SA(1, 0), Ab + k1, 0);
        CHECK(rdyB, o2);
        WAIT_L(8); BAR; WAIT_L(0); MMA(0, 0, At, B0); BAR; SCHED;
        LDB(B1, 0, 1); STAGE(SB(0, 0), Bb + k2, 0);
        PEEKED; BAR; WAIT_L(0); MMA(0, 1, At, B1); BAR;
        LDA(At, 0, 1); STAGE(SA(1, 1), Ab + k1, 1);
        BAR; WAIT_L(0); MMA(1, 0, At, B0); BAR; SCHED;
        PEEK; STAGE(SB(0, 1), Bb + k2, 1);
        CHECK(rdyA, o2);
        WAIT_PK; BAR; MMA(1, 1, At, B1); BAR;
        LDB(B0, 1, 0); SCHED; LDA(At, 1, 0); STAGE(SA(0, 0), Ab + k2, 0);
        CHECK(rdyB, o3);
        WAIT_L(8); BAR; WAIT_L(0); MMA(0, 0, At, B0); BAR; SCHED;
        LDB(B1, 1, 1); STAGE(SB(1, 0), Bb + k3, 0);
        PEEKED; BAR; WAIT_L(0); MMA(0, 1, At, B1); BAR;
        LDA(At, 1, 1); STAGE(SA(0, 1), Ab + k2, 1);
        BAR; WAIT_L(0); MMA(1, 0, At, B0); BAR; SCHED;
        PEEK; STAGE(SB(1, 1), Bb + k3, 1);
        CHECK(rdyA, o3);
        WAIT_PK; BAR; MMA(1, 1, At, B1); BAR;
        k1 = k3;
    }
    asm volatile("s_waitcnt vmcnt(2)" : "+v"(pa), "+v"(pb) :: "memory");
    if (wr == 0) BAR;
    if (!fillm) STAMP(2);
    int tz = threadIdx.x; asm volatile("" : "+v"(tz));
    lane = tz & 63;
    const int tile = pm * 8 + pn;
    volatile LAS u32* red = (volatile LAS u32*)(lds + LDS_MISC + 1024);
    if (!fillm) {
        float mx = 0.f;
#pragma unroll
        for (int a = 0; a < 2; ++a)
#pragma unroll
            for (int b = 0; b < 2; ++b)
#pragma unroll
                for (int m = 0; m < 4; ++m)
#pragma unroll
                    for (int n = 0; n < 2; ++n)
#pragma unroll
                        for (int r = 0; r < 4; ++r) mx = fmaxf(mx, fabsf(acc[a][b][m][n][r]));
        const u32 wm = wave_umax(__float_as_uint(mx));
        if (lane == 0) red[wid] = wm;
        WAIT_L(0); BAR; CFENCE;
        if (tz < NR) {
            const u32x4 r0 = *(volatile LAS u32x4*)red, r1 = *(volatile LAS u32x4*)(red + 4);
            const u32 t = max(max(max(r0.x, r0.y), max(r0.z, r0.w)), max(max(r1.x, r1.y), max(r1.z, r1.w)));
            st8_sys(peer_ws(lds, (u32)tz) + W_AMAX + (size_t)(p.rank * 128 + tile) * 8, (u32x2){t, call});
        }
    }
    bool dead = false;
    float amax;
    {
        const char* src = p.ws + W_AMAX + tz * 16;
        const u64 t0 = now();
        u32x4 gq;
        for (;;) {
            u32 t; poll16(gq, t, src, p.ws + W_TMO);
            dead = rfl(t) == call;
            if (__all(gq.y == call && gq.w == call)) break;
            if (spin_fail(p.ws, call, t0, t)) { dead = true; break; }
        }
        const u32 wm = wave_umax(max(gq.x, gq.z));
        if (lane == 0) red[8 + wid] = dead ? 0xffffffffu : wm;
        WAIT_L(0); BAR; CFENCE;
        const u32x4 r0 = *(volatile LAS u32x4*)(red + 8), r1 = *(volatile LAS u32x4*)(red + 12);
        const u32 t = rfl(max(max(max(r0.x, r0.y), max(r0.z, r0.w)), max(max(r1.x, r1.y), max(r1.z, r1.w))));
        dead = t == 0xffffffffu;
        amax = __uint_as_float(t);
    }
    if (!fillm) STAMP(3);
    if (wid < 2) __builtin_amdgcn_s_setprio(3);
    const float inv = amax > 0.f ? 127.0f / amax : 0.f;
    u32 q[32];
#pragma unroll
    for (int m = 0; m < 4; ++m)
#pragma unroll
        for (int a = 0; a < 2; ++a)
#pragma unroll
            for (int r = 0; r < 4; ++r) {
                const u32 t0 = __float_as_uint(fmaf(acc[a][0][m][0][r], inv, 12582912.f)), t1 = __float_as_uint(fmaf(acc[a][0][m][1][r], inv, 12582912.f));
                const u32 t2 = __float_as_uint(fmaf(acc[a][1][m][0][r], inv, 12582912.f)), t3 = __float_as_uint(fmaf(acc[a][1][m][1][r], inv, 12582912.f));
                q[(m * 2 + a) * 4 + r] = __builtin_amdgcn_perm(t1, t0, 0x0c0c0400u) | __builtin_amdgcn_perm(t3, t2, 0x04000c0cu);
            }
    if (dead) {
#pragma unroll
        for (int k = 0; k < 32; ++k) q[k] = 0x80808080u;
    }
    int col = p.rank * NS + pn * 256 + wc * 64;
    if (!fillm) {
        char* slot = p.ws + W_PO + (size_t)(tile * 8 + wid) * USLOT;
        LAS unsigned char* enc = lds + LDS_ENC + wid * 2048;
        *(volatile LAS u32x4*)(enc + lane * 16) = (u32x4){0u, 0u, 0u, 0u};
        *(volatile LAS u32x4*)(enc + 1024 + lane * 16) = (u32x4){0u, 0u, 0u, 0u};
        char* dp = slot + lane * 16;
        u32 hdr;
        if (wid < 2 && pn == 0 && pm < NRAW) {
            st_unit(dp, q);
            hdr = H_RAW | 512u;
        } else {
        u32 ac[8], m6[4], m7[4];
#pragma unroll
        for (int g = 0; g < 8; ++g) {
            ac[g] = 0;
#pragma unroll
            for (int i = 0; i < 4; ++i) { const u32 v = q[4 * g + i]; ac[g] |= ((v ^ (v << 1)) & 0xc0c0c0c0u) >> (2 * i); }
        }
#pragma unroll
        for (int h = 0; h < 4; ++h) { m6[h] = bfi(0x55555555u, ac[2 * h], ac[2 * h + 1] << 1); m7[h] = bfi(0xaaaaaaaau, ac[2 * h], ac[2 * h + 1] >> 1); }
        const u32 fl = nib_flags(m6[0]) | (nib_flags(m6[1]) << 1) | (nib_flags(m6[2]) << 2) | (nib_flags(m6[3]) << 3);
        const u32 cn = (u32)__popc(fl) | ((u32)(__popc(m7[0]) + __popc(m7[1]) + __popc(m7[2]) + __popc(m7[3])) << 16);
        const u32 inc = scan_add(cn), tot = rdl(inc, 63), exc = inc - cn;
        const u32 sdw = ((tot & 0xffffu) + 7u) >> 3, nexc = tot >> 16;
        if (nexc > 128u) {
            st_unit(dp, q);
            hdr = H_RAW | 512u;
        } else {
            u32 w[24];
#pragma unroll
            for (int g = 0; g < 8; ++g)
#pragma unroll
                for (int i = 0; i < 3; ++i) w[3 * g + i] = bfi(0x3f3f3f3fu, q[4 * g + i], q[4 * g + 3] << (6 - 2 * i));
            st16_sys<0>(dp, (u32x4){w[0], w[1], w[2], w[3]}); st16_sys<1024>(dp, (u32x4){w[4], w[5], w[6], w[7]});
            st16_sys<2048>(dp, (u32x4){w[8], w[9], w[10], w[11]}); st16_sys<3072>(dp, (u32x4){w[12], w[13], w[14], w[15]});
            st16_sys<0>(dp + 4096, (u32x4){w[16], w[17], w[18], w[19]}); st16_sys<1024>(dp + 4096, (u32x4){w[20], w[21], w[22], w[23]});
            ((volatile LAS u32*)enc)[lane] = fl;
            sparse_put(m6, fl, exc & 0xffffu, (LAS u32*)(enc + 256));
            volatile LAS unsigned short* list = (volatile LAS unsigned short*)(enc + 256 + sdw * 4u);
            u32 s = exc >> 16;
#pragma unroll
            for (int h = 0; h < 4; ++h) {
                u32 mm = m7[h];
                while (mm) { list[s] = (unsigned short)((lane * 4 + h) * 32 + __builtin_ctz(mm)); mm &= mm - 1u; ++s; }
            }
            const u32 n16 = (64u + sdw + ((nexc + 1u) >> 1) + 3u) >> 2;
            tail_out(slot + 6144, enc, n16, lane);
            hdr = (384u + n16) | (nexc << 17);
        }
        }
        if (lane == 0) red[16 + wid] = hdr;
        WAIT_V(0);
        const bool early = pn < NEARLY;
        if (early && wid < 2 && lane < NR && lane != p.rank)
            st8_sys(peer_ws(lds, (u32)lane) + W_HO + (size_t)((p.rank * 128 + tile) * 8 + wid) * 8, (u32x2){hdr, call});
        __builtin_amdgcn_s_setprio(0);
        WAIT_L(0); BAR; CFENCE;
        if (tz < 64 && (tz >> 3) != p.rank && (!early || (tz & 7) >= 2))
            st8_sys(peer_ws(lds, (u32)(tz >> 3)) + W_HO + (size_t)((p.rank * 128 + tile) * 8 + (tz & 7)) * 8, (u32x2){red[16 + (tz & 7)], call});
        STAMP(4);
    } else {
        const int g = pn * 4 + wc;
        col = g < 7 * FC ? ((p.rank + 1 + g / FC) & 7) * NS + 7 * 256 + (4 - FC + g % FC) * 64 : -1;
    }
    if (col >= 0) {
        signed char* o = p.out + (size_t)(pm * 256 + wr * 128 + (lane & 15)) * GN + col + (lane >> 4) * 16;
#pragma unroll
        for (int k = 0; k < 8; ++k)
            st16_plain(o + (size_t)k * 16 * GN, (u32x4){q[4 * k], q[4 * k + 1], q[4 * k + 2], q[4 * k + 3]});
    }
#undef SA
#undef SB
#undef STAGE
#undef LDA
#undef LDB
#undef MMA
#undef CHECK
#undef KOFF
#undef PEEK
#undef PEEKED
#undef WAIT_PK
}

__global__ void __launch_bounds__(BLOCK, 2) fused_gemm_q8(Params p) {
    __shared__ __attribute__((aligned(16))) unsigned char smem[LDS_BYTES];
    LAS unsigned char* lds = (LAS unsigned char*)smem;
    const int tid = threadIdx.x, wid = (int)rfl((u32)tid >> 6), lane = tid & 63;
    const int xcd = blockIdx.x & 7, idx = blockIdx.x >> 3;
    STAMP(0);
    const u64 tstart = now();
    volatile LAS u32* misc = (volatile LAS u32*)(lds + LDS_MISC);
    LAS unsigned char* enc = lds + LDS_ENC + wid * 2048;
    int i0 = -1, u0 = (int)blockIdx.x & 3;
    if (wid < 2) {
        if (idx < 16) { const int n = (wid * 128 + (int)blockIdx.x) >> 2; i0 = n < 28 ? (n >> 2) * 8 + 4 + (n & 3) : (n < 52 ? n + 28 : -1); }
        else if (idx >= 24) { const int n = (wid * 64 + (int)blockIdx.x - 192) >> 2; i0 = n < 28 ? (n >> 2) * 8 + (n & 3) : -1; }
    }
    if (i0 >= 0) {
        const PanelId d0 = panel_of(0, i0, p.rank);
        const int mode0 = idx < 16 ? 3 : 5;
        f32x4 f[16];
        unit_load(p, d0, u0, lane, f);
        WAIT_V(0); PIN_F(f);
        if (STAMPS == 1) STAMP(10);
        BAR;
        const u32 hdr = unit_pack(p, enc, d0, u0, lane, f, mode0);
        if (STAMPS == 1) STAMP(11);
        const u32 call0 = misc[0];
        WAIT_V(0);
        if (lane < NR) unit_announce(p, lds, d0, u0, lane, hdr, call0, mode0);
    } else {
        if (wid == 7) {
            u32 tk; u32x2 pw;
            ticket_issue(tk, p.ws + W_SEQ + (size_t)blockIdx.x * 128);
            ld8_sys(pw, (const char*)p.ws_all + (lane & 7) * 8);
            asm volatile("s_waitcnt vmcnt(0)" : "+v"(tk), "+v"(pw) :: "memory");
            if (lane == 0) misc[0] = tk + 1u;
            if (lane < NR) *(volatile LAS u32x2*)(lds + LDS_MISC + 64 + lane * 8) = pw;
            if (lane < 16) ring_words(lds)[lane] = 0u;
            ((volatile LAS u32*)(lds + LDS_MISC + 1280))[lane] = 0u;
            const u32 call7 = rfl(tk) + 1u;
            if (blockIdx.x == 0) {
                if (lane == 0) st8_sys(p.ws + W_ROT, (u32x2){(u32)__builtin_amdgcn_s_getreg(20 | (3 << 11)) & 7u, call7});
            } else if (idx >= 16 && idx < 24) {
                const u64 t0 = now();
                u32x2 ro;
                for (;;) {
                    u32 t; poll8(ro, t, p.ws + W_ROT, p.ws + W_TMO);
                    if (rfl(ro.y) == call7) break;
                    if (spin_fail(p.ws, call7, t0, t)) { ro.x = 8u; break; }
                }
                if (lane == 0) misc[1] = ro.x;
            }
            WAIT_L(0);
        }
        BAR; CFENCE;
    }
    const u32 call = rfl(misc[0]);
    STAMP(6);

    const bool is_gemm = idx < 16;
    const int c = idx - 16;
    const u32 rot0 = is_gemm || idx >= 24 ? 0u : rfl(misc[1]);
    if (rot0 > 7u) return;
    const int jl = (peer_of((int)((xcd + rot0) & 7u), p.rank) - p.rank - 1) & 7;
    if (!is_gemm && jl < 7 && c < NPW) {
        if (wid == 7) pull_schedule(p, lds, call, jl, c, wid, lane);
        else if (wid == 6) pull_load(p, lds, wid, lane);
        else pull_decode(p, lds, wid, lane);
        STAMP(5);
        return;
    }
    const int pm = (xcd >> 1) * 4 + (idx >> 2), pn = (xcd & 1) * 4 + (idx & 3);
    {
        const int me = idx >= 24 ? (int)blockIdx.x - 192 : 64 + c;
        const int lim = is_gemm ? 7 * 96 : 7 * 292, mode = is_gemm ? 2 : 1, estep = is_gemm ? lim : 8 * NSPARE;
        int e = is_gemm ? (wid >= 2 ? (wid - 2) * 128 + (int)blockIdx.x : lim) : ((wid >> 2) * NSPARE + me) * 4 + (wid & 3);
        volatile LAS u32* hb = (volatile LAS u32*)(lds + LDS_MISC + 1280 + (wid >> 2) * 128);
        auto unit_of = [&](int v, int& unit) {
            int r, i;
            if (is_gemm) { r = v / 96; i = 224 + v - r * 96; }
            else { r = v / 292; i = v - r * 292; if (i >= 288) i += 28; }
            unit = i & 3;
            return panel_of(1 + r, i >> 2, p.rank);
        };
        if (e < lim) {
            int unit;
            PanelId d = unit_of(e, unit);
            f32x4 f[16];
            if (!is_gemm) { while (now() - tstart < (wid < 4 ? SPD0 : SPD1)) __builtin_amdgcn_s_sleep(4); }
            unit_load(p, d, unit, lane, f);
            bool first = true;
            for (;;) {
                WAIT_V(0); PIN_F(f);
                if (STAMPS == 1 && first) STAMP(7);
                const u32 hdr = unit_pack(p, enc, d, unit, lane, f, mode);
                if (STAMPS == 1 && first) STAMP(8);
                e += estep;
                int unitn;
                const PanelId dn = unit_of(e, unitn);
                if (e < lim) { unit_load(p, dn, unitn, lane, f); WAIT_V(16); } else { WAIT_V(0); }
                if (is_gemm) {
                    if (lane < NR) unit_announce(p, lds, d, unit, lane, hdr, call, mode);
                } else {
                    u32 arr = 0;
                    if (lane == 0) { hb[4 + unit] = hdr; arr = __hip_atomic_fetch_add((LAS u32*)hb, 1u, __ATOMIC_RELAXED, __HIP_MEMORY_SCOPE_WORKGROUP); }
                    if (rfl(arr) == 3u && lane < 4 * NR) unit_announce(p, lds, d, lane & 3, lane >> 2, hb[4 + (lane & 3)], call, mode);
                    hb += 8;
                }
                if (STAMPS == 1 && first) STAMP(9);
                first = false;
                if (e >= lim) break;
                d = dn; unit = unitn;
            }
        }
        STAMP(1);
        if (!is_gemm) return;
    }
#pragma clang loop unroll(disable)
    for (int pass = 0; pass < (pn < NVP ? 2 : 1); ++pass) {
        int tz = threadIdx.x; asm volatile("" : "+v"(tz));
        gemm_tile(p, lds, call, pass == 1, pm, pn, wid, tz & 63);
        if (pass == 0) STAMP(5); else STAMP(13);
    }
}

extern "C" void kernel_launch(int rank, int nranks, ncclComm_t comm, hipStream_t stream,
                              void* const* d_in, const int* in_sizes, int n_in,
                              void* d_out, int out_size,
                              const int* in_dtypes, int out_dtype,
                              void* const* d_out_all,
                              void* const* d_ws_all, size_t ws_bytes) {
    (void)comm; (void)in_dtypes; (void)out_dtype; (void)d_out_all;
    if (nranks != NR || n_in < 2 || ws_bytes < W_END || in_sizes[0] != GM * KS || in_sizes[1] != KS * GN || out_size != GM * GN ||
        (unsigned)rank >= (unsigned)NR) {
        fprintf(stderr, "unexpected problem geometry\n");
        return;
    }
    Params p;
    p.x = (const float*)d_in[0];
    p.w = (const float*)d_in[1];
    p.out = (signed char*)d_out;
    p.ws = (char*)d_ws_all[rank];
    p.ws_all = (char* const*)d_ws_all;
    p.rank = rank;
    p.pad = 0;
    hipLaunchKernelGGL(fused_gemm_q8, dim3(GRID), dim3(BLOCK), 0, stream, p);
}
```

```cpp
#include <hip/hip_runtime.h>
#include <rccl/rccl.h>
#include <cstdio>
#include <cstdint>

#define STAMPS 0

#define LAS __attribute__((address_space(3)))
typedef unsigned u32;
typedef unsigned long long u64;
typedef u32 u32x4 __attribute__((ext_vector_type(4)));
typedef u32 u32x2 __attribute__((ext_vector_type(2)));
typedef float f32x4 __attribute__((ext_vector_type(4)));
typedef short bf16x8 __attribute__((ext_vector_type(8)));

constexpr int NR = 8, GM = 4096, GN = 16384, KS = 512, NS = 2048;
constexpr int GRID = 256, BLOCK = 512;
constexpr int PANEL = 32768;
constexpr int USLOT = 8192;
constexpr size_t W_TMO = 128;
constexpr size_t W_ROT = 256;
constexpr size_t W_Q = 1024;
constexpr size_t W_ACNT = 4096;
constexpr size_t W_BCNT = 8192;
constexpr size_t W_XCNT = 10240;
constexpr size_t W_AMAX = 16384;
constexpr size_t W_GMAX = 20480;
constexpr size_t W_HI = 32768;
constexpr size_t W_HX = 81920;
constexpr size_t W_HO = 98304;
constexpr size_t W_SEQ = 256 << 10;
constexpr size_t W_DBG = 512 << 10;
constexpr size_t W_A = 1ull << 20;
constexpr size_t W_B = 33ull << 20;
constexpr size_t W_X = 49ull << 20;
constexpr size_t W_PX = 65ull << 20;
constexpr size_t W_PW = 69ull << 20;
constexpr size_t W_PO = 85ull << 20;
constexpr size_t W_END = 93ull << 20;
constexpr u64 DEADLINE = 40000000ull;

#define NPW 8
#define DEPTH 7
#define DEPTH2 6
#define RUNAHEAD 2
#define GDLY 100
#define EDLY 30
#define GCOPY 8
#define SPD0 450
#define SPD1 700
#define NRAW 8
#define RATES 0
#define NEARLY 1
#define NRAWE 8
constexpr int ND = 6, NSLOT = 12;
constexpr int NSPARE = 7 * (16 - NPW) + 16;
#define FC 3
constexpr int NVP = (7 * FC + 3) / 4;
static_assert(7 * 292 <= 4 * 8 * NSPARE && NPW == 8 && DEPTH <= 11 && DEPTH2 <= 11 && FC >= 2 && FC <= 4, "roles");

constexpr int LDS_BM = 98304;
constexpr int LDS_ENC = 114688;
constexpr int LDS_MISC = 131072;
constexpr int LDS_BYTES = LDS_MISC + 2048;

struct Params {
    const float* x;
    const float* w;
    signed char* out;
    char* ws;
    char* const* ws_all;
    int rank;
    int pad;
};

#define WAIT_V(n) asm volatile("s_waitcnt vmcnt(" #n ")" ::: "memory")
#define WAIT_L(n) asm volatile("s_waitcnt lgkmcnt(" #n ")" ::: "memory")
#define BAR __builtin_amdgcn_s_barrier()
#define SCHED __builtin_amdgcn_sched_barrier(0)
#define CFENCE asm volatile("" ::: "memory")

template <int OFF> __device__ __forceinline__ void st16_sys(void* p, u32x4 x) {
    asm volatile("global_store_dwordx4 %0, %1, off offset:%2 sc0 sc1\n\ts_nop 1" :: "v"(p), "v"(x), "i"(OFF) : "memory");
}
__device__ __forceinline__ void ld8_sys(u32x2& x, const void* p) {
    asm volatile("global_load_dwordx2 %0, %1, off sc0 sc1" : "=v"(x) : "v"(p) : "memory");
}
__device__ __forceinline__ void st8_sys(void* p, u32x2 x) {
    asm volatile("global_store_dwordx2 %0, %1, off sc0 sc1\n\ts_nop 1" :: "v"(p), "v"(x) : "memory");
}
__device__ __forceinline__ void st4_sys(void* p, u32 x) {
    asm volatile("global_store_dword %0, %1, off sc0 sc1\n\ts_nop 1" :: "v"(p), "v"(x) : "memory");
}
__device__ __forceinline__ void add4_sys(void* p, u32 x) {
    asm volatile("global_atomic_add %0, %1, off sc1\n\ts_nop 1" :: "v"(p), "v"(x) : "memory");
}
__device__ __forceinline__ void ld16x4(f32x4& a, f32x4& b, f32x4& c, f32x4& d, const void* p0, const void* p1, const void* p2, const void* p3, int nt) {
    asm volatile("s_cmp_lg_u32 %8, 0\n\ts_cbranch_scc1 .Lnt%=\n\t"
                 "global_load_dwordx4 %0, %4, off\n\tglobal_load_dwordx4 %1, %5, off\n\tglobal_load_dwordx4 %2, %6, off\n\tglobal_load_dwordx4 %3, %7, off\n\t"
                 "s_branch .Lnd%=\n.Lnt%=:\n\t"
                 "global_load_dwordx4 %0, %4, off nt\n\tglobal_load_dwordx4 %1, %5, off nt\n\tglobal_load_dwordx4 %2, %6, off nt\n\tglobal_load_dwordx4 %3, %7, off nt\n.Lnd%=:"
                 : "=&v"(a), "=&v"(b), "=&v"(c), "=&v"(d) : "v"(p0), "v"(p1), "v"(p2), "v"(p3), "s"(nt) : "memory", "scc");
}
__device__ __forceinline__ void ticket_issue(u32& x, void* p) {
    u64 sv;
    asm volatile("s_mov_b64 %1, exec\n\ts_mov_b64 exec, 1\n\tglobal_atomic_add %0, %2, %3, off sc0 sc1\n\ts_mov_b64 exec, %1"
                 : "=&v"(x), "=&s"(sv) : "v"(p), "v"(1u) : "memory");
}
__device__ __forceinline__ void poll8(u32x2& x, u32& t, const void* p, const void* tp) {
    asm volatile("global_load_dwordx2 %0, %2, off sc0 sc1\n\tglobal_load_dword %1, %3, off sc0 sc1\n\ts_waitcnt vmcnt(0)"
                 : "=&v"(x), "=&v"(t) : "v"(p), "v"(tp) : "memory");
}
__device__ __forceinline__ void poll_win(u32& x, const void* p, u64 lanes) {
    u64 sv;
    asm volatile("s_mov_b64 %1, exec\n\ts_mov_b64 exec, %3\n\tglobal_load_dword %0, %2, off sc0 sc1\n\ts_mov_b64 exec, %1\n\ts_waitcnt vmcnt(0)"
                 : "+v"(x), "=&s"(sv) : "v"(p), "s"(lanes) : "memory");
}
__device__ __forceinline__ u32x2 ld8_wait(const void* p) {
    u32x2 x; asm volatile("global_load_dwordx2 %0, %1, off sc0 sc1\n\ts_waitcnt vmcnt(0)" : "=&v"(x) : "v"(p) : "memory"); return x;
}
__device__ __forceinline__ u32x4 ld16_wait(const void* p) {
    u32x4 x; asm volatile("global_load_dwordx4 %0, %1, off sc0 sc1\n\ts_waitcnt vmcnt(0)" : "=&v"(x) : "v"(p) : "memory"); return x;
}
__device__ __forceinline__ u32 ld4_wait(const void* p) {
    u32 x; asm volatile("global_load_dword %0, %1, off sc0 sc1\n\ts_waitcnt vmcnt(0)" : "=&v"(x) : "v"(p) : "memory"); return x;
}
__device__ __forceinline__ u32 cvt_pk_bf16(float lo, float hi) {
    u32 r; asm("v_cvt_pk_bf16_f32 %0, %1, %2" : "=v"(r) : "v"(lo), "v"(hi)); return r;
}
__device__ __forceinline__ u32 bfi(u32 m, u32 a, u32 b) { return __builtin_amdgcn_bitop3_b32(m, a, b, 0xCA); }
__device__ __forceinline__ u32 lshl_or(u32 a, u32 sh, u32 b) { u32 r; asm("v_lshl_or_b32 %0, %1, %2, %3" : "=v"(r) : "v"(a), "v"(sh), "v"(b)); return r; }
template <int SH> __device__ __forceinline__ u32 lshl_or_i(u32 a, u32 b) { u32 r; asm("v_lshl_or_b32 %0, %1, %2, %3" : "=v"(r) : "v"(a), "n"(SH), "v"(b)); return r; }
__device__ __forceinline__ u32 times15(u32 t) { u32 r = t << 4; asm("" : "+v"(r)); return r - t; }
__device__ __forceinline__ u32 rfl(u32 x) { return (u32)__builtin_amdgcn_readfirstlane((int)x); }
__device__ __forceinline__ u32 rdl(u32 x, int l) { return (u32)__builtin_amdgcn_readlane((int)x, l); }
__device__ __forceinline__ u64 now() { return __builtin_amdgcn_s_memrealtime(); }

__device__ __forceinline__ bool spin_fail(char* ws, u32 call, u64 t0, u32 t) {
    if (rfl(t) == call) return true;
    if (now() - t0 > DEADLINE) { st4_sys(ws + W_TMO, call); WAIT_V(0); return true; }
    __builtin_amdgcn_s_sleep(1);
    return false;
}
#define DPP0(v, ctl, rm, bc) (u32)__builtin_amdgcn_update_dpp(0, (int)(v), ctl, rm, 0xf, bc)
__device__ __forceinline__ u32 scan_add(u32 v) {
    v += DPP0(v, 0x111, 0xf, true); v += DPP0(v, 0x112, 0xf, true); v += DPP0(v, 0x114, 0xf, true); v += DPP0(v, 0x118, 0xf, true);
    v += DPP0(v, 0x142, 0xa, false); v += DPP0(v, 0x143, 0xc, false);
    return v;
}
__device__ __forceinline__ u32 wave_umax(u32 v) {
    v = max(v, DPP0(v, 0x111, 0xf, true)); v = max(v, DPP0(v, 0x112, 0xf, true)); v = max(v, DPP0(v, 0x114, 0xf, true)); v = max(v, DPP0(v, 0x118, 0xf, true));
    v = max(v, DPP0(v, 0x142, 0xa, false)); v = max(v, DPP0(v, 0x143, 0xc, false));
    return rdl(v, 63);
}
__device__ __forceinline__ int rot_of(int j) { return (j * 16) / 7; }
__device__ __forceinline__ int kt_of(int o, int rank, int j0) {
    if (o < 8) return rank * 8 + o;
    const int r = (o - 8) / 7, s = o - 8 - r * 7;
    int j = j0 - s; if (j < 0) j += 7;
    return ((rank + 1 + j) & 7) * 8 + r;
}
__device__ __forceinline__ int peer_of(int x, int rank) {
    const u32 ga = 0x6320u, gb = 0x7541u;
    const bool ina = (0x4du >> rank) & 1u;
    const u32 own = ina ? ga : gb, oth = ina ? gb : ga;
    if (x >= 4) return (int)((oth >> (4 * (x - 4))) & 7u);
    if (x == 1) return rank;
    const int want = x == 0 ? 0 : x - 1;
    int n = 0, res = rank;
#pragma unroll
    for (int i = 0; i < 4; ++i) { const int q = (int)((own >> (4 * i)) & 7u); if (q != rank) { if (n == want) res = q; ++n; } }
    return res;
}
__device__ __forceinline__ char* peer_ws(LAS unsigned char* lds, u32 pr) {
    return (char*)((volatile LAS u64*)(lds + LDS_MISC + 64))[pr & 7u];
}

#define STAMP(i) do {} while (0)

struct Msk { u32 m, ms, m12, m2, o[8]; };
template <bool HI> __device__ __forceinline__ Msk msk(u32 m) {
    Msk k; k.m = m; k.ms = m << 1;
    const u32 c2 = m - ((m >> 1) & 0x55555555u), cl = c2 & 0x33333333u;
    if (HI) {
        const u32 n2 = (m << 2) & k.ms & 0x44444444u, n12 = ((m << 2) | k.ms) & 0x44444444u;
        k.m2 = lshl_or_i<1>(n2, n2); k.m12 = lshl_or_i<1>(n12, n12);
    } else {
        const u32 n2 = (m >> 1) & m & 0x11111111u, n12 = ((m >> 1) | m) & 0x11111111u;
        k.m2 = lshl_or_i<1>(n2, n2); k.m12 = lshl_or_i<1>(n12, n12);
    }
    const u32 c4 = cl + ((c2 >> 2) & 0x33333333u), ce = c4 & 0x0f0f0f0fu, cb = ce + ((c4 >> 4) & 0x0f0f0f0fu);
    u32 pe; asm("v_mul_u32_u24 %0, 0x10101, %1" : "=v"(pe) : "v"(cb));
    const u32 po = pe + (ce >> 8);
    k.o[0] = 0; k.o[1] = ce; k.o[2] = pe; k.o[3] = po; k.o[4] = pe >> 8; k.o[5] = po >> 8; k.o[6] = pe >> 16; k.o[7] = po >> 16;
    return k;
}
__device__ __forceinline__ u32 press(u32 x, const Msk& k) {
    const u32 y = bfi(0x55555555u, x | ((x >> 1) & ~k.m), x & k.ms);
    const u32 hi = (y >> 2) & 0x33333333u;
    const u32 z = (y & 0x33333333u) + hi + (hi & k.m12) + ((hi & k.m2) << 1);
    u32 c = z & 15u, d = (z >> 28) << (k.o[7] & 31u);
#pragma unroll
    for (int i = 1; i < 7; i += 2) { c = lshl_or(__builtin_amdgcn_ubfe(z, 4 * i, 4u), k.o[i], c); d = lshl_or(__builtin_amdgcn_ubfe(z, 4 * i + 4, 4u), k.o[i + 1], d); }
    return c | d;
}
__device__ __forceinline__ u32 spread(u32 c, const Msk& k) {
    u32 z = c & 15u, w = (c >> (k.o[7] & 31u)) << 28;
    z = lshl_or_i<4>(__builtin_amdgcn_ubfe(c, k.o[1], 4u), z); w = lshl_or_i<8>(__builtin_amdgcn_ubfe(c, k.o[2], 4u), w);
    z = lshl_or_i<12>(__builtin_amdgcn_ubfe(c, k.o[3], 4u), z); w = lshl_or_i<16>(__builtin_amdgcn_ubfe(c, k.o[4], 4u), w);
    z = lshl_or_i<20>(__builtin_amdgcn_ubfe(c, k.o[5], 4u), z); w = lshl_or_i<24>(__builtin_amdgcn_ubfe(c, k.o[6], 4u), w);
    z |= w;
    const u32 h = bfi(k.m12, bfi(k.m2, z, z << 1), z << 2);
    const u32 y = bfi(0x33333333u, z, h);
    return bfi(0x55555555u, y, bfi(k.ms, y, y << 1)) & k.m;
}
__device__ __forceinline__ void bit_put(LAS u32* st, u32 off, u32 c) {
    const u64 x = (u64)c << (off & 31u);
    __hip_atomic_fetch_or(st + (off >> 5), (u32)x, __ATOMIC_RELAXED, __HIP_MEMORY_SCOPE_WORKGROUP);
    __hip_atomic_fetch_or(st + (off >> 5) + 1, (u32)(x >> 32), __ATOMIC_RELAXED, __HIP_MEMORY_SCOPE_WORKGROUP);
}
__device__ __forceinline__ u32 bit_get(const LAS u32* st, u32 off) {
    const u32 w = min(off >> 5, 500u);
    return __builtin_amdgcn_alignbit(st[w + 1], st[w], off & 31u);
}
__device__ __forceinline__ u32 popc(u32 x) { return (u32)__popc(x); }
__device__ __forceinline__ void to_planes(const u32* n, u32& p0, u32& p1, u32& p2, u32& p3) {
    const u32 A = bfi(0x33333333u, n[0], n[2] << 2), B = bfi(0x33333333u, n[0] >> 2, n[2]);
    const u32 C = bfi(0x33333333u, n[1], n[3] << 2), D = bfi(0x33333333u, n[1] >> 2, n[3]);
    p0 = bfi(0x55555555u, A, C << 1); p1 = bfi(0x55555555u, A >> 1, C); p2 = bfi(0x55555555u, B, D << 1); p3 = bfi(0x55555555u, B >> 1, D);
}
__device__ __forceinline__ void from_planes(u32* n, u32 p0, u32 p1, u32 p2, u32 p3) {
    const u32 A = bfi(0x55555555u, p0, p1 << 1), C = bfi(0x55555555u, p0 >> 1, p1);
    const u32 B = bfi(0x55555555u, p2, p3 << 1), D = bfi(0x55555555u, p2 >> 1, p3);
    n[0] = bfi(0x33333333u, A, B << 2); n[2] = bfi(0x33333333u, A >> 2, B); n[1] = bfi(0x33333333u, C, D << 2); n[3] = bfi(0x33333333u, C >> 2, D);
}
constexpr u32 MAXSDW = 384;
__device__ __forceinline__ void tail_out(char* dst, LAS unsigned char* enc, u32 n16, int lane) {
    CFENCE;
    const u32x4 a = *(volatile LAS u32x4*)(enc + lane * 16), b = *(volatile LAS u32x4*)(enc + 1024 + lane * 16);
    if ((u32)lane < n16) st16_sys<0>(dst + lane * 16, a);
    if ((u32)lane + 64u < n16) st16_sys<1024>(dst + lane * 16, b);
}
constexpr u32 H_RAW = 0x80000000u;

struct PanelId { int r, dest, pi; bool isB; };
__device__ __forceinline__ PanelId panel_of(int r, int i, int rank) {
    PanelId d; d.r = r; d.isB = i < 56 || i >= 72;
    d.dest = i < 56 ? ((rank + 1 + (i >> 3)) & 7) : rank;
    d.pi = d.isB ? (i & 7) : i - 56;
    return d;
}
__device__ __forceinline__ bool is_fill(const PanelId& d, int unit) { return d.isB && d.pi == 7 && unit >= 4 - FC; }
__device__ __forceinline__ void unit_load(const Params& p, const PanelId& d, int unit, int lane, f32x4 (&f)[16]) {
    const int fr = lane & 15, fq = lane >> 4;
    const float* b = d.isB ? p.w + (size_t)(d.r * 64 + 4 * fq) * GN + d.dest * NS + d.pi * 256 + unit * 64 + 4 * fr
                           : p.x + (size_t)(d.pi * 256 + unit * 64 + (lane >> 2)) * KS + d.r * 64 + 4 * ((lane & 3) ^ (fq & 2));
    const int s1 = d.isB ? 16 * GN : 16 * KS, s2 = d.isB ? GN : 16;
    const int nt = (int)rfl(d.isB ? 1u : 0u);
#pragma unroll
    for (int j = 0; j < 4; ++j) ld16x4(f[4 * j], f[4 * j + 1], f[4 * j + 2], f[4 * j + 3], b + j * s1, b + j * s1 + s2, b + j * s1 + 2 * s2, b + j * s1 + 3 * s2, nt);
}
#define PIN_F(f) do { asm volatile("" : "+v"(f[0]), "+v"(f[1]), "+v"(f[2]), "+v"(f[3]), "+v"(f[4]), "+v"(f[5]), "+v"(f[6]), "+v"(f[7]) :: "memory"); \
    asm volatile("" : "+v"(f[8]), "+v"(f[9]), "+v"(f[10]), "+v"(f[11]), "+v"(f[12]), "+v"(f[13]), "+v"(f[14]), "+v"(f[15]) :: "memory"); } while (0)
__device__ __forceinline__ void st_unit(char* dp, const u32 (&u)[32]) {
    st16_sys<0>(dp, (u32x4){u[0], u[1], u[2], u[3]}); st16_sys<1024>(dp, (u32x4){u[4], u[5], u[6], u[7]});
    st16_sys<2048>(dp, (u32x4){u[8], u[9], u[10], u[11]}); st16_sys<3072>(dp, (u32x4){u[12], u[13], u[14], u[15]});
    dp += 4096;
    st16_sys<0>(dp, (u32x4){u[16], u[17], u[18], u[19]}); st16_sys<1024>(dp, (u32x4){u[20], u[21], u[22], u[23]});
    st16_sys<2048>(dp, (u32x4){u[24], u[25], u[26], u[27]}); st16_sys<3072>(dp, (u32x4){u[28], u[29], u[30], u[31]});
}
__device__ __forceinline__ size_t local_off(const PanelId& d, int unit, int rank, int kt, size_t& cnt) {
    if (d.dest == rank) { cnt = (d.isB ? W_BCNT : W_ACNT) + (size_t)(d.pi * 64 + (kt & 7) * 8 + (kt >> 3)) * 4; return (d.isB ? W_B : W_A) + (size_t)(d.pi * 64 + kt) * PANEL + unit * USLOT; }
    const int g = ((d.dest - rank - 1) & 7) * FC + unit - (4 - FC);
    cnt = W_XCNT + (size_t)((g >> 2) * 64 + (kt & 7) * 8 + (kt >> 3)) * 4;
    return W_X + (size_t)((g >> 2) * 64 + kt) * PANEL + (g & 3) * USLOT;
}
__device__ __forceinline__ size_t export_off(const PanelId& d, int unit) {
    return (d.isB ? W_PW + (size_t)((d.dest * 8 + d.r) * 8 + d.pi) * PANEL : W_PX + (size_t)(d.r * 16 + d.pi) * PANEL) + unit * USLOT;
}
__device__ __forceinline__ u32 unit_pack(const Params& p, LAS unsigned char* enc, const PanelId& d, int unit, int lane, const f32x4 (&f)[16], int mode) {
    u32 u[32];
    if (d.isB) {
#pragma unroll
        for (int h = 0; h < 2; ++h)
#pragma unroll
            for (int c = 0; c < 4; ++c)
#pragma unroll
                for (int t = 0; t < 4; ++t) u[16 * h + 4 * c + t] = cvt_pk_bf16(f[8 * h + 2 * t][c], f[8 * h + 2 * t + 1][c]);
    } else {
#pragma unroll
        for (int c = 0; c < 16; ++c) { u[2 * c] = cvt_pk_bf16(f[c][0], f[c][1]); u[2 * c + 1] = cvt_pk_bf16(f[c][2], f[c][3]); }
    }
    const bool own = d.dest == p.rank, fill = is_fill(d, unit);
    u32 hdr = 0;
    if (mode & 4) {
        st_unit(p.ws + export_off(d, unit) + lane * 16, u);
        hdr = H_RAW | 512u;
    } else if ((mode & 1) && (!(d.isB && own) || fill)) {
        char* slot = p.ws + export_off(d, unit);
        *(volatile LAS u32x4*)(enc + lane * 16) = (u32x4){0u, 0u, 0u, 0u};
        *(volatile LAS u32x4*)(enc + 1024 + lane * 16) = (u32x4){0u, 0u, 0u, 0u};
        u32 hi[16], t[16], sum = 0;
#pragma unroll
        for (int k = 0; k < 16; ++k) {
            hi[k] = __builtin_amdgcn_perm(u[2 * k + 1], u[2 * k], 0x07050301u);
            t[k] = hi[k] | 0x80808080u;
            sum = __builtin_amdgcn_sad_u8(t[k], 0u, sum);
        }
        u32 a = ((rdl(scan_add(sum), 63) + 4095u) >> 12) - 128u;
        a = min(max(a, 6u), 126u);
        const u32 b4 = (a - 6u) * 0x01010101u;
        u32 bad = 0, n[8];
#pragma unroll
        for (int k = 0; k < 16; ++k) { t[k] -= b4; bad |= (t[k] & 0xf8f8f8f8u) ^ 0x80808080u; }
#pragma unroll
        for (int k = 0; k < 8; ++k)
            n[k] = __builtin_amdgcn_perm(0x06000204u, 0x080a0c0eu, t[2 * k] & 0x07070707u) | __builtin_amdgcn_perm(0x60002040u, 0x80a0c0e0u, t[2 * k + 1] & 0x07070707u)
                 | ((hi[2 * k] >> 7) & 0x01010101u) | ((hi[2 * k + 1] >> 3) & 0x10101010u);
        u32 sg[2], c0[2], c1[2], p1[2], p2[2], p3[2], p4[2];
#pragma unroll
        for (int h = 0; h < 2; ++h) {
            to_planes(n + 4 * h, sg[h], c0[h], c1[h], p4[h]);
            p2[h] = c1[h] | p4[h]; p1[h] = p2[h] | c0[h]; p3[h] = p4[h] | (c1[h] & c0[h]);
        }
        const u32 ca = (popc(p1[0]) + popc(p1[1])) | ((popc(p2[0]) + popc(p2[1])) << 16), cb = (popc(p3[0]) + popc(p3[1])) | ((popc(p4[0]) + popc(p4[1])) << 17);
        const u32 ia = scan_add(ca), ib = scan_add(cb), ta = rdl(ia, 63), tb = rdl(ib, 63);
        const u32 b3 = ta & 0xffffu, b4s = b3 + (ta >> 16), b5 = b4s + (tb & 0xffffu);
        const u32 sdw = (b5 + (tb >> 16) + 31u) >> 5;
        if (sdw > MAXSDW || __any(max(popc(p4[0]), popc(p4[1])) > 16u)) hdr = H_RAW | 512u;
        u32 nexc = 0;
        u64 anyb = __ballot(bad != 0);
        if (anyb && !hdr) {
            u32 bm[2] = {0u, 0u};
#pragma unroll
            for (int k = 0; k < 16; ++k) {
                const u32 y = (t[k] & 0xf8f8f8f8u) ^ 0x80808080u;
                bm[k >> 3] |= ((((y & 0x78787878u) + 0x78787878u) | y) & 0x80808080u) >> (7 - (k & 7));
            }
            if (rdl(scan_add((u32)(__popc(bm[0]) + __popc(bm[1]))), 63) > 64u) {
                hdr = H_RAW | 512u;
            } else {
                while (anyb) {
                    const int l = __builtin_ctzll(anyb);
                    const u32 w0 = rdl(bm[0], l), w1 = rdl(bm[1], l);
                    const u32 pos = (u32)__builtin_ctz(w0 ? w0 : w1), k = (w0 ? 0u : 8u) + (pos & 7u);
                    u32 hv = 0;
#pragma unroll
                    for (int kk = 0; kk < 16; ++kk) hv = k == (u32)kk ? hi[kk] : hv;
                    const u32 e = (rdl(hv, l) >> (pos & 24u)) & 0x7fu;
                    if (lane == 0) ((volatile LAS u32*)enc)[sdw + nexc] = ((u32)l << 13) | (k << 9) | ((pos >> 3) << 7) | e;
                    ++nexc;
                    if (lane == l) { if (w0) bm[0] &= bm[0] - 1u; else bm[1] &= bm[1] - 1u; }
                    anyb = __ballot((bm[0] | bm[1]) != 0);
                }
            }
        }
        char* dp = slot + lane * 16;
        if (hdr) {
            st_unit(dp, u);
        } else {
            u32 lo[16];
#pragma unroll
            for (int k = 0; k < 16; ++k) lo[k] = __builtin_amdgcn_perm(u[2 * k + 1], u[2 * k], 0x06040200u);
            st16_sys<0>(dp, (u32x4){lo[0], lo[1], lo[2], lo[3]}); st16_sys<1024>(dp, (u32x4){lo[4], lo[5], lo[6], lo[7]});
            st16_sys<2048>(dp, (u32x4){lo[8], lo[9], lo[10], lo[11]}); st16_sys<3072>(dp, (u32x4){lo[12], lo[13], lo[14], lo[15]});
            st16_sys<0>(dp + 4096, (u32x4){sg[0], sg[1], p1[0], p1[1]});
            u32 o2 = (ia - ca) & 0xffffu, o3 = b3 + ((ia - ca) >> 16), o4 = b4s + ((ib - cb) & 0xffffu), o5 = b5 + ((ib - cb) >> 16);
#pragma unroll
            for (int h = 0; h < 2; ++h) {
                bit_put((LAS u32*)enc, o2, press(p2[h], msk<false>(p1[h]))); o2 += popc(p1[h]);
                bit_put((LAS u32*)enc, o3, press(p3[h], msk<false>(p2[h]))); o3 += popc(p2[h]);
                bit_put((LAS u32*)enc, o4, press(p4[h], msk<false>(p3[h]))); o4 += popc(p3[h]);
                const Msk k = msk<false>(p4[h]);
                bit_put((LAS u32*)enc, o5, press(c0[h] & p4[h], k) | (press(c1[h] & p4[h], k) << popc(p4[h]))); o5 += 2u * popc(p4[h]);
            }
            const u32 n16 = max((sdw + nexc + 3u) >> 2, 1u);
            tail_out(slot + 5120, enc, n16, lane);
            hdr = (320u + n16) | (a << 10) | (nexc << 17);
        }
    }
    if (own ? (mode & 2) : fill) {
        size_t cnt;
        st_unit(p.ws + local_off(d, unit, p.rank, p.rank * 8 + d.r, cnt) + lane * 16, u);
    }
    return hdr;
}
__device__ __forceinline__ void unit_announce(const Params& p, LAS unsigned char* lds, const PanelId& d, int unit, int pr, u32 hdr, u32 call, int mode) {
    const bool fill = is_fill(d, unit);
    if (pr == p.rank) {
        if (d.dest == p.rank ? (mode & 2) : fill) { size_t cnt; local_off(d, unit, p.rank, p.rank * 8 + d.r, cnt); add4_sys(p.ws + cnt, 1u); }
    } else if (!(mode & 1)) {
    } else if (!d.isB || pr == d.dest) {
        st8_sys(peer_ws(lds, (u32)pr) + W_HI + (size_t)(((p.rank * 8 + d.r) * 24 + (d.isB ? d.pi : 8 + d.pi)) * 4 + unit) * 8, (u32x2){hdr, call});
    } else if (fill) {
        st8_sys(peer_ws(lds, (u32)pr) + W_HX + (size_t)(((p.rank * 8 + d.r) * 8 + d.dest) * 4 + unit) * 8, (u32x2){hdr, call});
    }
}

constexpr u32 T_IN = 0, T_OUT = 1, T_END = 2, T_DEAD = 3, T_SKIP = 4;
constexpr u32 NP_IN = 384, NP_X = 28 * FC, NP_ROW = 32 - FC, NP_OUT = 16 * NP_ROW, NP_ALL = NP_IN + NP_X + NP_OUT;
#define R_POSTED 0
#define R_TAKEN 1
#define R_FULL 2
#define R_DONE 4
__device__ __forceinline__ volatile LAS u32* ring_words(LAS unsigned char* lds) { return (volatile LAS u32*)(lds + LDS_MISC + 256); }
__device__ __forceinline__ volatile LAS u32x4* ring_desc(LAS unsigned char* lds, u32 n) { return (volatile LAS u32x4*)(lds + LDS_MISC + 512 + (n % (u32)NSLOT) * 32u); }

struct Pair { u32 type, so, ho, cnt, sstep, hstep, cstep, dstep; char* dst; };
__device__ __forceinline__ Pair pair_of(const Params& p, u32 z, int j, int peer, int rot) {
    Pair q; q.type = T_END; q.so = 0; q.ho = (u32)W_TMO; q.cnt = 0; q.sstep = USLOT; q.hstep = 8; q.cstep = 0; q.dstep = USLOT; q.dst = nullptr;
    size_t cnt;
    if (z < NP_IN) {
        PanelId d;
        const int P = (int)(z >> 1), r = P / 24, i = P - r * 24, unit = (int)(z & 1u) * 2;
        d.r = r; d.isB = i < 8; d.dest = p.rank; d.pi = d.isB ? i : ((i - 8 + rot) & 15);
        q.type = T_IN; q.so = (u32)export_off(d, unit);
        q.ho = (u32)W_HI + (u32)(((peer * 8 + r) * 24 + (d.isB ? d.pi : 8 + d.pi)) * 4 + unit) * 8u;
        q.dst = p.ws + local_off(d, unit, p.rank, peer * 8 + r, cnt);
        q.cnt = (u32)cnt;
    } else if (z < NP_IN + NP_X) {
        const int y = (int)(z - NP_IN), rp = y / (7 * FC), yy = y - rp * (7 * FC), jj = yy / FC, k = yy - jj * FC;
        const bool across = (FC & 1) && k == FC - 1;
        PanelId d; d.r = 2 * rp + (across ? 0 : (k & 1)); d.isB = true; d.dest = (p.rank + 1 + jj) & 7; d.pi = 7;
        const int unit = across ? 4 - FC : 2 - (k & 2);
        PanelId e = d; e.r = d.r + (across ? 1 : 0);
        const int unit1 = across ? unit : unit + 1;
        size_t cnt1;
        const u32 l0 = (u32)local_off(d, unit, p.rank, peer * 8 + d.r, cnt), l1 = (u32)local_off(e, unit1, p.rank, peer * 8 + e.r, cnt1);
        q.type = T_IN; q.so = (u32)export_off(d, unit); q.sstep = (u32)export_off(e, unit1) - q.so;
        q.ho = (u32)W_HX + (u32)(((peer * 8 + d.r) * 8 + d.dest) * 4 + unit) * 8u; q.hstep = across ? 256u : 8u;
        q.dst = p.ws + l0; q.dstep = l1 - l0;
        q.cnt = (u32)cnt; q.cstep = (u32)cnt1 - (u32)cnt;
    } else if (z < NP_ALL) {
        int y = (int)(z - NP_IN - NP_X), pm, pn, unit;
        bool across = false;
        if (y < 112) { pn = y >> 4; pm = y & 15; unit = 0; }
        else {
            y -= 112; pm = y / (int)(NP_ROW - 7); const int i = y - pm * (int)(NP_ROW - 7);
            pn = i < 21 ? i / 3 : 7; unit = i < 21 ? 2 + (i - pn * 3) * 2 : (i - 21) * 4;
            across = FC == 3 && i >= 21;
        }
        q.type = T_OUT; q.so = (u32)W_PO + (u32)((pm * 8 + pn) * 8 + unit) * (u32)USLOT;
        q.sstep = across ? 4u * USLOT : (u32)USLOT; q.hstep = across ? 32u : 8u; q.dstep = across ? 128u * GN : 64u;
        q.ho = (u32)W_HO + (u32)((peer * 128 + pm * 8 + pn) * 8 + unit) * 8u;
        q.dst = (char*)p.out + (size_t)(pm * 256 + (unit >> 2) * 128) * GN + peer * NS + pn * 256 + (unit & 3) * 64;
    }
    return q;
}
__device__ __forceinline__ void pull_schedule(const Params& p, LAS unsigned char* lds, u32 call, int j, int c, int wid, int lane) {
    __builtin_amdgcn_s_setprio(2);
    const int peer = (p.rank + 1 + j) & 7, rot = rot_of(j);
    const u32 sub = (u32)lane & 1u;
    const u32 subm = 0u - sub;
    const char* pws = peer_ws(lds, (u32)peer);
    volatile LAS u32* rw = ring_words(lds);
    const u32 qbase = (call - 1u) * NP_ALL - 2u * (u32)NPW;
    char* qp = p.ws + W_Q + j * 128;
    u32 tk;
    u32 z = (u32)c, zn = (u32)(NPW + c), n = 0, slot = 0, dec = 0, lap = 0;
    Pair cur = pair_of(p, z, j, peer, rot);
    u32x2 h = (u32x2){0u, 0u};
    bool dead = false;
    u32 ends = 0;
    for (;;) {
        const bool more = z < NP_ALL;
        const Pair nxt = pair_of(p, zn, j, peer, rot);
        const char* hnp = p.ws + (nxt.ho + (nxt.hstep & subm));
        const u64 src = (u64)(pws + (cur.so + (cur.sstep & subm))), d64 = (u64)(cur.dst + (cur.dstep & subm));
        asm volatile("" :: "v"(hnp), "v"(src), "v"(d64));
        if (more && !dead && !__all(h.y == call)) {
            const u64 t0 = now();
            for (;;) {
                u32 t; poll8(h, t, p.ws + (cur.ho + (cur.hstep & subm)), p.ws + W_TMO);
                if (__all(h.y == call)) { dead = rfl(t) == call; break; }
                if (spin_fail(p.ws, call, t0, t)) { dead = true; break; }
            }
        }
        u32x2 hn;
        ld8_sys(hn, hnp);
        tk = 0xffffffffu;
        if (more) ticket_issue(tk, qp);
        for (;;) {
            const u32 tkn = rw[R_TAKEN], dn = rw[R_DONE + dec + sub];
            if (__all(n < tkn + (u32)RUNAHEAD && dn + (u32)(NSLOT / ND) > lap)) break;
            __builtin_amdgcn_s_sleep(1);
        }
        if (lane < 2) {
            const u32 type = !dead || !more ? cur.type : (cur.type == T_OUT ? T_DEAD : T_SKIP);
            volatile LAS u32x4* ds = (volatile LAS u32x4*)(lds + LDS_MISC + 512 + (slot + sub) * 32u);
            ds[0] = (u32x4){(u32)src, (u32)(src >> 32), h.x | (STAMPS ? (z < NP_IN / 2 ? 0u : z < NP_IN ? 1u : z < NP_IN + NP_X ? 2u : z < NP_IN + NP_X + NP_OUT / 2 ? 3u : 4u) << 26 : 0u), type};
            ds[1] = (u32x4){(u32)d64, (u32)(d64 >> 32), cur.cnt + (cur.cstep & subm), 0u};
        }
        n += 2;
        rw[R_POSTED] = n;
        slot = slot == (u32)NSLOT - 2u ? 0u : slot + 2u;
        if (dec == (u32)ND - 2u) { dec = 0; ++lap; } else dec += 2u;
        asm volatile("s_waitcnt vmcnt(0)" : "+v"(tk), "+v"(hn) :: "memory");
        if (more) { z = zn; zn = rfl(tk) - qbase; cur = nxt; h = hn; }
        else if (++ends == (u32)ND / 2u) break;
    }
}

#define GLDS(src, off) "global_load_lds_dwordx4 " src ", off offset:" #off " sc0 sc1\n\ts_waitcnt vmcnt(%[d])\n\t"
template <int D> __device__ __forceinline__ void glds_part(const void* src, u32 dst, u32 rem) {
    const u64 mask = rem >= 64u ? ~0ull : (1ull << rem) - 1ull;
    u32 keep;
    asm volatile("s_mov_b32 %0, m0\n\ts_mov_b32 m0, %2\n\ts_mov_b64 exec, %3\n\t" GLDS("%1", 0) "s_mov_b64 exec, -1\n\ts_mov_b32 m0, %0"
                 : "=&s"(keep) : "v"(src), "s"(dst), "s"(mask), [d] "i"(D - 1) : "memory");
}
template <int D> __device__ __forceinline__ void load_unit(LAS unsigned char* lds, const char* src, u32 dst, u32 n16, u32 n, u32 nslot, u32& full, u32& posted, u32x4& ds) {
    volatile LAS u32* rw = ring_words(lds);
    u32 keep;
    asm volatile("s_mov_b32 %0, m0\n\ts_mov_b32 m0, %2\n\ts_nop 0\n\t" GLDS("%1", 0) GLDS("%1", 1024) "s_mov_b32 m0, %0"
                 : "=&s"(keep) : "v"(src), "s"(dst), [d] "i"(D - 1) : "memory");
    if (D <= 9 && full + 1u < n) { full = n - 1u; rw[R_FULL] = full; }
    asm volatile("s_mov_b32 %0, m0\n\ts_mov_b32 m0, %2\n\ts_nop 0\n\t" GLDS("%1", 2048) GLDS("%1", 3072) "s_mov_b32 m0, %0"
                 : "=&s"(keep) : "v"(src), "s"(dst), [d] "i"(D - 1) : "memory");
    if (D > 9 && full + 1u < n) { full = n - 1u; rw[R_FULL] = full; }
    if (D <= 5) { full = n; rw[R_FULL] = n; }
    posted = rw[R_POSTED];
    ds = *(volatile LAS u32x4*)(lds + LDS_MISC + 512 + nslot * 32u);
    src += 4096;
    glds_part<D>(src, dst + 4096u, 64u);
    if (D == 6) { full = n; rw[R_FULL] = n; }
    glds_part<D>(src + 1024, dst + 5120u, n16 - 320u);
    if (D == 7) { full = n; rw[R_FULL] = n; }
    if (n16 > 384u) {
        glds_part<D>(src + 2048, dst + 6144u, n16 - 384u);
        if (D == 8) { full = n; rw[R_FULL] = n; }
        if (n16 > 448u) glds_part<D>(src + 3072, dst + 7168u, n16 - 448u);
    }
    rw[R_TAKEN] = n + 1u;
}
__device__ __forceinline__ void pull_load(const Params& p, LAS unsigned char* lds, int wid, int lane) {
    __builtin_amdgcn_s_setprio(3);
    volatile LAS u32* rw = ring_words(lds);
    const u32 lbase = (u32)(u64)lds;
    u32 ends = 0, slot = 0, full = 0;
    u32 posted = 0;
    u32x4 ds = (u32x4){0u, 0u, 0u, 0u};
    for (u32 n = 0;; ++n) {
        volatile LAS u32x4* dsp = (volatile LAS u32x4*)(lds + LDS_MISC + 512 + slot * 32u);
        if (rfl(posted) <= n) {
            posted = rw[R_POSTED];
            if (rfl(posted) <= n) {
                WAIT_V(0);
                full = n; rw[R_FULL] = n;
                while (rfl(rw[R_POSTED]) <= n) __builtin_amdgcn_s_sleep(1);
            }
            ds = dsp[0];
        }
        const u32 type = rfl(ds.w);
        const u32 dst = lbase + slot * (u32)USLOT;
        slot = slot == (u32)NSLOT - 1u ? 0u : slot + 1u;
        if (type <= T_OUT) {
            const u32 n16 = rfl(min(max(ds.z & 1023u, 321u), 512u));
            const char* src = (const char*)(((u64)ds.y << 32) | ds.x) + lane * 16;
            if (type == T_IN) load_unit<DEPTH>(lds, src, dst, n16, n, slot, full, posted, ds);
            else load_unit<DEPTH2>(lds, src, dst, n16, n, slot, full, posted, ds);
        } else {
            WAIT_V(0);
            full = n + 1u; rw[R_FULL] = full; rw[R_TAKEN] = full;
            posted = 0;
            if (type == T_END && ++ends == (u32)ND) break;
        }
    }
}

__device__ __forceinline__ void pull_decode(const Params& p, LAS unsigned char* lds, int wid, int lane) {
    volatile LAS u32* rw = ring_words(lds);
    LAS u32* bm = (LAS u32*)(lds + LDS_BM + wid * 1024);
    u32 done = 0;
    for (u32 n = (u32)wid;; n += (u32)ND) {
        while (rfl(rw[R_FULL]) <= n) __builtin_amdgcn_s_sleep(1);
        CFENCE;
        const u32x4 d0 = ring_desc(lds, n)[0], d1 = ring_desc(lds, n)[1];
        const u32 type = rfl(d0.w), hdr = rfl(d0.z);
        if (type == T_END) break;
        char* dst = (char*)(((u64)d1.y << 32) | d1.x);
        LAS unsigned char* sl = lds + (n % (u32)NSLOT) * (u32)USLOT;
        u32 u[32];
        if (type <= T_OUT && (hdr & H_RAW)) {
#pragma unroll
            for (int k = 0; k < 8; ++k) { const u32x4 v = *(const LAS u32x4*)(sl + k * 1024 + lane * 16); u[4 * k] = v.x; u[4 * k + 1] = v.y; u[4 * k + 2] = v.z; u[4 * k + 3] = v.w; }
        } else if (type == T_IN) {
            const u32x4 dn = *(const LAS u32x4*)(sl + 4096 + lane * 16);
            u32x4 lo[4];
#pragma unroll
            for (int k = 0; k < 4; ++k) lo[k] = *(const LAS u32x4*)(sl + k * 1024 + lane * 16);
            const LAS u32* st = (const LAS u32*)(sl + 5120);
            u32 pl[4][2], c0[2], c1[2], base = 0;
            pl[0][0] = dn.z; pl[0][1] = dn.w;
#pragma unroll
            for (int lv = 0; lv < 4; ++lv) {
                const u32 n0 = popc(pl[lv][0]), n1 = popc(pl[lv][1]), cn = lv < 3 ? n0 + n1 : 2u * (n0 + n1);
                const u32 inc = scan_add(cn), o = base + inc - cn;
                const u32 g0 = bit_get(st, o), g1 = bit_get(st, o + (lv < 3 ? n0 : 2u * n0));
                base += rdl(inc, 63);
                const Msk k0 = msk<true>(pl[lv][0]), k1 = msk<true>(pl[lv][1]);
                if (lv < 3) { pl[lv + 1][0] = spread(g0, k0); pl[lv + 1][1] = spread(g1, k1); }
                else { c0[0] = spread(g0, k0); c1[0] = spread(g0 >> (n0 & 31u), k0); c0[1] = spread(g1, k1); c1[1] = spread(g1 >> (n1 & 31u), k1); }
            }
            const u32 b4 = (((hdr >> 10) & 0x7fu) - 6u) * 0x01010101u;
            const u32 tl = b4 + 0x07040506u, th = b4 + 0x00010203u;
            u32 hi[16];
#pragma unroll
            for (int h = 0; h < 2; ++h) {
                u32 nn[4];
                from_planes(nn, dn[h], c0[h] | (pl[0][h] & ~pl[1][h]) | (pl[2][h] & ~pl[3][h]), c1[h] | (pl[1][h] & ~pl[3][h]), pl[3][h]);
#pragma unroll
                for (int k = 0; k < 4; ++k) {
                    hi[8 * h + 2 * k] = bfi(0x80808080u, nn[k] << 7, __builtin_amdgcn_perm(th, tl, (nn[k] >> 1) & 0x07070707u));
                    hi[8 * h + 2 * k + 1] = bfi(0x80808080u, nn[k] << 3, __builtin_amdgcn_perm(th, tl, (nn[k] >> 5) & 0x07070707u));
                }
            }
            const u32 nexc = (hdr >> 17) & 0xffu;
            if (nexc) {
                const u32 ev = st[min((base + 31u) >> 5, MAXSDW) + (u32)lane];
                for (u32 e = 0; e < nexc; ++e) {
                    const u32 en = rdl(ev, (int)e);
                    const u32 sh = ((en >> 7) & 3u) * 8u, kk = (en >> 9) & 15u;
                    const bool me = (en >> 13) == (u32)lane;
#pragma unroll
                    for (int k = 0; k < 16; ++k)
                        if (me && kk == (u32)k) hi[k] = bfi(0x7fu << sh, (en & 0x7fu) << sh, hi[k]);
                }
            }
#pragma unroll
            for (int k = 0; k < 16; ++k) {
                u[2 * k] = __builtin_amdgcn_perm(hi[k], lo[k >> 2][k & 3], 0x05010400u);
                u[2 * k + 1] = __builtin_amdgcn_perm(hi[k], lo[k >> 2][k & 3], 0x07030602u);
            }
        } else if (type == T_OUT) {
            *(volatile LAS u32x4*)(bm + lane * 4) = (u32x4){0u, 0u, 0u, 0u};
            const u32x4 d0v = *(const LAS u32x4*)(sl + 4096 + lane * 16), d1v = *(const LAS u32x4*)(sl + 5120 + lane * 16);
            u32x4 lo[4];
#pragma unroll
            for (int k = 0; k < 4; ++k) lo[k] = *(const LAS u32x4*)(sl + k * 1024 + lane * 16);
            const LAS u32* st = (const LAS u32*)(sl + 6144);
            u32 f5[4], g[4];
            const u32x4 e3 = d0v & d1v;
            u32 cn = popc(e3.x) + popc(e3.y) + popc(e3.z) + popc(e3.w), inc = scan_add(cn), o = inc - cn;
#pragma unroll
            for (int h = 0; h < 4; ++h) { g[h] = bit_get(st, o); o += popc(e3[h]); }
            u32 base = rdl(inc, 63);
#pragma unroll
            for (int h = 0; h < 4; ++h) f5[h] = spread(g[h], msk<true>(e3[h]));
            cn = 2u * (popc(f5[0]) + popc(f5[1]) + popc(f5[2]) + popc(f5[3])); inc = scan_add(cn); o = base + inc - cn;
#pragma unroll
            for (int h = 0; h < 4; ++h) { g[h] = bit_get(st, o); o += 2u * popc(f5[h]); }
            base += rdl(inc, 63);
            const u32 nexc = (hdr >> 17) & 0xffu;
            const u32 ev = st[min((base + 31u) >> 5, MAXSDW) + (u32)lane];
            if (2u * (u32)lane < nexc) __hip_atomic_fetch_or(bm + ((ev >> 5) & 255u), 1u << (ev & 31u), __ATOMIC_RELAXED, __HIP_MEMORY_SCOPE_WORKGROUP);
            if (2u * (u32)lane + 1u < nexc) __hip_atomic_fetch_or(bm + ((ev >> 21) & 255u), 1u << ((ev >> 16) & 31u), __ATOMIC_RELAXED, __HIP_MEMORY_SCOPE_WORKGROUP);
            CFENCE;
            const u32x4 m7 = *(volatile LAS u32x4*)(bm + lane * 4);
#pragma unroll
            for (int h = 0; h < 4; ++h) {
                const Msk k = msk<true>(f5[h]);
                const u32 f3 = spread(g[h], k) | (d0v[h] & ~f5[h]), f4 = spread(g[h] >> (popc(f5[h]) & 31u), k) | (d1v[h] & ~f5[h]);
                u32 nn[4];
                from_planes(nn, f4 ^ f3, f5[h] ^ f3, f3, m7[h] ^ f3);
#pragma unroll
                for (int k2 = 0; k2 < 4; ++k2) {
                    const u32 L = lo[h][k2], t = (L >> 3) & 0x11111111u, H = nn[k2] ^ times15(t);
                    u[8 * h + 2 * k2] = bfi(0x0f0f0f0fu, L, H << 4); u[8 * h + 2 * k2 + 1] = bfi(0x0f0f0f0fu, L >> 4, H);
                }
            }
        } else {
#pragma unroll
            for (int k = 0; k < 32; ++k) u[k] = 0x80808080u;
        }
        WAIT_L(0);
        ++done;
        if (lane == 0) rw[R_DONE + wid] = done;
        if (type == T_IN) {
            st_unit(dst + lane * 16, u);
            WAIT_V(0);
            if (lane == 0) add4_sys(p.ws + d1.z, 1u);
        } else if (type != T_SKIP) {
            char* o = dst + (size_t)(lane & 15) * GN + (lane >> 4) * 16;
#pragma unroll
            for (int k = 0; k < 8; ++k)
                st16_sys<0>(o + (size_t)k * 16 * GN, (u32x4){u[4 * k], u[4 * k + 1], u[4 * k + 2], u[4 * k + 3]});
        }
    }
}

__device__ __forceinline__ void gemm_tile(const Params& p, LAS unsigned char* lds, u32 call, bool fillm, int pm, int pn, int wid, int lane) {
    const int wr = wid >> 2, wc = wid & 3;
    int j0 = 0;
#pragma unroll
    for (int j = 1; j < 7; ++j) if (rot_of(j) <= pm) j0 = j;
    const char* Ab = p.ws + W_A + (size_t)pm * 64 * PANEL;
    const char* Bb = p.ws + (fillm ? W_X : W_B) + (size_t)pn * 64 * PANEL;
    const char* cA = p.ws + W_ACNT + (size_t)pm * 256;
    const char* cB = p.ws + (fillm ? W_XCNT : W_BCNT) + (size_t)pn * 256;
    const u32 voff = (u32)(wid * 4096 + lane * 16);
    const u32 ldsw = (u32)wid * 2048u;
    const int aoff = wr * 8192 + ((lane & 15) * 4 + ((lane >> 4) ^ ((lane >> 2) & 2))) * 16;
    int boff = 65536 + wc * 4096 + lane * 16;
    asm volatile("" : "+v"(boff));
    int ktv = kt_of(lane, p.rank, j0);
    asm volatile("" : "+v"(ktv));
    const u32 cix = (u32)((ktv & 7) * 8 + (ktv >> 3)) * 4u;
    u64 rdyA = wid == 4 ? 0ull : ~0ull, rdyB = rdyA;
    const u32 needA = call * 4u, needB = (fillm && pn == NVP - 1) ? call * (u32)(7 * FC - 4 * (NVP - 1)) : needA;
#define SA(b, h) (((b) * 2 + (h)) * 16384)
#define SB(b, h) ((4 + (b) * 2 + (h)) * 16384)
#define STAGE(buf, gbase, h) do { \
        __builtin_amdgcn_global_load_lds((const u32*)((gbase) + voff + (h) * 2048), (LAS u32*)(lds + (buf) + ldsw), 16, 0, 17); \
        __builtin_amdgcn_global_load_lds((const u32*)((gbase) + voff + (h) * 2048), (LAS u32*)(lds + (buf) + ldsw), 16, 1024, 17); } while (0)
#define LDA(dst, b, h) do { _Pragma("unroll") for (int m = 0; m < 4; ++m) _Pragma("unroll") for (int k = 0; k < 2; ++k) \
        dst[m][k] = *(const LAS bf16x8*)(lds + SA(b, h) + aoff + m * 2048 + k * 1024); } while (0)
#define LDB(dst, b, h) do { _Pragma("unroll") for (int n = 0; n < 2; ++n) _Pragma("unroll") for (int k = 0; k < 2; ++k) \
        dst[n][k] = *(const LAS bf16x8*)(lds + (SB(b, h) - 65536) + boff + k * 2048 + n * 1024); } while (0)
#define MMA(ai, bj, At, Bt) do { __builtin_amdgcn_s_setprio(1); \
        _Pragma("unroll") for (int m = 0; m < 4; ++m) _Pragma("unroll") for (int n = 0; n < 2; ++n) _Pragma("unroll") for (int k = 0; k < 2; ++k) \
            acc[ai][bj][m][n] = __builtin_amdgcn_mfma_f32_16x16x32_bf16(Bt[n][k], At[m][k], acc[ai][bj][m][n], 0, 0, 0); \
        __builtin_amdgcn_s_setprio(0); } while (0)
#define CHECK(RDY, CP, NEED, o) do { if (!((RDY >> (o)) & 1ull)) { \
        const u64 t0_ = now(); __builtin_amdgcn_s_waitcnt(0xC07F); \
        for (u32 it_ = 1;; ++it_) { \
            u32 a_ = 0; poll_win(a_, CP + cix, 0xffffull << (o)); \
            RDY |= __ballot(a_ >= NEED); \
            if ((RDY >> (o)) & 1ull) break; \
            if (spin_fail(p.ws, call, t0_, (it_ & 63u) ? 0u : ld4_wait(p.ws + W_TMO))) { rdyA = rdyB = ~0ull; gave_up = true; break; } \
        } if (STAMPS > 1) { wacc += now() - t0_; __builtin_amdgcn_s_waitcnt(0xC07F); } } } while (0)
#define CHECKA(o) CHECK(rdyA, cA, needA, o)
#define CHECKB(o) CHECK(rdyB, cB, needB, o)
#define PEEK(o) do { pk = (u32)((~rdyA >> (((o) + 1) & 63)) | (~rdyB >> (((o) + 2) & 63))) & 1u; u64 sv_; \
        asm volatile("s_cmp_eq_u32 %5, 0\n\ts_cbranch_scc1 .Lpk%=\n\ts_mov_b64 %2, exec\n\ts_mov_b64 exec, %6\n\t" \
                     "global_load_dword %0, %3, off sc0 sc1\n\tglobal_load_dword %1, %4, off sc0 sc1\n\ts_mov_b64 exec, %2\n.Lpk%=:" \
                     : "+v"(pa), "+v"(pb), "=&s"(sv_) : "v"(cA + cix), "v"(cB + cix), "s"(pk), "s"(0xffffull << (o)) : "memory", "scc"); } while (0)
#define WAIT_PK asm volatile("s_cmp_lg_u32 %0, 0\n\ts_cbranch_scc1 .Lpw%=\n\ts_waitcnt vmcnt(6)\n.Lpw%=:\n\ts_waitcnt vmcnt(8)" :: "s"(pk) : "memory", "scc")
#define PEEKED do { asm volatile("s_waitcnt vmcnt(6)" : "+v"(pa), "+v"(pb) :: "memory"); rdyA |= __ballot(pa >= needA); rdyB |= __ballot(pb >= needB); } while (0)
#define KOFF(o) ((size_t)__builtin_amdgcn_readlane(ktv, (o)) * PANEL)

    u64 wacc = 0;
    f32x4 acc[2][2][4][2];
#pragma unroll
    for (int a = 0; a < 2; ++a)
#pragma unroll
        for (int b = 0; b < 2; ++b)
#pragma unroll
            for (int m = 0; m < 4; ++m)
#pragma unroll
                for (int n = 0; n < 2; ++n) acc[a][b][m][n] = (f32x4){0.f, 0.f, 0.f, 0.f};
    bf16x8 At[4][2], B0[2][2], B1[2][2];
    u32 pa = 0, pb = 0, pk;
    bool gave_up = false;

    CHECKA(0); CHECKB(0); CHECKB(1); CHECKA(1);
    BAR; CFENCE;
    size_t k1 = KOFF(1);
    {
        const size_t k0 = KOFF(0);
        STAGE(SB(0, 0), Bb + k0, 0); STAGE(SB(0, 1), Bb + k0, 1); STAGE(SA(0, 0), Ab + k0, 0);
        STAGE(SB(1, 0), Bb + k1, 0); STAGE(SA(0, 1), Ab + k0, 1); STAGE(SB(1, 1), Bb + k1, 1);
        WAIT_V(6); BAR;
        if (wr == 1) BAR;
    }
    for (int t = 0; t < 64; t += 2) {
        const int o2 = (t + 2) & 63, o3 = o2 + 1;
        const size_t k2 = KOFF(o2), k3 = KOFF(o3);
        LDB(B0, 0, 0); SCHED; LDA(At, 0, 0); STAGE(SA(1, 0), Ab + k1, 0);
        CHECKB(o2);
        WAIT_L(8); BAR; WAIT_L(0); MMA(0, 0, At, B0); BAR; SCHED;
        LDB(B1, 0, 1); STAGE(SB(0, 0), Bb + k2, 0);
        PEEKED; BAR; WAIT_L(0); MMA(0, 1, At, B1); BAR;
        LDA(At, 0, 1); STAGE(SA(1, 1), Ab + k1, 1);
        BAR; WAIT_L(0); MMA(1, 0, At, B0); BAR; SCHED;
        CHECKA(o2);
        PEEK(o2); STAGE(SB(0, 1), Bb + k2, 1);
        WAIT_PK; BAR; MMA(1, 1, At, B1); BAR;
        LDB(B0, 1, 0); SCHED; LDA(At, 1, 0); STAGE(SA(0, 0), Ab + k2, 0);
        CHECKB(o3);
        WAIT_L(8); BAR; WAIT_L(0); MMA(0, 0, At, B0); BAR; SCHED;
        LDB(B1, 1, 1); STAGE(SB(1, 0), Bb + k3, 0);
        PEEKED; BAR; WAIT_L(0); MMA(0, 1, At, B1); BAR;
        LDA(At, 1, 1); STAGE(SA(0, 1), Ab + k2, 1);
        BAR; WAIT_L(0); MMA(1, 0, At, B0); BAR; SCHED;
        CHECKA(o3);
        PEEK(o3); STAGE(SB(1, 1), Bb + k3, 1);
        WAIT_PK; BAR; MMA(1, 1, At, B1); BAR;
        k1 = k3;
    }
    asm volatile("s_waitcnt vmcnt(2)" : "+v"(pa), "+v"(pb) :: "memory");
    if (wr == 0) BAR;
    if (!fillm) STAMP(2);
    int tz = threadIdx.x; asm volatile("" : "+v"(tz));
    lane = tz & 63;
    const int tile = pm * 8 + pn;
    volatile LAS u32* red = (volatile LAS u32*)(lds + LDS_MISC + 1024);
    if (!fillm) {
        float mx = 0.f;
#pragma unroll
        for (int a = 0; a < 2; ++a)
#pragma unroll
            for (int b = 0; b < 2; ++b)
#pragma unroll
                for (int m = 0; m < 4; ++m)
#pragma unroll
                    for (int n = 0; n < 2; ++n)
#pragma unroll
                        for (int r = 0; r < 4; ++r) mx = fmaxf(mx, fabsf(acc[a][b][m][n][r]));
        const u32 wm = wave_umax(__float_as_uint(mx));
        if (lane == 0) red[wid] = wm;
        WAIT_L(0); BAR; CFENCE;
        if (tz == 0) {
            const u32x4 r0 = *(volatile LAS u32x4*)red, r1 = *(volatile LAS u32x4*)(red + 4);
            const u32 t = max(max(max(r0.x, r0.y), max(r0.z, r0.w)), max(max(r1.x, r1.y), max(r1.z, r1.w)));
            st8_sys(p.ws + W_AMAX + (size_t)tile * 8, (u32x2){t, call});
        }
    }
    if (wid == 0) {
        const char* src = p.ws + W_GMAX + (blockIdx.x & (GCOPY - 1)) * 256 + (lane & 7) * 8;
        const u64 t0 = now();
        u32x2 gq;
        bool dd = false;
        for (u32 it = 1;; ++it) {
            gq = ld8_wait(src);
            if (__all(gq.y == call)) break;
            if (spin_fail(p.ws, call, t0, (it & 63u) ? 0u : ld4_wait(p.ws + W_TMO))) { dd = true; break; }
        }
        const u32 wm = wave_umax(gq.x);
        if (lane == 0) red[8] = dd ? 0xffffffffu : wm;
    } else if (wid == 4) {
        if (lane == 0) red[9] = gave_up ? 1u : 0u;
        WAIT_V(0);
    } else {
        WAIT_V(0);
    }
    WAIT_L(0); BAR; CFENCE;
    const u32 am = rfl(red[8]);
    const bool dead = am == 0xffffffffu || rfl(red[9]) != 0u;
    const float amax = __uint_as_float(am);
    if (!fillm) STAMP(3);
    if (wid < 2) __builtin_amdgcn_s_setprio(3);
    const float inv = amax > 0.f ? 127.0f / amax : 0.f;
    u32 q[32];
#pragma unroll
    for (int m = 0; m < 4; ++m)
#pragma unroll
        for (int a = 0; a < 2; ++a)
#pragma unroll
            for (int r = 0; r < 4; ++r) {
                const u32 t0 = __float_as_uint(fmaf(acc[a][0][m][0][r], inv, 12582912.f)), t1 = __float_as_uint(fmaf(acc[a][0][m][1][r], inv, 12582912.f));
                const u32 t2 = __float_as_uint(fmaf(acc[a][1][m][0][r], inv, 12582912.f)), t3 = __float_as_uint(fmaf(acc[a][1][m][1][r], inv, 12582912.f));
                q[(m * 2 + a) * 4 + r] = __builtin_amdgcn_perm(t1, t0, 0x0c0c0400u) | __builtin_amdgcn_perm(t3, t2, 0x04000c0cu);
            }
    if (dead) {
#pragma unroll
        for (int k = 0; k < 32; ++k) q[k] = 0x80808080u;
    }
    int col = p.rank * NS + pn * 256 + wc * 64;
    if (!fillm) {
        char* slot = p.ws + W_PO + (size_t)(tile * 8 + wid) * USLOT;
        LAS unsigned char* enc = lds + LDS_ENC + wid * 2048;
        *(volatile LAS u32x4*)(enc + lane * 16) = (u32x4){0u, 0u, 0u, 0u};
        *(volatile LAS u32x4*)(enc + 1024 + lane * 16) = (u32x4){0u, 0u, 0u, 0u};
        char* dp = slot + lane * 16;
        u32 hdr;
        const int yrow = pn < 7 ? pn * 3 + (wid >> 1) - 1 : 21;
        if ((wid < 2 && pn == 0 && pm < NRAW) || (pm == 15 && (wid >= 2 || pn == 7) && yrow >= (int)NP_ROW - 7 - NRAWE)) {
            st_unit(dp, q);
            hdr = H_RAW | 512u;
        } else {
        u32 L[16], G[16], f3[4], f4[4], f5[4], m7[4];
#pragma unroll
        for (int j = 0; j < 16; ++j) {
            L[j] = bfi(0x0f0f0f0fu, q[2 * j], q[2 * j + 1] << 4);
            const u32 t = (L[j] >> 3) & 0x11111111u;
            G[j] = bfi(0x0f0f0f0fu, q[2 * j] >> 4, q[2 * j + 1]) ^ times15(t);
        }
        u32 ca = 0, cb = 0, wide = 0;
#pragma unroll
        for (int h = 0; h < 4; ++h) {
            to_planes(G + 4 * h, f4[h], f5[h], f3[h], m7[h]);
            f4[h] ^= f3[h]; f5[h] ^= f3[h]; m7[h] ^= f3[h];
            ca += popc(f5[h] | (f4[h] & f3[h])) | (popc(m7[h]) << 16); cb += 2u * popc(f5[h]); wide = max(wide, popc(f5[h]));
        }
        const u32 ia = scan_add(ca), ib = scan_add(cb), ta = rdl(ia, 63), exc = ia - ca;
        const u32 sdw = ((ta & 0xffffu) + rdl(ib, 63) + 31u) >> 5, nexc = ta >> 16;
        if (nexc > 128u || sdw > MAXSDW || __any(wide > 16u)) {
            st_unit(dp, q);
            hdr = H_RAW | 512u;
        } else {
            st16_sys<0>(dp, (u32x4){L[0], L[1], L[2], L[3]}); st16_sys<1024>(dp, (u32x4){L[4], L[5], L[6], L[7]});
            st16_sys<2048>(dp, (u32x4){L[8], L[9], L[10], L[11]}); st16_sys<3072>(dp, (u32x4){L[12], L[13], L[14], L[15]});
            st16_sys<0>(dp + 4096, (u32x4){f3[0] | f5[0], f3[1] | f5[1], f3[2] | f5[2], f3[3] | f5[3]});
            st16_sys<1024>(dp + 4096, (u32x4){f4[0] | f5[0], f4[1] | f5[1], f4[2] | f5[2], f4[3] | f5[3]});
            u32 oa = exc & 0xffffu, ob = (ta & 0xffffu) + ib - cb;
#pragma unroll
            for (int h = 0; h < 4; ++h) {
                const u32 e3 = f5[h] | (f4[h] & f3[h]);
                bit_put((LAS u32*)enc, oa, press(f5[h], msk<false>(e3))); oa += popc(e3);
                const Msk k = msk<false>(f5[h]);
                bit_put((LAS u32*)enc, ob, press(f3[h] & f5[h], k) | (press(f4[h] & f5[h], k) << popc(f5[h]))); ob += 2u * popc(f5[h]);
            }
            volatile LAS unsigned short* list = (volatile LAS unsigned short*)(enc + sdw * 4u);
            u32 s = exc >> 16;
#pragma unroll
            for (int h = 0; h < 4; ++h) {
                u32 mm = m7[h];
                while (mm) { list[s] = (unsigned short)((lane * 4 + h) * 32 + __builtin_ctz(mm)); mm &= mm - 1u; ++s; }
            }
            const u32 n16 = (sdw + ((nexc + 1u) >> 1) + 3u) >> 2;
            tail_out(slot + 6144, enc, n16, lane);
            hdr = (384u + n16) | (nexc << 17);
        }
        }
        if (lane == 0) red[16 + wid] = hdr;
        WAIT_V(0);
        const bool early = pn < NEARLY;
        if (early && wid < 2 && lane < NR && lane != p.rank)
            st8_sys(peer_ws(lds, (u32)lane) + W_HO + (size_t)((p.rank * 128 + tile) * 8 + wid) * 8, (u32x2){hdr, call});
        __builtin_amdgcn_s_setprio(0);
        WAIT_L(0); BAR; CFENCE;
        if (tz < 64 && (tz >> 3) != p.rank && (!early || (tz & 7) >= 2))
            st8_sys(peer_ws(lds, (u32)(tz >> 3)) + W_HO + (size_t)((p.rank * 128 + tile) * 8 + (tz & 7)) * 8, (u32x2){red[16 + (tz & 7)], call});
        STAMP(4);
    } else {
        const int g = pn * 4 + wc;
        col = g < 7 * FC ? ((p.rank + 1 + g / FC) & 7) * NS + 7 * 256 + (4 - FC + g % FC) * 64 : -1;
    }
    if (col >= 0) {
        signed char* o = p.out + (size_t)(pm * 256 + wr * 128 + (lane & 15)) * GN + col + (lane >> 4) * 16;
#pragma unroll
        for (int k = 0; k < 8; ++k)
            st16_sys<0>(o + (size_t)k * 16 * GN, (u32x4){q[4 * k], q[4 * k + 1], q[4 * k + 2], q[4 * k + 3]});
    }
#undef SA
#undef SB
#undef STAGE
#undef LDA
#undef LDB
#undef MMA
#undef CHECK
#undef CHECKA
#undef CHECKB
#undef PEEK
#undef PEEKED
#undef WAIT_PK
#undef KOFF
}

__global__ void __launch_bounds__(BLOCK, 2) fused_gemm_q8(Params p) {
    __shared__ __attribute__((aligned(16))) unsigned char smem[LDS_BYTES];
    LAS unsigned char* lds = (LAS unsigned char*)smem;
    const int tid = threadIdx.x, wid = (int)rfl((u32)tid >> 6), lane = tid & 63;
    const int xcd = blockIdx.x & 7, idx = blockIdx.x >> 3;
    STAMP(0);
    const u64 tstart = now();
    volatile LAS u32* misc = (volatile LAS u32*)(lds + LDS_MISC);
    LAS unsigned char* enc = lds + LDS_ENC + wid * 2048;
    int i0 = -1, r0 = 0, mode0 = 2;
    const int u0 = (int)blockIdx.x & 3;
    if (idx < 16) {
        if (wid < 6) { const int n = (wid * 128 + (int)blockIdx.x) >> 2; r0 = n / 24; i0 = 56 + n - r0 * 24; }
    } else if (idx >= 24 && wid < 5) {
        const int n = (wid * 64 + (int)blockIdx.x - 192) >> 2;
        if (n < 32) { i0 = n < 28 ? (n >> 2) * 8 + (n & 3) : -1; mode0 = 5; }
        else { const int m = n - 32; i0 = m < 28 ? (m >> 2) * 8 + 4 + (m & 3) : (m < 44 ? m + 28 : (m == 44 ? 79 : -1)); mode0 = 1; }
    }
    if (i0 >= 0) {
        const PanelId d0 = panel_of(r0, i0, p.rank);
        if (mode0 != 5) { while (now() - tstart < (mode0 == 2 ? GDLY : EDLY)) __builtin_amdgcn_s_sleep(1); }
        f32x4 f[16];
        unit_load(p, d0, u0, lane, f);
        BAR;
        WAIT_V(0); PIN_F(f);
        if (STAMPS == 1) STAMP(10);
        const u32 hdr = unit_pack(p, enc, d0, u0, lane, f, mode0);
        if (STAMPS == 1) STAMP(11);
        const u32 call0 = misc[0];
        WAIT_V(0);
        if (lane < NR) unit_announce(p, lds, d0, u0, lane, hdr, call0, mode0);
    } else {
        if (wid == 7) {
            u32 tk; u32x2 pw;
            ticket_issue(tk, p.ws + W_SEQ + (size_t)blockIdx.x * 128);
            ld8_sys(pw, (const char*)p.ws_all + (lane & 7) * 8);
            asm volatile("s_waitcnt vmcnt(0)" : "+v"(tk), "+v"(pw) :: "memory");
            if (lane == 0) misc[0] = tk + 1u;
            if (lane < NR) *(volatile LAS u32x2*)(lds + LDS_MISC + 64 + lane * 8) = pw;
            if (lane < 16) ring_words(lds)[lane] = 0u;
            ((volatile LAS u32*)(lds + LDS_MISC + 1280))[lane] = 0u;
            const u32 call7 = rfl(tk) + 1u;
            if (blockIdx.x == 0) {
                if (lane == 0) st8_sys(p.ws + W_ROT, (u32x2){(u32)__builtin_amdgcn_s_getreg(20 | (3 << 11)) & 7u, call7});
            } else if (idx >= 16 && idx < 24) {
                const u64 t0 = now();
                u32x2 ro;
                for (;;) {
                    u32 t; poll8(ro, t, p.ws + W_ROT, p.ws + W_TMO);
                    if (rfl(ro.y) == call7) break;
                    if (spin_fail(p.ws, call7, t0, t)) { ro.x = 8u; break; }
                }
                if (lane == 0) misc[1] = ro.x;
            }
            WAIT_L(0);
        }
        BAR; CFENCE;
    }
    const u32 call = rfl(misc[0]);
    STAMP(6);

    const bool is_gemm = idx < 16;
    const int c = idx - 16;
    const u32 rot0 = is_gemm || idx >= 24 ? 0u : rfl(misc[1]);
    if (rot0 > 7u) return;
    const int jl = (peer_of((int)((xcd + rot0) & 7u), p.rank) - p.rank - 1) & 7;
    if (!is_gemm && jl < 7 && c < NPW) {
        if (wid == 7) pull_schedule(p, lds, call, jl, c, wid, lane);
        else if (wid == 6) pull_load(p, lds, wid, lane);
        else pull_decode(p, lds, wid, lane);
        STAMP(5);
        return;
    }
    const int pm = (xcd >> 1) * 4 + (idx >> 2), pn = (xcd & 1) * 4 + (idx & 3);
    if (!is_gemm) {
        const int me = idx >= 24 ? (int)blockIdx.x - 192 : 64 + c, unit = wid & 3;
        int e = (wid >> 2) * NSPARE + me;
        volatile LAS u32* hb = (volatile LAS u32*)(lds + LDS_MISC + 1280 + (wid >> 2) * 128);
        auto panel_at = [&](int v) { const int r = v / 73, i = v - r * 73; return panel_of(1 + r, i < 72 ? i : 79, p.rank); };
        PanelId d = panel_at(e);
        f32x4 f[16];
        while (now() - tstart < (wid < 4 ? SPD0 : SPD1)) __builtin_amdgcn_s_sleep(4);
        unit_load(p, d, unit, lane, f);
        bool first = true;
        for (;;) {
            WAIT_V(0); PIN_F(f);
            if (STAMPS == 1 && first) STAMP(7);
            const u32 hdr = unit_pack(p, enc, d, unit, lane, f, 1);
            if (STAMPS == 1 && first) STAMP(8);
            e += 2 * NSPARE;
            const PanelId dn = panel_at(e);
            if (e < 7 * 73) { unit_load(p, dn, unit, lane, f); WAIT_V(16); } else { WAIT_V(0); }
            u32 arr = 0;
            if (lane == 0) { hb[4 + unit] = hdr; arr = __hip_atomic_fetch_add((LAS u32*)hb, 1u, __ATOMIC_RELAXED, __HIP_MEMORY_SCOPE_WORKGROUP); }
            if (rfl(arr) == 3u && lane < 4 * NR) unit_announce(p, lds, d, lane & 3, lane >> 2, hb[4 + (lane & 3)], call, 1);
            hb += 8;
            if (STAMPS == 1 && first) STAMP(9);
            first = false;
            if (e >= 7 * 73) break;
            d = dn;
        }
        STAMP(1);
        if (blockIdx.x == GRID - 1 && wid == 0) {
            const u64 t0 = now();
            while (now() - tstart < 6000) __builtin_amdgcn_s_sleep(16);
            for (u32 it = 1;; ++it) {
                const u32x4 gq = ld16_wait(p.ws + W_AMAX + lane * 16);
                if (__all(gq.y == call && gq.w == call)) {
                    const u32 wm = wave_umax(max(gq.x, gq.z));
                    if (lane < NR * GCOPY) st8_sys(peer_ws(lds, (u32)lane & 7u) + W_GMAX + (lane >> 3) * 256 + p.rank * 8, (u32x2){wm, call});
                    break;
                }
                if (spin_fail(p.ws, call, t0, (it & 63u) ? 0u : ld4_wait(p.ws + W_TMO))) break;
            }
        }
        return;
    }
#pragma clang loop unroll(disable)
    for (int pass = 0; pass < (pn < NVP ? 2 : 1); ++pass) {
        int tz = threadIdx.x; asm volatile("" : "+v"(tz));
        gemm_tile(p, lds, call, pass == 1, pm, pn, wid, tz & 63);
        if (pass == 0) STAMP(5); else STAMP(13);
    }
}

extern "C" void kernel_launch(int rank, int nranks, ncclComm_t comm, hipStream_t stream,
                              void* const* d_in, const int* in_sizes, int n_in,
                              void* d_out, int out_size,
                              const int* in_dtypes, int out_dtype,
                              void* const* d_out_all,
                              void* const* d_ws_all, size_t ws_bytes) {
    (void)comm; (void)in_dtypes; (void)out_dtype; (void)d_out_all;
    if (nranks != NR || n_in < 2 || ws_bytes < W_END || in_sizes[0] != GM * KS || in_sizes[1] != KS * GN || out_size != GM * GN ||
        (unsigned)rank >= (unsigned)NR) {
        fprintf(stderr, "unexpected problem geometry\n");
        return;
    }
    Params p;
    p.x = (const float*)d_in[0];
    p.w = (const float*)d_in[1];
    p.out = (signed char*)d_out;
    p.ws = (char*)d_ws_all[rank];
    p.ws_all = (char* const*)d_ws_all;
    p.rank = rank;
    p.pad = 0;
    hipLaunchKernelGGL(fused_gemm_q8, dim3(GRID), dim3(BLOCK), 0, stream, p);
}
```

```cpp
#include <hip/hip_runtime.h>
#include <rccl/rccl.h>
#include <cstdio>
#include <cstdint>

#define STAMPS 0

#define LAS __attribute__((address_space(3)))
typedef unsigned u32;
typedef unsigned long long u64;
typedef u32 u32x4 __attribute__((ext_vector_type(4)));
typedef u32 u32x2 __attribute__((ext_vector_type(2)));
typedef float f32x4 __attribute__((ext_vector_type(4)));
typedef short bf16x8 __attribute__((ext_vector_type(8)));

constexpr int NR = 8, GM = 4096, GN = 16384, KS = 512, NS = 2048;
constexpr int GRID = 256, BLOCK = 512;
constexpr int PANEL = 32768;
constexpr int USLOT = 8192;
constexpr size_t W_TMO = 128;
constexpr size_t W_ROT = 256;
constexpr size_t W_Q = 1024;
constexpr size_t W_ACNT = 4096;
constexpr size_t W_BCNT = 8192;
constexpr size_t W_XCNT = 10240;
constexpr size_t W_AMAX = 16384;
constexpr size_t W_GMAX = 20480;
constexpr size_t W_HI = 32768;
constexpr size_t W_HX = 81920;
constexpr size_t W_HO = 98304;
constexpr size_t W_SEQ = 256 << 10;
constexpr size_t W_DBG = 512 << 10;
constexpr size_t W_A = 1ull << 20;
constexpr size_t W_B = 33ull << 20;
constexpr size_t W_X = 49ull << 20;
constexpr size_t W_PX = 65ull << 20;
constexpr size_t W_PW = 69ull << 20;
constexpr size_t W_PO = 85ull << 20;
constexpr size_t W_END = 93ull << 20;
constexpr u64 DEADLINE = 40000000ull;

#define NPW 8
#define DEPTH 7
#define DEPTH2 6
#define RUNAHEAD 2
#define GDLY 100
#define NPUSH 3
#define GCOPY 8
#define NRX 5
#define SPD0 450
#define SPD1 700
#define NRAW 12
#define RATES 0
#define NEARLY 2
#define NRAWE 8
constexpr int ND = 6, NSLOT = 12;
constexpr int NSPARE = 7 * (16 - NPW) + 16;
#define FC 3
constexpr int NVP = (7 * FC + 3) / 4;
static_assert(NPUSH <= 7 && NRX <= 7 && 7 * 292 <= 4 * 8 * NSPARE && NPW == 8 && DEPTH <= 11 && DEPTH2 <= 11 && FC >= 2 && FC <= 4, "roles");

constexpr int LDS_BM = 98304;
constexpr int LDS_ENC = 114688;
constexpr int LDS_MISC = 131072;
constexpr int LDS_BYTES = LDS_MISC + 2048;

struct Params {
    const float* x;
    const float* w;
    signed char* out;
    char* ws;
    char* const* ws_all;
    int rank;
    int pad;
};

#define WAIT_V(n) asm volatile("s_waitcnt vmcnt(" #n ")" ::: "memory")
#define WAIT_L(n) asm volatile("s_waitcnt lgkmcnt(" #n ")" ::: "memory")
#define BAR __builtin_amdgcn_s_barrier()
#define SCHED __builtin_amdgcn_sched_barrier(0)
#define CFENCE asm volatile("" ::: "memory")

template <int OFF> __device__ __forceinline__ void st16_sys(void* p, u32x4 x) {
    asm volatile("global_store_dwordx4 %0, %1, off offset:%2 sc0 sc1\n\ts_nop 1" :: "v"(p), "v"(x), "i"(OFF) : "memory");
}
__device__ __forceinline__ void ld8_sys(u32x2& x, const void* p) {
    asm volatile("global_load_dwordx2 %0, %1, off sc0 sc1" : "=v"(x) : "v"(p) : "memory");
}
__device__ __forceinline__ void st8_sys(void* p, u32x2 x) {
    asm volatile("global_store_dwordx2 %0, %1, off sc0 sc1\n\ts_nop 1" :: "v"(p), "v"(x) : "memory");
}
__device__ __forceinline__ void st4_sys(void* p, u32 x) {
    asm volatile("global_store_dword %0, %1, off sc0 sc1\n\ts_nop 1" :: "v"(p), "v"(x) : "memory");
}
__device__ __forceinline__ void add4_sys(void* p, u32 x) {
    asm volatile("global_atomic_add %0, %1, off sc1\n\ts_nop 1" :: "v"(p), "v"(x) : "memory");
}
__device__ __forceinline__ void ld16x4(f32x4& a, f32x4& b, f32x4& c, f32x4& d, const void* p0, const void* p1, const void* p2, const void* p3, int nt) {
    asm volatile("s_cmp_lg_u32 %8, 0\n\ts_cbranch_scc1 .Lnt%=\n\t"
                 "global_load_dwordx4 %0, %4, off\n\tglobal_load_dwordx4 %1, %5, off\n\tglobal_load_dwordx4 %2, %6, off\n\tglobal_load_dwordx4 %3, %7, off\n\t"
                 "s_branch .Lnd%=\n.Lnt%=:\n\t"
                 "global_load_dwordx4 %0, %4, off nt\n\tglobal_load_dwordx4 %1, %5, off nt\n\tglobal_load_dwordx4 %2, %6, off nt\n\tglobal_load_dwordx4 %3, %7, off nt\n.Lnd%=:"
                 : "=&v"(a), "=&v"(b), "=&v"(c), "=&v"(d) : "v"(p0), "v"(p1), "v"(p2), "v"(p3), "s"(nt) : "memory", "scc");
}
__device__ __forceinline__ void ticket_issue(u32& x, void* p) {
    u64 sv;
    asm volatile("s_mov_b64 %1, exec\n\ts_mov_b64 exec, 1\n\tglobal_atomic_add %0, %2, %3, off sc0 sc1\n\ts_mov_b64 exec, %1"
                 : "=&v"(x), "=&s"(sv) : "v"(p), "v"(1u) : "memory");
}
__device__ __forceinline__ void poll8(u32x2& x, u32& t, const void* p, const void* tp) {
    asm volatile("global_load_dwordx2 %0, %2, off sc0 sc1\n\tglobal_load_dword %1, %3, off sc0 sc1\n\ts_waitcnt vmcnt(0)"
                 : "=&v"(x), "=&v"(t) : "v"(p), "v"(tp) : "memory");
}
__device__ __forceinline__ void poll_win(u32& x, const void* p, u64 lanes) {
    u64 sv;
    asm volatile("s_mov_b64 %1, exec\n\ts_mov_b64 exec, %3\n\tglobal_load_dword %0, %2, off sc0 sc1\n\ts_mov_b64 exec, %1\n\ts_waitcnt vmcnt(0)"
                 : "+v"(x), "=&s"(sv) : "v"(p), "s"(lanes) : "memory");
}
__device__ __forceinline__ u32x2 ld8_wait(const void* p) {
    u32x2 x; asm volatile("global_load_dwordx2 %0, %1, off sc0 sc1\n\ts_waitcnt vmcnt(0)" : "=&v"(x) : "v"(p) : "memory"); return x;
}
__device__ __forceinline__ u32x4 ld16_wait(const void* p) {
    u32x4 x; asm volatile("global_load_dwordx4 %0, %1, off sc0 sc1\n\ts_waitcnt vmcnt(0)" : "=&v"(x) : "v"(p) : "memory"); return x;
}
__device__ __forceinline__ u32 ld4_wait(const void* p) {
    u32 x; asm volatile("global_load_dword %0, %1, off sc0 sc1\n\ts_waitcnt vmcnt(0)" : "=&v"(x) : "v"(p) : "memory"); return x;
}
__device__ __forceinline__ u32 cvt_pk_bf16(float lo, float hi) {
    u32 r; asm("v_cvt_pk_bf16_f32 %0, %1, %2" : "=v"(r) : "v"(lo), "v"(hi)); return r;
}
__device__ __forceinline__ u32 bfi(u32 m, u32 a, u32 b) { return __builtin_amdgcn_bitop3_b32(m, a, b, 0xCA); }
__device__ __forceinline__ u32 lshl_or(u32 a, u32 sh, u32 b) { u32 r; asm("v_lshl_or_b32 %0, %1, %2, %3" : "=v"(r) : "v"(a), "v"(sh), "v"(b)); return r; }
template <int SH> __device__ __forceinline__ u32 lshl_or_i(u32 a, u32 b) { u32 r; asm("v_lshl_or_b32 %0, %1, %2, %3" : "=v"(r) : "v"(a), "n"(SH), "v"(b)); return r; }
__device__ __forceinline__ u32 times15(u32 t) { u32 r = t << 4; asm("" : "+v"(r)); return r - t; }
__device__ __forceinline__ u32 rfl(u32 x) { return (u32)__builtin_amdgcn_readfirstlane((int)x); }
__device__ __forceinline__ u32 rdl(u32 x, int l) { return (u32)__builtin_amdgcn_readlane((int)x, l); }
__device__ __forceinline__ u64 now() { return __builtin_amdgcn_s_memrealtime(); }

__device__ __forceinline__ bool spin_fail(char* ws, u32 call, u64 t0, u32 t) {
    if (rfl(t) == call) return true;
    if (now() - t0 > DEADLINE) { st4_sys(ws + W_TMO, call); WAIT_V(0); return true; }
    __builtin_amdgcn_s_sleep(1);
    return false;
}
#define DPP0(v, ctl, rm, bc) (u32)__builtin_amdgcn_update_dpp(0, (int)(v), ctl, rm, 0xf, bc)
__device__ __forceinline__ u32 scan_add(u32 v) {
    v += DPP0(v, 0x111, 0xf, true); v += DPP0(v, 0x112, 0xf, true); v += DPP0(v, 0x114, 0xf, true); v += DPP0(v, 0x118, 0xf, true);
    v += DPP0(v, 0x142, 0xa, false); v += DPP0(v, 0x143, 0xc, false);
    return v;
}
__device__ __forceinline__ u32 wave_umax(u32 v) {
    v = max(v, DPP0(v, 0x111, 0xf, true)); v = max(v, DPP0(v, 0x112, 0xf, true)); v = max(v, DPP0(v, 0x114, 0xf, true)); v = max(v, DPP0(v, 0x118, 0xf, true));
    v = max(v, DPP0(v, 0x142, 0xa, false)); v = max(v, DPP0(v, 0x143, 0xc, false));
    return rdl(v, 63);
}
__device__ __forceinline__ int rot_of(int j) { return (j * 16) / 7; }
__device__ __forceinline__ int kt_of(int o, int rank, int j0) {
    if (o < 8) return rank * 8 + o;
    const int r = (o - 8) / 7, s = o - 8 - r * 7;
    int j = j0 - s; if (j < 0) j += 7;
    return ((rank + 1 + j) & 7) * 8 + r;
}
__device__ __forceinline__ int peer_of(int x, int rank) {
    const u32 ga = 0x6320u, gb = 0x7541u;
    const bool ina = (0x4du >> rank) & 1u;
    const u32 own = ina ? ga : gb, oth = ina ? gb : ga;
    if (x >= 4) return (int)((oth >> (4 * (x - 4))) & 7u);
    if (x == 1) return rank;
    const int want = x == 0 ? 0 : x - 1;
    int n = 0, res = rank;
#pragma unroll
    for (int i = 0; i < 4; ++i) { const int q = (int)((own >> (4 * i)) & 7u); if (q != rank) { if (n == want) res = q; ++n; } }
    return res;
}
__device__ __forceinline__ char* peer_ws(LAS unsigned char* lds, u32 pr) {
    return (char*)((volatile LAS u64*)(lds + LDS_MISC + 64))[pr & 7u];
}

#define STAMP(i) do {} while (0)

struct Msk { u32 m, ms, m12, m2, o[8]; };
template <bool HI> __device__ __forceinline__ Msk msk(u32 m) {
    Msk k; k.m = m; k.ms = m << 1;
    const u32 c2 = m - ((m >> 1) & 0x55555555u), cl = c2 & 0x33333333u;
    if (HI) {
        const u32 n2 = (m << 2) & k.ms & 0x44444444u, n12 = ((m << 2) | k.ms) & 0x44444444u;
        k.m2 = lshl_or_i<1>(n2, n2); k.m12 = lshl_or_i<1>(n12, n12);
    } else {
        const u32 n2 = (m >> 1) & m & 0x11111111u, n12 = ((m >> 1) | m) & 0x11111111u;
        k.m2 = lshl_or_i<1>(n2, n2); k.m12 = lshl_or_i<1>(n12, n12);
    }
    const u32 c4 = cl + ((c2 >> 2) & 0x33333333u), ce = c4 & 0x0f0f0f0fu, cb = ce + ((c4 >> 4) & 0x0f0f0f0fu);
    u32 pe; asm("v_mul_u32_u24 %0, 0x10101, %1" : "=v"(pe) : "v"(cb));
    const u32 po = pe + (ce >> 8);
    k.o[0] = 0; k.o[1] = ce; k.o[2] = pe; k.o[3] = po; k.o[4] = pe >> 8; k.o[5] = po >> 8; k.o[6] = pe >> 16; k.o[7] = po >> 16;
    return k;
}
__device__ __forceinline__ u32 press(u32 x, const Msk& k) {
    const u32 y = bfi(0x55555555u, x | ((x >> 1) & ~k.m), x & k.ms);
    const u32 hi = (y >> 2) & 0x33333333u;
    const u32 z = (y & 0x33333333u) + hi + (hi & k.m12) + ((hi & k.m2) << 1);
    u32 c = z & 15u, d = (z >> 28) << (k.o[7] & 31u);
#pragma unroll
    for (int i = 1; i < 7; i += 2) { c = lshl_or(__builtin_amdgcn_ubfe(z, 4 * i, 4u), k.o[i], c); d = lshl_or(__builtin_amdgcn_ubfe(z, 4 * i + 4, 4u), k.o[i + 1], d); }
    return c | d;
}
__device__ __forceinline__ u32 spread(u32 c, const Msk& k) {
    u32 z = c & 15u, w = (c >> (k.o[7] & 31u)) << 28;
    z = lshl_or_i<4>(__builtin_amdgcn_ubfe(c, k.o[1], 4u), z); w = lshl_or_i<8>(__builtin_amdgcn_ubfe(c, k.o[2], 4u), w);
    z = lshl_or_i<12>(__builtin_amdgcn_ubfe(c, k.o[3], 4u), z); w = lshl_or_i<16>(__builtin_amdgcn_ubfe(c, k.o[4], 4u), w);
    z = lshl_or_i<20>(__builtin_amdgcn_ubfe(c, k.o[5], 4u), z); w = lshl_or_i<24>(__builtin_amdgcn_ubfe(c, k.o[6], 4u), w);
    z |= w;
    const u32 h = bfi(k.m12, bfi(k.m2, z, z << 1), z << 2);
    const u32 y = bfi(0x33333333u, z, h);
    return bfi(0x55555555u, y, bfi(k.ms, y, y << 1)) & k.m;
}
__device__ __forceinline__ void bit_put(LAS u32* st, u32 off, u32 c) {
    const u64 x = (u64)c << (off & 31u);
    __hip_atomic_fetch_or(st + (off >> 5), (u32)x, __ATOMIC_RELAXED, __HIP_MEMORY_SCOPE_WORKGROUP);
    __hip_atomic_fetch_or(st + (off >> 5) + 1, (u32)(x >> 32), __ATOMIC_RELAXED, __HIP_MEMORY_SCOPE_WORKGROUP);
}
__device__ __forceinline__ u32 bit_get(const LAS u32* st, u32 off) {
    const u32 w = min(off >> 5, 500u);
    return __builtin_amdgcn_alignbit(st[w + 1], st[w], off & 31u);
}
__device__ __forceinline__ u32 popc(u32 x) { return (u32)__popc(x); }
__device__ __forceinline__ void to_planes(const u32* n, u32& p0, u32& p1, u32& p2, u32& p3) {
    const u32 A = bfi(0x33333333u, n[0], n[2] << 2), B = bfi(0x33333333u, n[0] >> 2, n[2]);
    const u32 C = bfi(0x33333333u, n[1], n[3] << 2), D = bfi(0x33333333u, n[1] >> 2, n[3]);
    p0 = bfi(0x55555555u, A, C << 1); p1 = bfi(0x55555555u, A >> 1, C); p2 = bfi(0x55555555u, B, D << 1); p3 = bfi(0x55555555u, B >> 1, D);
}
__device__ __forceinline__ void from_planes(u32* n, u32 p0, u32 p1, u32 p2, u32 p3) {
    const u32 A = bfi(0x55555555u, p0, p1 << 1), C = bfi(0x55555555u, p0 >> 1, p1);
    const u32 B = bfi(0x55555555u, p2, p3 << 1), D = bfi(0x55555555u, p2 >> 1, p3);
    n[0] = bfi(0x33333333u, A, B << 2); n[2] = bfi(0x33333333u, A >> 2, B); n[1] = bfi(0x33333333u, C, D << 2); n[3] = bfi(0x33333333u, C >> 2, D);
}
constexpr u32 MAXSDW = 384;
__device__ __forceinline__ void tail_out(char* dst, LAS unsigned char* enc, u32 n16, int lane) {
    CFENCE;
    const u32x4 a = *(volatile LAS u32x4*)(enc + lane * 16), b = *(volatile LAS u32x4*)(enc + 1024 + lane * 16);
    if ((u32)lane < n16) st16_sys<0>(dst + lane * 16, a);
    if ((u32)lane + 64u < n16) st16_sys<1024>(dst + lane * 16, b);
}
constexpr u32 H_RAW = 0x80000000u;

struct PanelId { int r, dest, pi; bool isB; };
__device__ __forceinline__ PanelId panel_of(int r, int i, int rank) {
    PanelId d; d.r = r; d.isB = i < 56 || i >= 72;
    d.dest = i < 56 ? ((rank + 1 + (i >> 3)) & 7) : rank;
    d.pi = d.isB ? (i & 7) : i - 56;
    return d;
}
__device__ __forceinline__ bool is_fill(const PanelId& d, int unit) { return d.isB && d.pi == 7 && unit >= 4 - FC; }
__device__ __forceinline__ void unit_load(const Params& p, const PanelId& d, int unit, int lane, f32x4 (&f)[16]) {
    const int fr = lane & 15, fq = lane >> 4;
    const float* b = d.isB ? p.w + (size_t)(d.r * 64 + 4 * fq) * GN + d.dest * NS + d.pi * 256 + unit * 64 + 4 * fr
                           : p.x + (size_t)(d.pi * 256 + unit * 64 + (lane >> 2)) * KS + d.r * 64 + 4 * ((lane & 3) ^ (fq & 2));
    const int s1 = d.isB ? 16 * GN : 16 * KS, s2 = d.isB ? GN : 16;
    const int nt = (int)rfl(d.isB ? 1u : 0u);
#pragma unroll
    for (int j = 0; j < 4; ++j) ld16x4(f[4 * j], f[4 * j + 1], f[4 * j + 2], f[4 * j + 3], b + j * s1, b + j * s1 + s2, b + j * s1 + 2 * s2, b + j * s1 + 3 * s2, nt);
}
#define PIN_F(f) do { asm volatile("" : "+v"(f[0]), "+v"(f[1]), "+v"(f[2]), "+v"(f[3]), "+v"(f[4]), "+v"(f[5]), "+v"(f[6]), "+v"(f[7]) :: "memory"); \
    asm volatile("" : "+v"(f[8]), "+v"(f[9]), "+v"(f[10]), "+v"(f[11]), "+v"(f[12]), "+v"(f[13]), "+v"(f[14]), "+v"(f[15]) :: "memory"); } while (0)
__device__ __forceinline__ void st_unit(char* dp, const u32 (&u)[32]) {
    st16_sys<0>(dp, (u32x4){u[0], u[1], u[2], u[3]}); st16_sys<1024>(dp, (u32x4){u[4], u[5], u[6], u[7]});
    st16_sys<2048>(dp, (u32x4){u[8], u[9], u[10], u[11]}); st16_sys<3072>(dp, (u32x4){u[12], u[13], u[14], u[15]});
    dp += 4096;
    st16_sys<0>(dp, (u32x4){u[16], u[17], u[18], u[19]}); st16_sys<1024>(dp, (u32x4){u[20], u[21], u[22], u[23]});
    st16_sys<2048>(dp, (u32x4){u[24], u[25], u[26], u[27]}); st16_sys<3072>(dp, (u32x4){u[28], u[29], u[30], u[31]});
}
__device__ __forceinline__ size_t local_off(const PanelId& d, int unit, int rank, int kt, size_t& cnt) {
    if (d.dest == rank) { cnt = (d.isB ? W_BCNT : W_ACNT) + (size_t)(d.pi * 64 + (kt & 7) * 8 + (kt >> 3)) * 4; return (d.isB ? W_B : W_A) + (size_t)(d.pi * 64 + kt) * PANEL + unit * USLOT; }
    const int g = ((d.dest - rank - 1) & 7) * FC + unit - (4 - FC);
    cnt = W_XCNT + (size_t)((g >> 2) * 64 + (kt & 7) * 8 + (kt >> 3)) * 4;
    return W_X + (size_t)((g >> 2) * 64 + kt) * PANEL + (g & 3) * USLOT;
}
__device__ __forceinline__ size_t export_off(const PanelId& d, int unit) {
    return (d.isB ? W_PW + (size_t)((d.dest * 8 + d.r) * 8 + d.pi) * PANEL : W_PX + (size_t)(d.r * 16 + d.pi) * PANEL) + unit * USLOT;
}
__device__ __forceinline__ u32 unit_pack(const Params& p, LAS unsigned char* lds, LAS unsigned char* enc, const PanelId& d, int unit, int lane, const f32x4 (&f)[16], int mode) {
    u32 u[32];
    if (d.isB) {
#pragma unroll
        for (int h = 0; h < 2; ++h)
#pragma unroll
            for (int c = 0; c < 4; ++c)
#pragma unroll
                for (int t = 0; t < 4; ++t) u[16 * h + 4 * c + t] = cvt_pk_bf16(f[8 * h + 2 * t][c], f[8 * h + 2 * t + 1][c]);
    } else {
#pragma unroll
        for (int c = 0; c < 16; ++c) { u[2 * c] = cvt_pk_bf16(f[c][0], f[c][1]); u[2 * c + 1] = cvt_pk_bf16(f[c][2], f[c][3]); }
    }
    const bool own = d.dest == p.rank, fill = is_fill(d, unit);
    u32 hdr = 0;
    if (mode & 4) {
        size_t cnt;
        while (rfl(((volatile LAS u32*)(lds + LDS_MISC))[2]) == 0u) __builtin_amdgcn_s_sleep(1);
        st_unit(peer_ws(lds, (u32)d.dest) + local_off(d, unit, d.dest, p.rank * 8 + d.r, cnt) + lane * 16, u);
    } else if (mode == 1 && d.r == 0 && d.isB && d.pi < NRX) {
        st_unit(p.ws + export_off(d, unit) + lane * 16, u);
        hdr = H_RAW | 512u;
    } else if ((mode & 1) && (!(d.isB && own) || fill)) {
        char* slot = p.ws + export_off(d, unit);
        *(volatile LAS u32x4*)(enc + lane * 16) = (u32x4){0u, 0u, 0u, 0u};
        *(volatile LAS u32x4*)(enc + 1024 + lane * 16) = (u32x4){0u, 0u, 0u, 0u};
        u32 hi[16], t[16], sum = 0;
#pragma unroll
        for (int k = 0; k < 16; ++k) {
            hi[k] = __builtin_amdgcn_perm(u[2 * k + 1], u[2 * k], 0x07050301u);
            t[k] = hi[k] | 0x80808080u;
            sum = __builtin_amdgcn_sad_u8(t[k], 0u, sum);
        }
        u32 a = ((rdl(scan_add(sum), 63) + 4095u) >> 12) - 128u;
        a = min(max(a, 6u), 126u);
        const u32 b4 = (a - 6u) * 0x01010101u;
        u32 bad = 0, n[8];
#pragma unroll
        for (int k = 0; k < 16; ++k) { t[k] -= b4; bad |= (t[k] & 0xf8f8f8f8u) ^ 0x80808080u; }
#pragma unroll
        for (int k = 0; k < 8; ++k)
            n[k] = __builtin_amdgcn_perm(0x06000204u, 0x080a0c0eu, t[2 * k] & 0x07070707u) | __builtin_amdgcn_perm(0x60002040u, 0x80a0c0e0u, t[2 * k + 1] & 0x07070707u)
                 | ((hi[2 * k] >> 7) & 0x01010101u) | ((hi[2 * k + 1] >> 3) & 0x10101010u);
        u32 sg[2], c0[2], c1[2], p1[2], p2[2], p3[2], p4[2];
#pragma unroll
        for (int h = 0; h < 2; ++h) {
            to_planes(n + 4 * h, sg[h], c0[h], c1[h], p4[h]);
            p2[h] = c1[h] | p4[h]; p1[h] = p2[h] | c0[h]; p3[h] = p4[h] | (c1[h] & c0[h]);
        }
        const u32 ca = (popc(p1[0]) + popc(p1[1])) | ((popc(p2[0]) + popc(p2[1])) << 16), cb = (popc(p3[0]) + popc(p3[1])) | ((popc(p4[0]) + popc(p4[1])) << 17);
        const u32 ia = scan_add(ca), ib = scan_add(cb), ta = rdl(ia, 63), tb = rdl(ib, 63);
        const u32 b3 = ta & 0xffffu, b4s = b3 + (ta >> 16), b5 = b4s + (tb & 0xffffu);
        const u32 sdw = (b5 + (tb >> 16) + 31u) >> 5;
        if (sdw > MAXSDW || __any(max(popc(p4[0]), popc(p4[1])) > 16u)) hdr = H_RAW | 512u;
        u32 nexc = 0;
        u64 anyb = __ballot(bad != 0);
        if (anyb && !hdr) {
            u32 bm[2] = {0u, 0u};
#pragma unroll
            for (int k = 0; k < 16; ++k) {
                const u32 y = (t[k] & 0xf8f8f8f8u) ^ 0x80808080u;
                bm[k >> 3] |= ((((y & 0x78787878u) + 0x78787878u) | y) & 0x80808080u) >> (7 - (k & 7));
            }
            if (rdl(scan_add((u32)(__popc(bm[0]) + __popc(bm[1]))), 63) > 64u) {
                hdr = H_RAW | 512u;
            } else {
                while (anyb) {
                    const int l = __builtin_ctzll(anyb);
                    const u32 w0 = rdl(bm[0], l), w1 = rdl(bm[1], l);
                    const u32 pos = (u32)__builtin_ctz(w0 ? w0 : w1), k = (w0 ? 0u : 8u) + (pos & 7u);
                    u32 hv = 0;
#pragma unroll
                    for (int kk = 0; kk < 16; ++kk) hv = k == (u32)kk ? hi[kk] : hv;
                    const u32 e = (rdl(hv, l) >> (pos & 24u)) & 0x7fu;
                    if (lane == 0) ((volatile LAS u32*)enc)[sdw + nexc] = ((u32)l << 13) | (k << 9) | ((pos >> 3) << 7) | e;
                    ++nexc;
                    if (lane == l) { if (w0) bm[0] &= bm[0] - 1u; else bm[1] &= bm[1] - 1u; }
                    anyb = __ballot((bm[0] | bm[1]) != 0);
                }
            }
        }
        char* dp = slot + lane * 16;
        if (hdr) {
            st_unit(dp, u);
        } else {
            u32 lo[16];
#pragma unroll
            for (int k = 0; k < 16; ++k) lo[k] = __builtin_amdgcn_perm(u[2 * k + 1], u[2 * k], 0x06040200u);
            st16_sys<0>(dp, (u32x4){lo[0], lo[1], lo[2], lo[3]}); st16_sys<1024>(dp, (u32x4){lo[4], lo[5], lo[6], lo[7]});
            st16_sys<2048>(dp, (u32x4){lo[8], lo[9], lo[10], lo[11]}); st16_sys<3072>(dp, (u32x4){lo[12], lo[13], lo[14], lo[15]});
            st16_sys<0>(dp + 4096, (u32x4){sg[0], sg[1], p1[0], p1[1]});
            u32 o2 = (ia - ca) & 0xffffu, o3 = b3 + ((ia - ca) >> 16), o4 = b4s + ((ib - cb) & 0xffffu), o5 = b5 + ((ib - cb) >> 16);
#pragma unroll
            for (int h = 0; h < 2; ++h) {
                bit_put((LAS u32*)enc, o2, press(p2[h], msk<false>(p1[h]))); o2 += popc(p1[h]);
                bit_put((LAS u32*)enc, o3, press(p3[h], msk<false>(p2[h]))); o3 += popc(p2[h]);
                bit_put((LAS u32*)enc, o4, press(p4[h], msk<false>(p3[h]))); o4 += popc(p3[h]);
                const Msk k = msk<false>(p4[h]);
                bit_put((LAS u32*)enc, o5, press(c0[h] & p4[h], k) | (press(c1[h] & p4[h], k) << popc(p4[h]))); o5 += 2u * popc(p4[h]);
            }
            const u32 n16 = max((sdw + nexc + 3u) >> 2, 1u);
            tail_out(slot + 5120, enc, n16, lane);
            hdr = (320u + n16) | (a << 10) | (nexc << 17);
        }
    }
    if (own ? (mode & 2) : fill) {
        size_t cnt;
        st_unit(p.ws + local_off(d, unit, p.rank, p.rank * 8 + d.r, cnt) + lane * 16, u);
    }
    return hdr;
}
__device__ __forceinline__ void unit_announce(const Params& p, LAS unsigned char* lds, const PanelId& d, int unit, int pr, u32 hdr, u32 call, int mode) {
    const bool fill = is_fill(d, unit);
    if (mode & 4) {
        if (pr == d.dest) { size_t cnt; local_off(d, unit, d.dest, p.rank * 8 + d.r, cnt); add4_sys(peer_ws(lds, (u32)pr) + cnt, 1u); }
    } else if (pr == p.rank) {
        if (d.dest == p.rank ? (mode & 2) : fill) { size_t cnt; local_off(d, unit, p.rank, p.rank * 8 + d.r, cnt); add4_sys(p.ws + cnt, 1u); }
    } else if (!(mode & 1)) {
    } else if (!d.isB || pr == d.dest) {
        st8_sys(peer_ws(lds, (u32)pr) + W_HI + (size_t)(((p.rank * 8 + d.r) * 24 + (d.isB ? d.pi : 8 + d.pi)) * 4 + unit) * 8, (u32x2){hdr, call});
    } else if (fill) {
        st8_sys(peer_ws(lds, (u32)pr) + W_HX + (size_t)(((p.rank * 8 + d.r) * 8 + d.dest) * 4 + unit) * 8, (u32x2){hdr, call});
    }
}

constexpr u32 T_IN = 0, T_OUT = 1, T_END = 2, T_DEAD = 3, T_SKIP = 4;
constexpr u32 NP_IN = 384, NP_X = 28 * FC, NP_ROW = 32 - FC, NP_OUT = 16 * NP_ROW, NP_ALL = NP_IN + NP_X + NP_OUT;
#define R_POSTED 0
#define R_TAKEN 1
#define R_FULL 2
#define R_DONE 4
__device__ __forceinline__ volatile LAS u32* ring_words(LAS unsigned char* lds) { return (volatile LAS u32*)(lds + LDS_MISC + 256); }
__device__ __forceinline__ volatile LAS u32x4* ring_desc(LAS unsigned char* lds, u32 n) { return (volatile LAS u32x4*)(lds + LDS_MISC + 512 + (n % (u32)NSLOT) * 32u); }

struct Pair { u32 type, so, ho, cnt, sstep, hstep, cstep, dstep; char* dst; };
__device__ __forceinline__ Pair pair_of(const Params& p, u32 z, int j, int peer, int rot) {
    Pair q; q.type = T_END; q.so = 0; q.ho = (u32)W_TMO; q.cnt = 0; q.sstep = USLOT; q.hstep = 8; q.cstep = 0; q.dstep = USLOT; q.dst = nullptr;
    size_t cnt;
    if (z < NP_IN) {
        PanelId d;
        const int P = (int)(z >> 1), r = P / 24, i = P - r * 24, unit = (int)(z & 1u) * 2;
        d.r = r; d.isB = i < 8; d.dest = p.rank; d.pi = d.isB ? i : ((i - 8 + rot) & 15);
        q.type = T_IN; q.so = (u32)export_off(d, unit);
        q.ho = (u32)W_HI + (u32)(((peer * 8 + r) * 24 + (d.isB ? d.pi : 8 + d.pi)) * 4 + unit) * 8u;
        q.dst = p.ws + local_off(d, unit, p.rank, peer * 8 + r, cnt);
        q.cnt = (u32)cnt;
    } else if (z < NP_IN + NP_X) {
        const int y = (int)(z - NP_IN), rp = y / (7 * FC), yy = y - rp * (7 * FC), jj = yy / FC, k = yy - jj * FC;
        const bool across = (FC & 1) && k == FC - 1;
        PanelId d; d.r = 2 * rp + (across ? 0 : (k & 1)); d.isB = true; d.dest = (p.rank + 1 + jj) & 7; d.pi = 7;
        const int unit = across ? 4 - FC : 2 - (k & 2);
        PanelId e = d; e.r = d.r + (across ? 1 : 0);
        const int unit1 = across ? unit : unit + 1;
        size_t cnt1;
        const u32 l0 = (u32)local_off(d, unit, p.rank, peer * 8 + d.r, cnt), l1 = (u32)local_off(e, unit1, p.rank, peer * 8 + e.r, cnt1);
        q.type = T_IN; q.so = (u32)export_off(d, unit); q.sstep = (u32)export_off(e, unit1) - q.so;
        q.ho = (u32)W_HX + (u32)(((peer * 8 + d.r) * 8 + d.dest) * 4 + unit) * 8u; q.hstep = across ? 256u : 8u;
        q.dst = p.ws + l0; q.dstep = l1 - l0;
        q.cnt = (u32)cnt; q.cstep = (u32)cnt1 - (u32)cnt;
    } else if (z < NP_ALL) {
        int y = (int)(z - NP_IN - NP_X), pm, pn, unit;
        bool across = false;
        if (y < 112) { pn = y >> 4; pm = y & 15; unit = 0; }
        else {
            y -= 112; pm = y / (int)(NP_ROW - 7); const int i = y - pm * (int)(NP_ROW - 7);
            pn = i < 21 ? i / 3 : 7; unit = i < 21 ? 2 + (i - pn * 3) * 2 : (i - 21) * 4;
            across = FC == 3 && i >= 21;
        }
        q.type = T_OUT; q.so = (u32)W_PO + (u32)((pm * 8 + pn) * 8 + unit) * (u32)USLOT;
        q.sstep = across ? 4u * USLOT : (u32)USLOT; q.hstep = across ? 32u : 8u; q.dstep = across ? 128u * GN : 64u;
        q.ho = (u32)W_HO + (u32)((peer * 128 + pm * 8 + pn) * 8 + unit) * 8u;
        q.dst = (char*)p.out + (size_t)(pm * 256 + (unit >> 2) * 128) * GN + peer * NS + pn * 256 + (unit & 3) * 64;
    }
    return q;
}
__device__ __forceinline__ void pull_schedule(const Params& p, LAS unsigned char* lds, u32 call, int j, int c, int wid, int lane) {
    __builtin_amdgcn_s_setprio(2);
    const int peer = (p.rank + 1 + j) & 7, rot = rot_of(j);
    const u32 sub = (u32)lane & 1u;
    const u32 subm = 0u - sub;
    const char* pws = peer_ws(lds, (u32)peer);
    volatile LAS u32* rw = ring_words(lds);
    constexpr u32 Z0 = 2 * NPUSH;
    const u32 qbase = (call - 1u) * (NP_ALL - Z0) - 2u * (u32)NPW - Z0;
    char* qp = p.ws + W_Q + j * 128;
    u32 tk;
    u32 z = Z0 + (u32)c, zn = Z0 + (u32)(NPW + c), n = 0, slot = 0, dec = 0, lap = 0;
    Pair cur = pair_of(p, z, j, peer, rot);
    u32x2 h = (u32x2){0u, 0u};
    bool dead = false;
    u32 ends = 0;
    for (;;) {
        const bool more = z < NP_ALL;
        const Pair nxt = pair_of(p, zn, j, peer, rot);
        const char* hnp = p.ws + (nxt.ho + (nxt.hstep & subm));
        const u64 src = (u64)(pws + (cur.so + (cur.sstep & subm))), d64 = (u64)(cur.dst + (cur.dstep & subm));
        asm volatile("" :: "v"(hnp), "v"(src), "v"(d64));
        if (more && !dead && !__all(h.y == call)) {
            const u64 t0 = now();
            for (;;) {
                u32 t; poll8(h, t, p.ws + (cur.ho + (cur.hstep & subm)), p.ws + W_TMO);
                if (__all(h.y == call)) { dead = rfl(t) == call; break; }
                if (spin_fail(p.ws, call, t0, t)) { dead = true; break; }
            }
        }
        u32x2 hn;
        ld8_sys(hn, hnp);
        tk = 0xffffffffu;
        if (more) ticket_issue(tk, qp);
        for (;;) {
            const u32 tkn = rw[R_TAKEN], dn = rw[R_DONE + dec + sub];
            if (__all(n < tkn + (u32)RUNAHEAD && dn + (u32)(NSLOT / ND) > lap)) break;
            __builtin_amdgcn_s_sleep(1);
        }
        if (lane < 2) {
            const u32 type = !dead || !more ? cur.type : (cur.type == T_OUT ? T_DEAD : T_SKIP);
            volatile LAS u32x4* ds = (volatile LAS u32x4*)(lds + LDS_MISC + 512 + (slot + sub) * 32u);
            ds[0] = (u32x4){(u32)src, (u32)(src >> 32), h.x | (STAMPS ? (z < NP_IN / 2 ? 0u : z < NP_IN ? 1u : z < NP_IN + NP_X ? 2u : z < NP_IN + NP_X + NP_OUT / 2 ? 3u : 4u) << 26 : 0u), type};
            ds[1] = (u32x4){(u32)d64, (u32)(d64 >> 32), cur.cnt + (cur.cstep & subm), 0u};
        }
        n += 2;
        rw[R_POSTED] = n;
        slot = slot == (u32)NSLOT - 2u ? 0u : slot + 2u;
        if (dec == (u32)ND - 2u) { dec = 0; ++lap; } else dec += 2u;
        asm volatile("s_waitcnt vmcnt(0)" : "+v"(tk), "+v"(hn) :: "memory");
        if (more) { z = zn; zn = rfl(tk) - qbase; cur = nxt; h = hn; }
        else if (++ends == (u32)ND / 2u) break;
    }
}

#define GLDS(src, off) "global_load_lds_dwordx4 " src ", off offset:" #off " sc0 sc1\n\ts_waitcnt vmcnt(%[d])\n\t"
template <int D> __device__ __forceinline__ void glds_part(const void* src, u32 dst, u32 rem) {
    const u64 mask = rem >= 64u ? ~0ull : (1ull << rem) - 1ull;
    u32 keep;
    asm volatile("s_mov_b32 %0, m0\n\ts_mov_b32 m0, %2\n\ts_mov_b64 exec, %3\n\t" GLDS("%1", 0) "s_mov_b64 exec, -1\n\ts_mov_b32 m0, %0"
                 : "=&s"(keep) : "v"(src), "s"(dst), "s"(mask), [d] "i"(D - 1) : "memory");
}
template <int D> __device__ __forceinline__ void load_unit(LAS unsigned char* lds, const char* src, u32 dst, u32 n16, u32 n, u32 nslot, u32& full, u32& posted, u32x4& ds) {
    volatile LAS u32* rw = ring_words(lds);
    u32 keep;
    asm volatile("s_mov_b32 %0, m0\n\ts_mov_b32 m0, %2\n\ts_nop 0\n\t" GLDS("%1", 0) GLDS("%1", 1024) "s_mov_b32 m0, %0"
                 : "=&s"(keep) : "v"(src), "s"(dst), [d] "i"(D - 1) : "memory");
    if (D <= 9 && full + 1u < n) { full = n - 1u; rw[R_FULL] = full; }
    asm volatile("s_mov_b32 %0, m0\n\ts_mov_b32 m0, %2\n\ts_nop 0\n\t" GLDS("%1", 2048) GLDS("%1", 3072) "s_mov_b32 m0, %0"
                 : "=&s"(keep) : "v"(src), "s"(dst), [d] "i"(D - 1) : "memory");
    if (D > 9 && full + 1u < n) { full = n - 1u; rw[R_FULL] = full; }
    if (D <= 5) { full = n; rw[R_FULL] = n; }
    posted = rw[R_POSTED];
    ds = *(volatile LAS u32x4*)(lds + LDS_MISC + 512 + nslot * 32u);
    src += 4096;
    glds_part<D>(src, dst + 4096u, 64u);
    if (D == 6) { full = n; rw[R_FULL] = n; }
    glds_part<D>(src + 1024, dst + 5120u, n16 - 320u);
    if (D == 7) { full = n; rw[R_FULL] = n; }
    if (n16 > 384u) {
        glds_part<D>(src + 2048, dst + 6144u, n16 - 384u);
        if (D == 8) { full = n; rw[R_FULL] = n; }
        if (n16 > 448u) glds_part<D>(src + 3072, dst + 7168u, n16 - 448u);
    }
    rw[R_TAKEN] = n + 1u;
}
__device__ __forceinline__ void pull_load(const Params& p, LAS unsigned char* lds, int wid, int lane) {
    __builtin_amdgcn_s_setprio(3);
    volatile LAS u32* rw = ring_words(lds);
    const u32 lbase = (u32)(u64)lds;
    u32 ends = 0, slot = 0, full = 0;
    u32 posted = 0;
    u32x4 ds = (u32x4){0u, 0u, 0u, 0u};
    for (u32 n = 0;; ++n) {
        volatile LAS u32x4* dsp = (volatile LAS u32x4*)(lds + LDS_MISC + 512 + slot * 32u);
        if (rfl(posted) <= n) {
            posted = rw[R_POSTED];
            if (rfl(posted) <= n) {
                WAIT_V(0);
                full = n; rw[R_FULL] = n;
                while (rfl(rw[R_POSTED]) <= n) __builtin_amdgcn_s_sleep(1);
            }
            ds = dsp[0];
        }
        const u32 type = rfl(ds.w);
        const u32 dst = lbase + slot * (u32)USLOT;
        slot = slot == (u32)NSLOT - 1u ? 0u : slot + 1u;
        if (type <= T_OUT) {
            const u32 n16 = rfl(min(max(ds.z & 1023u, 321u), 512u));
            const char* src = (const char*)(((u64)ds.y << 32) | ds.x) + lane * 16;
            if (type == T_IN) load_unit<DEPTH>(lds, src, dst, n16, n, slot, full, posted, ds);
            else load_unit<DEPTH2>(lds, src, dst, n16, n, slot, full, posted, ds);
        } else {
            WAIT_V(0);
            full = n + 1u; rw[R_FULL] = full; rw[R_TAKEN] = full;
            posted = 0;
            if (type == T_END && ++ends == (u32)ND) break;
        }
    }
}

__device__ __forceinline__ void pull_decode(const Params& p, LAS unsigned char* lds, int wid, int lane) {
    volatile LAS u32* rw = ring_words(lds);
    LAS u32* bm = (LAS u32*)(lds + LDS_BM + wid * 1024);
    u32 done = 0;
    const u32 pos = (0x534120u >> (4 * wid)) & 7u;
    for (u32 n = pos;; n += (u32)ND) {
        while (rfl(rw[R_FULL]) <= n) __builtin_amdgcn_s_sleep(1);
        CFENCE;
        const u32x4 d0 = ring_desc(lds, n)[0], d1 = ring_desc(lds, n)[1];
        const u32 type = rfl(d0.w), hdr = rfl(d0.z);
        if (type == T_END) break;
        char* dst = (char*)(((u64)d1.y << 32) | d1.x);
        if (type == T_IN && rfl(d1.z) < (u32)W_XCNT) __builtin_amdgcn_s_setprio(1); else __builtin_amdgcn_s_setprio(0);
        LAS unsigned char* sl = lds + (n % (u32)NSLOT) * (u32)USLOT;
        u32 u[32];
        if (type <= T_OUT && (hdr & H_RAW)) {
#pragma unroll
            for (int k = 0; k < 8; ++k) { const u32x4 v = *(const LAS u32x4*)(sl + k * 1024 + lane * 16); u[4 * k] = v.x; u[4 * k + 1] = v.y; u[4 * k + 2] = v.z; u[4 * k + 3] = v.w; }
        } else if (type == T_IN) {
            const u32x4 dn = *(const LAS u32x4*)(sl + 4096 + lane * 16);
            u32x4 lo[4];
#pragma unroll
            for (int k = 0; k < 4; ++k) lo[k] = *(const LAS u32x4*)(sl + k * 1024 + lane * 16);
            const LAS u32* st = (const LAS u32*)(sl + 5120);
            u32 pl[4][2], c0[2], c1[2], base = 0;
            pl[0][0] = dn.z; pl[0][1] = dn.w;
#pragma unroll
            for (int lv = 0; lv < 4; ++lv) {
                const u32 n0 = popc(pl[lv][0]), n1 = popc(pl[lv][1]), cn = lv < 3 ? n0 + n1 : 2u * (n0 + n1);
                const u32 inc = scan_add(cn), o = base + inc - cn;
                const u32 g0 = bit_get(st, o), g1 = bit_get(st, o + (lv < 3 ? n0 : 2u * n0));
                base += rdl(inc, 63);
                const Msk k0 = msk<true>(pl[lv][0]), k1 = msk<true>(pl[lv][1]);
                if (lv < 3) { pl[lv + 1][0] = spread(g0, k0); pl[lv + 1][1] = spread(g1, k1); }
                else { c0[0] = spread(g0, k0); c1[0] = spread(g0 >> (n0 & 31u), k0); c0[1] = spread(g1, k1); c1[1] = spread(g1 >> (n1 & 31u), k1); }
            }
            const u32 b4 = (((hdr >> 10) & 0x7fu) - 6u) * 0x01010101u;
            const u32 tl = b4 + 0x07040506u, th = b4 + 0x00010203u;
            u32 hi[16];
#pragma unroll
            for (int h = 0; h < 2; ++h) {
                u32 nn[4];
                from_planes(nn, dn[h], c0[h] | (pl[0][h] & ~pl[1][h]) | (pl[2][h] & ~pl[3][h]), c1[h] | (pl[1][h] & ~pl[3][h]), pl[3][h]);
#pragma unroll
                for (int k = 0; k < 4; ++k) {
                    hi[8 * h + 2 * k] = bfi(0x80808080u, nn[k] << 7, __builtin_amdgcn_perm(th, tl, (nn[k] >> 1) & 0x07070707u));
                    hi[8 * h + 2 * k + 1] = bfi(0x80808080u, nn[k] << 3, __builtin_amdgcn_perm(th, tl, (nn[k] >> 5) & 0x07070707u));
                }
            }
            const u32 nexc = (hdr >> 17) & 0xffu;
            if (nexc) {
                const u32 ev = st[min((base + 31u) >> 5, MAXSDW) + (u32)lane];
                for (u32 e = 0; e < nexc; ++e) {
                    const u32 en = rdl(ev, (int)e);
                    const u32 sh = ((en >> 7) & 3u) * 8u, kk = (en >> 9) & 15u;
                    const bool me = (en >> 13) == (u32)lane;
#pragma unroll
                    for (int k = 0; k < 16; ++k)
                        if (me && kk == (u32)k) hi[k] = bfi(0x7fu << sh, (en & 0x7fu) << sh, hi[k]);
                }
            }
#pragma unroll
            for (int k = 0; k < 16; ++k) {
                u[2 * k] = __builtin_amdgcn_perm(hi[k], lo[k >> 2][k & 3], 0x05010400u);
                u[2 * k + 1] = __builtin_amdgcn_perm(hi[k], lo[k >> 2][k & 3], 0x07030602u);
            }
        } else if (type == T_OUT) {
            *(volatile LAS u32x4*)(bm + lane * 4) = (u32x4){0u, 0u, 0u, 0u};
            const u32x4 d0v = *(const LAS u32x4*)(sl + 4096 + lane * 16), d1v = *(const LAS u32x4*)(sl + 5120 + lane * 16);
            u32x4 lo[4];
#pragma unroll
            for (int k = 0; k < 4; ++k) lo[k] = *(const LAS u32x4*)(sl + k * 1024 + lane * 16);
            const LAS u32* st = (const LAS u32*)(sl + 6144);
            u32 f5[4], g[4];
            const u32x4 e3 = d0v & d1v;
            u32 cn = popc(e3.x) + popc(e3.y) + popc(e3.z) + popc(e3.w), inc = scan_add(cn), o = inc - cn;
#pragma unroll
            for (int h = 0; h < 4; ++h) { g[h] = bit_get(st, o); o += popc(e3[h]); }
            u32 base = rdl(inc, 63);
#pragma unroll
            for (int h = 0; h < 4; ++h) f5[h] = spread(g[h], msk<true>(e3[h]));
            u32 a3[2], a4[2], u5[2];
            {
                u32 n0 = popc(f5[0]) + popc(f5[1]), n1 = popc(f5[2]) + popc(f5[3]);
                cn = n0 + n1; inc = scan_add(cn); o = base + inc - cn; base += rdl(inc, 63);
                u5[0] = __builtin_amdgcn_ubfe(bit_get(st, o), 0u, n0); u5[1] = __builtin_amdgcn_ubfe(bit_get(st, o + n0), 0u, n1);
                n0 = popc(u5[0]); cn = n0 + popc(u5[1]); inc = scan_add(cn); o = base + inc - cn; base += rdl(inc, 63);
                a4[0] = spread(bit_get(st, o), msk<true>(u5[0])); a4[1] = spread(bit_get(st, o + n0), msk<true>(u5[1]));
                n0 = popc(a4[0]); cn = n0 + popc(a4[1]); inc = scan_add(cn); o = base + inc - cn; base += rdl(inc, 63);
                a3[0] = spread(bit_get(st, o), msk<true>(a4[0])) | (u5[0] & ~a4[0]); a3[1] = spread(bit_get(st, o + n0), msk<true>(a4[1])) | (u5[1] & ~a4[1]);
            }
            const u32 nexc = (hdr >> 17) & 0xffu;
            const u32 ev = st[min((base + 31u) >> 5, MAXSDW) + (u32)lane];
            if (2u * (u32)lane < nexc) __hip_atomic_fetch_or(bm + ((ev >> 5) & 255u), 1u << (ev & 31u), __ATOMIC_RELAXED, __HIP_MEMORY_SCOPE_WORKGROUP);
            if (2u * (u32)lane + 1u < nexc) __hip_atomic_fetch_or(bm + ((ev >> 21) & 255u), 1u << ((ev >> 16) & 31u), __ATOMIC_RELAXED, __HIP_MEMORY_SCOPE_WORKGROUP);
            CFENCE;
            const u32x4 m7 = *(volatile LAS u32x4*)(bm + lane * 4);
#pragma unroll
            for (int h = 0; h < 4; ++h) {
                const Msk k = msk<true>(f5[h]);
                const u32 sh = (h & 1) ? popc(f5[h - 1]) & 31u : 0u;
                const u32 f3 = spread(a3[h >> 1] >> sh, k) | (d0v[h] & ~f5[h]), f4 = spread(a4[h >> 1] >> sh, k) | (d1v[h] & ~f5[h]);
                u32 nn[4];
                from_planes(nn, f4 ^ f3, f5[h] ^ f3, f3, m7[h] ^ f3);
#pragma unroll
                for (int k2 = 0; k2 < 4; ++k2) {
                    const u32 L = lo[h][k2], t = (L >> 3) & 0x11111111u, H = nn[k2] ^ times15(t);
                    u[8 * h + 2 * k2] = bfi(0x0f0f0f0fu, L, H << 4); u[8 * h + 2 * k2 + 1] = bfi(0x0f0f0f0fu, L >> 4, H);
                }
            }
        } else {
#pragma unroll
            for (int k = 0; k < 32; ++k) u[k] = 0x80808080u;
        }
        WAIT_L(0);
        ++done;
        if (lane == 0) rw[R_DONE + pos] = done;
        if (type == T_IN) {
            st_unit(dst + lane * 16, u);
            WAIT_V(0);
            if (lane == 0) add4_sys(p.ws + d1.z, 1u);
        } else if (type != T_SKIP) {
            char* o = dst + (size_t)(lane & 15) * GN + (lane >> 4) * 16;
#pragma unroll
            for (int k = 0; k < 8; ++k)
                st16_sys<0>(o + (size_t)k * 16 * GN, (u32x4){u[4 * k], u[4 * k + 1], u[4 * k + 2], u[4 * k + 3]});
        }
    }
}

__device__ __forceinline__ void gemm_tile(const Params& p, LAS unsigned char* lds, u32 call, bool fillm, int pm, int pn, int wid, int lane) {
    const int wr = wid >> 2, wc = wid & 3;
    int j0 = 0;
#pragma unroll
    for (int j = 1; j < 7; ++j) if (rot_of(j) <= pm) j0 = j;
    const char* Ab = p.ws + W_A + (size_t)pm * 64 * PANEL;
    const char* Bb = p.ws + (fillm ? W_X : W_B) + (size_t)pn * 64 * PANEL;
    const char* cA = p.ws + W_ACNT + (size_t)pm * 256;
    const char* cB = p.ws + (fillm ? W_XCNT : W_BCNT) + (size_t)pn * 256;
    const u32 voff = (u32)(wid * 4096 + lane * 16);
    const u32 ldsw = (u32)wid * 2048u;
    const int aoff = wr * 8192 + ((lane & 15) * 4 + ((lane >> 4) ^ ((lane >> 2) & 2))) * 16;
    int boff = 65536 + wc * 4096 + lane * 16;
    asm volatile("" : "+v"(boff));
    int ktv = kt_of(lane, p.rank, j0);
    asm volatile("" : "+v"(ktv));
    const u32 cix = (u32)((ktv & 7) * 8 + (ktv >> 3)) * 4u;
    u64 rdyA = wid == 4 ? 0ull : ~0ull, rdyB = rdyA;
    const u32 needA = call * 4u, needB = (fillm && pn == NVP - 1) ? call * (u32)(7 * FC - 4 * (NVP - 1)) : needA;
#define SA(b, h) (((b) * 2 + (h)) * 16384)
#define SB(b, h) ((4 + (b) * 2 + (h)) * 16384)
#define STAGE(buf, gbase, h) do { \
        __builtin_amdgcn_global_load_lds((const u32*)((gbase) + voff + (h) * 2048), (LAS u32*)(lds + (buf) + ldsw), 16, 0, 17); \
        __builtin_amdgcn_global_load_lds((const u32*)((gbase) + voff + (h) * 2048), (LAS u32*)(lds + (buf) + ldsw), 16, 1024, 17); } while (0)
#define LDA(dst, b, h) do { _Pragma("unroll") for (int m = 0; m < 4; ++m) _Pragma("unroll") for (int k = 0; k < 2; ++k) \
        dst[m][k] = *(const LAS bf16x8*)(lds + SA(b, h) + aoff + m * 2048 + k * 1024); } while (0)
#define LDB(dst, b, h) do { _Pragma("unroll") for (int n = 0; n < 2; ++n) _Pragma("unroll") for (int k = 0; k < 2; ++k) \
        dst[n][k] = *(const LAS bf16x8*)(lds + (SB(b, h) - 65536) + boff + k * 2048 + n * 1024); } while (0)
#define MMA(ai, bj, At, Bt) do { __builtin_amdgcn_s_setprio(1); \
        _Pragma("unroll") for (int m = 0; m < 4; ++m) _Pragma("unroll") for (int n = 0; n < 2; ++n) _Pragma("unroll") for (int k = 0; k < 2; ++k) \
            acc[ai][bj][m][n] = __builtin_amdgcn_mfma_f32_16x16x32_bf16(Bt[n][k], At[m][k], acc[ai][bj][m][n], 0, 0, 0); \
        __builtin_amdgcn_s_setprio(0); } while (0)
#define CHECK(RDY, CP, NEED, o) do { if (!((RDY >> (o)) & 1ull)) { \
        const u64 t0_ = now(); __builtin_amdgcn_s_waitcnt(0xC07F); \
        for (u32 it_ = 1;; ++it_) { \
            u32 a_ = 0; poll_win(a_, CP + cix, 0xffffull << (o)); \
            RDY |= __ballot(a_ >= NEED); \
            if ((RDY >> (o)) & 1ull) break; \
            if (spin_fail(p.ws, call, t0_, (it_ & 63u) ? 0u : ld4_wait(p.ws + W_TMO))) { rdyA = rdyB = ~0ull; gave_up = true; break; } \
        } if (STAMPS > 1) { wacc += now() - t0_; __builtin_amdgcn_s_waitcnt(0xC07F); } } } while (0)
#define CHECKA(o) CHECK(rdyA, cA, needA, o)
#define CHECKB(o) CHECK(rdyB, cB, needB, o)
#define PEEK(o) do { pk = (u32)((~rdyA >> (((o) + 1) & 63)) | (~rdyB >> (((o) + 2) & 63))) & 1u; u64 sv_; \
        asm volatile("s_cmp_eq_u32 %5, 0\n\ts_cbranch_scc1 .Lpk%=\n\ts_mov_b64 %2, exec\n\ts_mov_b64 exec, %6\n\t" \
                     "global_load_dword %0, %3, off sc0 sc1\n\tglobal_load_dword %1, %4, off sc0 sc1\n\ts_mov_b64 exec, %2\n.Lpk%=:" \
                     : "+v"(pa), "+v"(pb), "=&s"(sv_) : "v"(cA + cix), "v"(cB + cix), "s"(pk), "s"(0xffffull << (o)) : "memory", "scc"); } while (0)
#define WAIT_PK asm volatile("s_cmp_lg_u32 %0, 0\n\ts_cbranch_scc1 .Lpw%=\n\ts_waitcnt vmcnt(6)\n.Lpw%=:\n\ts_waitcnt vmcnt(8)" :: "s"(pk) : "memory", "scc")
#define KOFF(o) ((size_t)__builtin_amdgcn_readlane(ktv, (o)) * PANEL)

    u64 wacc = 0;
    f32x4 acc[2][2][4][2];
#pragma unroll
    for (int a = 0; a < 2; ++a)
#pragma unroll
        for (int b = 0; b < 2; ++b)
#pragma unroll
            for (int m = 0; m < 4; ++m)
#pragma unroll
                for (int n = 0; n < 2; ++n) acc[a][b][m][n] = (f32x4){0.f, 0.f, 0.f, 0.f};
    bf16x8 At[4][2], B0[2][2], B1[2][2];
    u32 pa = 0, pb = 0, pk;
    bool gave_up = false;

    CHECKA(0); CHECKB(0); CHECKB(1); CHECKA(1); CHECKB(2);
    BAR; CFENCE;
    size_t k1 = KOFF(1);
    {
        const size_t k0 = KOFF(0);
        STAGE(SB(0, 0), Bb + k0, 0); STAGE(SB(0, 1), Bb + k0, 1); STAGE(SA(0, 0), Ab + k0, 0);
        STAGE(SB(1, 0), Bb + k1, 0); STAGE(SA(0, 1), Ab + k0, 1); STAGE(SB(1, 1), Bb + k1, 1);
        WAIT_V(6); BAR;
        if (wr == 1) BAR;
    }
#define BOOK(oa) do { rdyA |= __ballot(pa >= needA); rdyB |= __ballot(pb >= needB); CHECKA(oa); CHECKB(((oa) + 1) & 63); PEEK(oa); } while (0)
#define WAIT_P2 asm volatile("s_waitcnt vmcnt(6)" : "+v"(pa), "+v"(pb) :: "memory")
    size_t k2 = KOFF(2);
    for (int t = 0; t < 64; t += 2) {
        const int o2 = (t + 2) & 63, o3 = o2 + 1;
        LDB(B0, 0, 0); SCHED; LDA(At, 0, 0); STAGE(SA(1, 0), Ab + k1, 0);
        WAIT_L(8); BAR; WAIT_L(0); MMA(0, 0, At, B0); BAR; SCHED;
        LDB(B1, 0, 1); STAGE(SB(0, 0), Bb + k2, 0);
        WAIT_P2; BAR; WAIT_L(0); MMA(0, 1, At, B1); BAR;
        LDA(At, 0, 1); STAGE(SA(1, 1), Ab + k1, 1);
        BAR; WAIT_L(0); MMA(1, 0, At, B0); BAR; SCHED;
        BOOK(o2); STAGE(SB(0, 1), Bb + k2, 1);
        const size_t k3 = KOFF(o3);
        WAIT_PK; BAR; MMA(1, 1, At, B1); BAR;
        LDB(B0, 1, 0); SCHED; LDA(At, 1, 0); STAGE(SA(0, 0), Ab + k2, 0);
        WAIT_L(8); BAR; WAIT_L(0); MMA(0, 0, At, B0); BAR; SCHED;
        LDB(B1, 1, 1); STAGE(SB(1, 0), Bb + k3, 0);
        WAIT_P2; BAR; WAIT_L(0); MMA(0, 1, At, B1); BAR;
        LDA(At, 1, 1); STAGE(SA(0, 1), Ab + k2, 1);
        BAR; WAIT_L(0); MMA(1, 0, At, B0); BAR; SCHED;
        BOOK(o3); STAGE(SB(1, 1), Bb + k3, 1);
        k1 = k3; k2 = KOFF((o3 + 1) & 63);
        WAIT_PK; BAR; MMA(1, 1, At, B1); BAR;
    }
    asm volatile("s_waitcnt vmcnt(2)" : "+v"(pa), "+v"(pb) :: "memory");
    if (wr == 0) BAR;
    if (!fillm) STAMP(2);
    int tz = threadIdx.x; asm volatile("" : "+v"(tz));
    lane = tz & 63;
    const int tile = pm * 8 + pn;
    volatile LAS u32* red = (volatile LAS u32*)(lds + LDS_MISC + 1024);
    if (!fillm) {
        float mx = 0.f;
#pragma unroll
        for (int a = 0; a < 2; ++a)
#pragma unroll
            for (int b = 0; b < 2; ++b)
#pragma unroll
                for (int m = 0; m < 4; ++m)
#pragma unroll
                    for (int n = 0; n < 2; ++n)
#pragma unroll
                        for (int r = 0; r < 4; ++r) mx = fmaxf(mx, fabsf(acc[a][b][m][n][r]));
        const u32 wm = wave_umax(__float_as_uint(mx));
        if (lane == 0) red[wid] = wm;
        WAIT_L(0); BAR; CFENCE;
        if (tz == 0) {
            const u32x4 r0 = *(volatile LAS u32x4*)red, r1 = *(volatile LAS u32x4*)(red + 4);
            const u32 t = max(max(max(r0.x, r0.y), max(r0.z, r0.w)), max(max(r1.x, r1.y), max(r1.z, r1.w)));
            st8_sys(p.ws + W_AMAX + (size_t)tile * 8, (u32x2){t, call});
        }
    }
    if (wid == 0) {
        const char* src = p.ws + W_GMAX + (blockIdx.x & (GCOPY - 1)) * 256 + (lane & 7) * 8;
        const u64 t0 = now();
        u32x2 gq;
        bool dd = false;
        for (u32 it = 1;; ++it) {
            gq = ld8_wait(src);
            if (__all(gq.y == call)) break;
            if (spin_fail(p.ws, call, t0, (it & 63u) ? 0u : ld4_wait(p.ws + W_TMO))) { dd = true; break; }
        }
        const u32 wm = wave_umax(gq.x);
        if (lane == 0) red[8] = dd ? 0xffffffffu : wm;
    } else if (wid == 4) {
        if (lane == 0) red[9] = gave_up ? 1u : 0u;
        WAIT_V(0);
    } else {
        WAIT_V(0);
    }
    WAIT_L(0); BAR; CFENCE;
    const u32 am = rfl(red[8]);
    const bool dead = am == 0xffffffffu || rfl(red[9]) != 0u;
    const float amax = __uint_as_float(am);
    if (!fillm) STAMP(3);
    if (wid < 2) __builtin_amdgcn_s_setprio(3);
    const float inv = amax > 0.f ? 127.0f / amax : 0.f;
    u32 q[32];
#pragma unroll
    for (int m = 0; m < 4; ++m)
#pragma unroll
        for (int a = 0; a < 2; ++a)
#pragma unroll
            for (int r = 0; r < 4; ++r) {
                const u32 t0 = __float_as_uint(fmaf(acc[a][0][m][0][r], inv, 12582912.f)), t1 = __float_as_uint(fmaf(acc[a][0][m][1][r], inv, 12582912.f));
                const u32 t2 = __float_as_uint(fmaf(acc[a][1][m][0][r], inv, 12582912.f)), t3 = __float_as_uint(fmaf(acc[a][1][m][1][r], inv, 12582912.f));
                q[(m * 2 + a) * 4 + r] = __builtin_amdgcn_perm(t1, t0, 0x0c0c0400u) | __builtin_amdgcn_perm(t3, t2, 0x04000c0cu);
            }
    if (dead) {
#pragma unroll
        for (int k = 0; k < 32; ++k) q[k] = 0x80808080u;
    }
    int col = p.rank * NS + pn * 256 + wc * 64;
    if (!fillm) {
        char* slot = p.ws + W_PO + (size_t)(tile * 8 + wid) * USLOT;
        LAS unsigned char* enc = lds + LDS_ENC + wid * 2048;
        *(volatile LAS u32x4*)(enc + lane * 16) = (u32x4){0u, 0u, 0u, 0u};
        *(volatile LAS u32x4*)(enc + 1024 + lane * 16) = (u32x4){0u, 0u, 0u, 0u};
        char* dp = slot + lane * 16;
        u32 hdr;
        const int yrow = pn < 7 ? pn * 3 + (wid >> 1) - 1 : 21;
        if ((wid < 2 && pn == 0 && pm < NRAW) || (pm == 15 && (wid >= 2 || pn == 7) && yrow >= (int)NP_ROW - 7 - NRAWE)) {
            st_unit(dp, q);
            hdr = H_RAW | 512u;
        } else {
        u32 L[16], G[16], f3[4], f4[4], f5[4], m7[4];
#pragma unroll
        for (int j = 0; j < 16; ++j) {
            L[j] = bfi(0x0f0f0f0fu, q[2 * j], q[2 * j + 1] << 4);
            const u32 t = (L[j] >> 3) & 0x11111111u;
            G[j] = bfi(0x0f0f0f0fu, q[2 * j] >> 4, q[2 * j + 1]) ^ times15(t);
        }
        u32 ca = 0, cb = 0, cc = 0, wide = 0;
#pragma unroll
        for (int h = 0; h < 4; ++h) {
            to_planes(G + 4 * h, f4[h], f5[h], f3[h], m7[h]);
            f4[h] ^= f3[h]; f5[h] ^= f3[h]; m7[h] ^= f3[h];
            ca += popc(f5[h] | (f4[h] & f3[h])) | (popc(m7[h]) << 16); cb += popc(f5[h]) | (popc(f5[h] & (f4[h] | f3[h])) << 16); cc += popc(f5[h] & f4[h]);
            wide = max(wide, popc(f5[h]));
        }
        const u32 ia = scan_add(ca), ib = scan_add(cb), ic = scan_add(cc), ta = rdl(ia, 63), tb = rdl(ib, 63), exc = ia - ca;
        const u32 bb = ta & 0xffffu, bc = bb + (tb & 0xffffu), bd = bc + (tb >> 16);
        const u32 sdw = (bd + rdl(ic, 63) + 31u) >> 5, nexc = ta >> 16;
        if (nexc > 128u || sdw > MAXSDW || __any(wide > 15u)) {
            st_unit(dp, q);
            hdr = H_RAW | 512u;
        } else {
            st16_sys<0>(dp, (u32x4){L[0], L[1], L[2], L[3]}); st16_sys<1024>(dp, (u32x4){L[4], L[5], L[6], L[7]});
            st16_sys<2048>(dp, (u32x4){L[8], L[9], L[10], L[11]}); st16_sys<3072>(dp, (u32x4){L[12], L[13], L[14], L[15]});
            st16_sys<0>(dp + 4096, (u32x4){f3[0] | f5[0], f3[1] | f5[1], f3[2] | f5[2], f3[3] | f5[3]});
            st16_sys<1024>(dp + 4096, (u32x4){f4[0] | f5[0], f4[1] | f5[1], f4[2] | f5[2], f4[3] | f5[3]});
            u32 oa = exc & 0xffffu, ob = bb + ((ib - cb) & 0xffffu), oc = bc + ((ib - cb) >> 16), od = bd + ic - cc;
#pragma unroll
            for (int h = 0; h < 4; ++h) {
                const u32 e3 = f5[h] | (f4[h] & f3[h]);
                bit_put((LAS u32*)enc, oa, press(f5[h], msk<false>(e3))); oa += popc(e3);
            }
#pragma unroll
            for (int h = 0; h < 4; h += 2) {
                const Msk k0 = msk<false>(f5[h]), k1 = msk<false>(f5[h + 1]);
                const u32 n0 = popc(f5[h]);
                const u32 a3 = press(f3[h] & f5[h], k0) | (press(f3[h + 1] & f5[h + 1], k1) << n0);
                const u32 a4 = press(f4[h] & f5[h], k0) | (press(f4[h + 1] & f5[h + 1], k1) << n0);
                bit_put((LAS u32*)enc, ob, a3 | a4); ob += n0 + popc(f5[h + 1]);
                bit_put((LAS u32*)enc, oc, press(a4, msk<false>(a3 | a4))); oc += popc(a3 | a4);
                bit_put((LAS u32*)enc, od, press(a3 & a4, msk<false>(a4))); od += popc(a4);
            }
            volatile LAS unsigned short* list = (volatile LAS unsigned short*)(enc + sdw * 4u);
            u32 s = exc >> 16;
#pragma unroll
            for (int h = 0; h < 4; ++h) {
                u32 mm = m7[h];
                while (mm) { list[s] = (unsigned short)((lane * 4 + h) * 32 + __builtin_ctz(mm)); mm &= mm - 1u; ++s; }
            }
            const u32 n16 = (sdw + ((nexc + 1u) >> 1) + 3u) >> 2;
            tail_out(slot + 6144, enc, n16, lane);
            hdr = (384u + n16) | (nexc << 17);
        }
        }
        if (lane == 0) red[16 + wid] = hdr;
        WAIT_V(0);
        const bool early = pn < NEARLY;
        if (early && wid < 2 && lane < NR && lane != p.rank)
            st8_sys(peer_ws(lds, (u32)lane) + W_HO + (size_t)((p.rank * 128 + tile) * 8 + wid) * 8, (u32x2){hdr, call});
        __builtin_amdgcn_s_setprio(0);
        WAIT_L(0); BAR; CFENCE;
        if (tz < 64 && (tz >> 3) != p.rank && (!early || (tz & 7) >= 2))
            st8_sys(peer_ws(lds, (u32)(tz >> 3)) + W_HO + (size_t)((p.rank * 128 + tile) * 8 + (tz & 7)) * 8, (u32x2){red[16 + (tz & 7)], call});
        STAMP(4);
    } else {
        const int g = pn * 4 + wc;
        col = g < 7 * FC ? ((p.rank + 1 + g / FC) & 7) * NS + 7 * 256 + (4 - FC + g % FC) * 64 : -1;
    }
    if (col >= 0) {
        signed char* o = p.out + (size_t)(pm * 256 + wr * 128 + (lane & 15)) * GN + col + (lane >> 4) * 16;
#pragma unroll
        for (int k = 0; k < 8; ++k)
            st16_sys<0>(o + (size_t)k * 16 * GN, (u32x4){q[4 * k], q[4 * k + 1], q[4 * k + 2], q[4 * k + 3]});
    }
#undef SA
#undef SB
#undef STAGE
#undef LDA
#undef LDB
#undef MMA
#undef CHECK
#undef CHECKA
#undef CHECKB
#undef PEEK
#undef BOOK
#undef WAIT_P2
#undef WAIT_PK
#undef KOFF
}

__global__ void __launch_bounds__(BLOCK, 2) fused_gemm_q8(Params p) {
    __shared__ __attribute__((aligned(16))) unsigned char smem[LDS_BYTES];
    LAS unsigned char* lds = (LAS unsigned char*)smem;
    const int tid = threadIdx.x, wid = (int)rfl((u32)tid >> 6), lane = tid & 63;
    const int xcd = blockIdx.x & 7, idx = blockIdx.x >> 3;
    STAMP(0);
    const u64 tstart = now();
    volatile LAS u32* misc = (volatile LAS u32*)(lds + LDS_MISC);
    LAS unsigned char* enc = lds + LDS_ENC + wid * 2048;
    if (wid == 7) {
        if (lane < 4) misc[lane] = 0u;
        if (lane < 16) ring_words(lds)[lane] = 0u;
        ((volatile LAS u32*)(lds + LDS_MISC + 1280))[lane] = 0u;
        WAIT_L(0);
    }
    BAR; CFENCE;
    int i0 = -1, r0 = 0, mode0 = 2;
    const int u0 = (int)blockIdx.x & 3;
    if (idx < 16) {
        if (wid < 6) { const int n = (wid * 128 + (int)blockIdx.x) >> 2; r0 = n / 24; i0 = 56 + n - r0 * 24; }
    } else if (wid < 3) {
        constexpr int NE = 7 * (8 - NPUSH) + 17;
        const int n = (wid * 128 + (int)blockIdx.x - 128) >> 2;
        if (n < NE) { mode0 = 1; i0 = n < NE - 17 ? (n % 7) * 8 + NPUSH + n / 7 : (n < NE - 1 ? 56 + n - (NE - 17) : 79); }
        else if (n < 73) { mode0 = 4; const int m = n - NE; i0 = (m % 7) * 8 + m / 7; }
    }
    if (i0 >= 0) {
        const PanelId d0 = panel_of(r0, i0, p.rank);
        if (mode0 == 2) { while (now() - tstart < GDLY) __builtin_amdgcn_s_sleep(1); }
        f32x4 f[16];
        unit_load(p, d0, u0, lane, f);
        WAIT_V(0); PIN_F(f);
        if (STAMPS == 1) STAMP(10);
        const u32 hdr = unit_pack(p, lds, enc, d0, u0, lane, f, mode0);
        if (STAMPS == 1) STAMP(11);
        u32 call0;
        while ((call0 = rfl(misc[2])) == 0u) __builtin_amdgcn_s_sleep(1);
        WAIT_V(0);
        if (lane < NR) unit_announce(p, lds, d0, u0, lane, hdr, call0, mode0);
    } else {
        if (wid == 7) {
            u32 tk; u32x2 pw;
            ticket_issue(tk, p.ws + W_SEQ + (size_t)blockIdx.x * 128);
            ld8_sys(pw, (const char*)p.ws_all + (lane & 7) * 8);
            asm volatile("s_waitcnt vmcnt(0)" : "+v"(tk), "+v"(pw) :: "memory");
            if (lane < NR) *(volatile LAS u32x2*)(lds + LDS_MISC + 64 + lane * 8) = pw;
            const u32 call7 = rfl(tk) + 1u;
            if (lane == 0) misc[2] = call7;
            if (blockIdx.x == 0) {
                if (lane == 0) st8_sys(p.ws + W_ROT, (u32x2){(u32)__builtin_amdgcn_s_getreg(20 | (3 << 11)) & 7u, call7});
            } else if (idx >= 16 && idx < 24) {
                const u64 t0 = now();
                u32x2 ro;
                for (;;) {
                    u32 t; poll8(ro, t, p.ws + W_ROT, p.ws + W_TMO);
                    if (rfl(ro.y) == call7) break;
                    if (spin_fail(p.ws, call7, t0, t)) { ro.x = 8u; break; }
                }
                if (lane == 0) misc[1] = ro.x;
            }
            if (lane == 0) misc[0] = call7;
        }
    }
    u32 call;
    while ((call = rfl(misc[0])) == 0u) __builtin_amdgcn_s_sleep(1);
    CFENCE;
    STAMP(6);

    const bool is_gemm = idx < 16;
    const int c = idx - 16;
    const u32 rot0 = is_gemm || idx >= 24 ? 0u : rfl(misc[1]);
    if (rot0 > 7u) return;
    const int jl = (peer_of((int)((xcd + rot0) & 7u), p.rank) - p.rank - 1) & 7;
    if (!is_gemm && jl < 7 && c < NPW) {
        if (wid == 7) pull_schedule(p, lds, call, jl, c, wid, lane);
        else if (wid == 6) pull_load(p, lds, wid, lane);
        else pull_decode(p, lds, wid, lane);
        STAMP(5);
        return;
    }
    const int pm = (xcd >> 1) * 4 + (idx >> 2), pn = (xcd & 1) * 4 + (idx & 3);
    if (!is_gemm) {
        const int me = idx >= 24 ? (int)blockIdx.x - 192 : 64 + c, unit = wid & 3;
        int e = (wid >> 2) * NSPARE + me;
        volatile LAS u32* hb = (volatile LAS u32*)(lds + LDS_MISC + 1280 + (wid >> 2) * 128);
        auto panel_at = [&](int v) { const int r = v / 73, i = v - r * 73; return panel_of(1 + r, i < 72 ? i : 79, p.rank); };
        PanelId d = panel_at(e);
        f32x4 f[16];
        while (now() - tstart < (wid < 4 ? SPD0 : SPD1)) __builtin_amdgcn_s_sleep(4);
        unit_load(p, d, unit, lane, f);
        bool first = true;
        for (;;) {
            WAIT_V(0); PIN_F(f);
            if (STAMPS == 1 && first) STAMP(7);
            const u32 hdr = unit_pack(p, lds, enc, d, unit, lane, f, 1);
            if (STAMPS == 1 && first) STAMP(8);
            e += 2 * NSPARE;
            const PanelId dn = panel_at(e);
            if (e < 7 * 73) { unit_load(p, dn, unit, lane, f); WAIT_V(16); } else { WAIT_V(0); }
            u32 arr = 0;
            if (lane == 0) { hb[4 + unit] = hdr; arr = __hip_atomic_fetch_add((LAS u32*)hb, 1u, __ATOMIC_RELAXED, __HIP_MEMORY_SCOPE_WORKGROUP); }
            if (rfl(arr) == 3u && lane < 4 * NR) unit_announce(p, lds, d, lane & 3, lane >> 2, hb[4 + (lane & 3)], call, 1);
            hb += 8;
            if (STAMPS == 1 && first) STAMP(9);
            first = false;
            if (e >= 7 * 73) break;
            d = dn;
        }
        STAMP(1);
        if (blockIdx.x == GRID - 1 && wid == 0) {
            const u64 t0 = now();
            while (now() - tstart < 6000) __builtin_amdgcn_s_sleep(16);
            for (u32 it = 1;; ++it) {
                const u32x4 gq = ld16_wait(p.ws + W_AMAX + lane * 16);
                if (__all(gq.y == call && gq.w == call)) {
                    const u32 wm = wave_umax(max(gq.x, gq.z));
                    if (lane < NR * GCOPY) st8_sys(peer_ws(lds, (u32)lane & 7u) + W_GMAX + (lane >> 3) * 256 + p.rank * 8, (u32x2){wm, call});
                    break;
                }
                if (spin_fail(p.ws, call, t0, (it & 63u) ? 0u : ld4_wait(p.ws + W_TMO))) break;
            }
        }
        return;
    }
#pragma clang loop unroll(disable)
    for (int pass = 0; pass < (pn < NVP ? 2 : 1); ++pass) {
        int tz = threadIdx.x; asm volatile("" : "+v"(tz));
        gemm_tile(p, lds, call, pass == 1, pm, pn, wid, tz & 63);
        if (pass == 0) STAMP(5); else STAMP(13);
    }
}

extern "C" void kernel_launch(int rank, int nranks, ncclComm_t comm, hipStream_t stream,
                              void* const* d_in, const int* in_sizes, int n_in,
                              void* d_out, int out_size,
                              const int* in_dtypes, int out_dtype,
                              void* const* d_out_all,
                              void* const* d_ws_all, size_t ws_bytes) {
    (void)comm; (void)in_dtypes; (void)out_dtype; (void)d_out_all;
    if (nranks != NR || n_in < 2 || ws_bytes < W_END || in_sizes[0] != GM * KS || in_sizes[1] != KS * GN || out_size != GM * GN ||
        (unsigned)rank >= (unsigned)NR) {
        fprintf(stderr, "unexpected problem geometry\n");
        return;
    }
    Params p;
    p.x = (const float*)d_in[0];
    p.w = (const float*)d_in[1];
    p.out = (signed char*)d_out;
    p.ws = (char*)d_ws_all[rank];
    p.ws_all = (char* const*)d_ws_all;
    p.rank = rank;
    p.pad = 0;
    hipLaunchKernelGGL(fused_gemm_q8, dim3(GRID), dim3(BLOCK), 0, stream, p);
}
```

```cpp
#include <hip/hip_runtime.h>
#include <rccl/rccl.h>
#include <cstdio>
#include <cstdint>

#define LAS __attribute__((address_space(3)))
typedef unsigned u32;
typedef unsigned long long u64;
typedef u32 u32x4 __attribute__((ext_vector_type(4)));
typedef u32 u32x2 __attribute__((ext_vector_type(2)));
typedef float f32x4 __attribute__((ext_vector_type(4)));
typedef short bf16x8 __attribute__((ext_vector_type(8)));

constexpr int NR = 8, GM = 4096, GN = 16384, KS = 512, NS = 2048;
constexpr int GRID = 256, BLOCK = 512;
constexpr int PANEL = 32768;
constexpr int USLOT = 8192;
constexpr size_t W_TMO = 128;
constexpr size_t W_ROT = 24576;
constexpr size_t W_SQ = 384;
constexpr size_t W_Q = 1024;
constexpr size_t W_ACNT = 4096;
constexpr size_t W_BCNT = 8192;
constexpr size_t W_XCNT = 10240;
constexpr size_t W_SCNT = 12288;
constexpr size_t W_AMAX = 16384;
constexpr size_t W_GMAX = 20480;
constexpr size_t W_HI = 32768;
constexpr size_t W_HX = 81920;
constexpr size_t W_HO = 114688;
constexpr size_t W_HS = 180224;
constexpr size_t W_SEQ = 256 << 10;
constexpr size_t W_A = 1ull << 20;
constexpr size_t W_B = 33ull << 20;
constexpr size_t W_X = 49ull << 20;
constexpr size_t W_PX = 65ull << 20;
constexpr size_t W_PW = 69ull << 20;
constexpr size_t W_PO = 87ull << 20;
constexpr size_t W_PS = 95ull << 20;
constexpr size_t W_SG = 96ull << 20;
constexpr size_t W_END = 104ull << 20;
constexpr u64 DEADLINE = 100000000ull;

constexpr int NPW = 8;
constexpr int DEPTH = 7;
constexpr int DEPTH2 = 4;
constexpr int RUNAHEAD = 2;
constexpr int GDLY = 150;
constexpr int NPUSH = 3;
constexpr int GCOPY = 8;
constexpr int NH = 4;
constexpr int NLAG = 8;
constexpr int RLAG = 3;
constexpr int SPD0 = 450;
constexpr int SPD1 = 700;
constexpr int RLAST = 10;
constexpr int ND = 6, NSLOT = 12;
constexpr int NSPARE = 7 * (16 - NPW) + 16;
constexpr int FC = 2;
constexpr int NVP = (7 * FC + 3) / 4;
static_assert(NH <= 8 && NPUSH <= 7 && 7 * 73 <= 8 * NSPARE && NPW == 8 && DEPTH <= 11 && DEPTH2 <= 11 && FC == 2, "roles");

constexpr int LDS_BM = 98304;
constexpr int LDS_ENC = 114688;
constexpr int LDS_MISC = 131072;
constexpr int LDS_BYTES = LDS_MISC + 2048;

struct Params {
    const float* x;
    const float* w;
    signed char* out;
    char* ws;
    char* const* ws_all;
    int rank;
    int pad;
};

#define WAIT_V(n) asm volatile("s_waitcnt vmcnt(" #n ")" ::: "memory")
#define WAIT_L(n) asm volatile("s_waitcnt lgkmcnt(" #n ")" ::: "memory")
#define BAR __builtin_amdgcn_s_barrier()
#define SCHED __builtin_amdgcn_sched_barrier(0)
#define CFENCE asm volatile("" ::: "memory")

template <int OFF> __device__ __forceinline__ void st16_sys(void* p, u32x4 x) {
    asm volatile("global_store_dwordx4 %0, %1, off offset:%2 sc0 sc1\n\ts_nop 1" :: "v"(p), "v"(x), "i"(OFF) : "memory");
}
__device__ __forceinline__ void ld8_sys(u32x2& x, const void* p) {
    asm volatile("global_load_dwordx2 %0, %1, off sc0 sc1" : "=v"(x) : "v"(p) : "memory");
}
__device__ __forceinline__ void st8_sys(void* p, u32x2 x) {
    asm volatile("global_store_dwordx2 %0, %1, off sc0 sc1\n\ts_nop 1" :: "v"(p), "v"(x) : "memory");
}
__device__ __forceinline__ void st4_sys(void* p, u32 x) {
    asm volatile("global_store_dword %0, %1, off sc0 sc1\n\ts_nop 1" :: "v"(p), "v"(x) : "memory");
}
__device__ __forceinline__ void add4_sys(void* p, u32 x) {
    asm volatile("global_atomic_add %0, %1, off sc1\n\ts_nop 1" :: "v"(p), "v"(x) : "memory");
}
__device__ __forceinline__ void ld16x4(f32x4& a, f32x4& b, f32x4& c, f32x4& d, const void* p0, const void* p1, const void* p2, const void* p3, int nt) {
    asm volatile("s_cmp_lg_u32 %8, 0\n\ts_cbranch_scc1 .Lnt%=\n\t"
                 "global_load_dwordx4 %0, %4, off\n\tglobal_load_dwordx4 %1, %5, off\n\tglobal_load_dwordx4 %2, %6, off\n\tglobal_load_dwordx4 %3, %7, off\n\t"
                 "s_branch .Lnd%=\n.Lnt%=:\n\t"
                 "global_load_dwordx4 %0, %4, off nt\n\tglobal_load_dwordx4 %1, %5, off nt\n\tglobal_load_dwordx4 %2, %6, off nt\n\tglobal_load_dwordx4 %3, %7, off nt\n.Lnd%=:"
                 : "=&v"(a), "=&v"(b), "=&v"(c), "=&v"(d) : "v"(p0), "v"(p1), "v"(p2), "v"(p3), "s"(nt) : "memory", "scc");
}
__device__ __forceinline__ void ticket_issue(u32& x, void* p) {
    u64 sv;
    asm volatile("s_mov_b64 %1, exec\n\ts_mov_b64 exec, 1\n\tglobal_atomic_add %0, %2, %3, off sc0 sc1\n\ts_mov_b64 exec, %1"
                 : "=&v"(x), "=&s"(sv) : "v"(p), "v"(1u) : "memory");
}
__device__ __forceinline__ void poll8(u32x2& x, u32& t, const void* p, const void* tp) {
    asm volatile("global_load_dwordx2 %0, %2, off sc0 sc1\n\tglobal_load_dword %1, %3, off sc0 sc1\n\ts_waitcnt vmcnt(0)"
                 : "=&v"(x), "=&v"(t) : "v"(p), "v"(tp) : "memory");
}
__device__ __forceinline__ void ld4_sys(u32& x, const void* p) {
    asm volatile("global_load_dword %0, %1, off sc0 sc1" : "=v"(x) : "v"(p) : "memory");
}
__device__ __forceinline__ void ld16_sys(u32x4& x, const void* p) {
    asm volatile("global_load_dwordx4 %0, %1, off sc0 sc1" : "=v"(x) : "v"(p) : "memory");
}
__device__ __forceinline__ void poll_win(u32& x, const void* p, u64 lanes) {
    u64 sv;
    asm volatile("s_mov_b64 %1, exec\n\ts_mov_b64 exec, %3\n\tglobal_load_dword %0, %2, off sc0 sc1\n\ts_mov_b64 exec, %1\n\ts_waitcnt vmcnt(0)"
                 : "+v"(x), "=&s"(sv) : "v"(p), "s"(lanes) : "memory");
}
__device__ __forceinline__ u32x2 ld8_wait(const void* p) {
    u32x2 x; asm volatile("global_load_dwordx2 %0, %1, off sc0 sc1\n\ts_waitcnt vmcnt(0)" : "=&v"(x) : "v"(p) : "memory"); return x;
}
__device__ __forceinline__ u32x4 ld16_wait(const void* p) {
    u32x4 x; asm volatile("global_load_dwordx4 %0, %1, off sc0 sc1\n\ts_waitcnt vmcnt(0)" : "=&v"(x) : "v"(p) : "memory"); return x;
}
__device__ __forceinline__ u32 ld4_wait(const void* p) {
    u32 x; asm volatile("global_load_dword %0, %1, off sc0 sc1\n\ts_waitcnt vmcnt(0)" : "=&v"(x) : "v"(p) : "memory"); return x;
}
__device__ __forceinline__ u32 cvt_pk_bf16(float lo, float hi) {
    u32 r; asm("v_cvt_pk_bf16_f32 %0, %1, %2" : "=v"(r) : "v"(lo), "v"(hi)); return r;
}
__device__ __forceinline__ u32 bfi(u32 m, u32 a, u32 b) { return __builtin_amdgcn_bitop3_b32(m, a, b, 0xCA); }
template <int SH> __device__ __forceinline__ u32 lshl_or_i(u32 a, u32 b) { u32 r; asm("v_lshl_or_b32 %0, %1, %2, %3" : "=v"(r) : "v"(a), "n"(SH), "v"(b)); return r; }
__device__ __forceinline__ u32 times15(u32 t) { u32 r = t << 4; asm("" : "+v"(r)); return r - t; }
__device__ __forceinline__ u32 rfl(u32 x) { return (u32)__builtin_amdgcn_readfirstlane((int)x); }
__device__ __forceinline__ u32 rdl(u32 x, int l) { return (u32)__builtin_amdgcn_readlane((int)x, l); }
__device__ __forceinline__ u64 now() { return __builtin_amdgcn_s_memrealtime(); }

__device__ __forceinline__ bool spin_fail(char* ws, u32 call, u64 t0, u32 t) {
    if (rfl(t) == call) return true;
    if (now() - t0 > DEADLINE) { st4_sys(ws + W_TMO, call); WAIT_V(0); return true; }
    __builtin_amdgcn_s_sleep(1);
    return false;
}
#define DPP0(v, ctl, rm, bc) (u32)__builtin_amdgcn_update_dpp(0, (int)(v), ctl, rm, 0xf, bc)
__device__ __forceinline__ u32 scan_add(u32 v) {
    v += DPP0(v, 0x111, 0xf, true); v += DPP0(v, 0x112, 0xf, true); v += DPP0(v, 0x114, 0xf, true); v += DPP0(v, 0x118, 0xf, true);
    v += DPP0(v, 0x142, 0xa, false); v += DPP0(v, 0x143, 0xc, false);
    return v;
}
__device__ __forceinline__ u32 wave_umax(u32 v) {
    v = max(v, DPP0(v, 0x111, 0xf, true)); v = max(v, DPP0(v, 0x112, 0xf, true)); v = max(v, DPP0(v, 0x114, 0xf, true)); v = max(v, DPP0(v, 0x118, 0xf, true));
    v = max(v, DPP0(v, 0x142, 0xa, false)); v = max(v, DPP0(v, 0x143, 0xc, false));
    return rdl(v, 63);
}
__device__ __forceinline__ int rot_of(int j) { return (j * 16) / 7; }
__device__ __forceinline__ int kt_of(int o, int rank, int j0) {
    if (o < 8) return rank * 8 + o;
    const int r = (o - 8) / 7, s = o - 8 - r * 7;
    int j = j0 - s; if (j < 0) j += 7;
    return ((rank + 1 + j) & 7) * 8 + r;
}
__device__ __forceinline__ int peer_of(int x, int rank) {
    const u32 t = rank < 4 ? (rank < 2 ? (rank == 0 ? 0x47513206u : 0x62305417u) : (rank == 2 ? 0x71546023u : 0x74150632u))
                           : (rank < 6 ? (rank == 4 ? 0x63027145u : 0x06231754u) : (rank == 6 ? 0x15472360u : 0x20364571u));
    return (int)((t >> (4 * x)) & 7u);
}
__device__ __forceinline__ char* peer_ws(LAS unsigned char* lds, u32 pr) {
    return (char*)((volatile LAS u64*)(lds + LDS_MISC + 64))[pr & 7u];
}

struct Msk { u32 m, ms, m12, m2, pe, oo; };
template <bool HI> __device__ __forceinline__ Msk msk(u32 m) {
    Msk k; k.m = m; k.ms = m << 1;
    const u32 c2 = m - ((m >> 1) & 0x55555555u), cl = c2 & 0x33333333u;
    if (HI) {
        const u32 n2 = (m << 2) & k.ms & 0x44444444u, n12 = ((m << 2) | k.ms) & 0x44444444u;
        k.m2 = lshl_or_i<1>(n2, n2); k.m12 = lshl_or_i<1>(n12, n12);
    } else {
        const u32 n2 = (m >> 1) & m & 0x11111111u, n12 = ((m >> 1) | m) & 0x11111111u;
        k.m2 = lshl_or_i<1>(n2, n2); k.m12 = lshl_or_i<1>(n12, n12);
    }
    const u32 c4 = cl + ((c2 >> 2) & 0x33333333u), ce = c4 & 0x0f0f0f0fu, cb = ce + ((c4 >> 4) & 0x0f0f0f0fu);
    asm("v_mul_u32_u24 %0, 0x10101, %1" : "=v"(k.pe) : "v"(cb));
    k.oo = (k.pe << 8) + ce;
    return k;
}
#define SHL_B(d, o, ob, v, vb) "v_lshlrev_b32_sdwa " d ", " o ", " v " dst_sel:DWORD dst_unused:UNUSED_PAD src0_sel:BYTE_" #ob " src1_sel:BYTE_" #vb "\n\t"
#define SHR_B(d, db, o, ob, v) "v_lshrrev_b32_sdwa " d ", " o ", " v " dst_sel:BYTE_" #db " dst_unused:UNUSED_PRESERVE src0_sel:BYTE_" #ob " src1_sel:DWORD\n\t"
__device__ __forceinline__ u32 press(u32 x, const Msk& k) {
    const u32 y = bfi(0x55555555u, x | ((x >> 1) & ~k.m), x & k.ms);
    const u32 hi = (y >> 2) & 0x33333333u;
    const u32 z = (y & 0x33333333u) + hi + (hi & k.m12) + ((hi & k.m2) << 1);
    const u32 ze = z & 0x0f0f0f0fu, zo = (z >> 4) & 0x0f0f0f0fu;
    u32 t1, t2, t3, t4, t5, t6, t7;
    asm(SHL_B("%0", "%8", 0, "%10", 0) SHL_B("%1", "%7", 0, "%9", 1) SHL_B("%2", "%8", 1, "%10", 1) SHL_B("%3", "%7", 1, "%9", 2)
        SHL_B("%4", "%8", 2, "%10", 2) SHL_B("%5", "%7", 2, "%9", 3) SHL_B("%6", "%8", 3, "%10", 3)
        : "=&v"(t1), "=&v"(t2), "=&v"(t3), "=&v"(t4), "=&v"(t5), "=&v"(t6), "=&v"(t7) : "v"(k.pe), "v"(k.oo), "v"(ze), "v"(zo));
    return ((z & 15u) | t1) | (t2 | t3 | t4) | (t5 | t6 | t7);
}
__device__ __forceinline__ u32 spread(u32 c, const Msk& k) {
    u32 ze = c, zo;
    asm("v_lshrrev_b32 %1, %3, %4\n\t" SHR_B("%0", 1, "%2", 0, "%4") SHR_B("%1", 1, "%3", 1, "%4") SHR_B("%0", 2, "%2", 1, "%4")
        SHR_B("%1", 2, "%3", 2, "%4") SHR_B("%0", 3, "%2", 2, "%4") SHR_B("%1", 3, "%3", 3, "%4") "s_nop 0"
        : "+&v"(ze), "=&v"(zo) : "v"(k.pe), "v"(k.oo), "v"(c));
    const u32 z = bfi(0x0f0f0f0fu, ze, zo << 4);
    const u32 h = bfi(k.m12, bfi(k.m2, z, z << 1), z << 2);
    const u32 y = bfi(0x33333333u, z, h);
    return bfi(0x55555555u, y, bfi(k.ms, y, y << 1)) & k.m;
}
__device__ __forceinline__ void bit_put(LAS u32* st, u32 off, u32 c) {
    const u64 x = (u64)c << (off & 31u);
    __hip_atomic_fetch_or(st + (off >> 5), (u32)x, __ATOMIC_RELAXED, __HIP_MEMORY_SCOPE_WORKGROUP);
    __hip_atomic_fetch_or(st + (off >> 5) + 1, (u32)(x >> 32), __ATOMIC_RELAXED, __HIP_MEMORY_SCOPE_WORKGROUP);
}
__device__ __forceinline__ u32 bit_get(const LAS u32* st, u32 off) {
    const u32 w = min(off >> 5, 500u);
    return __builtin_amdgcn_alignbit(st[w + 1], st[w], off & 31u);
}
__device__ __forceinline__ u32 popc(u32 x) { return (u32)__popc(x); }
__device__ __forceinline__ void to_planes(const u32* n, u32& p0, u32& p1, u32& p2, u32& p3) {
    const u32 A = bfi(0x33333333u, n[0], n[2] << 2), B = bfi(0x33333333u, n[0] >> 2, n[2]);
    const u32 C = bfi(0x33333333u, n[1], n[3] << 2), D = bfi(0x33333333u, n[1] >> 2, n[3]);
    p0 = bfi(0x55555555u, A, C << 1); p1 = bfi(0x55555555u, A >> 1, C); p2 = bfi(0x55555555u, B, D << 1); p3 = bfi(0x55555555u, B >> 1, D);
}
__device__ __forceinline__ void from_planes(u32* n, u32 p0, u32 p1, u32 p2, u32 p3) {
    const u32 A = bfi(0x55555555u, p0, p1 << 1), C = bfi(0x55555555u, p0 >> 1, p1);
    const u32 B = bfi(0x55555555u, p2, p3 << 1), D = bfi(0x55555555u, p2 >> 1, p3);
    n[0] = bfi(0x33333333u, A, B << 2); n[2] = bfi(0x33333333u, A >> 2, B); n[1] = bfi(0x33333333u, C, D << 2); n[3] = bfi(0x33333333u, C >> 2, D);
}
constexpr u32 MAXSDW = 384;
__device__ __forceinline__ void tail_out(char* dst, LAS unsigned char* enc, u32 n16, int lane) {
    CFENCE;
    const u32x4 a = *(volatile LAS u32x4*)(enc + lane * 16), b = *(volatile LAS u32x4*)(enc + 1024 + lane * 16);
    if ((u32)lane < n16) st16_sys<0>(dst + lane * 16, a);
    if ((u32)lane + 64u < n16) st16_sys<1024>(dst + lane * 16, b);
}
constexpr u32 H_PLAIN = 0x40000000u;
constexpr u32 H_RAW = 0x80000000u;

struct PanelId { int r, dest, pi; bool isB; };
__device__ __forceinline__ PanelId panel_of(int r, int i, int rank) {
    PanelId d; d.r = r; d.isB = i < 56 || i >= 72;
    d.dest = i < 56 ? ((rank + 1 + (i >> 3)) & 7) : rank;
    d.pi = d.isB ? (i & 7) : i - 56;
    return d;
}
__device__ __forceinline__ bool is_fill(const PanelId& d, int unit) { return d.isB && d.pi == 7 && unit >= 4 - FC; }
__device__ __forceinline__ void unit_load(const Params& p, const PanelId& d, int unit, int lane, f32x4 (&f)[16]) {
    const int fr = lane & 15, fq = lane >> 4;
    const float* b = d.isB ? p.w + (size_t)(d.r * 64 + 4 * fq) * GN + d.dest * NS + d.pi * 256 + unit * 64 + 4 * fr
                           : p.x + (size_t)(d.pi * 256 + unit * 64 + (lane >> 2)) * KS + d.r * 64 + 4 * ((lane & 3) ^ (fq & 2));
    const int s1 = d.isB ? 16 * GN : 16 * KS, s2 = d.isB ? GN : 16;
    const int nt = (int)rfl(d.isB ? 1u : 0u);
#pragma unroll
    for (int j = 0; j < 4; ++j) ld16x4(f[4 * j], f[4 * j + 1], f[4 * j + 2], f[4 * j + 3], b + j * s1, b + j * s1 + s2, b + j * s1 + 2 * s2, b + j * s1 + 3 * s2, nt);
}
#define PIN_F(f) do { asm volatile("" : "+v"(f[0]), "+v"(f[1]), "+v"(f[2]), "+v"(f[3]), "+v"(f[4]), "+v"(f[5]), "+v"(f[6]), "+v"(f[7]) :: "memory"); \
    asm volatile("" : "+v"(f[8]), "+v"(f[9]), "+v"(f[10]), "+v"(f[11]), "+v"(f[12]), "+v"(f[13]), "+v"(f[14]), "+v"(f[15]) :: "memory"); } while (0)
__device__ __forceinline__ void st_unit(char* dp, const u32 (&u)[32]) {
    st16_sys<0>(dp, (u32x4){u[0], u[1], u[2], u[3]}); st16_sys<1024>(dp, (u32x4){u[4], u[5], u[6], u[7]});
    st16_sys<2048>(dp, (u32x4){u[8], u[9], u[10], u[11]}); st16_sys<3072>(dp, (u32x4){u[12], u[13], u[14], u[15]});
    dp += 4096;
    st16_sys<0>(dp, (u32x4){u[16], u[17], u[18], u[19]}); st16_sys<1024>(dp, (u32x4){u[20], u[21], u[22], u[23]});
    st16_sys<2048>(dp, (u32x4){u[24], u[25], u[26], u[27]}); st16_sys<3072>(dp, (u32x4){u[28], u[29], u[30], u[31]});
}
__device__ __forceinline__ size_t local_off(const PanelId& d, int unit, int rank, int kt, size_t& cnt) {
    if (d.dest == rank) { cnt = (d.isB ? W_BCNT : W_ACNT) + (size_t)(d.pi * 64 + (kt & 7) * 8 + (kt >> 3)) * 4; return (d.isB ? W_B : W_A) + (size_t)(d.pi * 64 + kt) * PANEL + unit * USLOT; }
    const int g = ((d.dest - rank - 1) & 7) * FC + unit - (4 - FC);
    cnt = W_XCNT + (size_t)((g >> 2) * 64 + (kt & 7) * 8 + (kt >> 3)) * 4;
    return W_X + (size_t)((g >> 2) * 64 + kt) * PANEL + (g & 3) * USLOT;
}
__device__ __forceinline__ size_t export_off(const PanelId& d, int unit, u32 call) {
    return (d.isB ? W_PW + (size_t)((d.dest * 8 + d.r) * 9 + d.pi + (d.pi == 7 ? (int)(call & 1u) : 0)) * PANEL : W_PX + (size_t)(d.r * 16 + d.pi) * PANEL) + unit * USLOT;
}
__device__ __forceinline__ u32 unit_pack(const Params& p, LAS unsigned char* lds, LAS unsigned char* enc, const PanelId& d, int unit, int lane, const f32x4 (&f)[16], int mode) {
    u32 u[32];
    if (d.isB) {
#pragma unroll
        for (int h = 0; h < 2; ++h)
#pragma unroll
            for (int c = 0; c < 4; ++c)
#pragma unroll
                for (int t = 0; t < 4; ++t) u[16 * h + 4 * c + t] = cvt_pk_bf16(f[8 * h + 2 * t][c], f[8 * h + 2 * t + 1][c]);
    } else {
#pragma unroll
        for (int c = 0; c < 16; ++c) { u[2 * c] = cvt_pk_bf16(f[c][0], f[c][1]); u[2 * c + 1] = cvt_pk_bf16(f[c][2], f[c][3]); }
    }
    const bool own = d.dest == p.rank, fill = is_fill(d, unit);
    u32 hdr = 0;
    if (mode & 4) {
        size_t cnt;
        while (rfl(((volatile LAS u32*)(lds + LDS_MISC))[2]) == 0u) __builtin_amdgcn_s_sleep(1);
        st_unit(peer_ws(lds, (u32)d.dest) + local_off(d, unit, d.dest, p.rank * 8 + d.r, cnt) + lane * 16, u);
    } else if ((mode & 1) && (!(d.isB && own) || fill)) {
        u32 call;
        while ((call = rfl(((volatile LAS u32*)(lds + LDS_MISC))[2])) == 0u) __builtin_amdgcn_s_sleep(1);
        char* slot = p.ws + export_off(d, unit, call);
        *(volatile LAS u32x4*)(enc + lane * 16) = (u32x4){0u, 0u, 0u, 0u};
        *(volatile LAS u32x4*)(enc + 1024 + lane * 16) = (u32x4){0u, 0u, 0u, 0u};
        u32 hi[16], t[16], sum = 0;
#pragma unroll
        for (int k = 0; k < 16; ++k) {
            hi[k] = __builtin_amdgcn_perm(u[2 * k + 1], u[2 * k], 0x07050301u);
            t[k] = hi[k] | 0x80808080u;
            sum = __builtin_amdgcn_sad_u8(t[k], 0u, sum);
        }
        u32 a = ((rdl(scan_add(sum), 63) + 4095u) >> 12) - 128u;
        a = min(max(a, 6u), 126u);
        const u32 b4 = (a - 6u) * 0x01010101u;
        u32 bad = 0, n[8];
#pragma unroll
        for (int k = 0; k < 16; ++k) { t[k] -= b4; bad |= (t[k] & 0xf8f8f8f8u) ^ 0x80808080u; }
#pragma unroll
        for (int k = 0; k < 8; ++k)
            n[k] = __builtin_amdgcn_perm(0x06000204u, 0x080a0c0eu, t[2 * k] & 0x07070707u) | __builtin_amdgcn_perm(0x60002040u, 0x80a0c0e0u, t[2 * k + 1] & 0x07070707u)
                 | ((hi[2 * k] >> 7) & 0x01010101u) | ((hi[2 * k + 1] >> 3) & 0x10101010u);
        u32 sg[2], c0[2], c1[2], p1[2], p2[2], p3[2], p4[2];
#pragma unroll
        for (int h = 0; h < 2; ++h) {
            to_planes(n + 4 * h, sg[h], c0[h], c1[h], p4[h]);
            p2[h] = c1[h] | p4[h]; p1[h] = p2[h] | c0[h]; p3[h] = p4[h] | (c1[h] & c0[h]);
        }
        const u32 ca = (popc(p1[0]) + popc(p1[1])) | ((popc(p2[0]) + popc(p2[1])) << 16), cb = (popc(p3[0]) + popc(p3[1])) | ((popc(p4[0]) + popc(p4[1])) << 17);
        const u32 ia = scan_add(ca), ib = scan_add(cb), ta = rdl(ia, 63), tb = rdl(ib, 63);
        const u32 b3 = ta & 0xffffu, b4s = b3 + (ta >> 16), b5 = b4s + (tb & 0xffffu);
        u32 sdw = (b5 + (tb >> 16) + 31u) >> 5;
        const bool plain = (d.r == 0 && d.isB && d.pi < 7)   || sdw > 256u || __any(max(popc(p4[0]), popc(p4[1])) > 16u);
        if (plain) sdw = 0;
        u32 nexc = 0;
        u64 anyb = __ballot(bad != 0);
        if (anyb) {
            u32 bm[2] = {0u, 0u};
#pragma unroll
            for (int k = 0; k < 16; ++k) {
                const u32 y = (t[k] & 0xf8f8f8f8u) ^ 0x80808080u;
                bm[k >> 3] |= ((((y & 0x78787878u) + 0x78787878u) | y) & 0x80808080u) >> (7 - (k & 7));
            }
            if (rdl(scan_add((u32)(__popc(bm[0]) + __popc(bm[1]))), 63) > 64u) {
                hdr = H_RAW | 512u;
            } else {
                while (anyb) {
                    const int l = __builtin_ctzll(anyb);
                    const u32 w0 = rdl(bm[0], l), w1 = rdl(bm[1], l);
                    const u32 pos = (u32)__builtin_ctz(w0 ? w0 : w1), k = (w0 ? 0u : 8u) + (pos & 7u);
                    u32 hv = 0;
#pragma unroll
                    for (int kk = 0; kk < 16; ++kk) hv = k == (u32)kk ? hi[kk] : hv;
                    const u32 e = (rdl(hv, l) >> (pos & 24u)) & 0x7fu;
                    if (lane == 0) ((volatile LAS u32*)enc)[sdw + nexc] = ((u32)l << 13) | (k << 9) | ((pos >> 3) << 7) | e;
                    ++nexc;
                    if (lane == l) { if (w0) bm[0] &= bm[0] - 1u; else bm[1] &= bm[1] - 1u; }
                    anyb = __ballot((bm[0] | bm[1]) != 0);
                }
            }
        }
        char* dp = slot + lane * 16;
        if (hdr) {
            st_unit(dp, u);
        } else {
            u32 lo[16];
#pragma unroll
            for (int k = 0; k < 16; ++k) lo[k] = __builtin_amdgcn_perm(u[2 * k + 1], u[2 * k], 0x06040200u);
            st16_sys<0>(dp, (u32x4){lo[0], lo[1], lo[2], lo[3]}); st16_sys<1024>(dp, (u32x4){lo[4], lo[5], lo[6], lo[7]});
            st16_sys<2048>(dp, (u32x4){lo[8], lo[9], lo[10], lo[11]}); st16_sys<3072>(dp, (u32x4){lo[12], lo[13], lo[14], lo[15]});
            if (plain) {
                st16_sys<0>(dp + 4096, (u32x4){sg[0], sg[1], c0[0], c0[1]}); st16_sys<1024>(dp + 4096, (u32x4){c1[0], c1[1], p4[0], p4[1]});
                const u32 n16 = (nexc + 3u) >> 2;
                tail_out(slot + 6144, enc, n16, lane);
                hdr = H_PLAIN | (384u + n16) | (a << 10) | (nexc << 17);
            } else {
                st16_sys<0>(dp + 4096, (u32x4){sg[0], sg[1], p1[0], p1[1]});
                u32 o2 = (ia - ca) & 0xffffu, o3 = b3 + ((ia - ca) >> 16), o4 = b4s + ((ib - cb) & 0xffffu), o5 = b5 + ((ib - cb) >> 16);
#pragma unroll
                for (int h = 0; h < 2; ++h) {
                    bit_put((LAS u32*)enc, o2, press(p2[h], msk<false>(p1[h]))); o2 += popc(p1[h]);
                    bit_put((LAS u32*)enc, o3, press(p3[h], msk<false>(p2[h]))); o3 += popc(p2[h]);
                    bit_put((LAS u32*)enc, o4, press(p4[h], msk<false>(p3[h]))); o4 += popc(p3[h]);
                    const Msk k = msk<false>(p4[h]);
                    bit_put((LAS u32*)enc, o5, press(c0[h] & p4[h], k) | (press(c1[h] & p4[h], k) << popc(p4[h]))); o5 += 2u * popc(p4[h]);
                }
                const u32 n16 = max((sdw + nexc + 3u) >> 2, 1u);
                tail_out(slot + 5120, enc, n16, lane);
                hdr = (320u + n16) | (a << 10) | (nexc << 17);
            }
        }
    }
    if (own ? (mode & 2) : fill) {
        size_t cnt;
        st_unit(p.ws + local_off(d, unit, p.rank, p.rank * 8 + d.r, cnt) + lane * 16, u);
    }
    return hdr;
}
__device__ __forceinline__ void unit_announce(const Params& p, LAS unsigned char* lds, const PanelId& d, int unit, int pr, u32 hdr, u32 call, int mode) {
    const bool fill = is_fill(d, unit);
    if (mode & 4) {
        if (pr == d.dest) { size_t cnt; local_off(d, unit, d.dest, p.rank * 8 + d.r, cnt); add4_sys(peer_ws(lds, (u32)pr) + cnt, 1u); }
    } else if (pr == p.rank) {
        if (d.dest == p.rank ? (mode & 2) : fill) { size_t cnt; local_off(d, unit, p.rank, p.rank * 8 + d.r, cnt); add4_sys(p.ws + cnt, 1u); }
    } else if (!(mode & 1)) {
    } else if (!d.isB || pr == d.dest) {
        st8_sys(peer_ws(lds, (u32)pr) + W_HI + (size_t)(((p.rank * 8 + d.r) * 24 + (d.isB ? d.pi : 8 + d.pi)) * 4 + unit) * 8, (u32x2){hdr, call});
    } else if (fill) {
        st8_sys(peer_ws(lds, (u32)pr) + W_HX + (call & 1u) * 16384u + (size_t)(((p.rank * 8 + d.r) * 8 + d.dest) * 4 + unit) * 8, (u32x2){hdr, call});
    }
}

constexpr u32 T_IN = 0, T_OUT = 1, T_DEAD = 3, T_SKIP = 4;
constexpr u32 NP_IN = 384, NP_X = 28 * FC, NP_S = 56, NP_ROW = 32 - FC, NP_OUT = 16 * NP_ROW, NP_ALL = NP_IN + NP_X + NP_S + NP_OUT;
#define R_POSTED 0
#define R_TAKEN 1
#define R_FULL 2
#define R_STOP 3
#define R_DONE 4
__device__ __forceinline__ volatile LAS u32* ring_words(LAS unsigned char* lds) { return (volatile LAS u32*)(lds + LDS_MISC + 256); }
__device__ __forceinline__ volatile LAS u32x4* ring_desc(LAS unsigned char* lds, u32 n) { return (volatile LAS u32x4*)(lds + LDS_MISC + 512 + (n % (u32)NSLOT) * 32u); }

struct Pair { u32 type, so, ho, cnt, sstep, cstep, dstep, sc; bool sgn; char* dst; };
__device__ __forceinline__ Pair pair_of(const Params& p, u32 call, u32 z, int peer, int rot) {
    Pair q; q.type = T_SKIP; q.so = 0; q.ho = (u32)W_TMO; q.cnt = 0; q.sstep = USLOT; q.cstep = 0; q.dstep = USLOT; q.sc = 0; q.sgn = false; q.dst = nullptr;
    size_t cnt;
    if (z < NP_IN) {
        PanelId d;
        constexpr int NE = 16 - NLAG;
        int P = (int)(z >> 1);
        if (P >= RLAG * 24 + 8 + NE) P += NLAG;
        if (P >= 192) P += 8;
        int r = P / 24;
        const int i = P - r * 24, unit = (int)(z & 1u) * 2;
        d.isB = i < 8; d.dest = p.rank; d.pi = d.isB ? i : ((i - 8 + rot) & 15);
        if (NLAG && !d.isB && r >= RLAG) {
            if (r == RLAG) { d.pi = i - 8 + (rot < NE ? rot : 0); if (d.pi >= NE) d.pi -= NE; }
            else if (r == 8) { d.pi = i - 8 + (rot < NE ? NE : rot); if (d.pi >= 16) d.pi -= NLAG; }
            if (d.pi >= NE) --r;
        }
        d.r = r;
        q.type = T_IN; q.so = (u32)export_off(d, unit, call);
        q.ho = (u32)W_HI + (u32)(((peer * 8 + r) * 24 + (d.isB ? d.pi : 8 + d.pi)) * 4 + unit) * 8u;
        q.dst = p.ws + local_off(d, unit, p.rank, peer * 8 + r, cnt);
        q.cnt = (u32)cnt;
    } else if (z < NP_IN + NP_X) {
        const int y = (int)(z - NP_IN), r = y / 7;
        PanelId d; d.r = r; d.isB = true; d.dest = (p.rank + 1 + y - r * 7) & 7; d.pi = 7;
        q.type = T_IN; q.so = (u32)export_off(d, 2, call);
        q.ho = (u32)W_HX + (call & 1u) * 16384u + (u32)(((peer * 8 + r) * 8 + d.dest) * 4 + 2) * 8u;
        q.dst = p.ws + local_off(d, 2, p.rank, peer * 8 + r, cnt);
        q.cnt = (u32)cnt;
    } else if (z < NP_IN + NP_X + NP_S) {
        q.type = T_IN; q.sgn = true; q.so = (u32)W_PS; q.sstep = 0; q.dstep = 0;
        q.ho = (u32)W_HS + (u32)(peer * 128 + (int)(z - NP_IN - NP_X) * 2) * 8u;
        q.dst = p.ws + W_SG + (size_t)peer * 128 * USLOT;
        q.cnt = (u32)W_SCNT + (u32)peer * 512u;
    } else if (z < NP_ALL) {
        int y = (int)(z - NP_IN - NP_X - NP_S), pm, pn = 7, unit;
        if (y >= RLAST * 2 && y < RLAST * 2 + 448) { y -= RLAST * 2; pm = y / 28; y -= pm * 28; pn = y >> 2; unit = (y & 3) * 2; }
        else { if (y >= RLAST * 2) y -= 448; pm = y >> 1; unit = (y & 1) * 4; }
        q.cnt = (u32)W_SG + (u32)(peer * 128 + pm * 8 + pn) * (u32)USLOT + (u32)unit * 1024u; q.cstep = 1024u;
        if (pn < 7) q.sc = (u32)W_SCNT + (u32)(peer * 128 + pm * 8 + pn) * 4u;
        q.type = T_OUT; q.so = (u32)W_PO + (u32)((pm * 8 + pn) * 8 + unit) * (u32)USLOT;
        q.dstep = 64u;
        q.ho = (u32)W_HO + (u32)((peer * 128 + pm * 8 + pn) * 8 + unit) * 8u;
        q.dst = (char*)p.out + (size_t)(pm * 256 + (unit >> 2) * 128) * GN + peer * NS + pn * 256 + (unit & 3) * 64;
    }
    return q;
}
__device__ __forceinline__ void pull_schedule(const Params& p, LAS unsigned char* lds, u32 call, int j, int c, int wid, int lane) {
    __builtin_amdgcn_s_setprio(2);
    const int peer = (p.rank + 1 + j) & 7, rot = rot_of(j);
    const u32 sub = (u32)lane & 1u;
    const u32 subm = 0u - sub;
    const char* pws = peer_ws(lds, (u32)peer);
    volatile LAS u32* rw = ring_words(lds);
    constexpr u32 Z0 = 2 * NPUSH, NP1 = NP_ALL - NP_OUT;
    const u32 qbase2 = (call - 1u) * (NP_OUT + 2u * (u32)(NPW + NH)) - NP1;
    u32 qbase = (call - 1u) * (NP1 - Z0 - (u32)NPW) - 2u * (u32)NPW - Z0;
    char* qp = p.ws + W_Q + j * 128;
    u32 tk;
    u32 z = Z0 + (u32)c, zn = Z0 + (u32)(NPW + c), n = 0, slot = 0, dec = 0, lap = 0;
    bool second = c >= NPW;
    if (second) {
        u32 tk1;
        qp += 1024; qbase = qbase2;
        ticket_issue(tk, qp); ticket_issue(tk1, qp);
        asm volatile("s_waitcnt vmcnt(0)" : "+v"(tk), "+v"(tk1) :: "memory");
        z = rfl(tk) - qbase; zn = rfl(tk1) - qbase;
    }
    Pair cur = pair_of(p, call, z, peer, rot);
    u32x2 h = (u32x2){0u, 0u};
    u32 sc = 0;
    bool dead = false;
    while (z < NP_ALL) {
        const Pair nxt = pair_of(p, call, zn, peer, rot);
        const char* hnp = p.ws + (nxt.ho + sub * 8u);
        const u64 src = (u64)(pws + (cur.so + (cur.sstep & subm))), d64 = (u64)(cur.dst + (cur.dstep & subm));
        asm volatile("" :: "v"(hnp), "v"(src), "v"(d64));
        if (!dead && !__all(h.y == call)) {
            const u64 t0 = now();
            for (;;) {
                u32 t; poll8(h, t, p.ws + (cur.ho + sub * 8u), p.ws + W_TMO);
                if (__all(h.y == call)) { dead = rfl(t) == call; break; }
                if (spin_fail(p.ws, call, t0, t)) { dead = true; break; }
            }
        }
        if (!dead && cur.sc && rfl(sc) < call) {
            const u64 t0 = now();
            for (;;) {
                u32x2 x; u32 t; poll8(x, t, p.ws + (cur.sc & ~7u), p.ws + W_TMO);
                if (rfl((cur.sc & 4u) ? x.y : x.x) >= call) break;
                if (spin_fail(p.ws, call, t0, t)) { dead = true; break; }
            }
        }
        u32x2 hn; u32 scn;
        ld8_sys(hn, hnp);
        ld4_sys(scn, p.ws + nxt.sc);
        ticket_issue(tk, qp);
        for (;;) {
            const u32 tkn = rw[R_TAKEN], dn = rw[R_DONE + dec + sub];
            if (__all(n < tkn + (u32)RUNAHEAD && dn + (u32)(NSLOT / ND) > lap)) break;
            __builtin_amdgcn_s_sleep(1);
        }
        const u32 na = rdl(h.x, 0) & 1023u, nf = (na + 7u) & ~7u;
        if (lane < 2) {
            const u32 type = !dead ? cur.type : (cur.type == T_OUT ? T_DEAD : T_SKIP);
            volatile LAS u32x4* ds = (volatile LAS u32x4*)(lds + LDS_MISC + 512 + (slot + sub) * 32u);
            const u32 tl = cur.sgn ? (h.x >> 10) & 127u : 0u;
            const bool mg = cur.type == T_OUT;
            const u32 back = mg ? (512u - nf) & subm : 0u, at = (slot + sub) * 512u - back;
            const u32 hx = mg ? (sub ? h.x + na - nf : h.x + nf - na) : h.x;
            const u64 s64 = src + tl * (u32)USLOT - back * 16u, e64 = d64 + tl * (u32)USLOT;
            ds[0] = (u32x4){(u32)s64, (u32)(s64 >> 32), hx, type | (at << 8)};
            ds[1] = (u32x4){(u32)e64, (u32)(e64 >> 32), cur.cnt + (cur.cstep & subm) + tl * 4u, (mg && sub ? at + na - nf : at) * 16u};
        }
        n += 2;
        rw[R_POSTED] = n;
        if (zn >= NP_ALL) rw[R_STOP] = n + 1u;
        slot = slot == (u32)NSLOT - 2u ? 0u : slot + 2u;
        if (dec == (u32)ND - 2u) { dec = 0; ++lap; } else dec += 2u;
        u32 tv;
        asm volatile("s_waitcnt vmcnt(0)\n\tv_mov_b64 %0, %3\n\tv_mov_b32 %1, %4\n\tv_mov_b32 %2, %5\n\ts_nop 0"
                     : "=&v"(h), "=&v"(sc), "=&v"(tv) : "v"(hn), "v"(scn), "v"(tk) : "memory");
        z = zn; zn = rfl(tv) - qbase; cur = nxt;
        if (!second && zn >= NP1) {
            second = true; qp += 1024; qbase = qbase2;
            ticket_issue(tk, qp);
            asm volatile("s_waitcnt vmcnt(0)" : "+v"(tk) :: "memory");
            zn = rfl(tk) - qbase;
        }
    }
    rw[R_STOP] = n + 1u;
}

#define GLDS(src, off) "global_load_lds_dwordx4 " src ", off offset:" #off " sc0 sc1\n\ts_waitcnt vmcnt(%[d])\n\t"
template <int D> __device__ __forceinline__ void glds_part(const void* src, u32 dst, u32 rem) {
    const u64 mask = rem >= 64u ? ~0ull : (1ull << rem) - 1ull;
    u32 keep;
    asm volatile("s_mov_b32 %0, m0\n\ts_mov_b32 m0, %2\n\ts_mov_b64 exec, %3\n\t" GLDS("%1", 0) "s_mov_b64 exec, -1\n\ts_mov_b32 m0, %0"
                 : "=&s"(keep) : "v"(src), "s"(dst), "s"(mask), [d] "i"(D - 1) : "memory");
}
template <int D> __device__ __forceinline__ void load_unit(LAS unsigned char* lds, const char* src, u32 dst, u32 n16, u32 n, u32 nslot, u32& full, u32& posted, u32x4& ds) {
    volatile LAS u32* rw = ring_words(lds);
    u32 keep;
    asm volatile("s_mov_b32 %0, m0\n\ts_mov_b32 m0, %2\n\ts_nop 0\n\t" GLDS("%1", 0) GLDS("%1", 1024) "s_mov_b32 m0, %0"
                 : "=&s"(keep) : "v"(src), "s"(dst), [d] "i"(D - 1) : "memory");
    if (D <= 3) { full = n; rw[R_FULL] = n; }
    else if (D <= 9 && full + 1u < n) { full = n - 1u; rw[R_FULL] = full; }
    asm volatile("s_mov_b32 %0, m0\n\ts_mov_b32 m0, %2\n\ts_nop 0\n\t" GLDS("%1", 2048) GLDS("%1", 3072) "s_mov_b32 m0, %0"
                 : "=&s"(keep) : "v"(src), "s"(dst), [d] "i"(D - 1) : "memory");
    if (D > 9 && full + 1u < n) { full = n - 1u; rw[R_FULL] = full; }
    if (D == 4 || D == 5) { full = n; rw[R_FULL] = n; }
    posted = rw[R_POSTED];
    ds = *(volatile LAS u32x4*)(lds + LDS_MISC + 512 + nslot * 32u);
    src += 4096;
    glds_part<D>(src, dst + 4096u, 64u);
    if (D == 6) { full = n; rw[R_FULL] = n; }
    glds_part<D>(src + 1024, dst + 5120u, n16 - 320u);
    if (D == 7) { full = n; rw[R_FULL] = n; }
    if (n16 > 384u) {
        glds_part<D>(src + 2048, dst + 6144u, n16 - 384u);
        if (D == 8) { full = n; rw[R_FULL] = n; }
        if (n16 > 448u) glds_part<D>(src + 3072, dst + 7168u, n16 - 448u);
    }
    rw[R_TAKEN] = n + 1u;
}
__device__ __forceinline__ void pull_load(const Params& p, LAS unsigned char* lds, int wid, int lane) {
    __builtin_amdgcn_s_setprio(3);
    volatile LAS u32* rw = ring_words(lds);
    const u32 lbase = (u32)(u64)lds;
    u32 slot = 0, full = 0;
    u32 posted = 0;
    u32x4 ds = (u32x4){0u, 0u, 0u, 0u};
    for (u32 n = 0;; ++n) {
        volatile LAS u32x4* dsp = (volatile LAS u32x4*)(lds + LDS_MISC + 512 + slot * 32u);
        if (rfl(posted) <= n) {
            posted = rw[R_POSTED];
            if (rfl(posted) <= n) {
                WAIT_V(0);
                full = n; rw[R_FULL] = n;
                bool stop = false;
                while (rfl(rw[R_POSTED]) <= n && !(stop = rfl(rw[R_STOP]) == n + 1u)) __builtin_amdgcn_s_sleep(1);
                if (stop) break;
            }
            ds = dsp[0];
        }
        const u32 type = rfl(ds.w) & 255u;
        const u32 dst = lbase + min(rfl(ds.w) >> 8, (u32)(NSLOT - 1) * 512u) * 16u;
        slot = slot == (u32)NSLOT - 1u ? 0u : slot + 1u;
        if (type <= T_OUT) {
            const u32 n16 = rfl(min(max(ds.z & 1023u, 321u), 512u));
            const char* src = (const char*)(((u64)ds.y << 32) | ds.x) + lane * 16;
            if (type == T_IN) load_unit<DEPTH>(lds, src, dst, n16, n, slot, full, posted, ds);
            else load_unit<DEPTH2>(lds, src, dst, n16, n, slot, full, posted, ds);
        } else {
            WAIT_V(0);
            full = n + 1u; rw[R_FULL] = full; rw[R_TAKEN] = full;
            posted = 0;
        }
    }
}

__device__ __forceinline__ void pull_decode(const Params& p, LAS unsigned char* lds, int wid, int lane) {
    volatile LAS u32* rw = ring_words(lds);
    LAS u32* bm = (LAS u32*)(lds + LDS_BM + wid * 1024);
    u32 done = 0;
    const u32 pos = (0x534120u >> (4 * wid)) & 7u;
    for (u32 n = pos;; n += (u32)ND) {
        bool stop = false;
        while (rfl(rw[R_FULL]) <= n && !(stop = rfl(rw[R_STOP]) - 1u <= n)) __builtin_amdgcn_s_sleep(1);
        if (stop) break;
        CFENCE;
        const u32x4 d0 = ring_desc(lds, n)[0], d1 = ring_desc(lds, n)[1];
        const u32 type = rfl(d0.w) & 255u, hdr = rfl(d0.z);
        char* dst = (char*)(((u64)d1.y << 32) | d1.x);
        if (type == T_IN && rfl(d1.z) < (u32)W_XCNT) __builtin_amdgcn_s_setprio(1); else __builtin_amdgcn_s_setprio(0);
        LAS unsigned char* sl = lds + min(rfl(d1.w), (u32)(NSLOT - 1) * (u32)USLOT);
        u32 u[32];
        const int ol = type == T_OUT && (hdr & H_RAW) ? (lane & 3) * 16 + (lane >> 2) : lane;
        if (type <= T_OUT && (hdr & H_RAW)) {
#pragma unroll
            for (int k = 0; k < 8; ++k) { const u32x4 v = *(const LAS u32x4*)(sl + k * 1024 + ol * 16); u[4 * k] = v.x; u[4 * k + 1] = v.y; u[4 * k + 2] = v.z; u[4 * k + 3] = v.w; }
        } else if (type == T_IN) {
            const u32x4 dn = *(const LAS u32x4*)(sl + 4096 + lane * 16);
            u32x4 lo[4];
#pragma unroll
            for (int k = 0; k < 4; ++k) lo[k] = *(const LAS u32x4*)(sl + k * 1024 + lane * 16);
            const LAS u32* st = (const LAS u32*)(sl + 5120);
            u32 pl[4][2], c0[2], c1[2], base = 0;
            pl[0][0] = dn.z; pl[0][1] = dn.w;
            if (hdr & H_PLAIN) {
                const u32x4 e = *(const LAS u32x4*)(sl + 5120 + lane * 16);
                c0[0] = dn.z; c0[1] = dn.w; c1[0] = e.x; c1[1] = e.y; pl[3][0] = e.z; pl[3][1] = e.w;
                base = 8192u;
            } else {
#pragma unroll
                for (int lv = 0; lv < 4; ++lv) {
                    const u32 n0 = popc(pl[lv][0]), n1 = popc(pl[lv][1]), cn = lv < 3 ? n0 + n1 : 2u * (n0 + n1);
                    const u32 inc = scan_add(cn), o = base + inc - cn;
                    const u32 g0 = bit_get(st, o), g1 = bit_get(st, o + (lv < 3 ? n0 : 2u * n0));
                    base += rdl(inc, 63);
                    const Msk k0 = msk<true>(pl[lv][0]), k1 = msk<true>(pl[lv][1]);
                    if (lv < 3) { pl[lv + 1][0] = spread(g0, k0); pl[lv + 1][1] = spread(g1, k1); }
                    else { c0[0] = spread(g0, k0); c1[0] = spread(g0 >> (n0 & 31u), k0); c0[1] = spread(g1, k1); c1[1] = spread(g1 >> (n1 & 31u), k1); }
                }
#pragma unroll
                for (int h = 0; h < 2; ++h) { c0[h] |= (pl[0][h] & ~pl[1][h]) | (pl[2][h] & ~pl[3][h]); c1[h] |= pl[1][h] & ~pl[3][h]; }
            }
            const u32 b4 = (((hdr >> 10) & 0x7fu) - 6u) * 0x01010101u;
            const u32 tl = b4 + 0x07040506u, th = b4 + 0x00010203u;
            u32 hi[16];
#pragma unroll
            for (int h = 0; h < 2; ++h) {
                u32 nn[4];
                from_planes(nn, dn[h], c0[h], c1[h], pl[3][h]);
#pragma unroll
                for (int k = 0; k < 4; ++k) {
                    hi[8 * h + 2 * k] = bfi(0x80808080u, nn[k] << 7, __builtin_amdgcn_perm(th, tl, (nn[k] >> 1) & 0x07070707u));
                    hi[8 * h + 2 * k + 1] = bfi(0x80808080u, nn[k] << 3, __builtin_amdgcn_perm(th, tl, (nn[k] >> 5) & 0x07070707u));
                }
            }
            const u32 nexc = (hdr >> 17) & 0xffu;
            if (nexc) {
                const u32 ev = st[min((base + 31u) >> 5, MAXSDW) + (u32)lane];
                for (u32 e = 0; e < nexc; ++e) {
                    const u32 en = rdl(ev, (int)e);
                    const u32 sh = ((en >> 7) & 3u) * 8u, kk = (en >> 9) & 15u;
                    const bool me = (en >> 13) == (u32)lane;
#pragma unroll
                    for (int k = 0; k < 16; ++k)
                        if (me && kk == (u32)k) hi[k] = bfi(0x7fu << sh, (en & 0x7fu) << sh, hi[k]);
                }
            }
#pragma unroll
            for (int k = 0; k < 16; ++k) {
                u[2 * k] = __builtin_amdgcn_perm(hi[k], lo[k >> 2][k & 3], 0x05010400u);
                u[2 * k + 1] = __builtin_amdgcn_perm(hi[k], lo[k >> 2][k & 3], 0x07030602u);
            }
        } else if (type == T_OUT) {
            *(volatile LAS u32x4*)(bm + lane * 4) = (u32x4){0u, 0u, 0u, 0u};
            u32x4 sg;
            ld16_sys(sg, p.ws + (d1.z + (u32)lane * 16u));
            const u32x4 d0v = *(const LAS u32x4*)(sl + 3072 + lane * 16), d1v = *(const LAS u32x4*)(sl + 4096 + lane * 16);
            const u32x4 sgi = *(const LAS u32x4*)(sl + 5120 + lane * 16);
            u32x4 lo[3];
#pragma unroll
            for (int k = 0; k < 3; ++k) lo[k] = *(const LAS u32x4*)(sl + k * 1024 + lane * 16);
            const LAS u32* st = (const LAS u32*)(sl + ((hdr & H_PLAIN) ? 6144 : 5120));
            u32 f5[4], g[4];
            const u32x4 e3 = d0v & d1v;
            u32 cn = popc(e3.x) + popc(e3.y) + popc(e3.z) + popc(e3.w), inc = scan_add(cn), o = inc - cn;
#pragma unroll
            for (int h = 0; h < 4; ++h) { g[h] = bit_get(st, o); o += popc(e3[h]); }
            u32 base = rdl(inc, 63);
#pragma unroll
            for (int h = 0; h < 4; ++h) f5[h] = spread(g[h], msk<true>(e3[h]));
            u32 a3[2], a4[2], u5[2];
            {
                u32 n0 = popc(f5[0]) + popc(f5[1]), n1 = popc(f5[2]) + popc(f5[3]);
                cn = n0 + n1; inc = scan_add(cn); o = base + inc - cn; base += rdl(inc, 63);
                u5[0] = __builtin_amdgcn_ubfe(bit_get(st, o), 0u, n0); u5[1] = __builtin_amdgcn_ubfe(bit_get(st, o + n0), 0u, n1);
                n0 = popc(u5[0]); cn = n0 + popc(u5[1]); inc = scan_add(cn); o = base + inc - cn; base += rdl(inc, 63);
                a4[0] = spread(bit_get(st, o), msk<true>(u5[0])); a4[1] = spread(bit_get(st, o + n0), msk<true>(u5[1]));
                n0 = popc(a4[0]); cn = n0 + popc(a4[1]); inc = scan_add(cn); o = base + inc - cn; base += rdl(inc, 63);
                a3[0] = spread(bit_get(st, o), msk<true>(a4[0])) | (u5[0] & ~a4[0]); a3[1] = spread(bit_get(st, o + n0), msk<true>(a4[1])) | (u5[1] & ~a4[1]);
            }
            const u32 nexc = (hdr >> 17) & 0xffu;
            const LAS u32* lp = st + min((base + 31u) >> 5, MAXSDW);
            const u32 h0 = lp[0], h1 = lp[1], e0 = ((const LAS unsigned char*)lp)[8 + lane], e1 = ((const LAS unsigned char*)lp)[72 + lane];
            const bool mine = ((lane < 32 ? h0 : h1) >> (lane & 31)) & 1u;
            const u32 kth = (u32)__builtin_amdgcn_ds_permute((int)((mine ? __builtin_amdgcn_mbcnt_hi(h1, __builtin_amdgcn_mbcnt_lo(h0, 0u)) : 63u) * 4u), mine ? lane : 63);
            const u64 end0 = __ballot(e0 >= 128u), end1 = __ballot(e1 >= 128u);
            const u32 k0 = __builtin_amdgcn_mbcnt_hi((u32)(end0 >> 32), __builtin_amdgcn_mbcnt_lo((u32)end0, 0u));
            const u32 k1 = __builtin_amdgcn_mbcnt_hi((u32)(end1 >> 32), __builtin_amdgcn_mbcnt_lo((u32)end1, (u32)__popcll(end0)));
            const u32 l0 = (u32)__builtin_amdgcn_ds_bpermute((int)(k0 * 4u), (int)kth), l1 = (u32)__builtin_amdgcn_ds_bpermute((int)(k1 * 4u), (int)kth);
            if ((u32)lane < nexc) __hip_atomic_fetch_or(bm + (l0 & 63u) * 4u + ((e0 >> 5) & 3u), 1u << (e0 & 31u), __ATOMIC_RELAXED, __HIP_MEMORY_SCOPE_WORKGROUP);
            if ((u32)lane + 64u < nexc) __hip_atomic_fetch_or(bm + (l1 & 63u) * 4u + ((e1 >> 5) & 3u), 1u << (e1 & 31u), __ATOMIC_RELAXED, __HIP_MEMORY_SCOPE_WORKGROUP);
            CFENCE;
            const u32x4 m7 = *(volatile LAS u32x4*)(bm + lane * 4);
            asm volatile("s_waitcnt vmcnt(0)" : "+v"(sg) :: "memory");
            if (hdr & H_PLAIN) sg = sgi;
#pragma unroll
            for (int h = 0; h < 4; ++h) {
                const Msk k = msk<true>(f5[h]);
                const u32 sh = (h & 1) ? popc(f5[h - 1]) & 31u : 0u;
                const u32 f3 = spread(a3[h >> 1] >> sh, k) | (d0v[h] & ~f5[h]), f4 = spread(a4[h >> 1] >> sh, k) | (d1v[h] & ~f5[h]);
                u32 nn[4];
                from_planes(nn, f3, f4, f5[h], m7[h]);
                u32 L[4];
#pragma unroll
                for (int i = 0; i < 3; ++i) L[i] = lo[(3 * h + i) >> 2][(3 * h + i) & 3];
                L[3] = bfi(0x22222222u, L[0] << 1, bfi(0x44444444u, L[1] << 2, L[2] << 3));
#pragma unroll
                for (int i = 0; i < 4; ++i) {
                    const u32 t = (sg[h] >> i) & 0x11111111u, ng = times15(t);
                    const u32 me = bfi(0x07070707u, L[i] >> 1, nn[i] << 3) & 0x7f7f7f7fu, mo = bfi(0x07070707u, L[i] >> 5, nn[i] >> 1) & 0x7f7f7f7fu;
                    u[8 * h + 2 * i] = bfi(bfi(0x0f0f0f0fu, ng, ng << 4), (0x80808080u - me) ^ 0x80808080u, me);
                    u[8 * h + 2 * i + 1] = bfi(bfi(0x0f0f0f0fu, ng >> 4, ng), (0x80808080u - mo) ^ 0x80808080u, mo);
                }
            }
        } else {
#pragma unroll
            for (int k = 0; k < 32; ++k) u[k] = 0x80808080u;
        }
        WAIT_L(0);
        ++done;
        if (lane == 0) rw[R_DONE + pos] = done;
        if (type == T_IN) {
            st_unit(dst + lane * 16, u);
            WAIT_V(0);
            if (lane == 0) add4_sys(p.ws + d1.z, 1u);
        } else if (type != T_SKIP) {
            char* o = dst + (size_t)(ol & 15) * GN + (ol >> 4) * 16;
#pragma unroll
            for (int k = 0; k < 8; ++k)
                st16_sys<0>(o + (size_t)k * 16 * GN, (u32x4){u[4 * k], u[4 * k + 1], u[4 * k + 2], u[4 * k + 3]});

        }
    }
}

__device__ __forceinline__ void gemm_tile(const Params& p, LAS unsigned char* lds, u32 call, bool fillm, int pm, int pn, int wid, int lane) {
    const int wr = wid >> 2, wc = (wid + 2 * wr) & 3, un = wr * 4 + wc;
    int j0 = 0;
#pragma unroll
    for (int j = 1; j < 7; ++j) if (rot_of(j) <= pm) j0 = j;
    const char* Ab = p.ws + W_A + (size_t)pm * 64 * PANEL;
    const char* Bb = p.ws + (fillm ? W_X : W_B) + (size_t)pn * 64 * PANEL;
    const char* cA = p.ws + W_ACNT + (size_t)pm * 256;
    const char* cB = p.ws + (fillm ? W_XCNT : W_BCNT) + (size_t)pn * 256;
    const u32 voff = (u32)(wid * 4096 + lane * 16);
    const u32 ldsw = (u32)wid * 2048u;
    const int aoff = wr * 8192 + ((lane & 15) * 4 + ((lane >> 4) ^ ((lane >> 2) & 2))) * 16;
    int boff = 65536 + wc * 4096 + lane * 16;
    asm volatile("" : "+v"(boff));
    int ktv = kt_of(lane, p.rank, j0);
    asm volatile("" : "+v"(ktv));
    const u32 cix = (u32)((ktv & 7) * 8 + (ktv >> 3)) * 4u;
    u64 rdyA = wid == 4 ? 0ull : ~0ull, rdyB = rdyA;
    const u32 needA = call * 4u, needB = (fillm && pn == NVP - 1) ? call * (u32)(7 * FC - 4 * (NVP - 1)) : needA;
#define SA(b, h) (((b) * 2 + (h)) * 16384)
#define SB(b, h) ((4 + (b) * 2 + (h)) * 16384)
#define STAGE(buf, gbase, h) do { \
        __builtin_amdgcn_global_load_lds((const u32*)((gbase) + voff + (h) * 2048), (LAS u32*)(lds + (buf) + ldsw), 16, 0, 17); \
        __builtin_amdgcn_global_load_lds((const u32*)((gbase) + voff + (h) * 2048), (LAS u32*)(lds + (buf) + ldsw), 16, 1024, 17); } while (0)
#define LDA(dst, b, h) do { _Pragma("unroll") for (int m = 0; m < 4; ++m) _Pragma("unroll") for (int k = 0; k < 2; ++k) \
        dst[m][k] = *(const LAS bf16x8*)(lds + SA(b, h) + aoff + m * 2048 + k * 1024); } while (0)
#define LDB(dst, b, h) do { _Pragma("unroll") for (int n = 0; n < 2; ++n) _Pragma("unroll") for (int k = 0; k < 2; ++k) \
        dst[n][k] = *(const LAS bf16x8*)(lds + (SB(b, h) - 65536) + boff + k * 2048 + n * 1024); } while (0)
#define MMA(ai, bj, At, Bt) do { __builtin_amdgcn_s_setprio(1); \
        _Pragma("unroll") for (int m = 0; m < 4; ++m) _Pragma("unroll") for (int n = 0; n < 2; ++n) _Pragma("unroll") for (int k = 0; k < 2; ++k) \
            acc[ai][bj][m][n] = __builtin_amdgcn_mfma_f32_16x16x32_bf16(Bt[n][k], At[m][k], acc[ai][bj][m][n], 0, 0, 0); \
        __builtin_amdgcn_s_setprio(0); } while (0)
#define CHECK(RDY, CP, NEED, o) do { if (!((RDY >> (o)) & 1ull)) { \
        const u64 t0_ = now(); __builtin_amdgcn_s_waitcnt(0xC07F); \
        for (u32 it_ = 1;; ++it_) { \
            u32 a_ = 0; poll_win(a_, CP + cix, 0xffffull << (o)); \
            RDY |= __ballot(a_ >= NEED); \
            if ((RDY >> (o)) & 1ull) break; \
            if (spin_fail(p.ws, call, t0_, (it_ & 63u) ? 0u : ld4_wait(p.ws + W_TMO))) { rdyA = rdyB = ~0ull; gave_up = true; break; } \
        } } } while (0)
#define CHECKA(o) CHECK(rdyA, cA, needA, o)
#define CHECKB(o) CHECK(rdyB, cB, needB, o)
#define PEEK(o) do { if (((~rdyA >> (((o) + 1) & 63)) | (~rdyB >> (((o) + 2) & 63))) & 1ull) { pk = 1; u64 sv_; u32 k_; \
        asm volatile("s_mov_b64 %0, exec\n\ts_mov_b32 %1, m0\n\ts_mov_b32 m0, %4\n\ts_mov_b64 exec, %5\n\t" \
                     "global_load_lds_dword %2, off sc0 sc1\n\tglobal_load_lds_dword %3, off offset:256 sc0 sc1\n\ts_mov_b64 exec, %0\n\ts_mov_b32 m0, %1" \
                     : "=&s"(sv_), "=&s"(k_) : "v"(cA + cix), "v"(cB + cix - 256), "s"((u32)(u64)pkl), "s"(0xffffull << (o)) : "memory"); } } while (0)
#define WAIT_PK asm volatile("s_cmp_lg_u32 %0, 0\n\ts_cbranch_scc1 .Lpw%=\n\ts_waitcnt vmcnt(6)\n.Lpw%=:\n\ts_waitcnt vmcnt(8)" :: "s"(rfl(pk)) : "memory", "scc")
#define KOFF(o) ((size_t)__builtin_amdgcn_readlane(ktv, (o)) * PANEL)

    f32x4 acc[2][2][4][2];
#pragma unroll
    for (int a = 0; a < 2; ++a)
#pragma unroll
        for (int b = 0; b < 2; ++b)
#pragma unroll
            for (int m = 0; m < 4; ++m)
#pragma unroll
                for (int n = 0; n < 2; ++n) acc[a][b][m][n] = (f32x4){0.f, 0.f, 0.f, 0.f};
    bf16x8 At[4][2], B0[2][2], B1[2][2];
    u32 pk;
    volatile LAS u32* pkl = (volatile LAS u32*)(lds + LDS_MISC + 1536);
    if (wid == 4) { pkl[lane] = 0u; pkl[64 + lane] = 0u; }
    bool gave_up = false;

    CHECKA(0); CHECKB(0); CHECKB(1); CHECKA(1); CHECKB(2);
    BAR; CFENCE;
    size_t k1 = KOFF(1);
    {
        const size_t k0 = KOFF(0);
        STAGE(SB(0, 0), Bb + k0, 0); STAGE(SB(0, 1), Bb + k0, 1); STAGE(SA(0, 0), Ab + k0, 0);
        STAGE(SB(1, 0), Bb + k1, 0); STAGE(SA(0, 1), Ab + k0, 1); STAGE(SB(1, 1), Bb + k1, 1);
        WAIT_V(6); BAR;
        if (wr == 1) BAR;
    }
#define BOOK(oa) do { pk = 0; if (__builtin_expect((oa) >= lim, 0)) { \
        rdyA |= __ballot(pkl[lane] >= needA); rdyB |= __ballot(pkl[64 + lane] >= needB); CHECKA(oa); CHECKB(((oa) + 1) & 63); PEEK(oa); \
        lim = min(~rdyA ? __builtin_ctzll(~rdyA) - 1 : 64, ~rdyB ? __builtin_ctzll(~rdyB) - 2 : 64); } } while (0)
    size_t k2 = KOFF(2);
    int lim = wid == 4 ? 0 : 64;
    for (int t = 0; t < 64; t += 2) {
        const int o2 = (t + 2) & 63, o3 = o2 + 1;
        LDB(B0, 0, 0); SCHED; LDA(At, 0, 0); STAGE(SA(1, 0), Ab + k1, 0);
        WAIT_L(8); BAR; WAIT_L(0); MMA(0, 0, At, B0); BAR; SCHED;
        LDB(B1, 0, 1); STAGE(SB(0, 0), Bb + k2, 0);
        WAIT_V(6); BAR; WAIT_L(0); MMA(0, 1, At, B1); BAR;
        LDA(At, 0, 1); STAGE(SA(1, 1), Ab + k1, 1);
        BAR; WAIT_L(0); MMA(1, 0, At, B0); BAR; SCHED;
        BOOK(o2); STAGE(SB(0, 1), Bb + k2, 1);
        const size_t k3 = KOFF(o3);
        WAIT_PK; BAR; MMA(1, 1, At, B1); BAR;
        LDB(B0, 1, 0); SCHED; LDA(At, 1, 0); STAGE(SA(0, 0), Ab + k2, 0);
        WAIT_L(8); BAR; WAIT_L(0); MMA(0, 0, At, B0); BAR; SCHED;
        LDB(B1, 1, 1); STAGE(SB(1, 0), Bb + k3, 0);
        WAIT_V(6); BAR; WAIT_L(0); MMA(0, 1, At, B1); BAR;
        LDA(At, 1, 1); STAGE(SA(0, 1), Ab + k2, 1);
        BAR; WAIT_L(0); MMA(1, 0, At, B0); BAR; SCHED;
        BOOK(o3); STAGE(SB(1, 1), Bb + k3, 1);
        k1 = k3; k2 = KOFF((o3 + 1) & 63);
        WAIT_PK; BAR; MMA(1, 1, At, B1); BAR;
    }
    int tz = threadIdx.x; asm volatile("" : "+v"(tz));
    lane = tz & 63;
    const int tile = pm * 8 + pn;
    volatile LAS u32* red = (volatile LAS u32*)(lds + LDS_MISC + 1024);
    u32 sgn[4] = {0u, 0u, 0u, 0u};
    if (!fillm) {
#pragma unroll
        for (int m = 0; m < 4; ++m)
#pragma unroll
            for (int t = 7; t >= 0; --t)
#pragma unroll
                for (int i = 3; i >= 0; --i)
                    sgn[m] = __builtin_amdgcn_alignbit(sgn[m], __float_as_uint(acc[i >> 1][t >> 2][m][(t >> 1) & 1][(i & 1) * 2 + (t & 1)]), 31u);
        st16_sys<0>(p.ws + W_PS + (size_t)tile * USLOT + un * 1024 + lane * 16, (u32x4){sgn[0], sgn[1], sgn[2], sgn[3]});
        if (wr == 0) BAR;
        float mx = 0.f;
#pragma unroll
        for (int a = 0; a < 2; ++a)
#pragma unroll
            for (int b = 0; b < 2; ++b)
#pragma unroll
                for (int m = 0; m < 4; ++m)
#pragma unroll
                    for (int n = 0; n < 2; ++n)
#pragma unroll
                        for (int r = 0; r < 4; ++r) mx = fmaxf(mx, fabsf(acc[a][b][m][n][r]));
        const u32 wm = wave_umax(__float_as_uint(mx));
        if (lane == 0) red[wid] = wm;
        u32 tk = 0;
        if (wid == 1 && pn < 7) { ticket_issue(tk, p.ws + W_SQ); asm volatile("s_waitcnt vmcnt(0)" : "+v"(tk) :: "memory"); }
        const u32 ord = (rdl(tk, 0) - (call - 1u) * 112u) & 127u;
        WAIT_V(0); WAIT_L(0); BAR; CFENCE;
        if (wid == 1 && pn < 7 && lane < NR && lane != p.rank)
            st8_sys(peer_ws(lds, (u32)lane) + W_HS + (size_t)((u32)p.rank * 128u + ord) * 8, (u32x2){H_RAW | 512u | ((u32)tile << 10), call});
        if (tz == 0) {
            const u32x4 r0 = *(volatile LAS u32x4*)red, r1 = *(volatile LAS u32x4*)(red + 4);
            const u32 t = max(max(max(r0.x, r0.y), max(r0.z, r0.w)), max(max(r1.x, r1.y), max(r1.z, r1.w)));
            st8_sys(p.ws + W_AMAX + (size_t)tile * 8, (u32x2){t, call});
        }
    } else if (wr == 0) BAR;
    if (wid == 0) {
        const char* src = p.ws + W_GMAX + (blockIdx.x & (GCOPY - 1)) * 256 + (lane & 7) * 8;
        const u64 t0 = now();
        u32x2 gq;
        bool dd = false;
        for (u32 it = 1;; ++it) {
            gq = ld8_wait(src);
            if (__all(gq.y == call)) break;
            if (spin_fail(p.ws, call, t0, (it & 63u) ? 0u : ld4_wait(p.ws + W_TMO))) { dd = true; break; }
        }
        const u32 wm = wave_umax(gq.x);
        if (lane == 0) red[8] = dd ? 0xffffffffu : wm;
    } else if (wid == 4) {
        if (lane == 0) red[9] = gave_up ? 1u : 0u;
        if (lane < 8) red[16 + lane] = 0u;
        WAIT_V(0);
    } else {
        WAIT_V(0);
    }
    WAIT_L(0); BAR; CFENCE;
    const u32 am = rfl(red[8]);
    const bool dead = am == 0xffffffffu || rfl(red[9]) != 0u;
    float amax = __uint_as_float(am);
    if (fillm) {
#pragma unroll
        for (int a = 0; a < 2; ++a)
#pragma unroll
            for (int b = 0; b < 2; ++b)
#pragma unroll
                for (int m = 0; m < 4; ++m)
#pragma unroll
                    for (int n = 0; n < 2; ++n)
#pragma unroll
                        for (int r = 0; r < 4; ++r) acc[a][b][m][n][r] = __builtin_amdgcn_fmed3f(acc[a][b][m][n][r], -amax, amax);
    }
    if (amax < 1e-30f) {
        amax *= 0x1p64f;
#pragma unroll
        for (int a = 0; a < 2; ++a)
#pragma unroll
            for (int b = 0; b < 2; ++b)
#pragma unroll
                for (int m = 0; m < 4; ++m)
#pragma unroll
                    for (int n = 0; n < 2; ++n) acc[a][b][m][n] *= 0x1p64f;
    }
    const float inv = amax > 0.f ? 127.0f / amax : 0.f;
    u32 q[32];
#pragma unroll
    for (int m = 0; m < 4; ++m)
#pragma unroll
        for (int a = 0; a < 2; ++a)
#pragma unroll
            for (int r = 0; r < 4; ++r) {
                const u32 t0 = __float_as_uint(fmaf(acc[a][0][m][0][r], inv, 12582912.f)), t1 = __float_as_uint(fmaf(acc[a][0][m][1][r], inv, 12582912.f));
                const u32 t2 = __float_as_uint(fmaf(acc[a][1][m][0][r], inv, 12582912.f)), t3 = __float_as_uint(fmaf(acc[a][1][m][1][r], inv, 12582912.f));
                q[(m * 2 + a) * 4 + r] = __builtin_amdgcn_perm(t1, t0, 0x0c0c0400u) | __builtin_amdgcn_perm(t3, t2, 0x04000c0cu);
            }
    if (dead) {
#pragma unroll
        for (int k = 0; k < 32; ++k) q[k] = 0x80808080u;
    }
    int col = p.rank * NS + pn * 256 + wc * 64;
    if (!fillm) {
        char* slot = p.ws + W_PO + (size_t)(tile * 8 + un) * USLOT;
        LAS unsigned char* enc = lds + LDS_ENC + wid * 2048;
        *(volatile LAS u32x4*)(enc + lane * 16) = (u32x4){0u, 0u, 0u, 0u};
        *(volatile LAS u32x4*)(enc + 1024 + lane * 16) = (u32x4){0u, 0u, 0u, 0u};
        char* dp = slot + lane * 16;
        u32 hdr;
        if (pn == 7 && wc >= 4 - FC) {
            hdr = 0;
        } else if (dead || (pn == 7 && pm >= RLAST)) {
            st_unit(dp, q);
            hdr = H_RAW | 512u;
        } else {
            u32 L[16], G[16], f3[4], f4[4], f5[4], m7[4];
#pragma unroll
            for (int j = 0; j < 16; ++j) {
                u32 mg[2];
#pragma unroll
                for (int e = 0; e < 2; ++e) {
                    const u32 v = q[2 * j + e], t = v & 0x80808080u;
                    mg[e] = bfi((t - (t >> 7)) | t, 0x80808080u - (v & 0x7f7f7f7fu), v);
                }
                L[j] = bfi(0x0f0f0f0fu, mg[0] << 1, mg[1] << 5);
                G[j] = bfi(0x0f0f0f0fu, mg[0] >> 3, mg[1] << 1);
            }
            u32 ca = 0, cb = 0, cc = 0, wide = 0;
#pragma unroll
            for (int h = 0; h < 4; ++h) {
                to_planes(G + 4 * h, f3[h], f4[h], f5[h], m7[h]);
                ca += popc(f5[h] | (f4[h] & f3[h])) | (popc(m7[h]) << 16); cb += popc(f5[h]) | (popc(f5[h] & (f4[h] | f3[h])) << 16); cc += popc(f5[h] & f4[h]);
                wide = max(wide, popc(f5[h]));
            }
            const u32 ia = scan_add(ca), ib = scan_add(cb), ic = scan_add(cc), ta = rdl(ia, 63), tb = rdl(ib, 63), exc = ia - ca;
            const u32 bb = ta & 0xffffu, bc = bb + (tb & 0xffffu), bd = bc + (tb >> 16);
            const u32 sdw = (bd + rdl(ic, 63) + 31u) >> 5, nexc = ta >> 16;
            const bool raw = nexc > 128u || sdw > MAXSDW || __any(wide > 15u);
            const u32 n16 = (sdw + 2u + ((nexc + 3u) >> 2) + 3u) >> 2;
            hdr = raw ? H_RAW | 512u : (pn == 7 ? H_PLAIN | (384u + n16) : 320u + n16) | (nexc << 17);
            if (un & 1) {
                u32 ha;
                while ((ha = rfl(red[15 + un])) == 0u) __builtin_amdgcn_s_sleep(1);
                slot -= (512u - (ha & 1023u)) * 16u;
                dp = slot + lane * 16;
            } else if (lane == 0) red[16 + un] = hdr;
            if (raw) {
                st_unit(dp, q);
            } else {
                u32 w[12];
#pragma unroll
                for (int h = 0; h < 4; ++h)
#pragma unroll
                    for (int i = 0; i < 3; ++i) w[3 * h + i] = bfi(0x11111111u, L[4 * h + 3] >> (i + 1), L[4 * h + i]);
                st16_sys<0>(dp, (u32x4){w[0], w[1], w[2], w[3]}); st16_sys<1024>(dp, (u32x4){w[4], w[5], w[6], w[7]});
                st16_sys<2048>(dp, (u32x4){w[8], w[9], w[10], w[11]});
                st16_sys<3072>(dp, (u32x4){f3[0] | f5[0], f3[1] | f5[1], f3[2] | f5[2], f3[3] | f5[3]});
                st16_sys<0>(dp + 4096, (u32x4){f4[0] | f5[0], f4[1] | f5[1], f4[2] | f5[2], f4[3] | f5[3]});
                if (pn == 7) st16_sys<1024>(dp + 4096, (u32x4){sgn[0], sgn[1], sgn[2], sgn[3]});
                u32 oa = exc & 0xffffu, ob = bb + ((ib - cb) & 0xffffu), oc = bc + ((ib - cb) >> 16), od = bd + ic - cc;
#pragma unroll
                for (int h = 0; h < 4; ++h) {
                    const u32 e3 = f5[h] | (f4[h] & f3[h]);
                    bit_put((LAS u32*)enc, oa, press(f5[h], msk<false>(e3))); oa += popc(e3);
                }
#pragma unroll
                for (int h = 0; h < 4; h += 2) {
                    const Msk k0 = msk<false>(f5[h]), k1 = msk<false>(f5[h + 1]);
                    const u32 n0 = popc(f5[h]);
                    const u32 a3 = press(f3[h] & f5[h], k0) | (press(f3[h + 1] & f5[h + 1], k1) << n0);
                    const u32 a4 = press(f4[h] & f5[h], k0) | (press(f4[h + 1] & f5[h + 1], k1) << n0);
                    bit_put((LAS u32*)enc, ob, a3 | a4); ob += n0 + popc(f5[h + 1]);
                    bit_put((LAS u32*)enc, oc, press(a4, msk<false>(a3 | a4))); oc += popc(a3 | a4);
                    bit_put((LAS u32*)enc, od, press(a3 & a4, msk<false>(a4))); od += popc(a4);
                }
                volatile LAS unsigned char* list = (volatile LAS unsigned char*)(enc + sdw * 4u + 8u);
                const u64 has = __ballot((ca >> 16) != 0u);
                if (lane < 2) ((volatile LAS u32*)enc)[sdw + (u32)lane] = (u32)(has >> (32 * lane));
                u32 s = exc >> 16;
                const u32 last = (ia >> 16) - 1u;
#pragma unroll
                for (int h = 0; h < 4; ++h) {
                    u32 mm = m7[h];
                    while (mm) { list[s] = (unsigned char)(h * 32 + __builtin_ctz(mm) + (s == last ? 128 : 0)); mm &= mm - 1u; ++s; }
                }
                tail_out(slot + (pn == 7 ? 6144 : 5120), enc, n16, lane);
            }
        }
        if (lane == 0) red[16 + un] = hdr;
        WAIT_V(0);
        WAIT_L(0); BAR; CFENCE;
        if (tz < 64 && (tz >> 3) != p.rank)
            st8_sys(peer_ws(lds, (u32)(tz >> 3)) + W_HO + (size_t)((p.rank * 128 + tile) * 8 + (tz & 7)) * 8, (u32x2){red[16 + (tz & 7)], call});
    } else {
        const int g = pn * 4 + wc;
        col = g < 7 * FC ? ((p.rank + 1 + g / FC) & 7) * NS + 7 * 256 + (4 - FC + g % FC) * 64 : -1;
    }
    if (col >= 0) {
        signed char* o = p.out + (size_t)(pm * 256 + wr * 128 + (lane & 15)) * GN + col + (lane >> 4) * 16;
#pragma unroll
        for (int k = 0; k < 8; ++k)
            st16_sys<0>(o + (size_t)k * 16 * GN, (u32x4){q[4 * k], q[4 * k + 1], q[4 * k + 2], q[4 * k + 3]});
    }
#undef SA
#undef SB
#undef STAGE
#undef LDA
#undef LDB
#undef MMA
#undef CHECK
#undef CHECKA
#undef CHECKB
#undef PEEK
#undef BOOK
#undef WAIT_PK
#undef KOFF
}

__global__ void __launch_bounds__(BLOCK, 2) fused_gemm_q8(Params p) {
    __shared__ __attribute__((aligned(16))) unsigned char smem[LDS_BYTES];
    LAS unsigned char* lds = (LAS unsigned char*)smem;
    const int tid = threadIdx.x, wid = (int)rfl((u32)tid >> 6), lane = tid & 63;
    const int xcd = blockIdx.x & 7, idx = blockIdx.x >> 3;
    const u64 tstart = now();
    volatile LAS u32* misc = (volatile LAS u32*)(lds + LDS_MISC);
    LAS unsigned char* enc = lds + LDS_ENC + wid * 2048;
    if (wid == 7) {
        if (lane < 4) misc[lane] = 0u;
        if (lane < 16) ring_words(lds)[lane] = 0u;
        ((volatile LAS u32*)(lds + LDS_MISC + 1280))[lane] = 0u;
        WAIT_L(0);
    }
    BAR; CFENCE;
    int i0 = -1, r0 = 0, mode0 = 2;
    const int u0 = (int)blockIdx.x & 3;
    if (idx < 16) {
        const int n = (wid * 128 + (int)blockIdx.x) >> 2;
        if (n < 24 * 8) { r0 = n / 24; i0 = 56 + n - r0 * 24; }
    } else if (wid < 3) {
        constexpr int NE = 7 * (8 - NPUSH) + 17;
        const int n = (wid * 128 + (int)blockIdx.x - 128) >> 2;
        if (n < NE) { mode0 = 1; i0 = n < NE - 17 ? (n % 7) * 8 + NPUSH + n / 7 : (n < NE - 1 ? 56 + n - (NE - 17) : 79); }
        else if (n < 73) { mode0 = 4; const int m = n - NE; i0 = (m % 7) * 8 + m / 7; }
    }
    if (i0 >= 0) {
        const PanelId d0 = panel_of(r0, i0, p.rank);
        if (mode0 == 2) { while (now() - tstart < GDLY) __builtin_amdgcn_s_sleep(1); }
        f32x4 f[16];
        unit_load(p, d0, u0, lane, f);
        WAIT_V(0); PIN_F(f);
        const u32 hdr = unit_pack(p, lds, enc, d0, u0, lane, f, mode0);
        u32 call0;
        while ((call0 = rfl(misc[2])) == 0u) __builtin_amdgcn_s_sleep(1);
        WAIT_V(0);
        if (lane < NR) unit_announce(p, lds, d0, u0, lane, hdr, call0, mode0);
    } else {
        if (wid == 7) {
            u32 tk; u32x2 pw;
            ticket_issue(tk, p.ws + W_SEQ + (size_t)blockIdx.x * 128);
            ld8_sys(pw, (const char*)p.ws_all + (lane & 7) * 8);
            asm volatile("s_waitcnt vmcnt(0)" : "+v"(tk), "+v"(pw) :: "memory");
            if (lane < NR) *(volatile LAS u32x2*)(lds + LDS_MISC + 64 + lane * 8) = pw;
            const u32 call7 = rfl(tk) + 1u;
            if (lane == 0) misc[2] = call7;
            if (blockIdx.x == 0) {
                if (lane < 32) st8_sys(p.ws + W_ROT + lane * 256, (u32x2){(u32)__builtin_amdgcn_s_getreg(20 | (3 << 11)) & 7u, call7});
            } else if (idx >= 16) {
                const u64 t0 = now();
                if (idx >= 24) { while (now() - tstart < 300) __builtin_amdgcn_s_sleep(4); }
                u32x2 ro;
                for (u32 it = 1;; ++it) {
                    ro = ld8_wait(p.ws + W_ROT + (blockIdx.x & 31) * 256);
                    if (rfl(ro.y) == call7) break;
                    if (spin_fail(p.ws, call7, t0, (it & 63u) ? 0u : ld4_wait(p.ws + W_TMO))) { ro.x = 8u; break; }
                }
                if (lane == 0) misc[1] = ro.x;
            }
            if (lane == 0) misc[0] = call7;
        }
    }
    u32 call;
    while ((call = rfl(misc[0])) == 0u) __builtin_amdgcn_s_sleep(1);
    CFENCE;

    const bool is_gemm = idx < 16;
    const int c = idx - 16;
    const u32 rot0 = is_gemm ? 0u : rfl(misc[1]);
    if (rot0 > 7u) return;
    const int jl = (peer_of((int)((xcd + rot0) & 7u), p.rank) - p.rank - 1) & 7;
    const int pm = (xcd >> 1) * 4 + (idx >> 2), pn = (xcd & 1) * 4 + (idx & 3);
    if (!is_gemm && !(jl < 7 && c < NPW)) {
        const int me = idx >= 24 ? (int)blockIdx.x - 192 : 64 + c, unit = wid & 3;
        int e = (wid >> 2) * NSPARE + me;
        volatile LAS u32* hb = (volatile LAS u32*)(lds + LDS_MISC + 1280 + (wid >> 2) * 128);
        constexpr int NPAN = 7 * 73;
        auto panel_at = [&](int v) { const int r = v / 73, i = v - r * 73; return panel_of(1 + r, i < 72 ? i : 79, p.rank); };
        PanelId d = panel_at(e);
        f32x4 f[16];
        while (now() - tstart < (wid < 4 ? SPD0 : SPD1)) __builtin_amdgcn_s_sleep(4);
        unit_load(p, d, unit, lane, f);
        for (;;) {
            WAIT_V(0); PIN_F(f);
            const u32 hdr = unit_pack(p, lds, enc, d, unit, lane, f, 1);
            e += 2 * NSPARE;
            const PanelId dn = panel_at(e);
            if (e < NPAN) { unit_load(p, dn, unit, lane, f); WAIT_V(16); } else { WAIT_V(0); }
            u32 arr = 0;
            if (lane == 0) { hb[4 + unit] = hdr; arr = __hip_atomic_fetch_add((LAS u32*)hb, 1u, __ATOMIC_RELAXED, __HIP_MEMORY_SCOPE_WORKGROUP); }
            if (rfl(arr) == 3u && lane < 4 * NR) unit_announce(p, lds, d, lane & 3, lane >> 2, hb[4 + (lane & 3)], call, 1);
            hb += 8;
            if (e >= NPAN) break;
            d = dn;
        }
        WAIT_V(0);
        if (blockIdx.x == GRID - 1 && wid == 0) {
            const u64 t0 = now();
            while (now() - tstart < 6000) __builtin_amdgcn_s_sleep(16);
            for (u32 it = 1;; ++it) {
                const u32x4 gq = ld16_wait(p.ws + W_AMAX + lane * 16);
                if (__all(gq.y == call && gq.w == call)) {
                    const u32 wm = wave_umax(max(gq.x, gq.z));
                    if (lane < NR * GCOPY) st8_sys(peer_ws(lds, (u32)lane & 7u) + W_GMAX + (lane >> 3) * 256 + p.rank * 8, (u32x2){wm, call});
                    break;
                }
                if (spin_fail(p.ws, call, t0, (it & 63u) ? 0u : ld4_wait(p.ws + W_TMO))) break;
            }
        }
        if (jl == 7 || c >= NPW + NH) return;
        while (now() - tstart < 7500) __builtin_amdgcn_s_sleep(32);
    }
    if (!is_gemm) {
        if (wid == 7) pull_schedule(p, lds, call, jl, c, wid, lane);
        else if (wid == 6) pull_load(p, lds, wid, lane);
        else pull_decode(p, lds, wid, lane);
        return;
    }
#pragma clang loop unroll(disable)
    for (int pass = 0; pass < (pn < NVP ? 2 : 1); ++pass) {
        int tz = threadIdx.x; asm volatile("" : "+v"(tz));
        gemm_tile(p, lds, call, pass == 1, pm, pn, wid, tz & 63);
    }
}

extern "C" void kernel_launch(int rank, int nranks, ncclComm_t comm, hipStream_t stream,
                              void* const* d_in, const int* in_sizes, int n_in,
                              void* d_out, int out_size,
                              const int* in_dtypes, int out_dtype,
                              void* const* d_out_all,
                              void* const* d_ws_all, size_t ws_bytes) {
    (void)comm; (void)in_dtypes; (void)out_dtype; (void)d_out_all;
    if (nranks != NR || n_in < 2 || ws_bytes < W_END || in_sizes[0] != GM * KS || in_sizes[1] != KS * GN || out_size != GM * GN ||
        (unsigned)rank >= (unsigned)NR) {
        fprintf(stderr, "unexpected problem geometry\n");
        return;
    }
    Params p;
    p.x = (const float*)d_in[0];
    p.w = (const float*)d_in[1];
    p.out = (signed char*)d_out;
    p.ws = (char*)d_ws_all[rank];
    p.ws_all = (char* const*)d_ws_all;
    p.rank = rank;
    p.pad = 0;
    hipLaunchKernelGGL(fused_gemm_q8, dim3(GRID), dim3(BLOCK), 0, stream, p);
}
```
